```python
import numpy as np

kernel_name = "gpu_ag_gemm_m4096_k4096_n16384_f32_reluq8"


ABSMAX_RTOL = 0.02

COMPUTE_DTYPE = "float32"
EPILOGUE = "relu_quant_int8"
IN_QUANT = None
OUT_QUANT = "int8"
IN_DTYPES = {"x": "float32", "w_mat": "float32"}
WORLD_SIZE = 8

def setup_inputs():

    x = np.random.randn(4096, 4096).astype(np.float32)
    w_mat = (np.random.randn(4096, 16384) / np.sqrt(4096)).astype(np.float32)
    return {"x": x, "w_mat": w_mat}

def _quant_dequant(y, amax):
    scale = amax / 127.0
    q = np.clip(np.round(y / scale), -127.0, 127.0)
    return (q * scale).astype(np.float32)
def _epilogue(y):
    y = np.maximum(y, 0.0)
    return _quant_dequant(y, np.abs(y).max())

def reference(x, w_mat):
    y = (x @ w_mat).astype(np.float32)
    return _epilogue(y)
```

```cpp
#include <hip/hip_runtime.h>
#include <rccl/rccl.h>
#include <cstdio>

#define HIP_CHECK(expr) do { hipError_t _e = (expr); if (_e != hipSuccess) { \
    fprintf(stderr, "HIP error %s at %s:%d\n", hipGetErrorString(_e), __FILE__, __LINE__); \
    return; } } while (0)

namespace {

#define LAS __attribute__((address_space(3)))
#define GAS __attribute__((address_space(1)))
typedef unsigned u32x4 __attribute__((ext_vector_type(4)));
typedef unsigned u32x2 __attribute__((ext_vector_type(2)));
typedef float f32x4 __attribute__((ext_vector_type(4)));
typedef __bf16 bf16x8 __attribute__((ext_vector_type(8)));
typedef unsigned long long u64;

constexpr int NR = 8;
constexpr int KDIM = 4096, NL = 2048;
constexpr int KS = 64;
constexpr int NWG = 256;
constexpr int UNIT = 8192;

constexpr size_t XS_OFF = 0;
constexpr size_t XS_SRC = (size_t)8 * KS * UNIT;
constexpr size_t WB_OFF = XS_OFF + NR * XS_SRC;
constexpr size_t XF_OFF = WB_OFF + (size_t)8 * KS * 32768;
constexpr size_t WF_OFF = XF_OFF + (size_t)NR * 8 * KS * 8;
constexpr size_t SQ_OFF = WF_OFF + (size_t)8 * KS * 4 * 4;
constexpr size_t A1_OFF = SQ_OFF + (size_t)NWG * 64;
constexpr size_t A2_OFF = A1_OFF + (size_t)2 * NWG * 8;
constexpr size_t TM_OFF = A2_OFF + (size_t)2 * 8 * 128;
constexpr size_t DG_OFF = TM_OFF + 128;
constexpr size_t WS_NEED = DG_OFF + (size_t)NWG * 8 * 128 * 8;

constexpr int NST = 3;
constexpr int LA_OFF = 0, LA_SLOT = 16384;
constexpr int LB_OFF = NST * LA_SLOT, LB_SLOT = 32768;
constexpr int LC_OFF = LB_OFF + NST * LB_SLOT;
constexpr int C_FULL = 0;
constexpr int C_DONE = 4;
constexpr int C_MAX = 8, C_CNT = 9, C_GMAX = 10, C_RDY = 11;
constexpr int LDS_BYTES = LC_OFF + 64;

constexpr u64 DEADLINE = 40000000ull;

struct Params {
    const float* x; const float* w; signed char* out;
    char* ws[NR];
    int rank; int pad;
};

__device__ __forceinline__ u64 rt() { return __builtin_amdgcn_s_memrealtime(); }
__device__ __forceinline__ unsigned rfl(unsigned v) { return __builtin_amdgcn_readfirstlane(v); }
__device__ __forceinline__ unsigned cvt_pk(float lo, float hi) {
    typedef float f2 __attribute__((ext_vector_type(2))); typedef __bf16 b2 __attribute__((ext_vector_type(2)));
    f2 v = {lo, hi}; return __builtin_bit_cast(unsigned, __builtin_convertvector(v, b2));
}
__device__ __forceinline__ __amdgpu_buffer_rsrc_t mkrsrc(const void* p, unsigned bytes) {
    return __builtin_amdgcn_make_buffer_rsrc((void*)p, 0, bytes, 0x00020000);
}
#define VMWAIT0() asm volatile("s_waitcnt vmcnt(0)" ::: "memory")
#define LGKM0() asm volatile("s_waitcnt lgkmcnt(0)" ::: "memory")
#define SCHED() __builtin_amdgcn_sched_barrier(0)

struct Tmo {
    GAS unsigned* word; unsigned call;
    __device__ __forceinline__ bool expired(u64 t0) const {
        if (rfl(__hip_atomic_load(word, __ATOMIC_RELAXED, __HIP_MEMORY_SCOPE_AGENT)) == call || rt() - t0 > DEADLINE) {
            __hip_atomic_store(word, call, __ATOMIC_RELAXED, __HIP_MEMORY_SCOPE_AGENT);
            return true;
        }
        return false;
    }
};

#define STAMP(i) do {} while (0)

__device__ __forceinline__ void convert_w(const float* __restrict__ w, char* me, int ct, int t, int q, int lane, unsigned call) {
    const int n = lane & 15, g = lane >> 4;
    const float* src = w + (size_t)(t * 64 + g * 4) * NL + ct * 256 + q * 64 + n * 4;
    f32x4 v[16];
#pragma unroll
    for (int i = 0; i < 4; ++i)
#pragma unroll
        for (int c = 0; c < 4; ++c) v[i * 4 + c] = __builtin_nontemporal_load((const f32x4*)(src + (size_t)(i * 16 + c) * NL));
    const __amdgpu_buffer_rsrc_t rs = mkrsrc(me + WB_OFF + ((size_t)ct * KS + t) * 32768, 32768);
#pragma unroll
    for (int kh = 0; kh < 2; ++kh)
#pragma unroll
        for (int c = 0; c < 4; ++c) {
            u32x4 o;
            o.x = cvt_pk(v[kh * 8 + 0][c], v[kh * 8 + 1][c]); o.y = cvt_pk(v[kh * 8 + 2][c], v[kh * 8 + 3][c]);
            o.z = cvt_pk(v[kh * 8 + 4][c], v[kh * 8 + 5][c]); o.w = cvt_pk(v[kh * 8 + 6][c], v[kh * 8 + 7][c]);
            __builtin_amdgcn_raw_buffer_store_b128(o, rs, lane * 16 + (kh * 16 + q * 4 + c) * 1024, 0, 17);
        }
    VMWAIT0();
    if (lane == 0) __hip_atomic_store((GAS unsigned*)(me + WF_OFF) + (ct * KS + t) * 4 + q, call, __ATOMIC_RELAXED, __HIP_MEMORY_SCOPE_SYSTEM);
}

__global__ void __launch_bounds__(512, 2) ag_gemm(Params p) {
    extern __shared__ __attribute__((aligned(16))) char lds_raw[];
    LAS char* lds = (LAS char*)lds_raw;
    volatile LAS unsigned* ctl = (volatile LAS unsigned*)(lds + LC_OFF);
    const int lane = threadIdx.x & 63, wave = rfl(threadIdx.x >> 6);
    const int bx = blockIdx.x, xcd = bx & 7, idx = bx >> 3;
    const int src = idx & 7, sub = xcd >> 1, ct = (xcd & 1) * 4 + (idx >> 3), rtile = src * 4 + sub;
    const int rank = p.rank;
    char* me = p.ws[rank & 7];
    const bool local = (src == rank);

    if (threadIdx.x < 16) ctl[threadIdx.x] = 0u;
    LGKM0();
    __builtin_amdgcn_s_barrier();
    STAMP(0);
    GAS unsigned* seqw = (GAS unsigned*)(me + SQ_OFF + (size_t)bx * 64);

    if (wave == 7 && local) {
        const int pi = xcd * 4 + (idx >> 3), rb = pi & 7, t0 = pi >> 3;
        const float* xs = p.x + (size_t)(rb * 64 + (lane & 15)) * KDIM + (lane >> 4) * 4;
        f32x4 v[16];
#pragma unroll
        for (int b = 0; b < 4; ++b)
#pragma unroll
            for (int i = 0; i < 4; ++i) v[b * 4 + i] = *(const f32x4*)(xs + (size_t)b * 16 * KDIM + t0 * 64 + i * 16);
        const unsigned call = rfl(__hip_atomic_load(seqw, __ATOMIC_RELAXED, __HIP_MEMORY_SCOPE_AGENT)) + 1u;
        __amdgpu_buffer_rsrc_t rs[NR];
#pragma unroll
        for (int q = 0; q < NR; ++q) rs[q] = mkrsrc(p.ws[(rank + 1 + q) & 7] + XS_OFF + (size_t)rank * XS_SRC, (unsigned)XS_SRC);
        GAS u64* flag = (GAS u64*)(p.ws[lane & 7] + XF_OFF) + (rank * 8 + rb) * KS;
        u32x4 o[8];
        auto pack = [&]() {
#pragma unroll
            for (int kh = 0; kh < 2; ++kh)
#pragma unroll
                for (int b = 0; b < 4; ++b) {
                    const f32x4 lo = v[b * 4 + kh * 2], hi = v[b * 4 + kh * 2 + 1];
                    u32x4 r; r.x = cvt_pk(lo[0], lo[1]); r.y = cvt_pk(lo[2], lo[3]); r.z = cvt_pk(hi[0], hi[1]); r.w = cvt_pk(hi[2], hi[3]);
                    o[kh * 4 + b] = r;
                }
        };
        pack();
        for (int n = 0; n < 16; ++n) {
            const int t = t0 + n * 4, tn = (n < 15) ? t + 4 : t;
            SCHED();
#pragma unroll
            for (int b = 0; b < 4; ++b)
#pragma unroll
                for (int i = 0; i < 4; ++i) v[b * 4 + i] = *(const f32x4*)(xs + (size_t)b * 16 * KDIM + tn * 64 + i * 16);
            SCHED();
            const int voff = (rb * KS + t) * UNIT + lane * 16;
#pragma unroll
            for (int j = 0; j < 8; ++j)
#pragma unroll
                for (int q = 0; q < NR; ++q) __builtin_amdgcn_raw_buffer_store_b128(o[j], rs[q], voff + j * 1024, 0, 17);
            SCHED();
            pack();
            SCHED();
            VMWAIT0();
            if (lane < NR) __hip_atomic_store(flag + t, (u64)call, __ATOMIC_RELAXED, __HIP_MEMORY_SCOPE_SYSTEM);
            STAMP(1 + n);
        }
        return;
    }

    const unsigned call = rfl(__hip_atomic_load(seqw, __ATOMIC_RELAXED, __HIP_MEMORY_SCOPE_AGENT)) + 1u;
    const Tmo tmo{(GAS unsigned*)(me + TM_OFF), call};
    convert_w(p.w, me, ct, rtile * 2 + (wave >> 2), wave & 3, lane, call);
    if (local && wave == 6) convert_w(p.w, me, ct, rtile * 2 + 1, 3, lane, call);
    STAMP(1);
    const unsigned par = call & 1u;

    if (wave < 4) {
        const int wm = wave & 1, wn = wave >> 1;
        f32x4 acc[4][8];
#pragma unroll
        for (int b = 0; b < 4; ++b)
#pragma unroll
            for (int n = 0; n < 8; ++n) acc[b][n] = (f32x4){0.f, 0.f, 0.f, 0.f};
        bf16x8 pa[4], pb[8], qa[4], qb[8];
        const int abase = LA_OFF + wm * 8192 + lane * 16, bbase = LB_OFF + wn * 8192 + lane * 16;
        auto ready = [&](int t) -> bool {
            const u32x4 f = *(volatile LAS u32x4*)(lds + LC_OFF + C_FULL * 4);
            const unsigned fa = rfl(wm ? f.y : f.x), fb = rfl(f.z);
            return fa > (unsigned)t && fb > (unsigned)t;
        };
        auto wait_ready = [&](int t) {
            unsigned sp = 0; const u64 t0 = rt();
            while (!ready(t)) { __builtin_amdgcn_s_sleep(1); if ((++sp & 1023u) == 0u && tmo.expired(t0)) break; }
        };
#define LOADF(A, B, slot, kh) do { \
            _Pragma("unroll") for (int b = 0; b < 4; ++b) A[b] = *(const LAS bf16x8*)(lds + abase + (slot) * LA_SLOT + (kh) * 4096 + b * 1024); \
            _Pragma("unroll") for (int n = 0; n < 8; ++n) B[n] = *(const LAS bf16x8*)(lds + bbase + (slot) * LB_SLOT + (kh) * 16384 + n * 1024); } while (0)
#define MMA(A, B) do { \
            _Pragma("unroll") for (int n = 0; n < 8; ++n) \
            _Pragma("unroll") for (int b = 0; b < 4; ++b) acc[b][n] = __builtin_amdgcn_mfma_f32_16x16x32_bf16(B[n], A[b], acc[b][n], 0, 0, 0); } while (0)
#define INTERLEAVE() do { _Pragma("unroll") for (int z = 0; z < 12; ++z) { \
            __builtin_amdgcn_sched_group_barrier(0x8, 2, 0); __builtin_amdgcn_sched_group_barrier(0x100, 1, 0); } \
            __builtin_amdgcn_sched_group_barrier(0x8, 8, 0); } while (0)
        wait_ready(0);
        STAMP(2);
        asm volatile("" ::: "memory");
        LOADF(pa, pb, 0, 0);
        int slot = 0;
        for (int t = 0; t < KS; ++t) {
            SCHED();
            LOADF(qa, qb, slot, 1);
            MMA(pa, pb);
            INTERLEAVE();
            SCHED();
            LGKM0();
            ctl[C_DONE + wave] = (unsigned)(t + 1);
            const int nslot = (slot == NST - 1) ? 0 : slot + 1;
            const bool nrdy = (t + 1 >= KS) || ready(t + 1);
            asm volatile("" ::: "memory");
            SCHED();
            LOADF(pa, pb, nslot, 0);
            MMA(qa, qb);
            INTERLEAVE();
            SCHED();
            if (!nrdy) {
                wait_ready(t + 1);
                asm volatile("" ::: "memory");
                LOADF(pa, pb, nslot, 0);
            }
            slot = nslot;
        }
        STAMP(3);
        float mx = 0.f;
#pragma unroll
        for (int b = 0; b < 4; ++b)
#pragma unroll
            for (int n = 0; n < 8; ++n) mx = fmaxf(fmaxf(fmaxf(mx, acc[b][n][0]), fmaxf(acc[b][n][1], acc[b][n][2])), acc[b][n][3]);
#pragma unroll
        for (int s = 32; s >= 1; s >>= 1) mx = fmaxf(mx, __shfl_xor(mx, s));
        if (lane == 0) {
            __hip_atomic_fetch_max((LAS unsigned*)(lds + LC_OFF) + C_MAX, __float_as_uint(mx), __ATOMIC_RELAXED, __HIP_MEMORY_SCOPE_WORKGROUP);
            LGKM0();
            __hip_atomic_fetch_add((LAS unsigned*)(lds + LC_OFF) + C_CNT, 1u, __ATOMIC_RELAXED, __HIP_MEMORY_SCOPE_WORKGROUP);
        }
        unsigned rdy;
        { unsigned sp = 0; const u64 t0 = rt();
          while ((rdy = rfl(ctl[C_RDY])) == 0u) { if ((++sp & 4095u) == 0u && tmo.expired(t0)) { rdy = 2u; break; } } }
        STAMP(4);
        const float gmax = __uint_as_float(rfl(ctl[C_GMAX]));
        const float scale = gmax / 127.0f;
        const float inv = scale > 0.f ? 1.0f / scale : 0.f;
        signed char* orow = p.out + (size_t)(rtile * 128 + wm * 64 + (lane & 15)) * NL + ct * 256 + wn * 128 + (lane >> 4) * 16;
#pragma unroll
        for (int b = 0; b < 4; ++b)
#pragma unroll
            for (int h = 0; h < 2; ++h) {
                u32x4 o;
#pragma unroll
                for (int r = 0; r < 4; ++r) {
                    unsigned d = 0;
#pragma unroll
                    for (int c = 0; c < 4; ++c) {
                        const float qv = __builtin_amdgcn_fmed3f(__builtin_rintf(acc[b][h * 4 + c][r] * inv), 0.f, 127.f);
                        d = __builtin_amdgcn_cvt_pk_u8_f32(qv, c, d);
                    }
                    o[r] = (rdy == 1u) ? d : 0x80808080u;
                }
                *(u32x4*)(orow + (size_t)b * 16 * NL + h * 64) = o;
            }
        STAMP(5);
    } else if (wave < 6) {
        const int f = wave - 4, rb = sub * 2 + f;
        const __amdgpu_buffer_rsrc_t rsx = mkrsrc(me + XS_OFF + (size_t)src * XS_SRC + (size_t)rb * KS * UNIT, KS * UNIT);
        const __amdgpu_buffer_rsrc_t rsf = mkrsrc(me + XF_OFF + (size_t)(src * 8 + rb) * KS * 8, KS * 8);
        u32x4 d0[8], d1[8], d2[8]; u32x2 f0 = {0, 0}, f1 = {0, 0}, f2 = {0, 0};
        int u0 = -1, u1 = -1, u2 = -1;
        int issued = 0, avail = 0, fin = 0, slot = 0; unsigned sp = 0; bool dead = false;
        u64 tw = rt();
        auto issue = [&](u32x4 (&d)[8], u32x2& fl, int& u) {
            const bool go = issued < avail;
            u = go ? issued : -1;
            const unsigned voff = (go ? (unsigned)issued * UNIT : 0x40000000u) + lane * 16;
#pragma unroll
            for (int j = 0; j < 8; ++j) d[j] = __builtin_amdgcn_raw_buffer_load_b128(rsx, voff + j * 1024, 0, 17);
            fl = __builtin_amdgcn_raw_buffer_load_b64(rsf, lane * 8, 0, 17);
            issued += go ? 1 : 0;
        };
        auto consume = [&](u32x4 (&d)[8], const u32x2& fl, int u) {
            asm volatile("" :: "v"(fl.y));
            const u64 m = ~__builtin_amdgcn_ballot_w64(fl.x >= call);
            const int a = dead ? KS : (m ? __builtin_ctzll(m) : KS);
            avail = a > avail ? a : avail;
            if (u >= 0) {
                if (u >= NST) {
                    unsigned s2 = 0; const u64 t0 = rt();
                    for (;;) {
                        const u32x4 dn = *(volatile LAS u32x4*)(lds + LC_OFF + C_DONE * 4);
                        const unsigned a0 = rfl(f ? dn.y : dn.x), a1 = rfl(f ? dn.w : dn.z);
                        if (a0 + NST > (unsigned)u && a1 + NST > (unsigned)u) break;
                        __builtin_amdgcn_s_sleep(1);
                        if ((++s2 & 1023u) == 0u && tmo.expired(t0)) break;
                    }
                }
                asm volatile("" ::: "memory");
#pragma unroll
                for (int j = 0; j < 8; ++j) *(LAS u32x4*)(lds + LA_OFF + slot * LA_SLOT + f * 8192 + j * 1024 + lane * 16) = d[j];
                LGKM0();
                ctl[C_FULL + f] = (unsigned)(u + 1);
                slot = (slot == NST - 1) ? 0 : slot + 1;
                ++fin; tw = rt();
            } else if (issued >= avail) {
                __builtin_amdgcn_s_sleep(2);
                if ((++sp & 255u) == 0u && tmo.expired(tw)) dead = true;
            }
        };
        issue(d0, f0, u0); issue(d1, f1, u1);
        while (fin < KS) {
            issue(d2, f2, u2); SCHED(); consume(d0, f0, u0); SCHED();
            issue(d0, f0, u0); SCHED(); consume(d1, f1, u1); SCHED();
            issue(d1, f1, u1); SCHED(); consume(d2, f2, u2); SCHED();
        }
    } else if (wave == 6) {
        const char* wb = me + WB_OFF + (size_t)ct * KS * 32768;
        const GAS unsigned* wf = (const GAS unsigned*)(me + WF_OFF) + ct * KS * 4;
        const unsigned ldsb = (unsigned)(size_t)(lds + LB_OFF);
        int wavail = 0, slot = 0, pub = 0;
        for (int t = 0; t < KS; ++t) {
            bool parked = false;
            if (wavail <= t) {
                VMWAIT0(); if (pub < t) { pub = t; ctl[C_FULL + 2] = (unsigned)pub; }
                unsigned sp = 0; const u64 t0 = rt();
                while (wavail <= t) {
                    const int base = (t >> 4) << 4;
                    const unsigned v = __hip_atomic_load(wf + base * 4 + lane, __ATOMIC_RELAXED, __HIP_MEMORY_SCOPE_SYSTEM);
                    const u64 m = ~__builtin_amdgcn_ballot_w64(v >= call);
                    wavail = base + ((m ? __builtin_ctzll(m) : 64) >> 2);
                    if (wavail <= t) { __builtin_amdgcn_s_sleep(4); if ((++sp & 255u) == 0u && tmo.expired(t0)) { wavail = KS; } }
                }
            }
            if (t >= NST) {
                unsigned sp = 0; const u64 t0 = rt();
                for (;;) {
                    const u32x4 dn = *(volatile LAS u32x4*)(lds + LC_OFF + C_DONE * 4);
                    const unsigned mn = rfl(min(min(dn.x, dn.y), min(dn.z, dn.w)));
                    if (mn + NST > (unsigned)t) break;
                    if (!parked) { parked = true; VMWAIT0(); if (pub < t) { pub = t; ctl[C_FULL + 2] = (unsigned)pub; } }
                    __builtin_amdgcn_s_sleep(1);
                    if ((++sp & 1023u) == 0u && tmo.expired(t0)) break;
                }
            }
            const char* sb = wb + (size_t)t * 32768;
            const unsigned db = ldsb + slot * LB_SLOT;
            const unsigned vo = lane * 16;
#pragma unroll
            for (int j = 0; j < 8; ++j) {
                unsigned keep;
                asm volatile("s_mov_b32 %0, m0\n\ts_mov_b32 m0, %3\n\ts_nop 0\n\t"
                             "global_load_lds_dwordx4 %1, %2 sc0 sc1\n\t"
                             "global_load_lds_dwordx4 %1, %2 offset:1024 sc0 sc1\n\t"
                             "global_load_lds_dwordx4 %1, %2 offset:2048 sc0 sc1\n\t"
                             "global_load_lds_dwordx4 %1, %2 offset:3072 sc0 sc1\n\t"
                             "s_mov_b32 m0, %0"
                             : "=&s"(keep) : "v"(vo), "s"(sb + j * 4096), "s"(db + j * 4096) : "memory");
            }
            asm volatile("s_waitcnt vmcnt(32)" ::: "memory");
            if (pub < t) { pub = t; ctl[C_FULL + 2] = (unsigned)pub; }
            slot = (slot == NST - 1) ? 0 : slot + 1;
        }
        VMWAIT0();
        ctl[C_FULL + 2] = (unsigned)KS;
        STAMP(2);
        { unsigned sp = 0; const u64 t0 = rt();
          while (rfl(ctl[C_CNT]) < 4u) { if ((++sp & 4095u) == 0u && tmo.expired(t0)) break; } }
        const unsigned tm = rfl(ctl[C_MAX]);
        if (lane == 0) __hip_atomic_store((GAS u64*)(me + A1_OFF) + par * NWG + bx, ((u64)call << 32) | tm, __ATOMIC_RELAXED, __HIP_MEMORY_SCOPE_SYSTEM);
        STAMP(3);
        unsigned g = 0;
        { unsigned sp = 0; const u64 t0 = rt();
          const GAS u64* a2 = (const GAS u64*)(me + A2_OFF + (size_t)(par * 8 + (bx & 7)) * 128) + (lane & 7);
          for (;;) {
              const u64 v = __hip_atomic_load(a2, __ATOMIC_RELAXED, __HIP_MEMORY_SCOPE_SYSTEM);
              g = (unsigned)v;
              if (__all((unsigned)(v >> 32) == call)) break;
              if ((++sp & 255u) == 0u && tmo.expired(t0)) break;
          } }
        g = max(g, (unsigned)__shfl_xor((int)g, 1)); g = max(g, (unsigned)__shfl_xor((int)g, 2)); g = max(g, (unsigned)__shfl_xor((int)g, 4));
        const bool bad = rfl(__hip_atomic_load(tmo.word, __ATOMIC_RELAXED, __HIP_MEMORY_SCOPE_AGENT)) == call;
        ctl[C_GMAX] = g;
        LGKM0();
        ctl[C_RDY] = bad ? 2u : 1u;
        STAMP(4);
        if (lane == 0) __hip_atomic_store(seqw, call, __ATOMIC_RELAXED, __HIP_MEMORY_SCOPE_AGENT);
    } else {
        if (xcd == 0 && idx == ((rank + 1) & 7)) {
            { unsigned sp = 0; const u64 t0 = rt();
              while (rfl(ctl[C_DONE]) < (unsigned)(KS - 2)) { __builtin_amdgcn_s_sleep(8); if ((++sp & 1023u) == 0u && tmo.expired(t0)) break; } }
            const __amdgpu_buffer_rsrc_t rs = mkrsrc(me + A1_OFF + (size_t)par * NWG * 8, NWG * 8);
            unsigned g = 0, sp = 0; const u64 t0 = rt();
            for (;;) {
                const u32x4 a = __builtin_amdgcn_raw_buffer_load_b128(rs, lane * 16, 0, 17), b = __builtin_amdgcn_raw_buffer_load_b128(rs, 1024 + lane * 16, 0, 17);
                g = max(max(a.x, a.z), max(b.x, b.z));
                if (__all(a.y == call && a.w == call && b.y == call && b.w == call)) break;
                asm volatile("" ::: "memory");
                if ((++sp & 255u) == 0u && tmo.expired(t0)) break;
            }
#pragma unroll
            for (int s = 32; s >= 1; s >>= 1) g = max(g, (unsigned)__shfl_xor((int)g, s));
            GAS u64* dst = (GAS u64*)(p.ws[lane >> 3] + A2_OFF + (size_t)(par * 8 + (lane & 7)) * 128) + rank;
            __hip_atomic_store(dst, ((u64)call << 32) | g, __ATOMIC_RELAXED, __HIP_MEMORY_SCOPE_SYSTEM);
            STAMP(2);
        }
    }
}

}

extern "C" void kernel_launch(int rank, int nranks, ncclComm_t comm,
                              hipStream_t stream,
                              void* const* d_in, const int* in_sizes, int n_in,
                              void* d_out, int out_size,
                              const int* in_dtypes, int out_dtype,
                              void* const* d_out_all,
                              void* const* d_ws_all, size_t ws_bytes) {
    if (nranks != NR || (unsigned)rank >= (unsigned)NR || ws_bytes < WS_NEED || n_in < 2) {
        fprintf(stderr, "unsupported configuration\n");
        return;
    }
    HIP_CHECK(hipFuncSetAttribute((const void*)ag_gemm, hipFuncAttributeMaxDynamicSharedMemorySize, LDS_BYTES));
    Params p;
    p.x = (const float*)d_in[0]; p.w = (const float*)d_in[1]; p.out = (signed char*)d_out;
    for (int i = 0; i < NR; ++i) p.ws[i] = (char*)d_ws_all[i];
    p.rank = rank; p.pad = 0;
    hipLaunchKernelGGL(ag_gemm, dim3(NWG), dim3(512), LDS_BYTES, stream, p);
    HIP_CHECK(hipGetLastError());
}
```

```cpp
#include <hip/hip_runtime.h>
#include <rccl/rccl.h>
#include <cstdio>

#define HIP_CHECK(expr) do { hipError_t _e = (expr); if (_e != hipSuccess) { \
    fprintf(stderr, "HIP error %s at %s:%d\n", hipGetErrorString(_e), __FILE__, __LINE__); \
    return; } } while (0)

namespace {

#define LAS __attribute__((address_space(3)))
#define GAS __attribute__((address_space(1)))
typedef unsigned u32x4 __attribute__((ext_vector_type(4)));
typedef unsigned u32x2 __attribute__((ext_vector_type(2)));
typedef float f32x4 __attribute__((ext_vector_type(4)));
typedef float f32x16 __attribute__((ext_vector_type(16)));
typedef float f32x32 __attribute__((ext_vector_type(32)));
typedef __bf16 bf16x8 __attribute__((ext_vector_type(8)));
typedef unsigned long long u64;

constexpr int NR = 8;
constexpr int KDIM = 4096, NL = 2048;
constexpr int KS = 64;
constexpr int NWG = 256;
constexpr int UNIT = 8192;
constexpr int EXCAP = 128;

constexpr size_t XS_OFF = 0;
constexpr size_t XS_SRC = (size_t)8 * KS * UNIT;
constexpr size_t WB_OFF = XS_OFF + NR * XS_SRC;
constexpr size_t XF_OFF = WB_OFF + (size_t)32 * KS * 8192;
constexpr size_t WF_OFF = XF_OFF + (size_t)NR * 8 * KS * 8;
constexpr size_t SQ_OFF = WF_OFF + (size_t)32 * KS * 64;
constexpr size_t A1_OFF = SQ_OFF + (size_t)NWG * 128;
constexpr size_t A2_OFF = A1_OFF + (size_t)2 * NWG * 64;
constexpr size_t TK_OFF = A2_OFF + (size_t)2 * 8 * 128;
constexpr size_t TM_OFF = TK_OFF + 128;
constexpr size_t DG_OFF = TM_OFF + 128;
constexpr size_t WS_NEED = DG_OFF + (size_t)NWG * 8 * 128 * 8;

constexpr int NST = 8;
constexpr int LA_SLOT = 18432;
constexpr int LC_OFF = NST * LA_SLOT;
constexpr int C_FULL = 0;
constexpr int C_DONE = 4;
constexpr int C_MAX = 8, C_CNT = 9, C_GMAX = 10, C_RDY = 11, C_SENT = 12, C_ISS = 13, C_SEQ = 14;
constexpr int LE_OFF = LC_OFF + 64;
constexpr int LDS_BYTES = LE_OFF + 8192;

#define CPX 3
#define PW 6
#define WDELAY 60
#define WDELAY2 200
#define FPRIO 0
#define PWIN 2
#define PEARLY 1
#define SOLO 0
#define PSLEEP 8
#define WSTAG 5
#define WSTEP 8
#define PREFN 2
constexpr int CODE_BYTES = 29696;
#define XLG "0"
#define XVM "8"
constexpr int NCOMM = 8 * CPX;
constexpr float XSCALE = 0.5f;
constexpr unsigned DEADLINE = 40000000u;

struct Params {
    const float* x; const float* w; signed char* out;
    char* peer[NR];
    int rank; int pad;
};

__device__ __forceinline__ u64 rt() { return __builtin_amdgcn_s_memrealtime(); }
__device__ __forceinline__ unsigned rt32() { return (unsigned)__builtin_amdgcn_s_memrealtime(); }
__device__ __forceinline__ unsigned rfl(unsigned v) { return __builtin_amdgcn_readfirstlane(v); }
__device__ __forceinline__ unsigned cvt_pk(float lo, float hi) {
    typedef float f2 __attribute__((ext_vector_type(2))); typedef __bf16 b2 __attribute__((ext_vector_type(2)));
    f2 v = {lo, hi}; return __builtin_bit_cast(unsigned, __builtin_convertvector(v, b2));
}
__device__ __forceinline__ __amdgpu_buffer_rsrc_t mkrsrc(const void* p, unsigned bytes) {
    return __builtin_amdgcn_make_buffer_rsrc((void*)p, 0, bytes, 0x00020000);
}
__device__ __forceinline__ unsigned rotr(unsigned v, int n) { return __builtin_amdgcn_alignbit(v, v, n); }
#define VMWAIT0() asm volatile("s_waitcnt vmcnt(0)" ::: "memory")
#define LGKM0() asm volatile("s_waitcnt lgkmcnt(0)" ::: "memory")
#define SCHED() __builtin_amdgcn_sched_barrier(0)
#define SGB(m, n) __builtin_amdgcn_sched_group_barrier(m, n, 0)
constexpr int SYS = 17;

struct Tmo {
    GAS unsigned* word; unsigned call;
    __device__ __forceinline__ bool hit() const {
        unsigned v;
        asm volatile("v_mov_b32 %0, 0\n\tglobal_load_dword %0, %0, %1 sc1\n\ts_waitcnt vmcnt(0)" : "=&v"(v) : "s"(word) : "memory");
        return rfl(v) == call;
    }
    __device__ __forceinline__ bool expired(unsigned t0) const {
        if (hit() || rt32() - t0 > DEADLINE) {
            unsigned z, c;
            asm volatile("v_mov_b32 %0, 0\n\tv_mov_b32 %1, %3\n\tglobal_store_dword %0, %1, %2 sc1\n\ts_waitcnt vmcnt(0)" : "=&v"(z), "=&v"(c) : "s"(word), "s"(call) : "memory");
            return true;
        }
        return false;
    }
};

struct Spin {
    unsigned n = 0, t0 = 0;
    template <unsigned MASK> __device__ __forceinline__ bool over(const Tmo& tmo) {
        if ((++n & MASK) != 0u) return false;
        if (n == MASK + 1u) { t0 = rt32(); return false; }
        return tmo.expired(t0);
    }
};

#define PSTAMP(r) do {} while (0)
#define STAMP(i) do {} while (0)

__device__ __forceinline__ void convert_w(const float* __restrict__ w, char* me, int ct, int t, int q, int lane, unsigned call) {
    const int n = lane & 15, g = lane >> 4;
    const float* src = w + (size_t)(t * 64 + g * 4) * NL + ct * 256 + q * 64 + n * 4;
    f32x4 v[16];
#pragma unroll
    for (int i = 0; i < 4; ++i)
#pragma unroll
        for (int c = 0; c < 4; ++c) v[i * 4 + c] = __builtin_nontemporal_load((const f32x4*)(src + (size_t)(i * 16 + c) * NL));
    SCHED();
    const __amdgpu_buffer_rsrc_t rs = mkrsrc(me + WB_OFF + ((size_t)(ct * 4 + q) * KS + t) * 8192, 8192);
#pragma unroll
    for (int kh = 0; kh < 2; ++kh)
#pragma unroll
        for (int c = 0; c < 4; ++c) {
            u32x4 o;
            o.x = cvt_pk(v[kh * 8 + 0][c], v[kh * 8 + 1][c]); o.y = cvt_pk(v[kh * 8 + 2][c], v[kh * 8 + 3][c]);
            o.z = cvt_pk(v[kh * 8 + 4][c], v[kh * 8 + 5][c]); o.w = cvt_pk(v[kh * 8 + 6][c], v[kh * 8 + 7][c]);
            __builtin_amdgcn_raw_buffer_store_b128(o, rs, lane * 16 + c * 1024 + kh * 4096, 0, SYS);
        }
    VMWAIT0();
    if (lane == 0) __hip_atomic_store((GAS unsigned*)(me + WF_OFF) + ((ct * 4 + q) * KS + t) * 16, call, __ATOMIC_RELAXED, __HIP_MEMORY_SCOPE_SYSTEM);
}

__device__ __forceinline__ unsigned wave_max(unsigned g) {
    g = max(g, (unsigned)__builtin_amdgcn_update_dpp(0u, g, 0x111, 0xf, 0xf, true));
    g = max(g, (unsigned)__builtin_amdgcn_update_dpp(0u, g, 0x112, 0xf, 0xf, true));
    g = max(g, (unsigned)__builtin_amdgcn_update_dpp(0u, g, 0x114, 0xf, 0xf, true));
    g = max(g, (unsigned)__builtin_amdgcn_update_dpp(0u, g, 0x118, 0xf, 0xf, true));
    return max(max((unsigned)__builtin_amdgcn_readlane(g, 15), (unsigned)__builtin_amdgcn_readlane(g, 31)), max((unsigned)__builtin_amdgcn_readlane(g, 47), (unsigned)__builtin_amdgcn_readlane(g, 63)));
}

__device__ __forceinline__ int xpos16(int r, int g) { return g * 16 + (r ^ ((g & 1) * 2 + (g >> 1) * 12)); }

__device__ __forceinline__ void wait_free(LAS char* lds, int u, int& lim, const Tmo& tmo) {
    if (u >= lim) {
        Spin sp;
        for (;;) {
            const u32x4 dn = *(volatile LAS u32x4*)(lds + LC_OFF + C_DONE * 4);
            lim = (int)rfl(min(min(dn.x, dn.y), min(dn.z, dn.w))) + NST;
            if (u < lim) break;
            __builtin_amdgcn_s_sleep(4);
            if (sp.over<1023u>(tmo)) break;
        }
    }
    asm volatile("" ::: "memory");
}

__global__ void __launch_bounds__(512, 2) ag_gemm(Params p) {
    extern __shared__ __attribute__((aligned(16))) char lds_raw[];
    LAS char* lds = (LAS char*)lds_raw;
    volatile LAS unsigned* ctl = (volatile LAS unsigned*)(lds + LC_OFF);
    const int lane = threadIdx.x & 63, wave = rfl(threadIdx.x >> 6);
    const int bx = blockIdx.x, xcd = bx & 7, idx = bx >> 3;
    const int src = idx & 7, sub = xcd >> 1, ct = (xcd & 1) * 4 + (idx >> 3), rtile = src * 4 + sub;
    const int rank = p.rank;
    char* me = p.peer[NR - 1];
    const bool local = (src == rank);
    const int jj = ((idx >> 3) + sub) & 3;
    const bool comm = local && jj < CPX;
    const int h = ((src - rank - 1) & 7) * 4 + sub;
    const bool host = !local && h < 8 * CPX;
    const int xrow = (((h >> 3) - (ct & 3)) & 3) * 128 + (h & 7) * 16;

    if (threadIdx.x < 16) ctl[threadIdx.x] = (threadIdx.x == C_FULL + 2 || threadIdx.x == C_FULL + 3) ? SOLO + 1u : 0u;
    LGKM0();
    __builtin_amdgcn_s_barrier();
    STAMP(0);
    if (wave == 7 && idx < PREFN) {
        const __amdgpu_buffer_rsrc_t rc = mkrsrc((const void*)__builtin_amdgcn_s_getpc(), CODE_BYTES);
        unsigned acc = 0;
#pragma unroll
        for (int j = 0; j < (CODE_BYTES + 8191) / 8192; ++j) acc ^= __builtin_amdgcn_raw_buffer_load_b32(rc, j * 8192 + lane * 128, 0, 0);
        asm volatile("" :: "v"(acc));
    }
    const unsigned tstart = rt32();
    GAS unsigned* seqw = (GAS unsigned*)(me + SQ_OFF + (size_t)bx * 128);
    constexpr int NCW = 32 - CPX;
    static_assert(PEARLY <= PW && PW <= 6, "sender waves");
    const int ci = local ? 28 + (jj - CPX) : h;
    const int cid = jj * 8 + xcd;

#define NOPUSH false
#define GET_CALL() const unsigned call = rfl(__hip_atomic_load(seqw, __ATOMIC_RELAXED, __HIP_MEMORY_SCOPE_AGENT)) + 1u; \
    const Tmo tmo{(GAS unsigned*)(me + TM_OFF), call}; const unsigned par = call & 1u; const bool nopush = NOPUSH; (void)par; (void)nopush

    if (comm) {
        if (wave < PW) {
            f32x4 v[16];
            auto load_x = [&](int u) {
                const float* s0 = p.x + (size_t)((u & 7) * 64 + (lane >> 2)) * KDIM + (u >> 3) * 64 + (lane & 3) * 4;
#pragma unroll
                for (int b = 0; b < 4; ++b)
#pragma unroll
                    for (int i = 0; i < 4; ++i) v[b * 4 + i] = *(const f32x4*)(s0 + (size_t)b * 16 * KDIM + i * 16);
            };
            if (wave >= PEARLY) __builtin_amdgcn_s_sleep(PSLEEP);
            load_x(wave * NCOMM + cid);
            SCHED();
            GET_CALL();
            constexpr unsigned NTK = 8 * KS;
            __amdgpu_buffer_rsrc_t rs[NR];
#pragma unroll
            for (int q = 0; q < NR; ++q) rs[q] = mkrsrc(p.peer[q] + XS_OFF + (size_t)rank * XS_SRC, (q == NR - 1 || !nopush) ? (unsigned)XS_SRC : 0u);
            GAS u64* flag = (GAS u64*)(p.peer[lane & 7] + XF_OFF) + rank * 8 * KS;
            GAS unsigned* tk = (GAS unsigned*)(me + TK_OFF);
            LAS char* elist = lds + LE_OFF + wave * 1024;
            const unsigned ptag = (unsigned)xpos16(lane >> 2, lane & 3) << 3, vo = lane * 16;
            unsigned u = wave * NCOMM + cid;
            int k = wave;
            while (u < 8u * KS) {
                const unsigned fi = (u & 7u) * KS + (u >> 3), soff = fi * UNIT;
                u32x4 L[4], H[4], NA, T;
                unsigned sum = 0;
#pragma unroll
                for (int i = 0; i < 16; ++i) {
                    const f32x4 s = v[((i >> 1) & 3) * 4 + (i >> 3) * 2 + (i & 1)] * XSCALE;
                    const unsigned d0 = cvt_pk(s[0], s[1]), d1 = cvt_pk(s[2], s[3]);
                    L[i >> 2][i & 3] = __builtin_amdgcn_perm(d1, d0, 0x06040200u);
                    H[i >> 2][i & 3] = __builtin_amdgcn_perm(d1, d0, 0x07050301u);
                }
                SCHED();
                PSTAMP(2);
                if (k < PEARLY) {
#pragma unroll
                    for (int j = 0; j < 4; ++j)
#pragma unroll
                        for (int q = 0; q < NR; ++q) __builtin_amdgcn_raw_buffer_store_b128(L[j], rs[q], vo + j * 1024, soff, SYS);
                    SCHED(); asm volatile("s_nop 1" ::: "memory"); SCHED();
                }
#pragma unroll
                for (int i = 0; i < 16; ++i) sum = __builtin_amdgcn_sad_u8(H[i >> 2][i & 3] & 0x7f7f7f7fu, 0u, sum);
                sum += __builtin_amdgcn_update_dpp(0u, sum, 0x111, 0xf, 0xf, true);
                sum += __builtin_amdgcn_update_dpp(0u, sum, 0x112, 0xf, 0xf, true);
                sum += __builtin_amdgcn_update_dpp(0u, sum, 0x114, 0xf, 0xf, true);
                sum += __builtin_amdgcn_update_dpp(0u, sum, 0x118, 0xf, 0xf, true);
                const int tot = __builtin_amdgcn_readlane(sum, 15) + __builtin_amdgcn_readlane(sum, 31) + __builtin_amdgcn_readlane(sum, 47) + __builtin_amdgcn_readlane(sum, 63);
                int bs = (tot - 5939) >> 12;
                bs = bs < 0 ? 0 : (bs > 124 ? 124 : bs);
                const unsigned b4 = (unsigned)bs * 0x01010101u, nb4 = (unsigned)((-bs) & 3) * 0x01010101u;
                unsigned N[6], cnt = 0, nlo = 0;
#pragma unroll
                for (int g = 0; g < 2; ++g) {
                    unsigned c[8], sg[8], x[8], rest = 0;
                    auto emit = [&](int e) {
                        const u64 m = __builtin_amdgcn_ballot_w64(x[e] != 0u);
                        const unsigned sh = (unsigned)__builtin_ctz(x[e] | 0x80000000u) & 24u;
                        const unsigned pos = min(cnt, (unsigned)EXCAP + 1u) + __builtin_amdgcn_mbcnt_hi((unsigned)(m >> 32), __builtin_amdgcn_mbcnt_lo((unsigned)m, 0u));
                        const unsigned ent = (((x[e] >> sh) & 0x7cu) << 9) | ((unsigned)(e >> 1) << 9) | ptag | ((unsigned)(e & 1) << 2) | (sh >> 3);
                        *(LAS unsigned short*)(elist + (x[e] != 0u ? pos * 2 : 512u + lane * 2)) = (unsigned short)ent;
                        x[e] &= ~(0xffu << sh);
                        cnt += (unsigned)__builtin_popcountll(m);
                    };
#pragma unroll
                    for (int e = 0; e < 8; ++e) {
                        const int i = g * 8 + e;
                        const unsigned hh = H[i >> 2][i & 3], hm = hh & 0x7f7f7f7fu;
                        c[e] = (hm + nb4) & 0x03030303u;
                        sg[e] = hh & 0x80808080u;
                        x[e] = hm ^ (c[e] + b4);
                        emit(e);
                        rest |= x[e];
                    }
                    while (__builtin_amdgcn_ballot_w64(rest != 0u) != 0ull && cnt <= (unsigned)EXCAP) {
                        rest = 0;
#pragma unroll
                        for (int e = 0; e < 8; ++e) { if (__builtin_amdgcn_ballot_w64(x[e] != 0u) != 0ull) emit(e); rest |= x[e]; }
                    }
                    if (g == 0) nlo = cnt;
                    N[g * 3 + 0] = (c[0] | sg[0]) | rotr(c[1] | sg[1], 29) | (c[6] << 5);
                    N[g * 3 + 1] = (c[2] | sg[2]) | rotr(c[3] | sg[3], 29) | (c[7] << 5);
                    N[g * 3 + 2] = (c[4] | sg[4]) | rotr(c[5] | sg[5], 29) | (sg[6] >> 1) | (sg[7] >> 2);
                }
                NA = (u32x4){N[0], N[1], N[2], N[3]};
                const unsigned u4 = __builtin_amdgcn_permlane32_swap(N[4], N[4], false, false)[1], u5 = __builtin_amdgcn_permlane32_swap(N[5], N[5], false, false)[1];
                const u32x4 el = *(volatile LAS u32x4*)(elist + (lane & 15) * 16);
                T = lane < 32 ? (u32x4){N[4], N[5], u4, u5} : el;
                unsigned hdr = ((unsigned)bs | (nlo << 8) | (cnt << 16));
                if (cnt > (unsigned)EXCAP) { hdr = 0x80u; NA = H[0]; T = H[1]; }
                unsigned vt = ((hdr & 0x80u) || (unsigned)lane < 32u + (((hdr >> 16) + 7u) >> 3)) ? vo + 1024 : 0x40000000u;
                asm volatile("" : "+v"(NA), "+v"(T), "+v"(vt));
                PSTAMP(3);
                SCHED();
                if (k >= PEARLY) {
                    Spin sp;
                    for (;;) {
                        const u32x2 c = *(volatile LAS u32x2*)(lds + LC_OFF + C_SENT * 4);
                        if ((int)rfl(c.x) + PWIN > k && (int)rfl(c.y) >= k) break;
                        __builtin_amdgcn_s_sleep(1);
                        if (sp.over<1023u>(tmo)) break;
                    }
                    asm volatile("" ::: "memory");
                    PSTAMP(4);
#pragma unroll
                    for (int j = 0; j < 4; ++j)
#pragma unroll
                        for (int q = 0; q < NR; ++q) __builtin_amdgcn_raw_buffer_store_b128(L[j], rs[q], vo + j * 1024, soff, SYS);
                }
#pragma unroll
                for (int q = 0; q < NR; ++q) __builtin_amdgcn_raw_buffer_store_b128(NA, rs[q], vo, soff + 4096, SYS);
                if (lane == 0) __hip_atomic_fetch_add((LAS unsigned*)(lds + LC_OFF) + C_ISS, 1u, __ATOMIC_RELAXED, __HIP_MEMORY_SCOPE_WORKGROUP);
#pragma unroll
                for (int q = 0; q < NR; ++q) __builtin_amdgcn_raw_buffer_store_b128(T, rs[q], vt, soff + 4096, SYS);
                if (hdr & 0x80u) {
#pragma unroll
                    for (int j = 2; j < 4; ++j)
#pragma unroll
                        for (int q = 0; q < NR; ++q) __builtin_amdgcn_raw_buffer_store_b128(H[j], rs[q], vo + j * 1024, soff + 4096, SYS);
                }
                SCHED(); asm volatile("s_nop 1" ::: "memory"); SCHED();
                PSTAMP(5);
                unsigned tv = 0;
                if (lane == 0) tv = __hip_atomic_fetch_add(tk, 1u, __ATOMIC_RELAXED, __HIP_MEMORY_SCOPE_AGENT);
                VMWAIT0();
                PSTAMP(6);
                if (lane == 0) __hip_atomic_fetch_add((LAS unsigned*)(lds + LC_OFF) + C_SENT, 1u, __ATOMIC_RELAXED, __HIP_MEMORY_SCOPE_WORKGROUP);
                if (lane == NR - 1 || (lane < NR && !nopush)) __hip_atomic_store(flag + fi, ((u64)hdr << 32) | call, __ATOMIC_RELAXED, __HIP_MEMORY_SCOPE_SYSTEM);
                PSTAMP(7);
                u = PW * NCOMM + (rfl(tv) - (call - 1u) * NTK);
                if (u < 8u * KS) {
                    load_x((int)u);
                    unsigned kq = 0;
                    if (lane == 0) kq = __hip_atomic_fetch_add((LAS unsigned*)(lds + LC_OFF) + C_SEQ, 1u, __ATOMIC_RELAXED, __HIP_MEMORY_SCOPE_WORKGROUP);
                    k = PW + (int)rfl(kq);
                }
            }
            if (lane == 0) __hip_atomic_fetch_add((LAS unsigned*)(lds + LC_OFF) + C_CNT, 1u, __ATOMIC_RELAXED, __HIP_MEMORY_SCOPE_WORKGROUP);
        } else if (wave == 6) {
            GET_CALL();
            constexpr int NLEFT = (KS - 2 * NCW) * 32;
            unsigned due = 800;
            for (int e = cid; e < NLEFT; e += NCOMM, due += 300) {
                while (rt32() - tstart < due) __builtin_amdgcn_s_sleep(16);
                convert_w(p.w, me, (e >> 2) & 7, 2 * NCW + (e >> 5), e & 3, lane, call);
            }
            if (lane == 0) __hip_atomic_fetch_add((LAS unsigned*)(lds + LC_OFF) + C_CNT, 1u, __ATOMIC_RELAXED, __HIP_MEMORY_SCOPE_WORKGROUP);
        } else if (wave == 7) {
            GET_CALL();
            if (lane == 0) __hip_atomic_store((GAS u64*)(me + A1_OFF) + (par * NWG + bx) * 8, (u64)call << 32, __ATOMIC_RELAXED, __HIP_MEMORY_SCOPE_SYSTEM);
            { Spin sp; while (rfl(ctl[C_CNT]) < (unsigned)PW + 1u) { __builtin_amdgcn_s_sleep(8); if (sp.over<1023u>(tmo)) break; } }
            if (lane == 0) __hip_atomic_store(seqw, call, __ATOMIC_RELAXED, __HIP_MEMORY_SCOPE_AGENT);
            if (cid == 0) {
                const __amdgpu_buffer_rsrc_t rs = mkrsrc(me + A1_OFF + (size_t)par * NWG * 64, NWG * 64);
                GAS u64* dst = (GAS u64*)(p.peer[lane >> 3] + A2_OFF + (size_t)(par * 8 + (lane & 7)) * 128) + rank;
                unsigned g = 0; Spin sp;
                for (;;) {
                    u32x2 a[4];
#pragma unroll
                    for (int j = 0; j < 4; ++j) a[j] = __builtin_amdgcn_raw_buffer_load_b64(rs, lane * 64, j * 4096, SYS);
                    g = max(max(a[0].x, a[1].x), max(a[2].x, a[3].x));
                    if (__all(a[0].y == call && a[1].y == call && a[2].y == call && a[3].y == call)) break;
                    asm volatile("" ::: "memory");
                    if (sp.over<255u>(tmo)) break;
                }
                g = wave_max(g);
                __hip_atomic_store(dst, ((u64)call << 32) | g, __ATOMIC_RELAXED, __HIP_MEMORY_SCOPE_SYSTEM);
                STAMP(2);
            }
        }
        return;
    }

    GET_CALL();
    if (wave < 4) {
        while (rt32() - tstart < WDELAY + ci * WSTAG) __builtin_amdgcn_s_sleep(2);
        convert_w(p.w, me, ct, ci, wave, lane, call);
        STAMP(1);
        const __amdgpu_buffer_rsrc_t rsw = mkrsrc(me + WB_OFF + (size_t)(ct * 4 + wave) * KS * 8192, KS * 8192);
        const __amdgpu_buffer_rsrc_t rsf = mkrsrc(me + WF_OFF + (size_t)(ct * 4 + wave) * KS * 64, KS * 64);
        const unsigned ldsb = (unsigned)(size_t)lds;
        unsigned vo = lane * 16;
        const unsigned abase = ldsb + xpos16(lane & 15, lane >> 4) * 16;
        const unsigned polla = ldsb + LC_OFF + (C_FULL + (lane & 3)) * 4, donea = ldsb + LC_OFF + (C_DONE + wave) * 4;
        f32x32 r0, r1, r2, r3; f32x16 r4;
        unsigned s_t, s_wk, s_bo, s_bc, s_x, s_sp, s_t0, s_late; u64 s_c, s_ok, s_c0, s_r0;
        asm volatile(R"ASM(
.set AG_A, 8
.set AG_C, 112
.macro AG_EXPIRED lout
  s_add_u32 %[sp], %[sp], 1
  s_and_b32 %[x], %[sp], 0x3ff
  s_cbranch_scc1 8f
  v_mov_b32 v6, 0
  global_load_dword v3, v6, %[tmo] sc1
  s_memrealtime %[c]
  s_waitcnt vmcnt(0) lgkmcnt(0)
  v_readfirstlane_b32 %[x], v3
  s_cmp_eq_u32 %[x], %[call]
  s_cbranch_scc1 \lout
  s_mov_b64 vcc, %[c]
  s_sub_u32 %[x], vcc_lo, %[t0]
  s_cmp_le_u32 %[x], %[dl]
  s_cbranch_scc1 8f
  v_mov_b32 v3, %[call]
  global_store_dword v6, v3, %[tmo] sc1
  s_waitcnt vmcnt(0)
  s_branch \lout
8:
.endm
.macro AG_T0
  s_memrealtime %[c]
  s_mov_b32 %[sp], 0
  s_waitcnt lgkmcnt(0)
  s_mov_b64 vcc, %[c]
  s_mov_b32 %[t0], vcc_lo
.endm
; wait until w K-step (t + add) is converted
.macro AG_NEEDW add
  AG_T0
1:
  v_lshlrev_b32 v6, 2, v0
  buffer_load_dword v3, v6, %[rsf], 0 offen sc0 sc1
  s_waitcnt vmcnt(0)
  v_cmp_gt_u32 vcc, %[call], v3
  s_ff1_i32_b64 %[wk], vcc
  s_min_u32 %[wk], %[wk], 1000
  s_add_u32 %[x], %[t], \add
  s_cmp_gt_u32 %[wk], %[x]
  s_cbranch_scc1 3f
  s_sleep 2
  AG_EXPIRED 2f
  s_branch 1b
2:
  s_movk_i32 %[wk], 1000
3:
.endm
; wait until x K-step (t + add) is in the ring, then read its first half
.macro AG_NEEDX add, va
  AG_T0
  s_add_u32 %[late], %[late], 1
1:
  ds_read_b32 v3, v4
  s_add_u32 %[x], %[t], \add
  s_waitcnt lgkmcnt(0)
  v_cmp_lt_u32 vcc, %[x], v3
  s_cmp_eq_u64 vcc, exec
  s_cbranch_scc1 2f
  s_sleep 1
  AG_EXPIRED 2f
  s_branch 1b
2:
  .set ag_b, 0
  .rept 8
  ds_read_b128 v[AG_A+4*ag_b:AG_A+4*ag_b+3], \va offset:(ag_b/4)*8192+(ag_b%%4)*1024
  .set ag_b, ag_b+1
  .endr
  ds_read_b128 v[AG_A+32:AG_A+35], \va offset:16384
.endm
.macro AG_MF b, n, bc
  v_mfma_f32_16x16x32_bf16 v[AG_C+16*\b+4*\n:AG_C+16*\b+4*\n+3], v[\bc+4*\n:\bc+4*\n+3], v[AG_A+4*\b:AG_A+4*\b+3], v[AG_C+16*\b+4*\n:AG_C+16*\b+4*\n+3]
.endm
; half K-step: first = 1 for the first half of step (t + add)
.macro AG_HALF bc, bn, va, vnext, kh, first, add
  .if )ASM" XLG R"ASM(
  s_waitcnt vmcnt()ASM" XVM R"ASM() lgkmcnt(8)
  .else
  s_waitcnt vmcnt()ASM" XVM R"ASM() lgkmcnt(6)
  .endif
  .set ag_b, 0
  .rept 9
  .if )ASM" XLG R"ASM(
  .if ag_b > 0
  s_waitcnt lgkmcnt(8)
  .endif
  .if \first && (ag_b == 7)
  s_waitcnt lgkmcnt(7)
  .endif
  .else
  .if (ag_b == 3) || (ag_b == 6)
  s_waitcnt lgkmcnt(6)
  .endif
  .endif
  AG_MF ag_b, 0, \bc
  .if ag_b == 4
  s_add_u32 %[bo], %[bo], 4096
  .endif
  .if \first && (ag_b == 5)
  s_add_u32 %[x], %[t], \add+1
  .endif
  .if \first && (ag_b == 6)
  s_and_b32 %[sp], %[x], 7
  .endif
  .if \first && (ag_b == 7)
  v_cmp_lt_u32 %[ok], %[x], v3
  .endif
  .if \first && (ag_b == 8)
  s_add_u32 %[x], %[x], 1
  .endif
  AG_MF ag_b, 1, \bc
  .if ag_b < 4
  buffer_load_dwordx4 v[\bn+4*ag_b:\bn+4*ag_b+3], v0, %[rsw], %[bc] offen offset:ag_b*1024 sc0 sc1
  .endif
  .if ag_b == 4
  s_min_u32 %[bc], %[bo], 127*4096
  .endif
  .if \first && (ag_b == 5)
  v_mov_b32 v6, %[x]
  .endif
  .if \first && (ag_b == 6)
  s_mul_i32 %[sp], %[sp], 18432
  .endif
  .if \first && (ag_b == 7)
  v_add_u32 \vnext, %[sp], v1
  .endif
  .if \first && (ag_b == 8)
  s_cmp_le_u32 %[wk], %[x]
  .endif
  .if (!\first) && (ag_b == 8)
  s_cmp_eq_u64 %[ok], exec
  .endif
  AG_MF ag_b, 2, \bc
  .if \first && (ag_b == 0)
  ds_read_b32 v3, v4
  .endif
  AG_MF ag_b, 3, \bc
  .if ag_b < 8
  ds_read_b128 v[AG_A+4*ag_b:AG_A+4*ag_b+3], \va offset:(ag_b/4)*8192+\kh*4096+(ag_b%%4)*1024
  .else
  ds_read_b128 v[AG_A+32:AG_A+35], \va offset:16384+\kh*1024
  .endif
  .set ag_b, ag_b+1
  .endr
.endm
  s_mov_b32 %[t], 0
  s_mov_b32 %[late], 0
  AG_NEEDW 1
  .set ag_i, 0
  .rept 12
  s_movk_i32 %[bc], (ag_i/4)*4096
  buffer_load_dwordx4 v[44+4*ag_i:44+4*ag_i+3], v0, %[rsw], %[bc] offen offset:(ag_i%%4)*1024 sc0 sc1
  .set ag_i, ag_i+1
  .endr
  s_movk_i32 %[bo], 3*4096
  s_movk_i32 %[bc], 3*4096
  .set ag_i, 0
  .rept 144
  v_mov_b32 v[AG_C+ag_i], 0
  .set ag_i, ag_i+1
  .endr
  v_mov_b32 v2, v1
  AG_NEEDX 0, v2
  s_memtime %[c0]
  s_memrealtime %[r0]
  s_waitcnt lgkmcnt(0)
.Lag_loop:
  AG_HALF 44, 92, v2, v7, 1, 1, 0
  ds_write_b32 v5, v6
  s_cbranch_scc1 .Lag_w0
.Lag_w0r:
  AG_HALF 60, 44, v7, v7, 0, 0, 0
  s_cbranch_scc0 .Lag_x0
.Lag_x0r:
  AG_HALF 76, 60, v7, v2, 1, 1, 1
  ds_write_b32 v5, v6
  s_cbranch_scc1 .Lag_w1
.Lag_w1r:
  AG_HALF 92, 76, v2, v2, 0, 0, 1
  s_cbranch_scc0 .Lag_x1
.Lag_x1r:
  s_add_u32 %[t], %[t], 2
  s_cmp_lt_u32 %[t], 64
  s_cbranch_scc1 .Lag_loop
  s_branch .Lag_end
.Lag_w0:
  AG_NEEDW 2
  s_branch .Lag_w0r
.Lag_w1:
  AG_NEEDW 3
  s_branch .Lag_w1r
.Lag_x0:
  AG_NEEDX 1, v7
  s_branch .Lag_x0r
.Lag_x1:
  AG_NEEDX 2, v2
  s_branch .Lag_x1r
.Lag_end:
  s_waitcnt vmcnt(0) lgkmcnt(0)
  s_nop 7
)ASM"
            : "={v[112:143]}"(r0), "={v[144:175]}"(r1), "={v[176:207]}"(r2), "={v[208:239]}"(r3), "={v[240:255]}"(r4),
              [t] "=&s"(s_t), [wk] "=&s"(s_wk), [bo] "=&s"(s_bo), [bc] "=&s"(s_bc), [x] "=&s"(s_x), [sp] "=&s"(s_sp), [t0] "=&s"(s_t0),
              [late] "=&s"(s_late), [c] "=&s"(s_c), [ok] "=&s"(s_ok), [c0] "=&s"(s_c0), [r0] "=&s"(s_r0), "+{v0}"(vo)
            : "{v1}"(abase), "{v4}"(polla), "{v5}"(donea), [rsw] "s"(rsw), [rsf] "s"(rsf), [tmo] "s"(tmo.word), [call] "s"(call), [dl] "s"(DEADLINE)
            : "memory", "vcc", "scc", "v2", "v3", "v6", "v7", "v8", "v9", "v10", "v11", "v12", "v13", "v14", "v15", "v16", "v17", "v18", "v19", "v20", "v21", "v22", "v23", "v24", "v25", "v26", "v27", "v28", "v29", "v30", "v31", "v32", "v33", "v34", "v35", "v36", "v37", "v38", "v39", "v40", "v41", "v42", "v43", "v44", "v45", "v46", "v47", "v48", "v49", "v50", "v51", "v52", "v53", "v54", "v55", "v56", "v57", "v58", "v59", "v60", "v61", "v62", "v63", "v64", "v65", "v66", "v67", "v68", "v69", "v70", "v71", "v72", "v73", "v74", "v75", "v76", "v77", "v78", "v79", "v80", "v81", "v82", "v83", "v84", "v85", "v86", "v87", "v88", "v89", "v90", "v91", "v92", "v93", "v94", "v95", "v96", "v97", "v98", "v99", "v100", "v101", "v102", "v103", "v104", "v105", "v106", "v107");
        f32x4 acc[9][4];
#pragma unroll
        for (int i = 0; i < 144; ++i) acc[i >> 4][(i >> 2) & 3][i & 3] = i < 32 ? r0[i & 31] : i < 64 ? r1[i & 31] : i < 96 ? r2[i & 31] : i < 128 ? r3[i & 31] : r4[i & 15];
        STAMP(3);
        const int ln = (int)(vo >> 4);
        float mx = 0.f;
        if (host) {
#pragma unroll
            for (int n = 0; n < 4; ++n) mx = fmaxf(fmaxf(fmaxf(mx, acc[8][n][0]), fmaxf(acc[8][n][1], acc[8][n][2])), acc[8][n][3]);
        }
#pragma unroll
        for (int b = 0; b < 8; ++b)
#pragma unroll
            for (int n = 0; n < 4; ++n) mx = fmaxf(fmaxf(fmaxf(mx, acc[b][n][0]), fmaxf(acc[b][n][1], acc[b][n][2])), acc[b][n][3]);
#pragma unroll
        for (int s = 32; s >= 1; s >>= 1) mx = fmaxf(mx, __shfl_xor(mx, s));
        if (ln == 0) {
            __hip_atomic_fetch_max((LAS unsigned*)(lds + LC_OFF) + C_MAX, __float_as_uint(mx), __ATOMIC_RELAXED, __HIP_MEMORY_SCOPE_WORKGROUP);
            LGKM0();
            __hip_atomic_fetch_add((LAS unsigned*)(lds + LC_OFF) + C_CNT, 1u, __ATOMIC_RELAXED, __HIP_MEMORY_SCOPE_WORKGROUP);
        }
        unsigned rdy;
        { unsigned sp = 0; const unsigned t0 = rt32();
          while ((rdy = rfl(ctl[C_RDY])) == 0u) { if ((++sp & 4095u) == 0u && tmo.expired(t0)) { rdy = 2u; break; } } }
        STAMP(4);
        const float gmax = __uint_as_float(rfl(ctl[C_GMAX]));
        const float scale = gmax / 127.0f;
        const float inv = scale > 0.f ? 1.0f / scale : 0.f;
        signed char* ocol = p.out + (size_t)(ln & 15) * NL + ct * 256 + wave * 64 + (ln >> 4) * 16;
#pragma unroll
        for (int b = 0; b < 9; ++b) {
            u32x4 o;
#pragma unroll
            for (int r = 0; r < 4; ++r) {
                unsigned d = 0;
#pragma unroll
                for (int c = 0; c < 4; ++c) {
                    const float qv = __builtin_amdgcn_fmed3f(__builtin_rintf(acc[b][c][r] * inv), 0.f, 127.f);
                    d = __builtin_amdgcn_cvt_pk_u8_f32(qv, c, d);
                }
                o[r] = (rdy == 1u) ? d : 0x80808080u;
            }
            if (b < 8) *(u32x4*)(ocol + (size_t)(rtile * 128 + b * 16) * NL) = o;
            else if (host) *(u32x4*)(ocol + (size_t)(rank * 512 + xrow) * NL) = o;
        }
        STAMP(5);
    } else {
        const int f = wave & 1, ph = (wave >> 1) & 1, rb = sub * 2 + f;
        const int wpos = xpos16(lane >> 2, lane & 3) * 16;
        if (!host && ph == 0) {
#pragma unroll
            for (int j = 0; j < NST; ++j) *(LAS u32x4*)(lds + j * LA_SLOT + 16384 + f * 1024 + lane * 16) = (u32x4){0u, 0u, 0u, 0u};
        }
        __builtin_amdgcn_s_setprio(FPRIO);
        char* xb = me + XS_OFF + (size_t)src * XS_SRC + (size_t)rb * KS * UNIT;
        const float* eb = p.x + (size_t)xrow * KDIM + f * 32;
        const __amdgpu_buffer_rsrc_t rsf = mkrsrc(me + XF_OFF + (size_t)(src * 8 + rb) * KS * 8, KS * 8);
        const unsigned vo = lane * 16, vnb = (lane & 31) * 16 + (lane >> 5) * 8, evo = (unsigned)(lane >> 2) * KDIM * 4 + (lane & 3) * 16;
        unsigned M = 0x83838383u, M3 = 0x03030303u;
        asm volatile("" : "+v"(M), "+v"(M3));
        struct Set { u32x4 L[4], NA, E0, E1; u32x2 NB, fl; unsigned EX; int u; };
        Set s0, s1;
        auto next = [](int u) { return u < SOLO ? u + 1 : u + 2; };
        int nxt = ph ? SOLO + 1 : 0, cur = nxt, lim = NST;
        u64 seen = (nopush && !local) ? ~0ull : 0ull;
        auto issue = [&](Set& s) {
            if (nxt == cur && nxt < KS && !((seen >> nxt) & 1ull)) {
                Spin sp;
                for (;;) {
                    if (rfl(__builtin_amdgcn_raw_buffer_load_b32(rsf, 0, nxt * 8, SYS)) >= call) break;
                    __builtin_amdgcn_s_sleep(1);
                    asm volatile("" ::: "memory");
                    if (sp.over<255u>(tmo)) { seen = ~0ull; break; }
                }
                seen |= 1ull << nxt;
            }
            const bool go = nxt < KS && ((seen >> nxt) & 1ull);
            s.u = go ? nxt : -1;
            const int un = nxt & (KS - 1);
            const __amdgpu_buffer_rsrc_t rsx = mkrsrc(xb, go ? KS * UNIT : 0u), rse = mkrsrc(eb, (go && host) ? 16u * KDIM * 4u - 128u : 0u);
#pragma unroll
            for (int j = 0; j < 4; ++j) s.L[j] = __builtin_amdgcn_raw_buffer_load_b128(rsx, vo + j * 1024, un * UNIT, SYS);
            s.NA = __builtin_amdgcn_raw_buffer_load_b128(rsx, vo, un * UNIT + 4096, SYS);
            s.NB = __builtin_amdgcn_raw_buffer_load_b64(rsx, vnb + 1024, un * UNIT + 4096, SYS);
            s.EX = __builtin_amdgcn_raw_buffer_load_b16(rsx, lane * 2 + 1536, un * UNIT + 4096, SYS);
            s.E0 = __builtin_amdgcn_raw_buffer_load_b128(rse, evo, un * 256, 0);
            s.E1 = __builtin_amdgcn_raw_buffer_load_b128(rse, evo + 64, un * 256, 0);
            s.fl = __builtin_amdgcn_raw_buffer_load_b64(rsf, lane * 8, 0, SYS);
            if (go) nxt = next(nxt);
        };
        auto consume = [&](Set& s) {
            seen |= __builtin_amdgcn_ballot_w64(s.fl.x >= call);
            const int u = s.u;
            const unsigned hdr = __builtin_amdgcn_readlane(s.fl.y, u & 63);
            if (u >= 0) {
                unsigned H[16];
                const unsigned b4 = (hdr & 0x7cu) * 0x01010101u;
#pragma unroll
                for (int g = 0; g < 2; ++g) {
                    const unsigned n0 = g ? s.NA.w : s.NA.x, n1 = g ? s.NB.x : s.NA.y, n2 = g ? s.NB.y : s.NA.z;
                    H[g * 8 + 0] = (n0 & M) | b4; H[g * 8 + 1] = (rotr(n0, 3) & M) | b4;
                    H[g * 8 + 2] = (n1 & M) | b4; H[g * 8 + 3] = (rotr(n1, 3) & M) | b4;
                    H[g * 8 + 4] = (n2 & M) | b4; H[g * 8 + 5] = (rotr(n2, 3) & M) | b4;
                    H[g * 8 + 6] = ((n2 << 1) & 0x80808080u) | (((n0 >> 5) & M3) | b4);
                    H[g * 8 + 7] = ((n2 << 2) & 0x80808080u) | (((n1 >> 5) & M3) | b4);
                }
                if (hdr & 0x83u) {
                    if (hdr & 0x80u) {
                        const __amdgpu_buffer_rsrc_t rsx = mkrsrc(xb, KS * UNIT);
                        u32x4 r[4];
#pragma unroll
                        for (int j = 0; j < 4; ++j) r[j] = __builtin_amdgcn_raw_buffer_load_b128(rsx, vo + j * 1024, u * UNIT + 4096, SYS);
                        asm volatile("" : "+v"(r[0]), "+v"(r[1]), "+v"(r[2]), "+v"(r[3]));
#pragma unroll
                        for (int j = 0; j < 16; ++j) H[j] = r[j >> 2][j & 3];
                    } else {
#pragma unroll
                        for (int j = 0; j < 16; ++j) H[j] += (hdr & 3u) * 0x01010101u;
                    }
                }
                wait_free(lds, u, lim, tmo);
                LAS char* dst = lds + (u & (NST - 1)) * LA_SLOT + f * 8192;
#pragma unroll
                for (int j = 0; j < 8; ++j) {
                    u32x4 o;
                    o.x = __builtin_amdgcn_perm(H[2 * j], s.L[j >> 1][(2 * j) & 3], 0x05010400u); o.y = __builtin_amdgcn_perm(H[2 * j], s.L[j >> 1][(2 * j) & 3], 0x07030602u);
                    o.z = __builtin_amdgcn_perm(H[2 * j + 1], s.L[j >> 1][(2 * j + 1) & 3], 0x05010400u); o.w = __builtin_amdgcn_perm(H[2 * j + 1], s.L[j >> 1][(2 * j + 1) & 3], 0x07030602u);
                    *(LAS u32x4*)(dst + j * 1024 + wpos) = o;
                }
                asm volatile("" ::: "memory");
                const unsigned nlo = (hdr >> 8) & 0xffu, nex = (hdr >> 16) & 0xffu;
                auto patch = [&](unsigned e, unsigned j) {
                    if (j < nex) __hip_atomic_fetch_xor((LAS unsigned*)(dst + ((e & 0x7feu) << 1) + (j >= nlo ? 4096u : 0u)), (e >> 11) << (10u + ((e & 1u) << 4)), __ATOMIC_RELAXED, __HIP_MEMORY_SCOPE_WORKGROUP);
                };
                patch(s.EX, lane);
                if (nex > 64u) {
                    unsigned e2 = __builtin_amdgcn_raw_buffer_load_b16(mkrsrc(xb, KS * UNIT), lane * 2 + 1536 + 128, u * UNIT + 4096, SYS);
                    asm volatile("" : "+v"(e2));
                    patch(e2, lane + 64);
                }
                if (host) {
                    const f32x4 lo = __builtin_bit_cast(f32x4, s.E0) * XSCALE, hi = __builtin_bit_cast(f32x4, s.E1) * XSCALE;
                    u32x4 r; r.x = cvt_pk(lo[0], lo[1]); r.y = cvt_pk(lo[2], lo[3]); r.z = cvt_pk(hi[0], hi[1]); r.w = cvt_pk(hi[2], hi[3]);
                    *(LAS u32x4*)(dst + 16384 - f * 7168 + wpos) = r;
                }
                asm volatile("" ::: "memory");
                cur = next(u);
                ctl[C_FULL + (wave & 3)] = cur >= KS ? 1000u : (unsigned)cur;
            }
        };
        bool wtodo = true;
        issue(s0);
        while (cur < KS) {
            issue(s1); SCHED(); consume(s0); SCHED();
            issue(s0); SCHED(); consume(s1); SCHED();
            if (wtodo && cur >= WSTEP + ci) { wtodo = false; convert_w(p.w, me, ct, NCW + ci, wave & 3, lane, call); STAMP(1); }
        }
        if (wave != 4) return;
        { unsigned sp = 0; const unsigned t0 = rt32();
          while (rfl(ctl[C_DONE]) < (unsigned)KS) { __builtin_amdgcn_s_sleep(4); if ((++sp & 1023u) == 0u && tmo.expired(t0)) break; }
          while (rfl(ctl[C_CNT]) < 4u) { if ((++sp & 4095u) == 0u && tmo.expired(t0)) break; } }
        const unsigned tm = rfl(ctl[C_MAX]);
        if (lane == 0) __hip_atomic_store((GAS u64*)(me + A1_OFF) + (par * NWG + bx) * 8, ((u64)call << 32) | tm, __ATOMIC_RELAXED, __HIP_MEMORY_SCOPE_SYSTEM);
        STAMP(3);
        unsigned g = 0;
        { unsigned sp = 0; const unsigned t0 = rt32();
          const GAS u64* a2 = (const GAS u64*)(me + A2_OFF + (size_t)(par * 8 + (bx & 7)) * 128) + (lane & 7);
          for (;;) {
              const u64 v = __hip_atomic_load(a2, __ATOMIC_RELAXED, __HIP_MEMORY_SCOPE_SYSTEM);
              g = (unsigned)v;
              if (__all((unsigned)(v >> 32) == call)) break;
              if ((++sp & 255u) == 0u && tmo.expired(t0)) break;
          } }
        g = max(g, (unsigned)__shfl_xor((int)g, 1)); g = max(g, (unsigned)__shfl_xor((int)g, 2)); g = max(g, (unsigned)__shfl_xor((int)g, 4));
        const bool bad = tmo.hit();
        ctl[C_GMAX] = g;
        LGKM0();
        ctl[C_RDY] = bad ? 2u : 1u;
        if (lane == 0) __hip_atomic_store(seqw, call, __ATOMIC_RELAXED, __HIP_MEMORY_SCOPE_AGENT);
        STAMP(4);
    }
}

}

extern "C" void kernel_launch(int rank, int nranks, ncclComm_t comm,
                              hipStream_t stream,
                              void* const* d_in, const int* in_sizes, int n_in,
                              void* d_out, int out_size,
                              const int* in_dtypes, int out_dtype,
                              void* const* d_out_all,
                              void* const* d_ws_all, size_t ws_bytes) {
    if (nranks != NR || (unsigned)rank >= (unsigned)NR || ws_bytes < WS_NEED || n_in < 2) {
        fprintf(stderr, "unsupported configuration\n");
        return;
    }
    HIP_CHECK(hipFuncSetAttribute((const void*)ag_gemm, hipFuncAttributeMaxDynamicSharedMemorySize, LDS_BYTES));
    Params p;
    p.x = (const float*)d_in[0]; p.w = (const float*)d_in[1]; p.out = (signed char*)d_out;
    for (int i = 0; i < NR; ++i) p.peer[i] = (char*)d_ws_all[(rank + 1 + i) % NR];
    p.rank = rank; p.pad = 0;
    hipLaunchKernelGGL(ag_gemm, dim3(NWG), dim3(512), LDS_BYTES, stream, p);
    HIP_CHECK(hipGetLastError());
}
```

```cpp
#include <hip/hip_runtime.h>
#include <rccl/rccl.h>
#include <cstdio>

#define HIP_CHECK(expr) do { hipError_t _e = (expr); if (_e != hipSuccess) { \
    fprintf(stderr, "HIP error %s at %s:%d\n", hipGetErrorString(_e), __FILE__, __LINE__); \
    return; } } while (0)

namespace {

#define LAS __attribute__((address_space(3)))
#define GAS __attribute__((address_space(1)))
typedef unsigned u32x4 __attribute__((ext_vector_type(4)));
typedef unsigned u32x2 __attribute__((ext_vector_type(2)));
typedef float f32x4 __attribute__((ext_vector_type(4)));
typedef float f32x16 __attribute__((ext_vector_type(16)));
typedef float f32x32 __attribute__((ext_vector_type(32)));
typedef __bf16 bf16x8 __attribute__((ext_vector_type(8)));
typedef unsigned long long u64;

constexpr int NR = 8;
constexpr int KDIM = 4096, NL = 2048;
constexpr int KS = 64;
constexpr int NWG = 256;
constexpr int UNIT = 8192;
constexpr int EXCAP = 128;

constexpr size_t XS_OFF = 0;
constexpr size_t XS_SRC = (size_t)8 * KS * UNIT;
constexpr size_t WB_OFF = XS_OFF + NR * XS_SRC;
constexpr size_t XF_OFF = WB_OFF + (size_t)32 * KS * 8192;
constexpr size_t WF_OFF = XF_OFF + (size_t)NR * 8 * KS * 8;
constexpr size_t SQ_OFF = WF_OFF + (size_t)32 * KS * 64;
constexpr size_t A1_OFF = SQ_OFF + (size_t)NWG * 128;
constexpr size_t A2_OFF = A1_OFF + (size_t)2 * NWG * 64;
constexpr size_t TK_OFF = A2_OFF + (size_t)2 * 8 * 128;
constexpr size_t TM_OFF = TK_OFF + 128;
constexpr size_t DG_OFF = TM_OFF + 128;
constexpr size_t WS_NEED = DG_OFF + (size_t)NWG * 8 * 128 * 8;

constexpr int NST = 8;
constexpr int LA_SLOT = 18432;
constexpr int LC_OFF = NST * LA_SLOT;
constexpr int C_FULL = 0;
constexpr int C_DONE = 4;
constexpr int C_MAX = 8, C_CNT = 9, C_GMAX = 10, C_RDY = 11, C_SENT = 12, C_ISS = 13, C_SEQ = 14;
constexpr int LE_OFF = LC_OFF + 64;
constexpr int LDS_BYTES = LE_OFF + 8192;

#define CPX 3
#define PW 4
#define WDELAY 60
#define WDELAY2 300
#define FPRIO 0
#define PWIN 2
#define PEARLY 1
#define SOLO 0
#define PSLEEP 8
#define WSTAG 5
#define WSTEP 8
#define PREFN 2
constexpr int CODE_BYTES = 27648;
#define OAUX 17
#define WALL 1
#define XDG "0"
#define WWAVE 0
#define WAUX 17
#define XLG "0"
#define XVM "8"
constexpr int NCOMM = 8 * CPX;
constexpr float XSCALE = 0.5f;
constexpr unsigned DEADLINE = 40000000u;

struct Params {
    const float* x; const float* w; signed char* out;
    char* peer[NR];
    int rank; int pad;
};

__device__ __forceinline__ u64 rt() { return __builtin_amdgcn_s_memrealtime(); }
__device__ __forceinline__ unsigned rt32() { return (unsigned)__builtin_amdgcn_s_memrealtime(); }
__device__ __forceinline__ unsigned rfl(unsigned v) { return __builtin_amdgcn_readfirstlane(v); }
__device__ __forceinline__ unsigned cvt_pk(float lo, float hi) {
    typedef float f2 __attribute__((ext_vector_type(2))); typedef __bf16 b2 __attribute__((ext_vector_type(2)));
    f2 v = {lo, hi}; return __builtin_bit_cast(unsigned, __builtin_convertvector(v, b2));
}
__device__ __forceinline__ __amdgpu_buffer_rsrc_t mkrsrc(const void* p, unsigned bytes) {
    return __builtin_amdgcn_make_buffer_rsrc((void*)p, 0, bytes, 0x00020000);
}
__device__ __forceinline__ int fbits(float v) { return __float_as_int(v); }
__device__ __forceinline__ unsigned rotr(unsigned v, int n) { return __builtin_amdgcn_alignbit(v, v, n); }
#define VMWAIT0() asm volatile("s_waitcnt vmcnt(0)" ::: "memory")
#define LGKM0() asm volatile("s_waitcnt lgkmcnt(0)" ::: "memory")
#define SCHED() __builtin_amdgcn_sched_barrier(0)
#define SGB(m, n) __builtin_amdgcn_sched_group_barrier(m, n, 0)
constexpr int SYS = 17;

struct Tmo {
    GAS unsigned* word; unsigned call;
    __device__ __forceinline__ bool hit() const {
        unsigned v;
        asm volatile("v_mov_b32 %0, 0\n\tglobal_load_dword %0, %0, %1 sc1\n\ts_waitcnt vmcnt(0)" : "=&v"(v) : "s"(word) : "memory");
        return rfl(v) == call;
    }
    __device__ __forceinline__ bool expired(unsigned t0) const {
        if (hit() || rt32() - t0 > DEADLINE) {
            unsigned z, c;
            asm volatile("v_mov_b32 %0, 0\n\tv_mov_b32 %1, %3\n\tglobal_store_dword %0, %1, %2 sc1\n\ts_waitcnt vmcnt(0)" : "=&v"(z), "=&v"(c) : "s"(word), "s"(call) : "memory");
            return true;
        }
        return false;
    }
};

struct Spin {
    unsigned n = 0, t0 = 0;
    template <unsigned MASK> __device__ __forceinline__ bool over(const Tmo& tmo) {
        if ((++n & MASK) != 0u) return false;
        if (n == MASK + 1u) { t0 = rt32(); return false; }
        return tmo.expired(t0);
    }
};

#define PSTAMP(r) do {} while (0)
#define STAMP(i) do {} while (0)

__device__ __forceinline__ void convert_w(const float* __restrict__ w, char* me, int ct, int t, int q, int lane, unsigned call) {
    const int n = lane & 15, g = lane >> 4;
    const float* src = w + (size_t)(t * 64 + g * 4) * NL + ct * 256 + q * 64 + n * 4;
    f32x4 v[16];
#pragma unroll
    for (int i = 0; i < 4; ++i)
#pragma unroll
        for (int c = 0; c < 4; ++c) {
            v[i * 4 + c] = __builtin_nontemporal_load((const f32x4*)(src + (size_t)(i * 16 + c) * NL));
        }
    SCHED();
    const __amdgpu_buffer_rsrc_t rs = mkrsrc(me + WB_OFF + ((size_t)(ct * 4 + q) * KS + t) * 8192, 8192);
#pragma unroll
    for (int kh = 0; kh < 2; ++kh)
#pragma unroll
        for (int c = 0; c < 4; ++c) {
            u32x4 o;
            o.x = cvt_pk(v[kh * 8 + 0][c], v[kh * 8 + 1][c]); o.y = cvt_pk(v[kh * 8 + 2][c], v[kh * 8 + 3][c]);
            o.z = cvt_pk(v[kh * 8 + 4][c], v[kh * 8 + 5][c]); o.w = cvt_pk(v[kh * 8 + 6][c], v[kh * 8 + 7][c]);
            __builtin_amdgcn_raw_buffer_store_b128(o, rs, lane * 16 + c * 1024 + kh * 4096, 0, WAUX);
        }
    VMWAIT0();
    if (lane == 0) __hip_atomic_store((GAS unsigned*)(me + WF_OFF) + ((ct * 4 + q) * KS + t) * 16, call, __ATOMIC_RELAXED, __HIP_MEMORY_SCOPE_SYSTEM);
}

__device__ __forceinline__ char* pick_peer(const Params& p, int q) {
    char* r = p.peer[0];
#pragma unroll
    for (int i = 1; i < NR; ++i) { char* c = p.peer[i]; asm volatile("" : "+s"(c)); r = q == i ? c : r; }
    return r;
}

__device__ __forceinline__ void touch(const void* ptr, unsigned junk) {
    unsigned keep;
    asm volatile("s_mov_b32 %0, m0\n\ts_mov_b32 m0, %2\n\ts_nop 0\n\tglobal_load_lds_dword %1, off\n\ts_mov_b32 m0, %0" : "=&s"(keep) : "v"(ptr), "s"(junk) : "memory");
}

__device__ __forceinline__ unsigned wave_max(unsigned g) {
    g = max(g, (unsigned)__builtin_amdgcn_update_dpp(0u, g, 0x111, 0xf, 0xf, true));
    g = max(g, (unsigned)__builtin_amdgcn_update_dpp(0u, g, 0x112, 0xf, 0xf, true));
    g = max(g, (unsigned)__builtin_amdgcn_update_dpp(0u, g, 0x114, 0xf, 0xf, true));
    g = max(g, (unsigned)__builtin_amdgcn_update_dpp(0u, g, 0x118, 0xf, 0xf, true));
    return max(max((unsigned)__builtin_amdgcn_readlane(g, 15), (unsigned)__builtin_amdgcn_readlane(g, 31)), max((unsigned)__builtin_amdgcn_readlane(g, 47), (unsigned)__builtin_amdgcn_readlane(g, 63)));
}

__device__ __forceinline__ int xpos16(int r, int g) { return g * 16 + (r ^ ((g & 1) * 2 + (g >> 1) * 12)); }

__device__ __forceinline__ void wait_free(LAS char* lds, int u, int& lim, const Tmo& tmo) {
    if (u >= lim) {
        Spin sp;
        for (;;) {
            const u32x4 dn = *(volatile LAS u32x4*)(lds + LC_OFF + C_DONE * 4);
            lim = (int)rfl(min(min(dn.x, dn.y), min(dn.z, dn.w))) + NST;
            if (u < lim) break;
            __builtin_amdgcn_s_sleep(4);
            if (sp.over<1023u>(tmo)) break;
        }
    }
    asm volatile("" ::: "memory");
}

__global__ void __launch_bounds__(512, 2) ag_gemm(Params p) {
    extern __shared__ __attribute__((aligned(16))) char lds_raw[];
    LAS char* lds = (LAS char*)lds_raw;
    volatile LAS unsigned* ctl = (volatile LAS unsigned*)(lds + LC_OFF);
    const int lane = threadIdx.x & 63, wave = rfl(threadIdx.x >> 6);
    const int bx = blockIdx.x, xcd = bx & 7, idx = bx >> 3;
    const int src = idx & 7, sub = xcd >> 1, ct = (xcd & 1) * 4 + (idx >> 3), rtile = src * 4 + sub;
    const int rank = p.rank;
    char* me = p.peer[NR - 1];
    const bool local = (src == rank);
    const int jj = ((idx >> 3) + sub) & 3;
    const bool comm = local && jj < CPX;
    const int h = ((src - rank - 1) & 7) * 4 + sub;
    const bool host = !local && h < 8 * CPX;
    const int xrow = (((h >> 3) - (ct & 3)) & 3) * 128 + (h & 7) * 16;

    GAS unsigned* seqw = (GAS unsigned*)(me + SQ_OFF + (size_t)bx * 128);
    if (wave == 6) touch((const void*)seqw, (unsigned)(size_t)(lds + LE_OFF + 7 * 1024));
    if (threadIdx.x < 16) ctl[threadIdx.x] = (threadIdx.x == C_FULL + 2 || threadIdx.x == C_FULL + 3) ? SOLO + 1u : 0u;
    LGKM0();
    __builtin_amdgcn_s_barrier();
    STAMP(0);
    if (wave == 7 && idx < PREFN) {
        const __amdgpu_buffer_rsrc_t rc = mkrsrc((const void*)__builtin_amdgcn_s_getpc(), CODE_BYTES);
        unsigned acc = 0;
#pragma unroll
        for (int j = 0; j < (CODE_BYTES + 8191) / 8192; ++j) acc ^= __builtin_amdgcn_raw_buffer_load_b32(rc, j * 8192 + lane * 128, 0, 0);
        asm volatile("" :: "v"(acc));
    }
    {
        const unsigned junk = (unsigned)(size_t)(lds + LE_OFF + 7 * 1024);
        if (comm) {
            if (wave < PW) {
                const int u = wave * NCOMM + jj * 8 + xcd;
                const float* s0 = p.x + (size_t)((u & 7) * 64 + lane) * KDIM + (u >> 3) * 64;
                touch(s0, junk); touch(s0 + 32, junk);
            }
        } else if (local) {
            if (wave >= 4) {
                const char* pa = wave == 4 ? p.peer[0] : wave == 5 ? p.peer[2] : wave == 6 ? p.peer[4] : p.peer[6];
                const char* pb = wave == 4 ? p.peer[1] : wave == 5 ? p.peer[3] : wave == 6 ? p.peer[5] : p.peer[7];
                const int w = (lane >> 1) & 7;
                touch(((lane & 1) ? pb : pa) + (w < 4 ? XS_OFF + (size_t)rank * XS_SRC + (size_t)w * (XS_SRC / 4) : XF_OFF + (size_t)rank * 8 * KS * 8), junk);
            }
        } else if (wave == 5) {
            const size_t o1 = (size_t)rtile * 128 * NL, o2 = (size_t)(rank * 512 + xrow) * NL;
            touch(lane == 0 ? (const char*)p.out + o1 : lane == 1 ? (const char*)p.out + o2 : lane == 2 ? (const char*)(p.x + (size_t)xrow * KDIM) : me + XS_OFF + (size_t)rtile * (XS_SRC / 4), junk);
        }
    }
    const unsigned tstart = rt32();
    constexpr int NCW = 32 - CPX;
    static_assert(PEARLY <= PW && PW <= 6, "sender waves");
    const int ci = local ? 28 + (jj - CPX) : h;
    const int cid = jj * 8 + xcd;

#define NOPUSH false
#define GET_CALL(N) const unsigned call = rfl(__hip_atomic_load(seqw, __ATOMIC_RELAXED, __HIP_MEMORY_SCOPE_AGENT)) + 1u; \
    const Tmo tmo{(GAS unsigned*)(me + TM_OFF), call}; const unsigned par = call & 1u; const bool nopush = NOPUSH; (void)par; (void)nopush

    if (comm) {
        if (wave < PW) {
            f32x4 v[16];
            auto load_x = [&](int u) {
                const float* s0 = p.x + (size_t)((u & 7) * 64 + (lane >> 2)) * KDIM + (u >> 3) * 64 + (lane & 3) * 4;
#pragma unroll
                for (int b = 0; b < 4; ++b)
#pragma unroll
                    for (int i = 0; i < 4; ++i) v[b * 4 + i] = *(const f32x4*)(s0 + (size_t)b * 16 * KDIM + i * 16);
            };
            if (wave >= PEARLY) __builtin_amdgcn_s_sleep(PSLEEP);
            load_x(wave * NCOMM + cid);
            SCHED();
            GET_CALL(16);
            constexpr unsigned NTK = 8 * KS;
            __amdgpu_buffer_rsrc_t rs[NR];
#pragma unroll
            for (int q = 0; q < NR; ++q) rs[q] = mkrsrc(p.peer[q] + XS_OFF + (size_t)rank * XS_SRC, (q == NR - 1 || !nopush) ? (unsigned)XS_SRC : 0u);
            GAS u64* flag = (GAS u64*)(pick_peer(p, lane & 7) + XF_OFF) + rank * 8 * KS;
            GAS unsigned* tk = (GAS unsigned*)(me + TK_OFF);
            LAS char* elist = lds + LE_OFF + wave * 1024;
            const unsigned ptag = (unsigned)xpos16(lane >> 2, lane & 3) << 3, vo = lane * 16;
            unsigned u = wave * NCOMM + cid;
            int k = wave;
            while (u < 8u * KS) {
                const unsigned fi = (u & 7u) * KS + (u >> 3), soff = fi * UNIT;
                u32x4 L[4], H[4], NA, T;
                unsigned sum = 0;
#pragma unroll
                for (int i = 0; i < 16; ++i) {
                    const f32x4 s = v[((i >> 1) & 3) * 4 + (i >> 3) * 2 + (i & 1)] * XSCALE;
                    const unsigned d0 = cvt_pk(s[0], s[1]), d1 = cvt_pk(s[2], s[3]);
                    L[i >> 2][i & 3] = __builtin_amdgcn_perm(d1, d0, 0x06040200u);
                    H[i >> 2][i & 3] = __builtin_amdgcn_perm(d1, d0, 0x07050301u);
                }
                SCHED();
                PSTAMP(2);
                if (k < PEARLY) {
#pragma unroll
                    for (int j = 0; j < 4; ++j)
#pragma unroll
                        for (int q = 0; q < NR; ++q) __builtin_amdgcn_raw_buffer_store_b128(L[j], rs[q], vo + j * 1024, soff, SYS);
                    SCHED(); asm volatile("s_nop 1" ::: "memory"); SCHED();
                }
#pragma unroll
                for (int i = 0; i < 16; ++i) sum = __builtin_amdgcn_sad_u8(H[i >> 2][i & 3] & 0x7f7f7f7fu, 0u, sum);
                sum += __builtin_amdgcn_update_dpp(0u, sum, 0x111, 0xf, 0xf, true);
                sum += __builtin_amdgcn_update_dpp(0u, sum, 0x112, 0xf, 0xf, true);
                sum += __builtin_amdgcn_update_dpp(0u, sum, 0x114, 0xf, 0xf, true);
                sum += __builtin_amdgcn_update_dpp(0u, sum, 0x118, 0xf, 0xf, true);
                const int tot = __builtin_amdgcn_readlane(sum, 15) + __builtin_amdgcn_readlane(sum, 31) + __builtin_amdgcn_readlane(sum, 47) + __builtin_amdgcn_readlane(sum, 63);
                int bs = (tot - 5939) >> 12;
                bs = bs < 0 ? 0 : (bs > 124 ? 124 : bs);
                const unsigned b4 = (unsigned)bs * 0x01010101u, nb4 = (unsigned)((-bs) & 3) * 0x01010101u;
                unsigned N[6], cnt = 0, nlo = 0;
#pragma unroll
                for (int g = 0; g < 2; ++g) {
                    unsigned c[8], sg[8], x[8], rest = 0;
                    auto emit = [&](int e) {
                        const u64 m = __builtin_amdgcn_ballot_w64(x[e] != 0u);
                        const unsigned sh = (unsigned)__builtin_ctz(x[e] | 0x80000000u) & 24u;
                        const unsigned pos = min(cnt, (unsigned)EXCAP + 1u) + __builtin_amdgcn_mbcnt_hi((unsigned)(m >> 32), __builtin_amdgcn_mbcnt_lo((unsigned)m, 0u));
                        const unsigned ent = (((x[e] >> sh) & 0x7cu) << 9) | ((unsigned)(e >> 1) << 9) | ptag | ((unsigned)(e & 1) << 2) | (sh >> 3);
                        *(LAS unsigned short*)(elist + (x[e] != 0u ? pos * 2 : 512u + lane * 2)) = (unsigned short)ent;
                        x[e] &= ~(0xffu << sh);
                        cnt += (unsigned)__builtin_popcountll(m);
                    };
#pragma unroll
                    for (int e = 0; e < 8; ++e) {
                        const int i = g * 8 + e;
                        const unsigned hh = H[i >> 2][i & 3], hm = hh & 0x7f7f7f7fu;
                        c[e] = (hm + nb4) & 0x03030303u;
                        sg[e] = hh & 0x80808080u;
                        x[e] = hm ^ (c[e] + b4);
                        emit(e);
                        rest |= x[e];
                    }
                    while (__builtin_amdgcn_ballot_w64(rest != 0u) != 0ull && cnt <= (unsigned)EXCAP) {
                        rest = 0;
#pragma unroll
                        for (int e = 0; e < 8; ++e) { if (__builtin_amdgcn_ballot_w64(x[e] != 0u) != 0ull) emit(e); rest |= x[e]; }
                    }
                    if (g == 0) nlo = cnt;
                    N[g * 3 + 0] = (c[0] | sg[0]) | rotr(c[1] | sg[1], 29) | (c[6] << 5);
                    N[g * 3 + 1] = (c[2] | sg[2]) | rotr(c[3] | sg[3], 29) | (c[7] << 5);
                    N[g * 3 + 2] = (c[4] | sg[4]) | rotr(c[5] | sg[5], 29) | (sg[6] >> 1) | (sg[7] >> 2);
                }
                NA = (u32x4){N[0], N[1], N[2], N[3]};
                const unsigned u4 = __builtin_amdgcn_permlane32_swap(N[4], N[4], false, false)[1], u5 = __builtin_amdgcn_permlane32_swap(N[5], N[5], false, false)[1];
                const u32x4 el = *(volatile LAS u32x4*)(elist + (lane & 15) * 16);
                T = lane < 32 ? (u32x4){N[4], N[5], u4, u5} : el;
                unsigned hdr = ((unsigned)bs | (nlo << 8) | (cnt << 16));
                if (cnt > (unsigned)EXCAP) { hdr = 0x80u; NA = H[0]; T = H[1]; }
                unsigned vt = ((hdr & 0x80u) || (unsigned)lane < 32u + (((hdr >> 16) + 7u) >> 3)) ? vo + 1024 : 0x40000000u;
                asm volatile("" : "+v"(NA), "+v"(T), "+v"(vt));
                PSTAMP(3);
                SCHED();
                if (k >= PEARLY) {
                    Spin sp;
                    for (;;) {
                        const u32x2 c = *(volatile LAS u32x2*)(lds + LC_OFF + C_SENT * 4);
                        if ((int)rfl(c.x) + PWIN > k && (int)rfl(c.y) >= k) break;
                        __builtin_amdgcn_s_sleep(1);
                        if (sp.over<1023u>(tmo)) break;
                    }
                    asm volatile("" ::: "memory");
                    PSTAMP(4);
#pragma unroll
                    for (int j = 0; j < 4; ++j)
#pragma unroll
                        for (int q = 0; q < NR; ++q) __builtin_amdgcn_raw_buffer_store_b128(L[j], rs[q], vo + j * 1024, soff, SYS);
                }
#pragma unroll
                for (int q = 0; q < NR; ++q) __builtin_amdgcn_raw_buffer_store_b128(NA, rs[q], vo, soff + 4096, SYS);
                if (lane == 0) __hip_atomic_fetch_add((LAS unsigned*)(lds + LC_OFF) + C_ISS, 1u, __ATOMIC_RELAXED, __HIP_MEMORY_SCOPE_WORKGROUP);
#pragma unroll
                for (int q = 0; q < NR; ++q) __builtin_amdgcn_raw_buffer_store_b128(T, rs[q], vt, soff + 4096, SYS);
                if (hdr & 0x80u) {
#pragma unroll
                    for (int j = 2; j < 4; ++j)
#pragma unroll
                        for (int q = 0; q < NR; ++q) __builtin_amdgcn_raw_buffer_store_b128(H[j], rs[q], vo + j * 1024, soff + 4096, SYS);
                }
                SCHED(); asm volatile("s_nop 1" ::: "memory"); SCHED();
                PSTAMP(5);
                unsigned tv = 0;
                if (lane == 0) tv = __hip_atomic_fetch_add(tk, 1u, __ATOMIC_RELAXED, __HIP_MEMORY_SCOPE_AGENT);
                VMWAIT0();
                PSTAMP(6);
                if (lane == 0) __hip_atomic_fetch_add((LAS unsigned*)(lds + LC_OFF) + C_SENT, 1u, __ATOMIC_RELAXED, __HIP_MEMORY_SCOPE_WORKGROUP);
                if (lane == NR - 1 || (lane < NR && !nopush)) __hip_atomic_store(flag + fi, ((u64)hdr << 32) | call, __ATOMIC_RELAXED, __HIP_MEMORY_SCOPE_SYSTEM);
                PSTAMP(7);
                u = PW * NCOMM + (rfl(tv) - (call - 1u) * NTK);
                if (u < 8u * KS) {
                    load_x((int)u);
                    unsigned kq = 0;
                    if (lane == 0) kq = __hip_atomic_fetch_add((LAS unsigned*)(lds + LC_OFF) + C_SEQ, 1u, __ATOMIC_RELAXED, __HIP_MEMORY_SCOPE_WORKGROUP);
                    k = PW + (int)rfl(kq);
                }
            }
            if (lane == 0) __hip_atomic_fetch_add((LAS unsigned*)(lds + LC_OFF) + C_CNT, 1u, __ATOMIC_RELAXED, __HIP_MEMORY_SCOPE_WORKGROUP);
        } else if (wave == 7) {
            GET_CALL(0);
            if (lane == 0) __hip_atomic_store((GAS u64*)(me + A1_OFF) + (par * NWG + bx) * 8, (u64)call << 32, __ATOMIC_RELAXED, __HIP_MEMORY_SCOPE_SYSTEM);
            { Spin sp; while (rfl(ctl[C_CNT]) < (unsigned)PW) { __builtin_amdgcn_s_sleep(8); if (sp.over<1023u>(tmo)) break; } }
            if (lane == 0) __hip_atomic_store(seqw, call, __ATOMIC_RELAXED, __HIP_MEMORY_SCOPE_AGENT);
            if (cid == 0) {
                const __amdgpu_buffer_rsrc_t rs = mkrsrc(me + A1_OFF + (size_t)par * NWG * 64, NWG * 64);
                GAS u64* dst = (GAS u64*)(pick_peer(p, lane >> 3) + A2_OFF + (size_t)(par * 8 + (lane & 7)) * 128) + rank;
                unsigned g = 0; Spin sp;
                for (;;) {
                    u32x2 a[4];
#pragma unroll
                    for (int j = 0; j < 4; ++j) a[j] = __builtin_amdgcn_raw_buffer_load_b64(rs, lane * 64, j * 4096, SYS);
                    g = max(max(a[0].x, a[1].x), max(a[2].x, a[3].x));
                    if (__all(a[0].y == call && a[1].y == call && a[2].y == call && a[3].y == call)) break;
                    asm volatile("" ::: "memory");
                    if (sp.over<255u>(tmo)) break;
                }
                g = wave_max(g);
                __hip_atomic_store(dst, ((u64)call << 32) | g, __ATOMIC_RELAXED, __HIP_MEMORY_SCOPE_SYSTEM);
                STAMP(2);
            }
        }
        return;
    }

    GET_CALL(0);
    while (rt32() - tstart < (wave < 4 ? WDELAY2 : WDELAY) + (wave & 3) * WWAVE + ci * WSTAG) __builtin_amdgcn_s_sleep(2);
    convert_w(p.w, me, ct, wave < 4 ? NCW + ci : ci, wave & 3, lane, call);
    if (wave < 4) {
        STAMP(1);
        const __amdgpu_buffer_rsrc_t rsw = mkrsrc(me + WB_OFF + (size_t)(ct * 4 + wave) * KS * 8192, KS * 8192);
        const __amdgpu_buffer_rsrc_t rsf = mkrsrc(me + WF_OFF + (size_t)(ct * 4 + wave) * KS * 64, KS * 64);
        const unsigned ldsb = (unsigned)(size_t)lds;
        unsigned vo = lane * 16;
        const unsigned abase = ldsb + xpos16(lane & 15, lane >> 4) * 16;
        const unsigned polla = ldsb + LC_OFF + (C_FULL + (lane & 3)) * 4, donea = ldsb + LC_OFF + (C_DONE + wave) * 4;
        f32x32 r0, r1, r2, r3; f32x16 r4;
        unsigned s_t, s_wk, s_bo, s_bc, s_x, s_sp, s_t0, s_late; u64 s_c, s_ok, s_c0, s_r0; unsigned s_wx, s_ww;
        asm volatile(R"ASM(
.set AG_A, 8
.set AG_C, 112
.macro AG_EXPIRED lout
  s_add_u32 %[sp], %[sp], 1
  s_and_b32 %[x], %[sp], 0x3ff
  s_cbranch_scc1 8f
  v_mov_b32 v6, 0
  global_load_dword v3, v6, %[tmo] sc1
  s_memrealtime %[c]
  s_waitcnt vmcnt(0) lgkmcnt(0)
  v_readfirstlane_b32 %[x], v3
  s_cmp_eq_u32 %[x], %[call]
  s_cbranch_scc1 \lout
  s_mov_b64 vcc, %[c]
  s_sub_u32 %[x], vcc_lo, %[t0]
  s_cmp_le_u32 %[x], %[dl]
  s_cbranch_scc1 8f
  v_mov_b32 v3, %[call]
  global_store_dword v6, v3, %[tmo] sc1
  s_waitcnt vmcnt(0)
  s_branch \lout
8:
.endm
.macro AG_T0
  s_memrealtime %[c]
  s_mov_b32 %[sp], 0
  s_waitcnt lgkmcnt(0)
  s_mov_b64 vcc, %[c]
  s_mov_b32 %[t0], vcc_lo
.endm
; wait until w K-step (t + add) is converted
.macro AG_NEEDW add
  AG_T0
1:
  v_lshlrev_b32 v6, 2, v0
  buffer_load_dword v3, v6, %[rsf], 0 offen sc0 sc1
  s_waitcnt vmcnt(0)
  v_cmp_gt_u32 vcc, %[call], v3
  s_ff1_i32_b64 %[wk], vcc
  s_min_u32 %[wk], %[wk], 1000
  s_add_u32 %[x], %[t], \add
  s_cmp_gt_u32 %[wk], %[x]
  s_cbranch_scc1 3f
  s_sleep 2
  AG_EXPIRED 2f
  s_branch 1b
2:
  s_movk_i32 %[wk], 1000
3:
  .if )ASM" XDG R"ASM(
  s_memrealtime %[c]
  s_waitcnt lgkmcnt(0)
  s_mov_b64 vcc, %[c]
  s_sub_u32 %[x], vcc_lo, %[t0]
  s_add_u32 %[ww], %[ww], %[x]
  .endif
.endm
; wait until x K-step (t + add) is in the ring, then read its first half
.macro AG_NEEDX add, va
  AG_T0
  s_add_u32 %[late], %[late], 1
1:
  ds_read_b32 v3, v4
  s_add_u32 %[x], %[t], \add
  s_waitcnt lgkmcnt(0)
  v_cmp_lt_u32 vcc, %[x], v3
  s_cmp_eq_u64 vcc, exec
  s_cbranch_scc1 2f
  s_sleep 1
  AG_EXPIRED 2f
  s_branch 1b
2:
  .if )ASM" XDG R"ASM(
  s_memrealtime %[c]
  s_waitcnt lgkmcnt(0)
  s_mov_b64 vcc, %[c]
  s_sub_u32 %[x], vcc_lo, %[t0]
  s_add_u32 %[wx], %[wx], %[x]
  .endif
  .set ag_b, 0
  .rept 8
  ds_read_b128 v[AG_A+4*ag_b:AG_A+4*ag_b+3], \va offset:(ag_b/4)*8192+(ag_b%%4)*1024
  .set ag_b, ag_b+1
  .endr
  ds_read_b128 v[AG_A+32:AG_A+35], \va offset:16384
.endm
.macro AG_MF b, n, bc
  v_mfma_f32_16x16x32_bf16 v[AG_C+16*\b+4*\n:AG_C+16*\b+4*\n+3], v[\bc+4*\n:\bc+4*\n+3], v[AG_A+4*\b:AG_A+4*\b+3], v[AG_C+16*\b+4*\n:AG_C+16*\b+4*\n+3]
.endm
; half K-step: first = 1 for the first half of step (t + add)
.macro AG_HALF bc, bn, va, vnext, kh, first, add
  .if )ASM" XLG R"ASM(
  s_waitcnt vmcnt()ASM" XVM R"ASM() lgkmcnt(8)
  .else
  s_waitcnt vmcnt()ASM" XVM R"ASM() lgkmcnt(6)
  .endif
  .set ag_b, 0
  .rept 9
  .if )ASM" XLG R"ASM(
  .if ag_b > 0
  s_waitcnt lgkmcnt(8)
  .endif
  .if \first && (ag_b == 7)
  s_waitcnt lgkmcnt(7)
  .endif
  .else
  .if (ag_b == 3) || (ag_b == 6)
  s_waitcnt lgkmcnt(6)
  .endif
  .endif
  AG_MF ag_b, 0, \bc
  .if ag_b == 4
  s_add_u32 %[bo], %[bo], 4096
  .endif
  .if \first && (ag_b == 5)
  s_add_u32 %[x], %[t], \add+1
  .endif
  .if \first && (ag_b == 6)
  s_and_b32 %[sp], %[x], 7
  .endif
  .if \first && (ag_b == 7)
  v_cmp_lt_u32 %[ok], %[x], v3
  .endif
  .if \first && (ag_b == 8)
  s_add_u32 %[x], %[x], 1
  .endif
  AG_MF ag_b, 1, \bc
  .if ag_b < 4
  buffer_load_dwordx4 v[\bn+4*ag_b:\bn+4*ag_b+3], v0, %[rsw], %[bc] offen offset:ag_b*1024 sc0 sc1
  .endif
  .if ag_b == 4
  s_min_u32 %[bc], %[bo], 127*4096
  .endif
  .if \first && (ag_b == 5)
  v_mov_b32 v6, %[x]
  .endif
  .if \first && (ag_b == 6)
  s_mul_i32 %[sp], %[sp], 18432
  .endif
  .if \first && (ag_b == 7)
  v_add_u32 \vnext, %[sp], v1
  .endif
  .if \first && (ag_b == 8)
  s_cmp_le_u32 %[wk], %[x]
  .endif
  .if (!\first) && (ag_b == 8)
  s_cmp_eq_u64 %[ok], exec
  .endif
  AG_MF ag_b, 2, \bc
  .if \first && (ag_b == 0)
  ds_read_b32 v3, v4
  .endif
  AG_MF ag_b, 3, \bc
  .if ag_b < 8
  ds_read_b128 v[AG_A+4*ag_b:AG_A+4*ag_b+3], \va offset:(ag_b/4)*8192+\kh*4096+(ag_b%%4)*1024
  .else
  ds_read_b128 v[AG_A+32:AG_A+35], \va offset:16384+\kh*1024
  .endif
  .set ag_b, ag_b+1
  .endr
.endm
  s_mov_b32 %[t], 0
  s_mov_b32 %[late], 0
  AG_NEEDW 1
  .set ag_i, 0
  .rept 12
  s_movk_i32 %[bc], (ag_i/4)*4096
  buffer_load_dwordx4 v[44+4*ag_i:44+4*ag_i+3], v0, %[rsw], %[bc] offen offset:(ag_i%%4)*1024 sc0 sc1
  .set ag_i, ag_i+1
  .endr
  s_movk_i32 %[bo], 3*4096
  s_movk_i32 %[bc], 3*4096
  .set ag_i, 0
  .rept 144
  v_mov_b32 v[AG_C+ag_i], 0
  .set ag_i, ag_i+1
  .endr
  v_mov_b32 v2, v1
  AG_NEEDX 0, v2
  .if )ASM" XDG R"ASM(
  s_mov_b32 %[wx], 0
  s_mov_b32 %[ww], 0
  s_memtime %[c0]
  s_memrealtime %[r0]
  s_waitcnt lgkmcnt(0)
  .endif
.Lag_loop:
  AG_HALF 44, 92, v2, v7, 1, 1, 0
  ds_write_b32 v5, v6
  s_cbranch_scc1 .Lag_w0
.Lag_w0r:
  AG_HALF 60, 44, v7, v7, 0, 0, 0
  s_cbranch_scc0 .Lag_x0
.Lag_x0r:
  AG_HALF 76, 60, v7, v2, 1, 1, 1
  ds_write_b32 v5, v6
  s_cbranch_scc1 .Lag_w1
.Lag_w1r:
  AG_HALF 92, 76, v2, v2, 0, 0, 1
  s_cbranch_scc0 .Lag_x1
.Lag_x1r:
  s_add_u32 %[t], %[t], 2
  s_cmp_lt_u32 %[t], 64
  s_cbranch_scc1 .Lag_loop
  s_branch .Lag_end
.Lag_w0:
  AG_NEEDW 2
  s_branch .Lag_w0r
.Lag_w1:
  AG_NEEDW 3
  s_branch .Lag_w1r
.Lag_x0:
  AG_NEEDX 1, v7
  s_branch .Lag_x0r
.Lag_x1:
  AG_NEEDX 2, v2
  s_branch .Lag_x1r
.Lag_end:
  s_waitcnt vmcnt(0) lgkmcnt(0)
  s_nop 7
)ASM"
            : "={v[112:143]}"(r0), "={v[144:175]}"(r1), "={v[176:207]}"(r2), "={v[208:239]}"(r3), "={v[240:255]}"(r4),
              [t] "=&s"(s_t), [wk] "=&s"(s_wk), [bo] "=&s"(s_bo), [bc] "=&s"(s_bc), [x] "=&s"(s_x), [sp] "=&s"(s_sp), [t0] "=&s"(s_t0),
              [late] "=&s"(s_late), [c] "=&s"(s_c), [ok] "=&s"(s_ok), [c0] "=&s"(s_c0), [wx] "=&s"(s_wx), [ww] "=&s"(s_ww), [r0] "=&s"(s_r0), "+{v0}"(vo)
            : "{v1}"(abase), "{v4}"(polla), "{v5}"(donea), [rsw] "s"(rsw), [rsf] "s"(rsf), [tmo] "s"(tmo.word), [call] "s"(call), [dl] "s"(DEADLINE)
            : "memory", "vcc", "scc", "v2", "v3", "v6", "v7", "v8", "v9", "v10", "v11", "v12", "v13", "v14", "v15", "v16", "v17", "v18", "v19", "v20", "v21", "v22", "v23", "v24", "v25", "v26", "v27", "v28", "v29", "v30", "v31", "v32", "v33", "v34", "v35", "v36", "v37", "v38", "v39", "v40", "v41", "v42", "v43", "v44", "v45", "v46", "v47", "v48", "v49", "v50", "v51", "v52", "v53", "v54", "v55", "v56", "v57", "v58", "v59", "v60", "v61", "v62", "v63", "v64", "v65", "v66", "v67", "v68", "v69", "v70", "v71", "v72", "v73", "v74", "v75", "v76", "v77", "v78", "v79", "v80", "v81", "v82", "v83", "v84", "v85", "v86", "v87", "v88", "v89", "v90", "v91", "v92", "v93", "v94", "v95", "v96", "v97", "v98", "v99", "v100", "v101", "v102", "v103", "v104", "v105", "v106", "v107");
        f32x4 acc[9][4];
#pragma unroll
        for (int i = 0; i < 144; ++i) acc[i >> 4][(i >> 2) & 3][i & 3] = i < 32 ? r0[i & 31] : i < 64 ? r1[i & 31] : i < 96 ? r2[i & 31] : i < 128 ? r3[i & 31] : r4[i & 15];
        STAMP(3);
        const int ln = (int)(vo >> 4);
        int mx = 0;
        if (host) {
#pragma unroll
            for (int n = 0; n < 4; ++n) { mx = max(max(mx, fbits(acc[8][n][0])), fbits(acc[8][n][1])); mx = max(max(mx, fbits(acc[8][n][2])), fbits(acc[8][n][3])); }
        }
        int m2 = 0;
#pragma unroll
        for (int b = 0; b < 8; ++b)
#pragma unroll
            for (int n = 0; n < 4; ++n) { mx = max(max(mx, fbits(acc[b][n][0])), fbits(acc[b][n][1])); m2 = max(max(m2, fbits(acc[b][n][2])), fbits(acc[b][n][3])); }
        const unsigned wm = wave_max((unsigned)max(mx, m2));
        if (ln == 0) {
            __hip_atomic_fetch_max((LAS unsigned*)(lds + LC_OFF) + C_MAX, wm, __ATOMIC_RELAXED, __HIP_MEMORY_SCOPE_WORKGROUP);
            __hip_atomic_fetch_add((LAS unsigned*)(lds + LC_OFF) + C_CNT, 1u, __ATOMIC_RELAXED, __HIP_MEMORY_SCOPE_WORKGROUP);
        }
        const __amdgpu_buffer_rsrc_t rso = mkrsrc(p.out, 8u * 512u * NL);
        const unsigned ovo = (unsigned)(ln & 15) * NL + ct * 256 + wave * 64 + (ln >> 4) * 16;
        unsigned rdy, gm;
        { Spin sp;
          for (;;) { const u32x2 c = *(volatile LAS u32x2*)(lds + LC_OFF + C_GMAX * 4); gm = rfl(c.x); rdy = rfl(c.y); if (rdy != 0u) break; if (sp.over<4095u>(tmo)) { rdy = 2u; break; } } }
        STAMP(4);
        const float inv = __uint_as_float(gm);
#pragma unroll
        for (int b = 0; b < 9; ++b) {
            u32x4 o = {0u, 0u, 0u, 0u};
#pragma unroll
            for (int c = 0; c < 4; ++c) {
                f32x4 q = acc[b][c] * inv;
                asm volatile("" : "+v"(q));
#pragma unroll
                for (int r = 0; r < 4; ++r) o[r] = __builtin_amdgcn_cvt_pk_u8_f32(q[r], c, o[r]);
            }
            if (rdy != 1u) o = (u32x4){0x80808080u, 0x80808080u, 0x80808080u, 0x80808080u};
            if (b < 8 || host) __builtin_amdgcn_raw_buffer_store_b128(o, rso, ovo + (unsigned)(b < 8 ? rtile * 128 + b * 16 : rank * 512 + xrow) * NL, 0, OAUX);
        }
        STAMP(5);
    } else {
        const int f = wave & 1, ph = (wave >> 1) & 1, rb = sub * 2 + f;
        const int wpos = xpos16(lane >> 2, lane & 3) * 16;
        if (!host && ph == 0) {
#pragma unroll
            for (int j = 0; j < NST; ++j) *(LAS u32x4*)(lds + j * LA_SLOT + 16384 + f * 1024 + lane * 16) = (u32x4){0u, 0u, 0u, 0u};
        }
        STAMP(6);
        __builtin_amdgcn_s_setprio(FPRIO);
        char* xb = me + XS_OFF + (size_t)src * XS_SRC + (size_t)rb * KS * UNIT;
        const float* eb = p.x + (size_t)xrow * KDIM + f * 32;
        const __amdgpu_buffer_rsrc_t rsf = mkrsrc(me + XF_OFF + (size_t)(src * 8 + rb) * KS * 8, KS * 8);
        const unsigned vo = lane * 16, vnb = (lane & 31) * 16 + (lane >> 5) * 8, evo = (unsigned)(lane >> 2) * KDIM * 4 + (lane & 3) * 16;
        unsigned M = 0x83838383u, M3 = 0x03030303u;
        asm volatile("" : "+v"(M), "+v"(M3));
        struct Set { u32x4 L[4], NA, E0, E1; u32x2 NB, fl; unsigned EX; int u; };
        Set s0, s1;
        auto next = [](int u) { return u < SOLO ? u + 1 : u + 2; };
        int nxt = ph ? SOLO + 1 : 0, cur = nxt, lim = NST;
        u64 seen = (nopush && !local) ? ~0ull : 0ull;
        auto issue = [&](Set& s) {
            if (nxt == cur && nxt < KS && !((seen >> nxt) & 1ull)) {
                Spin sp;
                for (;;) {
                    if (rfl(__builtin_amdgcn_raw_buffer_load_b32(rsf, 0, nxt * 8, SYS)) >= call) break;
                    __builtin_amdgcn_s_sleep(1);
                    asm volatile("" ::: "memory");
                    if (sp.over<255u>(tmo)) { seen = ~0ull; break; }
                }
                seen |= 1ull << nxt;
            }
            const bool go = nxt < KS && ((seen >> nxt) & 1ull);
            s.u = go ? nxt : -1;
            const int un = nxt & (KS - 1);
            const __amdgpu_buffer_rsrc_t rsx = mkrsrc(xb, go ? KS * UNIT : 0u), rse = mkrsrc(eb, (go && host) ? 16u * KDIM * 4u - 128u : 0u);
#pragma unroll
            for (int j = 0; j < 4; ++j) s.L[j] = __builtin_amdgcn_raw_buffer_load_b128(rsx, vo + j * 1024, un * UNIT, SYS);
            s.NA = __builtin_amdgcn_raw_buffer_load_b128(rsx, vo, un * UNIT + 4096, SYS);
            s.NB = __builtin_amdgcn_raw_buffer_load_b64(rsx, vnb + 1024, un * UNIT + 4096, SYS);
            s.EX = __builtin_amdgcn_raw_buffer_load_b16(rsx, lane * 2 + 1536, un * UNIT + 4096, SYS);
            s.E0 = __builtin_amdgcn_raw_buffer_load_b128(rse, evo, un * 256, 0);
            s.E1 = __builtin_amdgcn_raw_buffer_load_b128(rse, evo + 64, un * 256, 0);
            s.fl = __builtin_amdgcn_raw_buffer_load_b64(rsf, lane * 8, 0, SYS);
            if (go) nxt = next(nxt);
        };
        auto consume = [&](Set& s) {
            seen |= __builtin_amdgcn_ballot_w64(s.fl.x >= call);
            const int u = s.u;
            const unsigned hdr = __builtin_amdgcn_readlane(s.fl.y, u & 63);
            if (u >= 0) {
                unsigned H[16];
                const unsigned b4 = (hdr & 0x7cu) * 0x01010101u;
#pragma unroll
                for (int g = 0; g < 2; ++g) {
                    const unsigned n0 = g ? s.NA.w : s.NA.x, n1 = g ? s.NB.x : s.NA.y, n2 = g ? s.NB.y : s.NA.z;
                    H[g * 8 + 0] = (n0 & M) | b4; H[g * 8 + 1] = (rotr(n0, 3) & M) | b4;
                    H[g * 8 + 2] = (n1 & M) | b4; H[g * 8 + 3] = (rotr(n1, 3) & M) | b4;
                    H[g * 8 + 4] = (n2 & M) | b4; H[g * 8 + 5] = (rotr(n2, 3) & M) | b4;
                    H[g * 8 + 6] = ((n2 << 1) & 0x80808080u) | (((n0 >> 5) & M3) | b4);
                    H[g * 8 + 7] = ((n2 << 2) & 0x80808080u) | (((n1 >> 5) & M3) | b4);
                }
                if (hdr & 0x83u) {
                    if (hdr & 0x80u) {
                        const __amdgpu_buffer_rsrc_t rsx = mkrsrc(xb, KS * UNIT);
                        u32x4 r[4];
#pragma unroll
                        for (int j = 0; j < 4; ++j) r[j] = __builtin_amdgcn_raw_buffer_load_b128(rsx, vo + j * 1024, u * UNIT + 4096, SYS);
                        asm volatile("" : "+v"(r[0]), "+v"(r[1]), "+v"(r[2]), "+v"(r[3]));
#pragma unroll
                        for (int j = 0; j < 16; ++j) H[j] = r[j >> 2][j & 3];
                    } else {
#pragma unroll
                        for (int j = 0; j < 16; ++j) H[j] += (hdr & 3u) * 0x01010101u;
                    }
                }
                wait_free(lds, u, lim, tmo);
                LAS char* dst = lds + (u & (NST - 1)) * LA_SLOT + f * 8192;
#pragma unroll
                for (int j = 0; j < 8; ++j) {
                    u32x4 o;
                    o.x = __builtin_amdgcn_perm(H[2 * j], s.L[j >> 1][(2 * j) & 3], 0x05010400u); o.y = __builtin_amdgcn_perm(H[2 * j], s.L[j >> 1][(2 * j) & 3], 0x07030602u);
                    o.z = __builtin_amdgcn_perm(H[2 * j + 1], s.L[j >> 1][(2 * j + 1) & 3], 0x05010400u); o.w = __builtin_amdgcn_perm(H[2 * j + 1], s.L[j >> 1][(2 * j + 1) & 3], 0x07030602u);
                    *(LAS u32x4*)(dst + j * 1024 + wpos) = o;
                }
                asm volatile("" ::: "memory");
                const unsigned nlo = (hdr >> 8) & 0xffu, nex = (hdr >> 16) & 0xffu;
                auto patch = [&](unsigned e, unsigned j) {
                    if (j < nex) __hip_atomic_fetch_xor((LAS unsigned*)(dst + ((e & 0x7feu) << 1) + (j >= nlo ? 4096u : 0u)), (e >> 11) << (10u + ((e & 1u) << 4)), __ATOMIC_RELAXED, __HIP_MEMORY_SCOPE_WORKGROUP);
                };
                patch(s.EX, lane);
                if (nex > 64u) {
                    unsigned e2 = __builtin_amdgcn_raw_buffer_load_b16(mkrsrc(xb, KS * UNIT), lane * 2 + 1536 + 128, u * UNIT + 4096, SYS);
                    asm volatile("" : "+v"(e2));
                    patch(e2, lane + 64);
                }
                if (host) {
                    const f32x4 lo = __builtin_bit_cast(f32x4, s.E0) * XSCALE, hi = __builtin_bit_cast(f32x4, s.E1) * XSCALE;
                    u32x4 r; r.x = cvt_pk(lo[0], lo[1]); r.y = cvt_pk(lo[2], lo[3]); r.z = cvt_pk(hi[0], hi[1]); r.w = cvt_pk(hi[2], hi[3]);
                    *(LAS u32x4*)(dst + 16384 - f * 7168 + wpos) = r;
                }
                asm volatile("" ::: "memory");
                cur = next(u);
                ctl[C_FULL + (wave & 3)] = cur >= KS ? 1000u : (unsigned)cur;
            }
        };
        int wtodo = (wave == 7 && ci < (KS - 2 * NCW) * 4) ? 2 : 1, wat = WSTEP + ci;
        --wtodo; wat += 8;
        issue(s0);
        while (cur < KS) {
            issue(s1); SCHED(); consume(s0); SCHED();
            issue(s0); SCHED(); consume(s1); SCHED();
            if (wtodo && cur >= wat) { const bool last = wat != WSTEP + ci; --wtodo; wat += 8; convert_w(p.w, me, ct, last ? 2 * NCW + (ci >> 2) : NCW + ci, last ? ci & 3 : wave & 3, lane, call); STAMP(1); }
        }
        if (wave != 4) return;
        { unsigned sp = 0; const unsigned t0 = rt32();
          while (rfl(ctl[C_DONE]) < (unsigned)KS) { __builtin_amdgcn_s_sleep(4); if ((++sp & 1023u) == 0u && tmo.expired(t0)) break; }
          while (rfl(ctl[C_CNT]) < 4u) { if ((++sp & 4095u) == 0u && tmo.expired(t0)) break; } }
        const unsigned tm = rfl(ctl[C_MAX]);
        if (lane == 0) __hip_atomic_store((GAS u64*)(me + A1_OFF) + (par * NWG + bx) * 8, ((u64)call << 32) | tm, __ATOMIC_RELAXED, __HIP_MEMORY_SCOPE_SYSTEM);
        STAMP(3);
        unsigned g = 0;
        { unsigned sp = 0; const unsigned t0 = rt32();
          const GAS u64* a2 = (const GAS u64*)(me + A2_OFF + (size_t)(par * 8 + (bx & 7)) * 128) + (lane & 7);
          for (;;) {
              const u64 v = __hip_atomic_load(a2, __ATOMIC_RELAXED, __HIP_MEMORY_SCOPE_SYSTEM);
              g = (unsigned)v;
              if (__all((unsigned)(v >> 32) == call)) break;
              if ((++sp & 255u) == 0u && tmo.expired(t0)) break;
          } }
        g = max(g, (unsigned)__shfl_xor((int)g, 1)); g = max(g, (unsigned)__shfl_xor((int)g, 2)); g = max(g, (unsigned)__shfl_xor((int)g, 4));
        const bool bad = tmo.hit();
        const float scale = __uint_as_float(g) / 127.0f;
        *(volatile LAS u32x2*)(lds + LC_OFF + C_GMAX * 4) = (u32x2){__float_as_uint(scale > 0.f ? 1.0f / scale : 0.f), bad ? 2u : 1u};
        if (lane == 0) __hip_atomic_store(seqw, call, __ATOMIC_RELAXED, __HIP_MEMORY_SCOPE_AGENT);
        STAMP(4);
    }
}

}

extern "C" void kernel_launch(int rank, int nranks, ncclComm_t comm,
                              hipStream_t stream,
                              void* const* d_in, const int* in_sizes, int n_in,
                              void* d_out, int out_size,
                              const int* in_dtypes, int out_dtype,
                              void* const* d_out_all,
                              void* const* d_ws_all, size_t ws_bytes) {
    if (nranks != NR || (unsigned)rank >= (unsigned)NR || ws_bytes < WS_NEED || n_in < 2) {
        fprintf(stderr, "unsupported configuration\n");
        return;
    }
    HIP_CHECK(hipFuncSetAttribute((const void*)ag_gemm, hipFuncAttributeMaxDynamicSharedMemorySize, LDS_BYTES));
    Params p;
    p.x = (const float*)d_in[0]; p.w = (const float*)d_in[1]; p.out = (signed char*)d_out;
    for (int i = 0; i < NR; ++i) p.peer[i] = (char*)d_ws_all[(rank + 1 + i) % NR];
    p.rank = rank; p.pad = 0;
    hipLaunchKernelGGL(ag_gemm, dim3(NWG), dim3(512), LDS_BYTES, stream, p);
    HIP_CHECK(hipGetLastError());
}
```

```cpp
#include <hip/hip_runtime.h>
#include <rccl/rccl.h>
#include <cstdio>

#define HIP_CHECK(expr) do { hipError_t _e = (expr); if (_e != hipSuccess) { \
    fprintf(stderr, "HIP error %s at %s:%d\n", hipGetErrorString(_e), __FILE__, __LINE__); \
    return; } } while (0)

namespace {

#define LAS __attribute__((address_space(3)))
#define GAS __attribute__((address_space(1)))
typedef unsigned u32x4 __attribute__((ext_vector_type(4)));
typedef unsigned u32x2 __attribute__((ext_vector_type(2)));
typedef float f32x4 __attribute__((ext_vector_type(4)));
typedef float f32x16 __attribute__((ext_vector_type(16)));
typedef float f32x32 __attribute__((ext_vector_type(32)));
typedef __bf16 bf16x8 __attribute__((ext_vector_type(8)));
typedef unsigned long long u64;

constexpr int NR = 8;
constexpr int KDIM = 4096, NL = 2048;
constexpr int KS = 64;
constexpr int NWG = 256;
constexpr int UNIT = 8192;
constexpr int EXCAP = 128;

constexpr size_t XS_OFF = 0;
constexpr size_t XS_SRC = (size_t)8 * KS * UNIT;
constexpr size_t WB_OFF = XS_OFF + NR * XS_SRC;
constexpr size_t XF_OFF = WB_OFF + (size_t)32 * KS * 8192;
constexpr size_t WF_OFF = XF_OFF + (size_t)NR * 8 * KS * 8;
constexpr size_t SQ_OFF = WF_OFF + (size_t)32 * KS * 64;
constexpr size_t A1_OFF = SQ_OFF + (size_t)NWG * 128;
constexpr size_t A2_OFF = A1_OFF + (size_t)2 * NWG * 64;
constexpr size_t A3_OFF = A2_OFF + (size_t)2 * 8 * 128;
constexpr size_t TK_OFF = A3_OFF + (size_t)2 * 8 * 128;
constexpr size_t TM_OFF = TK_OFF + 128;
constexpr size_t DG_OFF = TM_OFF + 128;
constexpr size_t WS_NEED = DG_OFF + (size_t)NWG * 8 * 128 * 8;

constexpr int NST = 6, LA_SLOT = 9216, LA_HALF = 3 * LA_SLOT, LA_PAR = 2 * LA_HALF;
constexpr int LC_OFF = 2 * LA_PAR;
constexpr int C_FULL = 0;
constexpr int C_DONE = 4;
constexpr int C_MAX = 8, C_CNT = 9, C_GMAX = 10, C_RDY = 11, C_SENT = 12, C_ISS = 13, C_SEQ = 14;
constexpr int LE_OFF = LC_OFF + 64;
constexpr int LDS_BYTES = LE_OFF + 8192;
static_assert(LDS_BYTES > 81920, "one workgroup per CU");

#define CPX 3
#define PW 4
#define WDELAY 20
#define WDELAY2 340
#define FPRIO 0
#define PWIN 2
#define PEARLY 1
#define SOLO 0
#define PSLEEP 8
#define WSTAG 12
#define WSTEP 4
#define PREFN 2
constexpr int CODE_BYTES = 32768;
#define OAUX 17
#define WALL 1
#define XDG "0"
#define WWAVE 0
#define WAUX 17
#define WDELAY3 100
#define WJIT 0
#define WSLOPE 2
#define SWIN 24
#define SREL 16
#define SPEC 1
#define VALIGN "0"
#define VBS "0"
#define MPRIO 0
#define WX 7
#define WNU 3
#define MDELAY 600
#define SPECN 208
#define DPRIO "0"
#define XLG "0"
#define XVM "8"
constexpr int NCOMM = 8 * CPX;
constexpr float XSCALE = 0.5f;
constexpr unsigned DEADLINE = 40000000u;

struct Params {
    const float* x; const float* w; signed char* out;
    char* peer[NR];
    int rank; int pad;
};

__device__ __forceinline__ u64 rt() { return __builtin_amdgcn_s_memrealtime(); }
__device__ __forceinline__ unsigned rt32() { return (unsigned)__builtin_amdgcn_s_memrealtime(); }
__device__ __forceinline__ unsigned rfl(unsigned v) { return __builtin_amdgcn_readfirstlane(v); }
__device__ __forceinline__ unsigned cvt_pk(float lo, float hi) {
    typedef float f2 __attribute__((ext_vector_type(2))); typedef __bf16 b2 __attribute__((ext_vector_type(2)));
    f2 v = {lo, hi}; return __builtin_bit_cast(unsigned, __builtin_convertvector(v, b2));
}
__device__ __forceinline__ __amdgpu_buffer_rsrc_t mkrsrc(const void* p, unsigned bytes) {
    return __builtin_amdgcn_make_buffer_rsrc((void*)p, 0, bytes, 0x00020000);
}
__device__ __forceinline__ int fbits(float v) { return __float_as_int(v); }
__device__ __forceinline__ unsigned rotr(unsigned v, int n) { return __builtin_amdgcn_alignbit(v, v, n); }
#define VMWAIT0() asm volatile("s_waitcnt vmcnt(0)" ::: "memory")
#define LGKM0() asm volatile("s_waitcnt lgkmcnt(0)" ::: "memory")
#define SCHED() __builtin_amdgcn_sched_barrier(0)
#define SGB(m, n) __builtin_amdgcn_sched_group_barrier(m, n, 0)
constexpr int SYS = 17;

struct Tmo {
    GAS unsigned* word; unsigned call;
    __device__ __forceinline__ bool hit() const {
        unsigned v;
        asm volatile("v_mov_b32 %0, 0\n\tglobal_load_dword %0, %0, %1 sc1\n\ts_waitcnt vmcnt(0)" : "=&v"(v) : "s"(word) : "memory");
        return rfl(v) == call;
    }
    __device__ __forceinline__ bool expired(unsigned t0) const {
        if (hit() || rt32() - t0 > DEADLINE) {
            unsigned z, c;
            asm volatile("v_mov_b32 %0, 0\n\tv_mov_b32 %1, %3\n\tglobal_store_dword %0, %1, %2 sc1\n\ts_waitcnt vmcnt(0)" : "=&v"(z), "=&v"(c) : "s"(word), "s"(call) : "memory");
            return true;
        }
        return false;
    }
};

struct Spin {
    unsigned n = 0, t0 = 0;
    template <unsigned MASK> __device__ __forceinline__ bool over(const Tmo& tmo) {
        if ((++n & MASK) != 0u) return false;
        if (n == MASK + 1u) { t0 = rt32(); return false; }
        return tmo.expired(t0);
    }
};

#define PSTAMP(r) do {} while (0)
#define STAMP(i) do {} while (0)

__device__ __forceinline__ void w_load(f32x4 (&v)[16], const float* __restrict__ w, bool on, int ct, int t, int q, int lane) {
    const __amdgpu_buffer_rsrc_t rs = mkrsrc(w + (size_t)t * 64 * NL + ct * 256 + q * 64, on ? 63u * NL * 4u + 256u : 0u);
    const int n = lane & 15, g = lane >> 4;
    const unsigned vo = (unsigned)((g >> 1) * 16 + (g & 1) * 2) * NL * 4 + n * 16;
#pragma unroll
    for (int i = 0; i < 16; ++i) v[i] = __builtin_bit_cast(f32x4, __builtin_amdgcn_raw_buffer_load_b128(rs, vo, ((i >> 3) * 32 + ((i >> 1) & 3) * 4 + (i & 1)) * NL * 4, 2));
}
__device__ __forceinline__ void w_store(const f32x4 (&v)[16], char* me, int ct, int t, int q, int lane) {
    const __amdgpu_buffer_rsrc_t rs = mkrsrc(me + WB_OFF + ((size_t)(ct * 4 + q) * KS + t) * 8192, 8192);
#pragma unroll
    for (int kh = 0; kh < 2; ++kh)
#pragma unroll
        for (int c = 0; c < 4; ++c) {
            u32x4 o;
            o.x = cvt_pk(v[kh * 8 + 0][c], v[kh * 8 + 1][c]); o.y = cvt_pk(v[kh * 8 + 2][c], v[kh * 8 + 3][c]);
            o.z = cvt_pk(v[kh * 8 + 4][c], v[kh * 8 + 5][c]); o.w = cvt_pk(v[kh * 8 + 6][c], v[kh * 8 + 7][c]);
            __builtin_amdgcn_raw_buffer_store_b128(o, rs, lane * 16 + c * 1024 + kh * 4096, 0, SYS);
        }
}

__device__ __forceinline__ char* pick_peer(const Params& p, int q) {
    char* r = p.peer[0];
#pragma unroll
    for (int i = 1; i < NR; ++i) { char* c = p.peer[i]; asm volatile("" : "+s"(c)); r = q == i ? c : r; }
    return r;
}

__device__ __forceinline__ void touch(const void* ptr, unsigned junk) {
    unsigned keep;
    asm volatile("s_mov_b32 %0, m0\n\ts_mov_b32 m0, %2\n\ts_nop 0\n\tglobal_load_lds_dword %1, off\n\ts_mov_b32 m0, %0" : "=&s"(keep) : "v"(ptr), "s"(junk) : "memory");
}

__device__ __forceinline__ unsigned wave_max(unsigned g) {
    g = max(g, (unsigned)__builtin_amdgcn_update_dpp(0u, g, 0x111, 0xf, 0xf, true));
    g = max(g, (unsigned)__builtin_amdgcn_update_dpp(0u, g, 0x112, 0xf, 0xf, true));
    g = max(g, (unsigned)__builtin_amdgcn_update_dpp(0u, g, 0x114, 0xf, 0xf, true));
    g = max(g, (unsigned)__builtin_amdgcn_update_dpp(0u, g, 0x118, 0xf, 0xf, true));
    return max(max((unsigned)__builtin_amdgcn_readlane(g, 15), (unsigned)__builtin_amdgcn_readlane(g, 31)), max((unsigned)__builtin_amdgcn_readlane(g, 47), (unsigned)__builtin_amdgcn_readlane(g, 63)));
}

__global__ void __launch_bounds__(512, 2) ag_gemm(Params p) {
    extern __shared__ __attribute__((aligned(16))) char lds_raw[];
    LAS char* lds = (LAS char*)lds_raw;
    volatile LAS unsigned* ctl = (volatile LAS unsigned*)(lds + LC_OFF);
    const int lane = threadIdx.x & 63, wave = rfl(threadIdx.x >> 6);
    const int bx = blockIdx.x, xcd = bx & 7, idx = bx >> 3;
    const unsigned junk = (unsigned)(size_t)(lds + LE_OFF + 7 * 1024);
    if (wave == 7 && idx < PREFN) {
        const char* pc = (const char*)__builtin_amdgcn_s_getpc();
#pragma unroll
        for (int j = 0; j < CODE_BYTES / 8192; ++j) touch(pc + j * 8192 + lane * 128, junk);
    }
    asm volatile("" :: "s"(p.x), "s"(p.w), "s"(p.out), "s"(p.rank), "s"(p.peer[0]), "s"(p.peer[1]), "s"(p.peer[2]), "s"(p.peer[3]),
                       "s"(p.peer[4]), "s"(p.peer[5]), "s"(p.peer[6]), "s"(p.peer[7]));
    const int rank = p.rank & 7;
    const int rel = idx & 7, src = (rank + 1 + rel) & 7, sub = xcd >> 1, ct = (xcd & 1) * 4 + (idx >> 3), rtile = src * 4 + sub;
    char* me = p.peer[NR - 1];
    const bool local = rel == NR - 1;
    const int jj = ((idx >> 3) + sub) & 3;
    const bool comm = local && jj < CPX;
    const int h = rel * 4 + sub;
    const bool host = h < 8 * CPX;
    const int xrow = (((h >> 3) - (ct & 3)) & 3) * 128 + (h & 7) * 16;
    constexpr int NCW = 32 - CPX;
    static_assert(PEARLY <= PW && PW <= 6, "roles");
    const int ci = local ? 28 + (jj - CPX) : h;
    const int cid = jj * 8 + xcd;
    GAS unsigned* seqw = (GAS unsigned*)(me + SQ_OFF + (size_t)bx * 128);
#define WG_START() do { if (threadIdx.x < 16) ctl[threadIdx.x] = (threadIdx.x == C_FULL + 2 || threadIdx.x == C_FULL + 3) ? 1u : 0u; \
        LGKM0(); __builtin_amdgcn_s_barrier(); } while (0)
#define NOPUSH false
#define GET_CALL() const unsigned call = rfl(__hip_atomic_load(seqw, __ATOMIC_RELAXED, __HIP_MEMORY_SCOPE_AGENT)) + 1u; \
    const Tmo tmo{(GAS unsigned*)(me + TM_OFF), call}; const unsigned par = call & 1u; const bool nopush = NOPUSH; (void)par; (void)nopush
    STAMP(0);

    if (__builtin_expect(comm, 1)) {
        if (__builtin_expect(wave < PW, 1)) {
            f32x4 v[16];
            auto load_x = [&](int u) {
                const float* s0 = p.x + (size_t)((u & 7) * 64 + (lane >> 2)) * KDIM + (u >> 3) * 64 + (lane & 3) * 4;
#pragma unroll
                for (int b = 0; b < 4; ++b)
#pragma unroll
                    for (int i = 0; i < 4; ++i) v[b * 4 + i] = *(const GAS f32x4*)(s0 + (size_t)b * 16 * KDIM + i * 16);
            };
            if (wave >= PEARLY) __builtin_amdgcn_s_sleep(PSLEEP);
            load_x(wave * NCOMM + cid);
            SCHED();
            WG_START();
            GET_CALL();
            constexpr unsigned NTK = 8 * KS;
            __amdgpu_buffer_rsrc_t rs[NR];
#pragma unroll
            for (int q = 0; q < NR; ++q) rs[q] = mkrsrc(p.peer[q] + XS_OFF + (size_t)rank * XS_SRC, (q == NR - 1 || !nopush) ? (unsigned)XS_SRC : 0u);
            GAS u64* flag = (GAS u64*)(pick_peer(p, lane & 7) + XF_OFF) + rank * 8 * KS;
            GAS unsigned* tk = (GAS unsigned*)(me + TK_OFF);
            LAS char* elist = lds + LE_OFF + wave * 1024;
            const unsigned ptag = (unsigned)lane << 1, vo = lane * 16;
            unsigned u = wave * NCOMM + cid;
            int k = wave;
            while (u < 8u * KS) {
                const unsigned fi = (u & 7u) * KS + (u >> 3), soff = fi * UNIT;
                u32x4 L[4], H[4], NA, T;
                unsigned sum = 0;
#pragma unroll
                for (int i = 0; i < 16; ++i) {
                    const f32x4 s = v[((i >> 1) & 3) * 4 + (i >> 3) * 2 + (i & 1)] * XSCALE;
                    const unsigned d0 = cvt_pk(s[0], s[1]), d1 = cvt_pk(s[2], s[3]);
                    L[i >> 2][i & 3] = __builtin_amdgcn_perm(d1, d0, 0x06040200u);
                    H[i >> 2][i & 3] = __builtin_amdgcn_perm(d1, d0, 0x07050301u);
                }
                SCHED();
                PSTAMP(2);
                if (k < PEARLY) {
#pragma unroll
                    for (int j = 0; j < 4; ++j)
#pragma unroll
                        for (int q = 0; q < NR; ++q) __builtin_amdgcn_raw_buffer_store_b128(L[j], rs[q], vo + j * 1024, soff, SYS);
                    SCHED(); asm volatile("s_nop 1" ::: "memory"); SCHED();
                }
#pragma unroll
                for (int i = 0; i < 16; ++i) sum = __builtin_amdgcn_sad_u8(H[i >> 2][i & 3] & 0x7f7f7f7fu, 0u, sum);
                sum += __builtin_amdgcn_update_dpp(0u, sum, 0x111, 0xf, 0xf, true);
                sum += __builtin_amdgcn_update_dpp(0u, sum, 0x112, 0xf, 0xf, true);
                sum += __builtin_amdgcn_update_dpp(0u, sum, 0x114, 0xf, 0xf, true);
                sum += __builtin_amdgcn_update_dpp(0u, sum, 0x118, 0xf, 0xf, true);
                const int tot = __builtin_amdgcn_readlane(sum, 15) + __builtin_amdgcn_readlane(sum, 31) + __builtin_amdgcn_readlane(sum, 47) + __builtin_amdgcn_readlane(sum, 63);
                int bs = (tot - 5939) >> 12;
                bs = bs < 0 ? 0 : (bs > 124 ? 124 : bs);
                const unsigned b4 = (unsigned)bs * 0x01010101u, nb4 = (unsigned)((-bs) & 3) * 0x01010101u;
                unsigned N[6], cnt = 0, nlo = 0;
#pragma unroll
                for (int g = 0; g < 2; ++g) {
                    unsigned c[8], sg[8], x[8], rest = 0;
                    auto emit = [&](int e) {
                        const u64 m = __builtin_amdgcn_ballot_w64(x[e] != 0u);
                        const unsigned sh = (unsigned)__builtin_ctz(x[e] | 0x80000000u) & 24u;
                        const unsigned pos = min(cnt, (unsigned)EXCAP + 1u) + __builtin_amdgcn_mbcnt_hi((unsigned)(m >> 32), __builtin_amdgcn_mbcnt_lo((unsigned)m, 0u));
                        const unsigned ent = (((x[e] >> sh) & 0x7cu) << 9) | ((unsigned)e << 8) | ((sh & 16u) << 3) | ptag | ((sh >> 3) & 1u);
                        *(LAS unsigned short*)(elist + (x[e] != 0u ? pos * 2 : 512u + lane * 2)) = (unsigned short)ent;
                        x[e] &= ~(0xffu << sh);
                        cnt += (unsigned)__builtin_popcountll(m);
                    };
#pragma unroll
                    for (int e = 0; e < 8; ++e) {
                        const int i = g * 8 + e;
                        const unsigned hh = H[i >> 2][i & 3], hm = hh & 0x7f7f7f7fu;
                        c[e] = (hm + nb4) & 0x03030303u;
                        sg[e] = hh & 0x80808080u;
                        x[e] = hm ^ (c[e] + b4);
                        emit(e);
                        rest |= x[e];
                    }
                    while (__builtin_amdgcn_ballot_w64(rest != 0u) != 0ull && cnt <= (unsigned)EXCAP) {
                        rest = 0;
#pragma unroll
                        for (int e = 0; e < 8; ++e) { if (__builtin_amdgcn_ballot_w64(x[e] != 0u) != 0ull) emit(e); rest |= x[e]; }
                    }
                    if (g == 0) nlo = cnt;
                    N[g * 3 + 0] = (c[0] | sg[0]) | rotr(c[1] | sg[1], 29) | (c[6] << 5);
                    N[g * 3 + 1] = (c[2] | sg[2]) | rotr(c[3] | sg[3], 29) | (c[7] << 5);
                    N[g * 3 + 2] = (c[4] | sg[4]) | rotr(c[5] | sg[5], 29) | (sg[6] >> 1) | (sg[7] >> 2);
                }
                NA = (u32x4){N[0], N[1], N[2], N[3]};
                const unsigned u4 = __builtin_amdgcn_permlane32_swap(N[4], N[4], false, false)[1], u5 = __builtin_amdgcn_permlane32_swap(N[5], N[5], false, false)[1];
                const u32x4 el = *(volatile LAS u32x4*)(elist + (lane & 15) * 16);
                T = lane < 32 ? (u32x4){N[4], N[5], u4, u5} : el;
                unsigned hdr = ((unsigned)bs | (nlo << 8) | (cnt << 16));
                if (cnt > (unsigned)EXCAP) { hdr = 0x80u; NA = H[0]; T = H[1]; }
                unsigned vt = ((hdr & 0x80u) || (unsigned)lane < 32u + (((hdr >> 16) + 7u) >> 3)) ? vo + 1024 : 0x40000000u;
                asm volatile("" : "+v"(NA), "+v"(T), "+v"(vt));
                PSTAMP(3);
                SCHED();
                if (k >= PEARLY) {
                    Spin sp;
                    for (;;) {
                        const u32x2 c = *(volatile LAS u32x2*)(lds + LC_OFF + C_SENT * 4);
                        if ((int)rfl(c.x) + PWIN > k && (int)rfl(c.y) >= k) break;
                        __builtin_amdgcn_s_sleep(1);
                        if (sp.over<1023u>(tmo)) break;
                    }
                    asm volatile("" ::: "memory");
                    PSTAMP(4);
#pragma unroll
                    for (int j = 0; j < 4; ++j) {
#pragma unroll
                        for (int q = 0; q < NR; ++q) __builtin_amdgcn_raw_buffer_store_b128(L[j], rs[q], vo + j * 1024, soff, SYS);
                        asm volatile("s_waitcnt vmcnt(%0)" :: "i"(SWIN) : "memory");
                    }
                }
#pragma unroll
                for (int q = 0; q < NR; ++q) __builtin_amdgcn_raw_buffer_store_b128(NA, rs[q], vo, soff + 4096, SYS);
                asm volatile("s_waitcnt vmcnt(%0)" :: "i"(SWIN) : "memory");
#pragma unroll
                for (int q = 0; q < NR; ++q) __builtin_amdgcn_raw_buffer_store_b128(T, rs[q], vt, soff + 4096, SYS);
                asm volatile("s_waitcnt vmcnt(%0)" :: "i"(SREL) : "memory");
                if (lane == 0) __hip_atomic_fetch_add((LAS unsigned*)(lds + LC_OFF) + C_ISS, 1u, __ATOMIC_RELAXED, __HIP_MEMORY_SCOPE_WORKGROUP);
                if (hdr & 0x80u) {
#pragma unroll
                    for (int j = 2; j < 4; ++j)
#pragma unroll
                        for (int q = 0; q < NR; ++q) __builtin_amdgcn_raw_buffer_store_b128(H[j], rs[q], vo + j * 1024, soff + 4096, SYS);
                }
                SCHED(); asm volatile("s_nop 1" ::: "memory"); SCHED();
                PSTAMP(5);
                unsigned tv = 0;
                if (lane == 0) tv = __hip_atomic_fetch_add(tk, 1u, __ATOMIC_RELAXED, __HIP_MEMORY_SCOPE_AGENT);
                VMWAIT0();
                PSTAMP(6);
                if (lane == 0) __hip_atomic_fetch_add((LAS unsigned*)(lds + LC_OFF) + C_SENT, 1u, __ATOMIC_RELAXED, __HIP_MEMORY_SCOPE_WORKGROUP);
                if (lane == NR - 1 || (lane < NR && !nopush)) __hip_atomic_store(flag + fi, ((u64)hdr << 32) | call, __ATOMIC_RELAXED, __HIP_MEMORY_SCOPE_SYSTEM);
                PSTAMP(7);
                u = PW * NCOMM + (rfl(tv) - (call - 1u) * NTK);
                if (u < 8u * KS) {
                    load_x((int)u);
                    unsigned kq = 0;
                    if (lane == 0) kq = __hip_atomic_fetch_add((LAS unsigned*)(lds + LC_OFF) + C_SEQ, 1u, __ATOMIC_RELAXED, __HIP_MEMORY_SCOPE_WORKGROUP);
                    k = PW + (int)rfl(kq);
                }
            }
            if (lane == 0) __hip_atomic_fetch_add((LAS unsigned*)(lds + LC_OFF) + C_CNT, 1u, __ATOMIC_RELAXED, __HIP_MEMORY_SCOPE_WORKGROUP);
        } else {
            WG_START();
            if (wave != 7) return;
            GET_CALL();
            if (lane == 0) __hip_atomic_store((GAS u64*)(me + A1_OFF) + (par * NWG + bx) * 8, (u64)call << 32, __ATOMIC_RELAXED, __HIP_MEMORY_SCOPE_SYSTEM);
            { Spin sp; while (rfl(ctl[C_CNT]) < (unsigned)PW) { __builtin_amdgcn_s_sleep(8); if (sp.over<1023u>(tmo)) break; } }
            if (lane == 0) __hip_atomic_store(seqw, call, __ATOMIC_RELAXED, __HIP_MEMORY_SCOPE_AGENT);
            if (cid < 2) {
                const __amdgpu_buffer_rsrc_t rs = mkrsrc(me + A1_OFF + (size_t)par * NWG * 64, NWG * 64);
                GAS u64* dst = (GAS u64*)(pick_peer(p, lane >> 3) + (cid ? A3_OFF : A2_OFF) + (size_t)(par * 8 + (lane & 7)) * 128) + rank;
                unsigned g = 0, sent = 0; Spin sp;
                for (;;) {
                    u32x2 a[4];
#pragma unroll
                    for (int j = 0; j < 4; ++j) a[j] = __builtin_amdgcn_raw_buffer_load_b64(rs, lane * 64, j * 4096, SYS);
                    g = 0;
#pragma unroll
                    for (int j = 0; j < 4; ++j) g = max(g, a[j].y == call ? a[j].x : 0u);
                    if (__all(a[0].y == call && a[1].y == call && a[2].y == call && a[3].y == call)) break;
                    int nin = 0;
#pragma unroll
                    for (int j = 0; j < 4; ++j) nin += __builtin_popcountll(__builtin_amdgcn_ballot_w64(a[j].y == call));
                    if (cid && nin >= SPECN && __any(g > sent)) { sent = wave_max(g); __hip_atomic_store(dst, ((u64)call << 32) | sent, __ATOMIC_RELAXED, __HIP_MEMORY_SCOPE_SYSTEM); }
                    asm volatile("" ::: "memory");
                    if (sp.over<255u>(tmo)) break;
                }
                if (cid == 0) {
                    g = wave_max(g);
                    __hip_atomic_store(dst, ((u64)call << 32) | g, __ATOMIC_RELAXED, __HIP_MEMORY_SCOPE_SYSTEM);
                    STAMP(2);
                }
            }
        }
        return;
    }

    if (local) {
        if (wave >= 4) {
            const char* pa = wave == 4 ? p.peer[0] : wave == 5 ? p.peer[2] : wave == 6 ? p.peer[4] : p.peer[6];
            const char* pb = wave == 4 ? p.peer[1] : wave == 5 ? p.peer[3] : wave == 6 ? p.peer[5] : p.peer[7];
            const int w = (lane >> 1) & 7;
            touch(((lane & 1) ? pb : pa) + (w < 4 ? XS_OFF + (size_t)rank * XS_SRC + (size_t)w * (XS_SRC / 4) : XF_OFF + (size_t)rank * 8 * KS * 8), junk);
        }
    } else if (wave == 0) {
        const size_t o1 = (size_t)rtile * 128 * NL, o2 = (size_t)(rank * 512 + xrow) * NL;
        touch(lane == 0 ? (const char*)p.out + o1 : lane == 1 ? (const char*)p.out + o2 : lane == 2 ? (const char*)(p.x + (size_t)xrow * KDIM) : me + XS_OFF + (size_t)rtile * (XS_SRC / 4), junk);
    }
    const unsigned tstart = rt32();
    WG_START();
    unsigned call, wfl = 0;
    {
        constexpr int NH = 8 * CPX, NN = NCW - NH, TH = KS - 4 * NN;
        static_assert(TH >= NH && TH <= 2 * NH, "at most one unit per wave of a host");
        const int t1 = host ? (wave < 4 ? NH + ci : ci) : TH + (ci - NH) * 4 + (wave >> 2) * 2, q = wave & 3;
        const bool one = !host || t1 < TH;
        while (rt32() - tstart < (host ? (wave < 4 ? WDELAY2 : WDELAY) + ci * WSTAG : WDELAY3)) __builtin_amdgcn_s_sleep(2);
        f32x4 va[16], vb[16];
        w_load(va, p.w, one, ct, t1, q, lane);
        if (!host) w_load(vb, p.w, true, ct, t1 + 1, q, lane);
        const unsigned sv = __hip_atomic_load(seqw, __ATOMIC_RELAXED, __HIP_MEMORY_SCOPE_AGENT);
        SCHED();
        if (one) w_store(va, me, ct, t1, q, lane);
        if (!host) w_store(vb, me, ct, t1 + 1, q, lane);
        if (wave < 4 && lane < NH) wfl = __hip_atomic_load((GAS unsigned*)(me + WF_OFF) + ((ct * 4 + wave) * KS + lane) * 16, __ATOMIC_RELAXED, __HIP_MEMORY_SCOPE_SYSTEM);
        VMWAIT0();
        call = rfl(sv) + 1u;
        if (lane < (host ? (one ? 1 : 0) : 2)) __hip_atomic_store((GAS unsigned*)(me + WF_OFF) + ((ct * 4 + q) * KS + t1 + lane) * 16, call, __ATOMIC_RELAXED, __HIP_MEMORY_SCOPE_SYSTEM);
    }
    const Tmo tmo{(GAS unsigned*)(me + TM_OFF), call};
    const unsigned par = call & 1u;
    const bool nopush = NOPUSH;
    if (wave < 4) {
        STAMP(1);
        const __amdgpu_buffer_rsrc_t rsw = mkrsrc(me + WB_OFF + (size_t)(ct * 4 + wave) * KS * 8192, KS * 8192);
        const __amdgpu_buffer_rsrc_t rsf = mkrsrc(me + WF_OFF + (size_t)(ct * 4 + wave) * KS * 64, KS * 64);
        const unsigned ldsb = (unsigned)(size_t)lds;
        unsigned vo = lane * 16;
        const unsigned abase = ldsb + lane * 16;
        const unsigned polla = ldsb + LC_OFF + (C_FULL + (lane & 3)) * 4, donea = ldsb + LC_OFF + (C_DONE + wave) * 4;
        f32x32 r0, r1, r2, r3; f32x16 r4;
        static_assert(LA_SLOT == 9216 && LA_HALF == 27648 && LA_PAR == 55296, "literals in the K loop");
        int s_wk = __builtin_ctzll(__builtin_amdgcn_ballot_w64(wfl < call)) - 1;
        unsigned s_so, s_t, s_bo, s_bc, s_x, s_sp, s_t0, s_late; u64 s_c, s_c0, s_r0; unsigned s_wx, s_ww;
        __builtin_amdgcn_s_setprio(MPRIO);
        asm volatile(R"ASM(
.set AG_A, 8
.set AG_C, 112
.set AG_DIAG, )ASM" XDG R"ASM(
.macro AG_EXPIRED lout
  s_add_u32 %[sp], %[sp], 1
  s_and_b32 %[x], %[sp], 0x3ff
  s_cbranch_scc1 8f
  v_mov_b32 v6, 0
  global_load_dword v3, v6, %[tmo] sc1
  s_memrealtime %[c]
  s_waitcnt vmcnt(0) lgkmcnt(0)
  v_readfirstlane_b32 %[x], v3
  s_cmp_eq_u32 %[x], %[call]
  s_cbranch_scc1 \lout
  s_mov_b64 vcc, %[c]
  s_cmp_eq_u32 %[sp], 0x400
  s_cselect_b32 %[t0], vcc_lo, %[t0]
  s_sub_u32 %[x], vcc_lo, %[t0]
  s_cmp_le_u32 %[x], 40000000
  s_cbranch_scc1 8f
  v_mov_b32 v3, %[call]
  global_store_dword v6, v3, %[tmo] sc1
  s_waitcnt vmcnt(0)
  s_branch \lout
8:
.endm
.macro AG_TIC
  s_mov_b32 %[sp], 0
  .if AG_DIAG
  s_memrealtime %[c]
  s_waitcnt lgkmcnt(0)
  s_mov_b64 vcc, %[c]
  s_mov_b32 %[t0], vcc_lo
  .endif
.endm
.macro AG_TOC acc
  .if AG_DIAG
  s_memrealtime %[c]
  s_waitcnt lgkmcnt(0)
  s_mov_b64 vcc, %[c]
  s_sub_u32 %[x], vcc_lo, %[t0]
  s_add_u32 \acc, \acc, %[x]
  .endif
.endm
; wait until w K-step (t + add + 1) is converted; wk = last K-step known to be converted with all before it
.macro AG_NEEDW add
  s_waitcnt lgkmcnt(0)
  AG_TIC
1:
  v_lshlrev_b32 v6, 2, v0
  buffer_load_dword v3, v6, %[rsf], 0 offen sc0 sc1
  s_waitcnt vmcnt(0)
  v_cmp_gt_u32 vcc, %[call], v3
  s_ff1_i32_b64 %[wk], vcc
  s_min_u32 %[wk], %[wk], 1000
  s_sub_i32 %[wk], %[wk], 1
  s_add_u32 %[x], %[t], \add
  s_cmp_gt_i32 %[wk], %[x]
  s_cbranch_scc1 3f
  s_sleep 2
  AG_EXPIRED 2f
  s_branch 1b
2:
  s_movk_i32 %[wk], 1000
3:
  AG_TOC %[ww]
  ds_read_b32 v3, v4
  s_add_u32 %[x], %[t], \add
.endm
; wait until x K-step (t + add) is in the ring, then read its first half
.macro AG_NEEDX add, va, nb
  AG_TIC
  s_add_u32 %[late], %[late], 1
1:
  ds_read_b32 v3, v4
  s_add_u32 %[x], %[t], \add
  s_waitcnt lgkmcnt(0)
  v_cmp_lt_u32 vcc, %[x], v3
  s_cmp_eq_u64 vcc, exec
  s_cbranch_scc1 2f
  s_sleep 1
  AG_EXPIRED 2f
  s_branch 1b
2:
  AG_TOC %[wx]
  .set ag_b, 0
  .rept 8
  ds_read_b128 v[AG_A+4*ag_b:AG_A+4*ag_b+3], \va offset:(ag_b/4)*27648+(ag_b%%4)*1024
  .set ag_b, ag_b+1
  .endr
  .if \nb == 9
  ds_read_b128 v[AG_A+32:AG_A+35], \va offset:8192
  .endif
.endm
.macro AG_MF b, m, bc
  .set ag_n, \m ^ ((\b & 1) * 3)
  v_mfma_f32_16x16x32_bf16 v[AG_C+16*\b+4*ag_n:AG_C+16*\b+4*ag_n+3], v[\bc+4*ag_n:\bc+4*ag_n+3], v[AG_A+4*\b:AG_A+4*\b+3], v[AG_C+16*\b+4*ag_n:AG_C+16*\b+4*ag_n+3]
.endm
; half K-step on nb row blocks: first = 1 for the first half of step (t + add)
.macro AG_HALF nb, bc, bn, va, vnext, kh, first, add
  s_waitcnt vmcnt(8) lgkmcnt(\nb-3)
  .set ag_b, 0
  .rept \nb
  .if (ag_b == 3) || (ag_b == 6)
  s_waitcnt lgkmcnt(\nb-3)
  .endif
  AG_MF ag_b, 0, \bc
  .if ag_b == 4
  s_add_u32 %[bo], %[bo], 4096
  .endif
  .if \first && (ag_b == 5)
  s_add_u32 %[x], %[t], \add+1
  .endif
  .if \first && \add && (ag_b == 6)
  s_cmp_eq_u32 %[so], 27648
  .endif
  .if (!\first) && (ag_b == 6)
  v_cmp_lt_u32 vcc, %[x], v3
  .endif
  .if \first && \add && (ag_b == 7)
  v_add_u32 \vnext, %[so], v1
  .endif
  .if \first && (!\add) && (ag_b == 7)
  v_add_u32 \vnext, 55296, \va
  .endif
  AG_MF ag_b, 1, \bc
  .if ag_b < 4
  buffer_load_dwordx4 v[\bn+4*ag_b:\bn+4*ag_b+3], v0, %[rsw], %[bc] offen offset:ag_b*1024 sc0 sc1
  .endif
  .if ag_b == 4
  s_min_u32 %[bc], %[bo], 127*4096
  .endif
  .if \first && (ag_b == 5)
  v_mov_b32 v6, %[x]
  .endif
  .if \first && \add && (ag_b == 6)
  s_cselect_b32 %[so], 0, %[so]
  .endif
  .if \first && (ag_b == 7)
  s_cmp_le_i32 %[wk], %[x]
  .endif
  .if (!\first) && (ag_b == 7)
  s_cmp_eq_u64 vcc, exec
  .endif
  AG_MF ag_b, 2, \bc
  .if \first && \add && (ag_b == 5)
  s_add_u32 %[so], %[so], 9216
  .endif
  .if \first && (ag_b == \nb-1)
  ds_read_b32 v3, v4
  .endif
  AG_MF ag_b, 3, \bc
  .if ag_b < 8
  ds_read_b128 v[AG_A+4*ag_b:AG_A+4*ag_b+3], \va offset:(ag_b/4)*27648+\kh*4096+(ag_b%%4)*1024
  .else
  ds_read_b128 v[AG_A+32:AG_A+35], \va offset:8192+\kh*27648
  .endif
  .set ag_b, ag_b+1
  .endr
.endm
.macro AG_LOOP nb
  AG_NEEDX 0, v2, \nb
  .if AG_DIAG
  s_mov_b32 %[wx], 0
  s_mov_b32 %[ww], 0
  s_memtime %[c0]
  s_memrealtime %[r0]
  s_waitcnt lgkmcnt(0)
  .endif
.Lag_loop\nb:
  AG_HALF \nb, 44, 92, v2, v7, 1, 1, 0
  ds_write_b32 v5, v6
  s_cbranch_scc1 .Lag_w0\nb
.Lag_w0r\nb:
  AG_HALF \nb, 60, 44, v7, v7, 0, 0, 0
  s_cbranch_scc0 .Lag_x0\nb
.Lag_x0r\nb:
  AG_HALF \nb, 76, 60, v7, v2, 1, 1, 1
  ds_write_b32 v5, v6
  s_cbranch_scc1 .Lag_w1\nb
.Lag_w1r\nb:
  AG_HALF \nb, 92, 76, v2, v2, 0, 0, 1
  s_cbranch_scc0 .Lag_x1\nb
.Lag_x1r\nb:
  s_add_u32 %[t], %[t], 2
  s_cmp_lt_u32 %[t], 64
  s_cbranch_scc1 .Lag_loop\nb
  s_branch .Lag_end
.Lag_w0\nb:
  AG_NEEDW 1
  s_branch .Lag_w0r\nb
.Lag_w1\nb:
  AG_NEEDW 2
  s_branch .Lag_w1r\nb
.Lag_x0\nb:
  AG_NEEDX 1, v7, \nb
  s_branch .Lag_x0r\nb
.Lag_x1\nb:
  AG_NEEDX 2, v2, \nb
  s_branch .Lag_x1r\nb
.endm
  s_nop 4
  s_mov_b32 %[t], 0
  s_mov_b32 %[late], 0
  s_mov_b32 %[so], 0
  s_cmp_gt_i32 %[wk], 0
  s_cbranch_scc1 .Lag_go
  AG_NEEDW 0
.Lag_go:
  .set ag_i, 0
  .rept 12
  s_movk_i32 %[bc], (ag_i/4)*4096
  buffer_load_dwordx4 v[44+4*ag_i:44+4*ag_i+3], v0, %[rsw], %[bc] offen offset:(ag_i%%4)*1024 sc0 sc1
  .set ag_i, ag_i+1
  .endr
  s_movk_i32 %[bo], 3*4096
  s_movk_i32 %[bc], 3*4096
  .set ag_i, 0
  .rept 72
  v_mov_b64 v[AG_C+ag_i:AG_C+ag_i+1], 0
  .set ag_i, ag_i+2
  .endr
  v_mov_b32 v2, v1
  s_cmp_eq_u32 %[host], 0
  s_cbranch_scc1 .Lag_eight
  AG_LOOP 9
.Lag_eight:
  AG_LOOP 8
.Lag_end:
  s_waitcnt vmcnt(0) lgkmcnt(0)
  s_nop 7
)ASM"
            : "={v[112:143]}"(r0), "={v[144:175]}"(r1), "={v[176:207]}"(r2), "={v[208:239]}"(r3), "={v[240:255]}"(r4),
              [t] "=&s"(s_t), [wk] "+s"(s_wk), [bo] "=&s"(s_bo), [bc] "=&s"(s_bc), [x] "=&s"(s_x), [sp] "=&s"(s_sp), [t0] "=&s"(s_t0),
              [late] "=&s"(s_late), [so] "=&s"(s_so), [c] "=&s"(s_c), [c0] "=&s"(s_c0), [wx] "=&s"(s_wx), [ww] "=&s"(s_ww), [r0] "=&s"(s_r0), "+{v0}"(vo)
            : "{v1}"(abase), "{v4}"(polla), "{v5}"(donea), [rsw] "s"(rsw), [rsf] "s"(rsf), [tmo] "s"(tmo.word), [call] "s"(call), [host] "s"(rfl(host ? 1u : 0u))
            : "memory", "vcc", "scc", "v2", "v3", "v6", "v7", "v8", "v9", "v10", "v11", "v12", "v13", "v14", "v15", "v16", "v17", "v18", "v19", "v20", "v21", "v22", "v23", "v24", "v25", "v26", "v27", "v28", "v29", "v30", "v31", "v32", "v33", "v34", "v35", "v36", "v37", "v38", "v39", "v40", "v41", "v42", "v43", "v44", "v45", "v46", "v47", "v48", "v49", "v50", "v51", "v52", "v53", "v54", "v55", "v56", "v57", "v58", "v59", "v60", "v61", "v62", "v63", "v64", "v65", "v66", "v67", "v68", "v69", "v70", "v71", "v72", "v73", "v74", "v75", "v76", "v77", "v78", "v79", "v80", "v81", "v82", "v83", "v84", "v85", "v86", "v87", "v88", "v89", "v90", "v91", "v92", "v93", "v94", "v95", "v96", "v97", "v98", "v99", "v100", "v101", "v102", "v103", "v104", "v105", "v106", "v107");
        f32x4 acc[9][4];
#pragma unroll
        for (int i = 0; i < 144; ++i) acc[i >> 4][(i >> 2) & 3][i & 3] = i < 32 ? r0[i & 31] : i < 64 ? r1[i & 31] : i < 96 ? r2[i & 31] : i < 128 ? r3[i & 31] : r4[i & 15];
        STAMP(3);
        const int ln = (int)(vo >> 4);
        int mx = 0;
#pragma unroll
        for (int n = 0; n < 4; ++n) acc[8][n] *= XSCALE;
        if (host) {
#pragma unroll
            for (int n = 0; n < 4; ++n) { mx = max(max(mx, fbits(acc[8][n][0])), fbits(acc[8][n][1])); mx = max(max(mx, fbits(acc[8][n][2])), fbits(acc[8][n][3])); }
        }
        int m2 = 0;
#pragma unroll
        for (int b = 0; b < 8; ++b)
#pragma unroll
            for (int n = 0; n < 4; ++n) { mx = max(max(mx, fbits(acc[b][n][0])), fbits(acc[b][n][1])); m2 = max(max(m2, fbits(acc[b][n][2])), fbits(acc[b][n][3])); }
        const unsigned wm = wave_max((unsigned)max(mx, m2));
        if (ln == 0) {
            __hip_atomic_fetch_max((LAS unsigned*)(lds + LC_OFF) + C_MAX, wm, __ATOMIC_RELAXED, __HIP_MEMORY_SCOPE_WORKGROUP);
            const unsigned arrived = __hip_atomic_fetch_add((LAS unsigned*)(lds + LC_OFF) + C_CNT, 1u, __ATOMIC_RELAXED, __HIP_MEMORY_SCOPE_WORKGROUP);
            const unsigned tm = ctl[C_MAX];
            if (arrived == 3u) __hip_atomic_store((GAS u64*)(me + A1_OFF) + (par * NWG + bx) * 8, ((u64)call << 32) | tm, __ATOMIC_RELAXED, __HIP_MEMORY_SCOPE_SYSTEM);
        }
        STAMP(6);
        const __amdgpu_buffer_rsrc_t rso = mkrsrc(p.out, 8u * 512u * NL);
        const unsigned ovo = (unsigned)(ln & 15) * NL + ct * 256 + wave * 64 + (ln >> 4) * 16;
        unsigned used = 0xffffffffu; Spin sp;
#pragma unroll 1
        for (;;) {
            const u32x2 c = *(volatile LAS u32x2*)(lds + LC_OFF + C_GMAX * 4);
            unsigned st = rfl(c.y);
            const unsigned run = rfl(c.x), g = max(run, wm);
            if (st == 0u && sp.over<4095u>(tmo)) st = 2u;
            if (st == 2u || (g != used && (st != 0u || (SPEC && run != 0u)))) {
                if (used != 0xffffffffu) VMWAIT0();
                used = g;
                const float scale = __uint_as_float(g) / 127.0f;
                const float inv = scale > 0.f ? 1.0f / scale : 0.f;
#pragma unroll
                for (int b = 0; b < 9; ++b) {
                    u32x4 o = {0u, 0u, 0u, 0u};
#pragma unroll
                    for (int cc = 0; cc < 4; ++cc) {
                        f32x4 q = acc[b][cc] * inv;
                        asm volatile("" : "+v"(q));
#pragma unroll
                        for (int r = 0; r < 4; ++r) o[r] = __builtin_amdgcn_cvt_pk_u8_f32(q[r], cc, o[r]);
                    }
                    if (st == 2u) o = (u32x4){0x80808080u, 0x80808080u, 0x80808080u, 0x80808080u};
                    if (b < 8 || host) __builtin_amdgcn_raw_buffer_store_b128(o, rso, ovo + (unsigned)(b < 8 ? rtile * 128 + b * 16 : rank * 512 + xrow) * NL, 0, OAUX);
                }
            }
            if (st != 0u) break;
            __builtin_amdgcn_s_sleep(1);
        }
        STAMP(5);
    } else {
        const int f = wave & 1, ph = (wave >> 1) & 1, rb = sub * 2 + f;
        STAMP(6);
        __builtin_amdgcn_s_setprio(FPRIO);
        const __amdgpu_buffer_rsrc_t rsx = mkrsrc(me + XS_OFF + (size_t)src * XS_SRC + (size_t)rb * KS * UNIT, KS * UNIT);
        const __amdgpu_buffer_rsrc_t rse = mkrsrc(p.x + (size_t)xrow * KDIM + f * 32, host ? 16u * KDIM * 4u - 128u : 0u);
        const __amdgpu_buffer_rsrc_t rsf = mkrsrc(me + XF_OFF + (size_t)(src * 8 + rb) * KS * 8, KS * 8);
        const unsigned ldsb = (unsigned)(size_t)lds;
        const unsigned mbase = ldsb + f * LA_HALF, vpub = ldsb + LC_OFF + (C_FULL + (wave & 3)) * 4, vdone = ldsb + LC_OFF + (C_DONE + (lane & 3)) * 4;
        const unsigned fcall = (nopush && !local) ? 0u : call, hostf = rfl(host ? 1u : 0u);
        unsigned s_cur = ph, s_lim = NST, s_so = 0, s_h, s_b, s_x, s_y, s_sp, s_t0, s_m0; u64 s_seen, s_c;
        const unsigned s_end = KS;
        asm volatile(R"ASM(
.macro FD_EXPIRED lout
  s_add_u32 %[sp], %[sp], 1
  s_and_b32 %[x], %[sp], 0xff
  s_cbranch_scc1 8f
  v_mov_b32 v9, 0
  global_load_dword v12, v9, %[tmo] sc1
  s_memrealtime %[c]
  s_waitcnt vmcnt(0) lgkmcnt(0)
  v_readfirstlane_b32 %[x], v12
  s_cmp_eq_u32 %[x], %[call]
  s_cbranch_scc1 \lout
  s_mov_b64 vcc, %[c]
  s_cmp_eq_u32 %[sp], 0x100
  s_cselect_b32 %[t0], vcc_lo, %[t0]
  s_sub_u32 %[x], vcc_lo, %[t0]
  s_cmp_le_u32 %[x], 40000000
  s_cbranch_scc1 8f
  v_mov_b32 v12, %[call]
  global_store_dword v9, v12, %[tmo] sc1
  s_waitcnt vmcnt(0)
  s_branch \lout
8:
.endm
.macro FD_FLAGS
  buffer_load_dwordx2 v[10:11], v6, %[rsf], 0 offen sc0 sc1
  s_waitcnt vmcnt(0)
  v_cmp_le_u32 %[seen], %[fcall], v10
.endm
; loads of unit %[y] into the set at r
.macro FD_ISSUE r
  s_lshl_b32 %[x], %[y], 13
  s_lshl_b32 %[b], %[y], 8
  buffer_load_dwordx4 v[\r:\r+3], v0, %[rsx], %[x] offen sc0 sc1
  buffer_load_dwordx4 v[\r+4:\r+7], v0, %[rsx], %[x] offen offset:1024 sc0 sc1
  buffer_load_dwordx4 v[\r+8:\r+11], v0, %[rsx], %[x] offen offset:2048 sc0 sc1
  buffer_load_dwordx4 v[\r+12:\r+15], v0, %[rsx], %[x] offen offset:3072 sc0 sc1
  s_bitset1_b32 %[x], 12
  buffer_load_dwordx4 v[\r+24:\r+27], v3, %[rse], %[b] offen
  buffer_load_dwordx4 v[\r+28:\r+31], v3, %[rse], %[b] offen offset:64
  buffer_load_dwordx4 v[\r+16:\r+19], v0, %[rsx], %[x] offen sc0 sc1
  buffer_load_dwordx2 v[\r+20:\r+21], v1, %[rsx], %[x] offen offset:1024 sc0 sc1
  buffer_load_ushort v[\r+22], v2, %[rsx], %[x] offen offset:1536 sc0 sc1
.endm
; wait for the flag of unit %[y]
.macro FD_WAITFLAG
  s_mov_b32 %[sp], 0
1:
  FD_FLAGS
  s_bitcmp1_b64 %[seen], %[y]
  s_cbranch_scc1 2f
  s_sleep 1
  FD_EXPIRED 2f
  s_branch 1b
2:
.endm
.macro FD_DEC h, n0, n1, n2
  v_and_or_b32 v[\h], v[\n0], v7, %[b]
  v_alignbit_b32 v12, v[\n0], v[\n0], 3
  v_and_or_b32 v[\h+2], v[\n1], v7, %[b]
  v_alignbit_b32 v13, v[\n1], v[\n1], 3
  v_and_or_b32 v[\h+4], v[\n2], v7, %[b]
  v_alignbit_b32 v14, v[\n2], v[\n2], 3
  v_and_or_b32 v[\h+1], v12, v7, %[b]
  v_lshrrev_b32 v12, 5, v[\n0]
  v_and_or_b32 v[\h+3], v13, v7, %[b]
  v_lshrrev_b32 v13, 5, v[\n1]
  v_and_or_b32 v[\h+5], v14, v7, %[b]
  v_lshlrev_b32 v14, 1, v[\n2]
  v_bfi_b32 v12, v8, v14, v12
  v_lshlrev_b32 v15, 2, v[\n2]
  v_and_or_b32 v[\h+6], v12, v7, %[b]
  v_bfi_b32 v13, v8, v15, v13
  v_and_or_b32 v[\h+7], v13, v7, %[b]
.endm
; exception entry in register e -> address v12, mask v13
.macro FD_EXC e
  v_and_b32 v12, 0x7fe, v[\e]
  v_and_b32 v14, 1, v[\e]
  v_lshrrev_b32 v13, 11, v[\e]
  v_lshl_or_b32 v14, v14, 4, 10
  v_lshl_add_u32 v12, v12, 1, %[x]
  v_lshlrev_b32 v13, v14, v13
.endm
; one unit: set r holds the loads of unit cur, set q takes the next one; ob = offset of this parity's ring
.macro FD_STEP r, q, ob, id
  s_add_u32 %[y], %[cur], 2
  s_cmp_ge_u32 %[y], %[end]
  s_cbranch_scc1 .Lfd_last\id
  s_bitcmp1_b64 %[seen], %[y]
  s_cbranch_scc0 .Lfd_look\id
  FD_ISSUE \q
  s_setprio 0
  s_waitcnt vmcnt(9)
.Lfd_go\id:
  v_readlane_b32 %[h], v11, %[cur]
  s_and_b32 %[b], %[h], 0x7c
  s_mul_i32 %[b], %[b], 0x01010101
  FD_DEC 80, \r+16, \r+17, \r+18
  s_add_u32 m0, %[mb], %[so]
  FD_DEC 88, \r+19, \r+20, \r+21
  s_and_b32 %[x], %[h], 0xc00083
  s_cbranch_scc1 .Lfd_fix\id
.Lfd_fixed\id:
  s_cmp_ge_u32 %[cur], %[lim]
  s_cbranch_scc1 .Lfd_free\id
.Lfd_freed\id:
  .set fd_i, 0
  .rept 16
  v_perm_b32 v[96+(fd_i%%2)*2], v[80+fd_i], v[\r+fd_i], %[p0]
  v_perm_b32 v[97+(fd_i%%2)*2], v[80+fd_i], v[\r+fd_i], %[p1]
  ds_write_addtid_b32 v[96+(fd_i%%2)*2] offset:\ob+fd_i*512
  ds_write_addtid_b32 v[97+(fd_i%%2)*2] offset:\ob+fd_i*512+256
  .set fd_i, fd_i+1
  .endr
  s_add_u32 %[x], m0, \ob
  FD_EXC \r+22
  s_bfe_u32 %[b], %[h], 0x60008
  s_bfm_b64 exec, %[b], 0
  s_bfe_u32 %[b], %[h], 0x60010
  ds_xor_b32 v12, v13
  s_bfm_b64 %[c], %[b], 0
  s_andn2_b64 exec, %[c], exec
  s_add_u32 %[so], %[so], 9216
  ds_xor_b32 v12, v13 offset:4096
  s_mov_b64 exec, -1
  s_and_b32 %[b], %[h], 0xff000000
  s_cbranch_scc1 .Lfd_many\id
.Lfd_manied\id:
  s_cmp_eq_u32 %[host], 0
  s_cbranch_scc1 .Lfd_pub\id
  v_cvt_pk_bf16_f32 v96, v[\r+24], v[\r+25]
  v_cvt_pk_bf16_f32 v97, v[\r+26], v[\r+27]
  v_cvt_pk_bf16_f32 v98, v[\r+28], v[\r+29]
  v_cvt_pk_bf16_f32 v99, v[\r+30], v[\r+31]
  ds_write_addtid_b32 v96 offset:\ob+8192
  ds_write_addtid_b32 v97 offset:\ob+8192+256
  ds_write_addtid_b32 v98 offset:\ob+8192+512
  ds_write_addtid_b32 v99 offset:\ob+8192+768
.Lfd_pub\id:
  s_add_u32 %[cur], %[cur], 2
  s_cmp_eq_u32 %[so], 27648
  s_cselect_b32 %[so], 0, %[so]
  s_cmp_ge_u32 %[cur], 64
  s_cselect_b32 %[x], 1000, %[cur]
  v_mov_b32 v9, %[x]
  s_cmp_ge_u32 %[cur], %[end]
  ds_write_b32 v4, v9
  s_cbranch_scc1 .Lfd_end
  s_cmp_eq_u32 %[y], %[cur]
  s_cbranch_scc0 .Lfd_late\id
.Lfd_next\id:
.endm
; out-of-line parts of a step
.macro FD_SLOW r, q, ob, id
.Lfd_last\id:
  s_waitcnt vmcnt(0)
  s_branch .Lfd_go\id
.Lfd_look\id:
  FD_FLAGS
  s_bitcmp1_b64 %[seen], %[y]
  s_cbranch_scc0 3f
  FD_ISSUE \q
  s_branch .Lfd_go\id
3:
  s_mov_b32 %[y], -1
  s_branch .Lfd_go\id
.Lfd_late\id:
  s_mov_b32 %[y], %[cur]
  FD_WAITFLAG
  s_setprio )ASM" DPRIO R"ASM(
  FD_ISSUE \q
  s_branch .Lfd_next\id
.Lfd_fix\id:
  s_bitcmp1_b32 %[h], 7
  s_cbranch_scc0 3f
  s_lshl_b32 %[x], %[cur], 13
  s_bitset1_b32 %[x], 12
  buffer_load_dwordx4 v[80:83], v0, %[rsx], %[x] offen sc0 sc1
  buffer_load_dwordx4 v[84:87], v0, %[rsx], %[x] offen offset:1024 sc0 sc1
  buffer_load_dwordx4 v[88:91], v0, %[rsx], %[x] offen offset:2048 sc0 sc1
  buffer_load_dwordx4 v[92:95], v0, %[rsx], %[x] offen offset:3072 sc0 sc1
  s_waitcnt vmcnt(0)
  s_branch .Lfd_fixed\id
3:
  s_and_b32 %[b], %[h], 3
  s_mul_i32 %[b], %[b], 0x01010101
  .set fd_i, 0
  .rept 16
  v_add_u32 v[80+fd_i], %[b], v[80+fd_i]
  .set fd_i, fd_i+1
  .endr
  s_and_b32 %[x], %[h], 0xc00000
  s_cbranch_scc0 .Lfd_fixed\id
  s_lshl_b32 %[h], %[h], 8          ; 64 entries or more: the counts move to bits 16-31, all are applied out of line
  s_and_b32 %[x], %[h], 0xffff0000
  s_lshr_b32 %[h], %[h], 16
  s_and_b32 %[h], %[h], 0xff
  s_and_b32 %[x], %[x], 0xff000000
  s_or_b32 %[h], %[h], %[x]
  s_branch .Lfd_fixed\id
.Lfd_many\id:
  v_lshrrev_b32 v15, 1, v2
  s_and_b32 %[sp], %[h], 0xff
  s_lshr_b32 %[b], %[h], 24
  v_cmp_gt_u32 vcc, %[sp], v15
  s_mov_b64 exec, vcc
  ds_xor_b32 v12, v13
  s_not_b64 exec, exec
  ds_xor_b32 v12, v13 offset:4096
  s_mov_b64 exec, -1
  s_lshl_b32 %[t0], %[cur], 13
  s_bitset1_b32 %[t0], 12
  buffer_load_ushort v9, v2, %[rsx], %[t0] offen offset:1664 sc0 sc1
  v_add_u32 v15, 64, v15
  s_waitcnt vmcnt(0)
  FD_EXC 9
  v_cmp_gt_u32 %[c], %[b], v15
  v_cmp_gt_u32 vcc, %[sp], v15
  s_mov_b64 exec, vcc
  ds_xor_b32 v12, v13
  s_andn2_b64 exec, %[c], vcc
  ds_xor_b32 v12, v13 offset:4096
  s_mov_b64 exec, -1
  s_branch .Lfd_manied\id
.Lfd_free\id:
  s_mov_b32 %[sp], 0
1:
  ds_read_b32 v9, v5
  s_waitcnt lgkmcnt(0)
  v_readlane_b32 %[lim], v9, 0
  v_readlane_b32 %[x], v9, 1
  s_min_u32 %[lim], %[lim], %[x]
  v_readlane_b32 %[x], v9, 2
  s_min_u32 %[lim], %[lim], %[x]
  v_readlane_b32 %[x], v9, 3
  s_min_u32 %[lim], %[lim], %[x]
  s_add_u32 %[lim], %[lim], 6
  s_cmp_lt_u32 %[cur], %[lim]
  s_cbranch_scc1 .Lfd_freed\id
  s_sleep 4
  FD_EXPIRED .Lfd_freed\id
  s_branch 1b
.endm
  s_nop 4
  s_mov_b32 %[m0s], m0
  v_mov_b32 v7, 0x83838383
  v_mov_b32 v8, 0x80808080
  s_mov_b32 %[y], %[cur]
  FD_WAITFLAG
  FD_ISSUE 16
  s_bitcmp1_b32 %[cur], 0
  s_cbranch_scc1 .Lfd_loop1
.Lfd_loop0:
  FD_STEP 16, 48, 0, 0
  FD_STEP 48, 16, 0, 1
  s_branch .Lfd_loop0
.Lfd_loop1:
  FD_STEP 16, 48, 55296, 2
  FD_STEP 48, 16, 55296, 3
  s_branch .Lfd_loop1
  FD_SLOW 16, 48, 0, 0
  FD_SLOW 48, 16, 0, 1
  FD_SLOW 16, 48, 55296, 2
  FD_SLOW 48, 16, 55296, 3
.Lfd_end:
  s_mov_b32 m0, %[m0s]
  s_waitcnt vmcnt(0) lgkmcnt(0)
)ASM"
            : [cur] "+s"(s_cur), [lim] "+s"(s_lim), [so] "+s"(s_so), [h] "=&s"(s_h), [b] "=&s"(s_b), [x] "=&s"(s_x), [y] "=&s"(s_y), [sp] "=&s"(s_sp),
              [t0] "=&s"(s_t0), [m0s] "=&s"(s_m0), [seen] "=&s"(s_seen), [c] "=&s"(s_c)
            : "{v0}"(lane * 16), "{v1}"((lane & 31) * 16 + (lane >> 5) * 8), "{v2}"(lane * 2), "{v3}"((lane >> 2) * KDIM * 4 + (lane & 3) * 16), "{v4}"(vpub), "{v5}"(vdone), "{v6}"(lane * 8),
              [rsx] "s"(rsx), [rse] "s"(rse), [rsf] "s"(rsf), [tmo] "s"(tmo.word), [call] "s"(call), [fcall] "s"(fcall), [mb] "s"(mbase), [end] "s"(s_end), [host] "s"(hostf),
              [p0] "s"(0x05010400u), [p1] "s"(0x07030602u)
            : "memory", "vcc", "scc", "v7", "v8", "v9", "v10", "v11", "v12", "v13", "v14", "v15", "v16", "v17", "v18", "v19", "v20", "v21", "v22", "v23", "v24", "v25", "v26", "v27", "v28", "v29", "v30", "v31", "v32", "v33", "v34", "v35", "v36", "v37", "v38", "v39", "v40", "v41", "v42", "v43", "v44", "v45", "v46", "v47", "v48", "v49", "v50", "v51", "v52", "v53", "v54", "v55", "v56", "v57", "v58", "v59", "v60", "v61", "v62", "v63", "v64", "v65", "v66", "v67", "v68", "v69", "v70", "v71", "v72", "v73", "v74", "v75", "v76", "v77", "v78", "v79", "v80", "v81", "v82", "v83", "v84", "v85", "v86", "v87", "v88", "v89", "v90", "v91", "v92", "v93", "v94", "v95", "v96", "v97", "v98", "v99");
        static_assert(DEADLINE == 40000000u && NST == 6, "literals in the feeder loop");
        if (wave != 4) return;
        {
            const GAS u64* a2 = lane < 16 ? (const GAS u64*)(me + (lane < 8 ? A2_OFF : A3_OFF) + (size_t)(par * 8 + (bx & 7)) * 128) + (lane & 7) : (const GAS u64*)(me + TM_OFF);
            unsigned told = 0; Spin sp;
            for (;;) {
                const u64 v = __hip_atomic_load(a2, __ATOMIC_RELAXED, __HIP_MEMORY_SCOPE_SYSTEM);
                const bool in = lane < 16 && (unsigned)(v >> 32) == call;
                unsigned g = in ? (unsigned)v : 0u;
                g = max(g, (unsigned)__builtin_amdgcn_update_dpp(0u, g, 0x111, 0xf, 0xf, true));
                g = max(g, (unsigned)__builtin_amdgcn_update_dpp(0u, g, 0x112, 0xf, 0xf, true));
                g = max(g, (unsigned)__builtin_amdgcn_update_dpp(0u, g, 0x114, 0xf, 0xf, true));
                g = max(g, (unsigned)__builtin_amdgcn_update_dpp(0u, g, 0x118, 0xf, 0xf, true));
                g = __builtin_amdgcn_readlane(g, 15);
                const unsigned im = (unsigned)__builtin_amdgcn_ballot_w64(in);
                unsigned st = (im & 0xffu) == 0xffu ? 1u : 0u;
                if (((im | (im >> 8)) & 0xffu) != 0xffu) g = 0u;
                if ((unsigned)__builtin_amdgcn_readlane((unsigned)v, 16) == call || (st == 0u && sp.over<255u>(tmo))) st = 2u;
                if (st != 0u || g != told) { told = g; *(volatile LAS u32x2*)(lds + LC_OFF + C_GMAX * 4) = (u32x2){g, st}; }
                if (st != 0u) break;
                __builtin_amdgcn_s_sleep(1);
            }
        }
        if (lane == 0) __hip_atomic_store(seqw, call, __ATOMIC_RELAXED, __HIP_MEMORY_SCOPE_AGENT);
        STAMP(4);
    }
}

}

extern "C" void kernel_launch(int rank, int nranks, ncclComm_t comm,
                              hipStream_t stream,
                              void* const* d_in, const int* in_sizes, int n_in,
                              void* d_out, int out_size,
                              const int* in_dtypes, int out_dtype,
                              void* const* d_out_all,
                              void* const* d_ws_all, size_t ws_bytes) {
    if (nranks != NR || (unsigned)rank >= (unsigned)NR || ws_bytes < WS_NEED || n_in < 2) {
        fprintf(stderr, "unsupported configuration\n");
        return;
    }
    HIP_CHECK(hipFuncSetAttribute((const void*)ag_gemm, hipFuncAttributeMaxDynamicSharedMemorySize, LDS_BYTES));
    Params p;
    p.x = (const float*)d_in[0]; p.w = (const float*)d_in[1]; p.out = (signed char*)d_out;
    for (int i = 0; i < NR; ++i) p.peer[i] = (char*)d_ws_all[(rank + 1 + i) % NR];
    p.rank = rank; p.pad = 0;
    hipLaunchKernelGGL(ag_gemm, dim3(NWG), dim3(512), LDS_BYTES, stream, p);
    HIP_CHECK(hipGetLastError());
}
```

```cpp
#include <hip/hip_runtime.h>
#include <rccl/rccl.h>
#include <cstdio>

#define HIP_CHECK(expr) do { hipError_t _e = (expr); if (_e != hipSuccess) { \
    fprintf(stderr, "HIP error %s at %s:%d\n", hipGetErrorString(_e), __FILE__, __LINE__); \
    return; } } while (0)

namespace {

#define LAS __attribute__((address_space(3)))
#define GAS __attribute__((address_space(1)))
typedef unsigned u32x4 __attribute__((ext_vector_type(4)));
typedef unsigned u32x2 __attribute__((ext_vector_type(2)));
typedef float f32x4 __attribute__((ext_vector_type(4)));
typedef float f32x16 __attribute__((ext_vector_type(16)));
typedef float f32x32 __attribute__((ext_vector_type(32)));
typedef unsigned long long u64;

constexpr int NR = 8;
constexpr int KDIM = 4096, NL = 2048;
constexpr int KS = 64;
constexpr int NWG = 256;
constexpr int UNIT = 8192;
constexpr int EXCAP = 63;

constexpr size_t XS_OFF = 0;
constexpr size_t XS_SRC = (size_t)8 * KS * UNIT;
constexpr size_t WB_OFF = XS_OFF + NR * XS_SRC;
constexpr size_t XF_OFF = WB_OFF + (size_t)32 * KS * 8192;
constexpr size_t WF_OFF = XF_OFF + (size_t)NR * 8 * KS * 16;
constexpr size_t SQ_OFF = WF_OFF + (size_t)32 * KS * 64;
constexpr size_t A1_OFF = SQ_OFF + (size_t)NWG * 128;
constexpr size_t A2_OFF = A1_OFF + (size_t)2 * NWG * 64;
constexpr size_t A3_OFF = A2_OFF + (size_t)2 * 8 * 128;
constexpr size_t TK_OFF = A3_OFF + (size_t)2 * 8 * 128;
constexpr size_t TM_OFF = TK_OFF + 128;
constexpr size_t DG_OFF = TM_OFF + 128;
constexpr size_t WS_NEED = DG_OFF + (size_t)NWG * 8 * 128 * 8;

constexpr int NST = 6, LA_SLOT = 9216, LA_HALF = 3 * LA_SLOT, LA_PAR = 2 * LA_HALF;
constexpr int LC_OFF = 2 * LA_PAR;
constexpr int C_FULL = 0;
constexpr int C_DONE = 4;
constexpr int C_MAX = 8, C_CNT = 9, C_GMAX = 10  , C_SENT = 12, C_ISS = 13, C_SEQ = 14;
constexpr int LJ_OFF = LC_OFF + 64;
constexpr int LDS_BYTES = LJ_OFF + 256;
static_assert(LDS_BYTES > 81920, "one workgroup per CU");

#define CPX 3
#define PW 6
#define PWIN 3
#define SREL 16
#define PEARLY 2
#define WDELAY 20
#define WDELAY2 340
#define WSTAG 12
#define SPECN 208
constexpr int CODE_BYTES = 29440;
#define XDG "0"
constexpr int NCOMM = 8 * CPX;
constexpr float XSCALE = 0.5f;
constexpr unsigned DEADLINE = 40000000u;

struct Params {
    const float* x; const float* w; signed char* out;
    char* peer[NR];
    int rank; int pad;
};

__device__ __forceinline__ u64 rt() { return __builtin_amdgcn_s_memrealtime(); }
__device__ __forceinline__ unsigned rt32() { return (unsigned)__builtin_amdgcn_s_memrealtime(); }
__device__ __forceinline__ unsigned rfl(unsigned v) { return __builtin_amdgcn_readfirstlane(v); }
__device__ __forceinline__ unsigned cvt_pk(float lo, float hi) {
    typedef float f2 __attribute__((ext_vector_type(2))); typedef __bf16 b2 __attribute__((ext_vector_type(2)));
    f2 v = {lo, hi}; return __builtin_bit_cast(unsigned, __builtin_convertvector(v, b2));
}
__device__ __forceinline__ __amdgpu_buffer_rsrc_t mkrsrc(const void* p, unsigned bytes) {
    return __builtin_amdgcn_make_buffer_rsrc((void*)p, 0, bytes, 0x00020000);
}
__device__ __forceinline__ int fbits(float v) { return __float_as_int(v); }
__device__ __forceinline__ unsigned rotr(unsigned v, int n) { return __builtin_amdgcn_alignbit(v, v, n); }
#define VMWAIT0() asm volatile("s_waitcnt vmcnt(0)" ::: "memory")
#define LGKM0() asm volatile("s_waitcnt lgkmcnt(0)" ::: "memory")
#define SCHED() __builtin_amdgcn_sched_barrier(0)
constexpr int SYS = 17;

struct Tmo {
    GAS unsigned* word; unsigned call;
    __device__ __forceinline__ bool hit() const {
        unsigned v;
        asm volatile("v_mov_b32 %0, 0\n\tglobal_load_dword %0, %0, %1 sc1\n\ts_waitcnt vmcnt(0)" : "=&v"(v) : "s"(word) : "memory");
        return rfl(v) == call;
    }
    __device__ __forceinline__ bool expired(unsigned t0) const {
        if (hit() || rt32() - t0 > DEADLINE) {
            unsigned z, c;
            asm volatile("v_mov_b32 %0, 0\n\tv_mov_b32 %1, %3\n\tglobal_store_dword %0, %1, %2 sc1\n\ts_waitcnt vmcnt(0)" : "=&v"(z), "=&v"(c) : "s"(word), "s"(call) : "memory");
            return true;
        }
        return false;
    }
};

struct Spin {
    unsigned n = 0, t0 = 0;
    template <unsigned MASK> __device__ __forceinline__ bool over(const Tmo& tmo) {
        if ((++n & MASK) != 0u) return false;
        if (n == MASK + 1u) { t0 = rt32(); return false; }
        return tmo.expired(t0);
    }
};

#define PSTAMP(r) do {} while (0)
#define STAMP(i) do {} while (0)

__device__ __forceinline__ void w_load(f32x4 (&v)[16], const float* __restrict__ w, bool on, int ct, int t, int q, int lane) {
    const __amdgpu_buffer_rsrc_t rs = mkrsrc(w + (size_t)t * 64 * NL + ct * 256 + q * 64, on ? 63u * NL * 4u + 256u : 0u);
    const int n = lane & 15, g = lane >> 4;
    const unsigned vo = (unsigned)((g >> 1) * 16 + (g & 1) * 2) * NL * 4 + n * 16;
#pragma unroll
    for (int i = 0; i < 16; ++i) v[i] = __builtin_bit_cast(f32x4, __builtin_amdgcn_raw_buffer_load_b128(rs, vo, ((i >> 3) * 32 + ((i >> 1) & 3) * 4 + (i & 1)) * NL * 4, 2));
}
__device__ __forceinline__ void w_store(const f32x4 (&v)[16], char* me, int ct, int t, int q, int lane) {
    const __amdgpu_buffer_rsrc_t rs = mkrsrc(me + WB_OFF + ((size_t)(ct * 4 + q) * KS + t) * 8192, 8192);
#pragma unroll
    for (int kh = 0; kh < 2; ++kh)
#pragma unroll
        for (int c = 0; c < 4; ++c) {
            u32x4 o;
            o.x = cvt_pk(v[kh * 8 + 0][c], v[kh * 8 + 1][c]); o.y = cvt_pk(v[kh * 8 + 2][c], v[kh * 8 + 3][c]);
            o.z = cvt_pk(v[kh * 8 + 4][c], v[kh * 8 + 5][c]); o.w = cvt_pk(v[kh * 8 + 6][c], v[kh * 8 + 7][c]);
            __builtin_amdgcn_raw_buffer_store_b128(o, rs, lane * 16 + c * 1024 + kh * 4096, 0, SYS);
        }
}

__device__ __forceinline__ char* pick_peer(const Params& p, int q) {
    char* r = p.peer[0];
#pragma unroll
    for (int i = 1; i < NR; ++i) { char* c = p.peer[i]; asm volatile("" : "+s"(c)); r = q == i ? c : r; }
    return r;
}

__device__ __forceinline__ void touch(const void* ptr, unsigned junk) {
    unsigned keep;
    asm volatile("s_mov_b32 %0, m0\n\ts_mov_b32 m0, %2\n\ts_nop 0\n\tglobal_load_lds_dword %1, off\n\ts_mov_b32 m0, %0" : "=&s"(keep) : "v"(ptr), "s"(junk) : "memory");
}

__device__ __forceinline__ unsigned wave_max(unsigned g) {
    g = max(g, (unsigned)__builtin_amdgcn_update_dpp(0u, g, 0x111, 0xf, 0xf, true));
    g = max(g, (unsigned)__builtin_amdgcn_update_dpp(0u, g, 0x112, 0xf, 0xf, true));
    g = max(g, (unsigned)__builtin_amdgcn_update_dpp(0u, g, 0x114, 0xf, 0xf, true));
    g = max(g, (unsigned)__builtin_amdgcn_update_dpp(0u, g, 0x118, 0xf, 0xf, true));
    return max(max((unsigned)__builtin_amdgcn_readlane(g, 15), (unsigned)__builtin_amdgcn_readlane(g, 31)), max((unsigned)__builtin_amdgcn_readlane(g, 47), (unsigned)__builtin_amdgcn_readlane(g, 63)));
}

__global__ void __launch_bounds__(512, 2) ag_gemm(Params p) {
    extern __shared__ __attribute__((aligned(16))) char lds_raw[];
    LAS char* lds = (LAS char*)lds_raw;
    volatile LAS unsigned* ctl = (volatile LAS unsigned*)(lds + LC_OFF);
    const int lane = threadIdx.x & 63, wave = rfl(threadIdx.x >> 6);
    const int bx = blockIdx.x, xcd = bx & 7, idx = bx >> 3;
    const unsigned junk = (unsigned)(size_t)(lds + LJ_OFF);
    if (wave == 7 && idx < 2) {
        const char* pc = (const char*)__builtin_amdgcn_s_getpc();
#pragma unroll
        for (int j = 0; j < (CODE_BYTES + 8191) / 8192; ++j) touch(pc + min(j * 8192 + lane * 128, CODE_BYTES), junk);
    }
    asm volatile("" :: "s"(p.x), "s"(p.w), "s"(p.out), "s"(p.rank), "s"(p.peer[0]), "s"(p.peer[1]), "s"(p.peer[2]), "s"(p.peer[3]),
                       "s"(p.peer[4]), "s"(p.peer[5]), "s"(p.peer[6]), "s"(p.peer[7]));
    const int rank = p.rank & 7;
    const int rel = idx & 7, src = (rank + 1 + rel) & 7, sub = xcd >> 1, ct = (xcd & 1) * 4 + (idx >> 3), rtile = src * 4 + sub;
    char* me = p.peer[NR - 1];
    const bool local = rel == NR - 1;
    const int jj = ((idx >> 3) + sub) & 3;
    const bool comm = local && jj < CPX;
    const int h = rel * 4 + sub;
    const bool host = h < 8 * CPX;
    const int xrow = (((h >> 3) - (ct & 3)) & 3) * 128 + (h & 7) * 16;
    constexpr int NCW = 32 - CPX;
    static_assert(PEARLY <= PW && PW <= 7, "roles");
    const int ci = local ? 28 + (jj - CPX) : h;
    const int cid = jj * 8 + xcd;
    GAS unsigned* seqw = (GAS unsigned*)(me + SQ_OFF + (size_t)bx * 128);
#define WG_START() do { if (threadIdx.x < 16) ctl[threadIdx.x] = (threadIdx.x == C_FULL + 2 || threadIdx.x == C_FULL + 3) ? 1u : 0u; \
        LGKM0(); __builtin_amdgcn_s_barrier(); } while (0)
#define NOPUSH false
#define GET_CALL() const unsigned call = rfl(__hip_atomic_load(seqw, __ATOMIC_RELAXED, __HIP_MEMORY_SCOPE_AGENT)) + 1u; \
    const Tmo tmo{(GAS unsigned*)(me + TM_OFF), call}; const unsigned par = call & 1u; const bool nopush = NOPUSH; (void)par; (void)nopush
    STAMP(0);

    if (__builtin_expect(comm, 1)) {
        if (__builtin_expect(wave < PW, 1)) {
            constexpr unsigned NHU = 16 * KS;
            f32x4 v[8];
            auto load_x = [&](int u) {
                const float* s0 = p.x + (size_t)((u & 7) * 64 + (lane >> 2)) * KDIM + (u >> 3) * 32 + (lane & 3) * 4;
#pragma unroll
                for (int e = 0; e < 8; ++e) v[e] = *(const GAS f32x4*)(s0 + (size_t)(e >> 1) * 16 * KDIM + (e & 1) * 16);
            };
            if (wave >= PEARLY) __builtin_amdgcn_s_sleep(8);
            load_x(wave * NCOMM + cid);
            SCHED();
            WG_START();
            GET_CALL();
            __amdgpu_buffer_rsrc_t rs[NR];
#pragma unroll
            for (int q = 0; q < NR; ++q) rs[q] = mkrsrc(p.peer[q] + XS_OFF + (size_t)rank * XS_SRC, (q == NR - 1 || !nopush) ? (unsigned)XS_SRC : 0u);
            GAS u64* flag = (GAS u64*)(pick_peer(p, lane & 7) + XF_OFF) + rank * 8 * KS * 2;
            GAS unsigned* tk = (GAS unsigned*)(me + TK_OFF);
            LAS char* tail = lds + wave * 2048;
            const unsigned ptag = (unsigned)lane << 1, vo = lane * 16;
            unsigned u = wave * NCOMM + cid;
            int k = wave;
            while (u < NHU) {
                const unsigned fi = (u & 7u) * KS * 2 + (u >> 3), soff = fi * (UNIT / 2);
                u32x4 L[2], H[2], T;
                unsigned sum = 0;
#pragma unroll
                for (int e = 0; e < 8; ++e) {
                    const f32x4 s = v[e] * XSCALE;
                    const unsigned d0 = cvt_pk(s[0], s[1]), d1 = cvt_pk(s[2], s[3]);
                    L[e >> 2][e & 3] = __builtin_amdgcn_perm(d1, d0, 0x06040200u);
                    H[e >> 2][e & 3] = __builtin_amdgcn_perm(d1, d0, 0x07050301u);
                }
                SCHED();
                PSTAMP(2);
                if (k < PEARLY) {
#pragma unroll
                    for (int j = 0; j < 2; ++j)
#pragma unroll
                        for (int q = 0; q < NR; ++q) __builtin_amdgcn_raw_buffer_store_b128(L[j], rs[q], vo + j * 1024, soff, SYS);
                    SCHED(); asm volatile("s_nop 1" ::: "memory"); SCHED();
                }
#pragma unroll
                for (int e = 0; e < 8; ++e) sum = __builtin_amdgcn_sad_u8(H[e >> 2][e & 3] & 0x7f7f7f7fu, 0u, sum);
                sum += __builtin_amdgcn_update_dpp(0u, sum, 0x111, 0xf, 0xf, true);
                sum += __builtin_amdgcn_update_dpp(0u, sum, 0x112, 0xf, 0xf, true);
                sum += __builtin_amdgcn_update_dpp(0u, sum, 0x114, 0xf, 0xf, true);
                sum += __builtin_amdgcn_update_dpp(0u, sum, 0x118, 0xf, 0xf, true);
                const int tot = __builtin_amdgcn_readlane(sum, 15) + __builtin_amdgcn_readlane(sum, 31) + __builtin_amdgcn_readlane(sum, 47) + __builtin_amdgcn_readlane(sum, 63);
                int bs = (tot - 2970) >> 11;
                bs = bs < 0 ? 0 : (bs > 124 ? 124 : bs);
                const unsigned b4 = (unsigned)bs * 0x01010101u, nb4 = (unsigned)((-bs) & 3) * 0x01010101u;
                unsigned cnt = 0, c[8], sg[8], x[8], rest = 0;
                auto emit = [&](int e) {
                    const u64 m = __builtin_amdgcn_ballot_w64(x[e] != 0u);
                    const unsigned sh = (unsigned)__builtin_ctz(x[e] | 0x80000000u) & 24u;
                    const unsigned pos = min(cnt, (unsigned)EXCAP + 1u) + __builtin_amdgcn_mbcnt_hi((unsigned)(m >> 32), __builtin_amdgcn_mbcnt_lo((unsigned)m, 0u));
                    const unsigned ent = (((x[e] >> sh) & 0x7cu) << 9) | ((unsigned)e << 8) | ((sh & 16u) << 3) | ptag | ((sh >> 3) & 1u);
                    *(LAS unsigned short*)(tail + (x[e] != 0u ? 768u + pos * 2 : 1024u + lane * 2)) = (unsigned short)ent;
                    x[e] &= ~(0xffu << sh);
                    cnt += (unsigned)__builtin_popcountll(m);
                };
#pragma unroll
                for (int e = 0; e < 8; ++e) {
                    const unsigned hh = H[e >> 2][e & 3], hm = hh & 0x7f7f7f7fu;
                    c[e] = (hm + nb4) & 0x03030303u;
                    sg[e] = hh & 0x80808080u;
                    x[e] = hm ^ (c[e] + b4);
                    emit(e);
                    rest |= x[e];
                }
                while (__builtin_amdgcn_ballot_w64(rest != 0u) != 0ull && cnt <= (unsigned)EXCAP) {
                    rest = 0;
#pragma unroll
                    for (int e = 0; e < 8; ++e) { if (__builtin_amdgcn_ballot_w64(x[e] != 0u) != 0ull) emit(e); rest |= x[e]; }
                }
                ((LAS unsigned*)tail)[lane * 3 + 0] = (c[0] | sg[0]) | rotr(c[1] | sg[1], 29) | (c[6] << 5);
                ((LAS unsigned*)tail)[lane * 3 + 1] = (c[2] | sg[2]) | rotr(c[3] | sg[3], 29) | (c[7] << 5);
                ((LAS unsigned*)tail)[lane * 3 + 2] = (c[4] | sg[4]) | rotr(c[5] | sg[5], 29) | (sg[6] >> 1) | (sg[7] >> 2);
                asm volatile("" ::: "memory");
                T = *(volatile LAS u32x4*)(tail + lane * 16);
                unsigned hdr = (unsigned)bs | (cnt << 8);
                if (cnt > (unsigned)EXCAP) { hdr = 0x80u; T = H[0]; }
                unsigned vt = ((hdr & 0x80u) || (unsigned)lane < 48u + ((cnt + 7u) >> 3)) ? vo + 2048 : 0x40000000u;
                asm volatile("" : "+v"(T), "+v"(vt));
                PSTAMP(3);
                SCHED();
                if (k >= PEARLY) {
                    Spin sp;
                    for (;;) {
                        const u32x2 cc = *(volatile LAS u32x2*)(lds + LC_OFF + C_SENT * 4);
                        if ((int)rfl(cc.x) + PWIN > k && (int)rfl(cc.y) >= k) break;
                        __builtin_amdgcn_s_sleep(1);
                        if (sp.over<1023u>(tmo)) break;
                    }
                    asm volatile("" ::: "memory");
                    PSTAMP(4);
#pragma unroll
                    for (int j = 0; j < 2; ++j)
#pragma unroll
                        for (int q = 0; q < NR; ++q) __builtin_amdgcn_raw_buffer_store_b128(L[j], rs[q], vo + j * 1024, soff, SYS);
                }
#pragma unroll
                for (int q = 0; q < NR; ++q) __builtin_amdgcn_raw_buffer_store_b128(T, rs[q], vt, soff, SYS);
                if (hdr & 0x80u) {
#pragma unroll
                    for (int q = 0; q < NR; ++q) __builtin_amdgcn_raw_buffer_store_b128(H[1], rs[q], vo + 3072, soff, SYS);
                }
                SCHED(); asm volatile("s_nop 1" ::: "memory"); SCHED();
                asm volatile("s_waitcnt vmcnt(%0)" :: "i"(SREL) : "memory");
                if (lane == 0) __hip_atomic_fetch_add((LAS unsigned*)(lds + LC_OFF) + C_ISS, 1u, __ATOMIC_RELAXED, __HIP_MEMORY_SCOPE_WORKGROUP);
                PSTAMP(5);
                unsigned tv = 0;
                if (lane == 0) tv = __hip_atomic_fetch_add(tk, 1u, __ATOMIC_RELAXED, __HIP_MEMORY_SCOPE_AGENT);
                VMWAIT0();
                PSTAMP(6);
                if (lane == 0) __hip_atomic_fetch_add((LAS unsigned*)(lds + LC_OFF) + C_SENT, 1u, __ATOMIC_RELAXED, __HIP_MEMORY_SCOPE_WORKGROUP);
                if (lane == NR - 1 || (lane < NR && !nopush)) __hip_atomic_store(flag + fi, ((u64)hdr << 32) | call, __ATOMIC_RELAXED, __HIP_MEMORY_SCOPE_SYSTEM);
                PSTAMP(7);
                u = PW * NCOMM + (rfl(tv) - (call - 1u) * NHU);
                if (u < NHU) {
                    load_x((int)u);
                    unsigned kq = 0;
                    if (lane == 0) kq = __hip_atomic_fetch_add((LAS unsigned*)(lds + LC_OFF) + C_SEQ, 1u, __ATOMIC_RELAXED, __HIP_MEMORY_SCOPE_WORKGROUP);
                    k = PW + (int)rfl(kq);
                }
            }
            if (lane == 0) __hip_atomic_fetch_add((LAS unsigned*)(lds + LC_OFF) + C_CNT, 1u, __ATOMIC_RELAXED, __HIP_MEMORY_SCOPE_WORKGROUP);
        } else {
            WG_START();
            if (wave != 7) return;
            GET_CALL();
            if (lane == 0) __hip_atomic_store((GAS u64*)(me + A1_OFF) + (par * NWG + bx) * 8, (u64)call << 32, __ATOMIC_RELAXED, __HIP_MEMORY_SCOPE_SYSTEM);
            { Spin sp; while (rfl(ctl[C_CNT]) < (unsigned)PW) { __builtin_amdgcn_s_sleep(8); if (sp.over<1023u>(tmo)) break; } }
            if (lane == 0) __hip_atomic_store(seqw, call, __ATOMIC_RELAXED, __HIP_MEMORY_SCOPE_AGENT);
            if (cid < 2) {
                const __amdgpu_buffer_rsrc_t rs = mkrsrc(me + A1_OFF + (size_t)par * NWG * 64, NWG * 64);
                GAS u64* dst = (GAS u64*)(pick_peer(p, lane >> 3) + (cid ? A3_OFF : A2_OFF) + (size_t)(par * 8 + (lane & 7)) * 128) + rank;
                unsigned g = 0, sent = 0, off[4]; Spin sp;
#pragma unroll
                for (int j = 0; j < 4; ++j) off[j] = lane * 64 + j * 4096;
                for (;;) {
                    u32x2 a[4];
#pragma unroll
                    for (int j = 0; j < 4; ++j) a[j] = __builtin_amdgcn_raw_buffer_load_b64(rs, off[j], 0, SYS);
                    int nin = 0;
#pragma unroll
                    for (int j = 0; j < 4; ++j) {
                        if (a[j].y == call) { g = max(g, a[j].x); off[j] = 0x40000000u; }
                        nin += __builtin_popcountll(__builtin_amdgcn_ballot_w64(off[j] == 0x40000000u));
                    }
                    if (nin == NWG) break;
                    if (cid && nin >= SPECN && __any(g > sent)) { sent = wave_max(g); __hip_atomic_store(dst, ((u64)call << 32) | sent, __ATOMIC_RELAXED, __HIP_MEMORY_SCOPE_SYSTEM); }
                    asm volatile("" ::: "memory");
                    if (sp.over<255u>(tmo)) break;
                }
                if (cid == 0) {
                    g = wave_max(g);
                    __hip_atomic_store(dst, ((u64)call << 32) | g, __ATOMIC_RELAXED, __HIP_MEMORY_SCOPE_SYSTEM);
                    STAMP(2);
                }
            }
        }
        return;
    }

    if (idx == 2 * CPX - 1 && wave < 4) {
        const char* pa = wave == 0 ? p.peer[0] : wave == 1 ? p.peer[2] : wave == 2 ? p.peer[4] : p.peer[6];
        const char* pb = wave == 0 ? p.peer[1] : wave == 1 ? p.peer[3] : wave == 2 ? p.peer[5] : p.peer[7];
        const int w = (lane >> 1) & 7;
        touch(((lane & 1) ? pb : pa) + (w < 4 ? XS_OFF + (size_t)rank * XS_SRC + (size_t)w * (XS_SRC / 4) : XF_OFF + (size_t)rank * 8 * KS * 16), junk);
    }
    if (wave == 3) {
        const size_t o1 = (size_t)rtile * 128 * NL, o2 = (size_t)(rank * 512 + xrow) * NL;
        touch(lane == 0 ? (const char*)p.out + o1 : lane == 1 ? (const char*)p.out + o2 : lane == 2 ? (const char*)(p.x + (size_t)xrow * KDIM) : me + XS_OFF + (size_t)rtile * (XS_SRC / 4), junk);
    }
    const unsigned tstart = rt32();
    WG_START();
    unsigned call, wfl = 0;
    {
        constexpr int NH = 8 * CPX, NN = NCW - NH, TH = KS - 4 * NN;
        static_assert(TH >= NH && TH <= 2 * NH, "at most one unit per wave of a host");
        const int t1 = host ? (wave < 4 ? NH + ci : ci) : TH + (ci - NH) * 4 + (wave >> 2) * 2, q = wave & 3;
        const bool one = !host || t1 < TH;
        while (rt32() - tstart < (host ? (wave < 4 ? WDELAY2 : WDELAY) + ci * WSTAG : WDELAY)) __builtin_amdgcn_s_sleep(2);
        f32x4 va[16], vb[16];
        w_load(va, p.w, one, ct, t1, q, lane);
        if (!host) w_load(vb, p.w, true, ct, t1 + 1, q, lane);
        const unsigned sv = __hip_atomic_load(seqw, __ATOMIC_RELAXED, __HIP_MEMORY_SCOPE_AGENT);
        SCHED();
        if (one) w_store(va, me, ct, t1, q, lane);
        if (!host) w_store(vb, me, ct, t1 + 1, q, lane);
        if (wave < 4 && lane < NH) wfl = __hip_atomic_load((GAS unsigned*)(me + WF_OFF) + ((ct * 4 + wave) * KS + lane) * 16, __ATOMIC_RELAXED, __HIP_MEMORY_SCOPE_SYSTEM);
        VMWAIT0();
        call = rfl(sv) + 1u;
        if (lane < (host ? (one ? 1 : 0) : 2)) __hip_atomic_store((GAS unsigned*)(me + WF_OFF) + ((ct * 4 + q) * KS + t1 + lane) * 16, call, __ATOMIC_RELAXED, __HIP_MEMORY_SCOPE_SYSTEM);
    }
    const Tmo tmo{(GAS unsigned*)(me + TM_OFF), call};
    const unsigned par = call & 1u;
    const bool nopush = NOPUSH;
    if (wave < 4) {
        STAMP(1);
        const __amdgpu_buffer_rsrc_t rsw = mkrsrc(me + WB_OFF + (size_t)(ct * 4 + wave) * KS * 8192, KS * 8192);
        const __amdgpu_buffer_rsrc_t rsf = mkrsrc(me + WF_OFF + (size_t)(ct * 4 + wave) * KS * 64, KS * 64);
        const unsigned ldsb = (unsigned)(size_t)lds;
        unsigned vo = lane * 16;
        const unsigned abase = ldsb + lane * 16;
        const unsigned polla = ldsb + LC_OFF + (C_FULL + (lane & 3)) * 4, donea = ldsb + LC_OFF + (C_DONE + wave) * 4;
        f32x32 r0, r1, r2, r3; f32x16 r4;
        static_assert(LA_SLOT == 9216 && LA_HALF == 27648 && LA_PAR == 55296, "literals in the K loop");
        int s_wk = __builtin_ctzll(__builtin_amdgcn_ballot_w64(wfl < call)) - 1;
        unsigned s_so, s_t, s_bo, s_bc, s_x, s_sp, s_t0, s_late; u64 s_c, s_c0, s_r0; unsigned s_wx, s_ww;
        asm volatile(R"ASM(
.set AG_A, 8
.set AG_C, 112
.set AG_DIAG, )ASM" XDG R"ASM(
.macro AG_EXPIRED lout
  s_add_u32 %[sp], %[sp], 1
  s_and_b32 %[x], %[sp], 0x3ff
  s_cbranch_scc1 8f
  v_mov_b32 v6, 0
  global_load_dword v3, v6, %[tmo] sc1
  s_memrealtime %[c]
  s_waitcnt vmcnt(0) lgkmcnt(0)
  v_readfirstlane_b32 %[x], v3
  s_cmp_eq_u32 %[x], %[call]
  s_cbranch_scc1 \lout
  s_mov_b64 vcc, %[c]
  s_cmp_eq_u32 %[sp], 0x400
  s_cselect_b32 %[t0], vcc_lo, %[t0]
  s_sub_u32 %[x], vcc_lo, %[t0]
  s_cmp_le_u32 %[x], 40000000
  s_cbranch_scc1 8f
  v_mov_b32 v3, %[call]
  global_store_dword v6, v3, %[tmo] sc1
  s_waitcnt vmcnt(0)
  s_branch \lout
8:
.endm
.macro AG_TIC
  s_mov_b32 %[sp], 0
  .if AG_DIAG
  s_memrealtime %[c]
  s_waitcnt lgkmcnt(0)
  s_mov_b64 vcc, %[c]
  s_mov_b32 %[t0], vcc_lo
  .endif
.endm
.macro AG_TOC acc
  .if AG_DIAG
  s_memrealtime %[c]
  s_waitcnt lgkmcnt(0)
  s_mov_b64 vcc, %[c]
  s_sub_u32 %[x], vcc_lo, %[t0]
  s_add_u32 \acc, \acc, %[x]
  .endif
.endm
; wait until w K-step (t + add + 1) is converted; wk = last K-step known to be converted with all before it
.macro AG_NEEDW add
  s_waitcnt lgkmcnt(0)
  AG_TIC
1:
  v_lshlrev_b32 v6, 2, v0
  buffer_load_dword v3, v6, %[rsf], 0 offen sc0 sc1
  s_waitcnt vmcnt(0)
  v_cmp_gt_u32 vcc, %[call], v3
  s_ff1_i32_b64 %[wk], vcc
  s_min_u32 %[wk], %[wk], 1000
  s_sub_i32 %[wk], %[wk], 1
  s_add_u32 %[x], %[t], \add
  s_cmp_gt_i32 %[wk], %[x]
  s_cbranch_scc1 3f
  s_sleep 2
  AG_EXPIRED 2f
  s_branch 1b
2:
  s_movk_i32 %[wk], 1000
3:
  AG_TOC %[ww]
  ds_read_b32 v3, v4
  s_add_u32 %[x], %[t], \add
.endm
; wait until x K-step (t + add) is in the ring, then read its first half
.macro AG_NEEDX add, va, nb
  AG_TIC
  s_add_u32 %[late], %[late], 1
1:
  ds_read_b32 v3, v4
  s_add_u32 %[x], %[t], \add
  s_waitcnt lgkmcnt(0)
  v_cmp_lt_u32 vcc, %[x], v3
  s_cmp_eq_u64 vcc, exec
  s_cbranch_scc1 2f
  s_sleep 1
  AG_EXPIRED 2f
  s_branch 1b
2:
  AG_TOC %[wx]
  .set ag_b, 0
  .rept 8
  ds_read_b128 v[AG_A+4*ag_b:AG_A+4*ag_b+3], \va offset:(ag_b/4)*27648+(ag_b%%4)*1024
  .set ag_b, ag_b+1
  .endr
  .if \nb == 9
  ds_read_b128 v[AG_A+32:AG_A+35], \va offset:8192
  .endif
.endm
.macro AG_MF b, m, bc
  .set ag_n, \m ^ ((\b & 1) * 3)
  v_mfma_f32_16x16x32_bf16 v[AG_C+16*\b+4*ag_n:AG_C+16*\b+4*ag_n+3], v[\bc+4*ag_n:\bc+4*ag_n+3], v[AG_A+4*\b:AG_A+4*\b+3], v[AG_C+16*\b+4*ag_n:AG_C+16*\b+4*ag_n+3]
.endm
; half K-step on nb row blocks: first = 1 for the first half of step (t + add)
.macro AG_HALF nb, bc, bn, va, vnext, kh, first, add
  s_waitcnt vmcnt(8) lgkmcnt(\nb-3)
  .set ag_b, 0
  .rept \nb
  .if (ag_b == 3) || (ag_b == 6)
  s_waitcnt lgkmcnt(\nb-3)
  .endif
  AG_MF ag_b, 0, \bc
  .if ag_b == 4
  s_add_u32 %[bo], %[bo], 4096
  .endif
  .if \first && (ag_b == 5)
  s_add_u32 %[x], %[t], \add+1
  .endif
  .if \first && \add && (ag_b == 6)
  s_cmp_eq_u32 %[so], 27648
  .endif
  .if (!\first) && (ag_b == 6)
  v_cmp_lt_u32 vcc, %[x], v3
  .endif
  .if \first && \add && (ag_b == 7)
  v_add_u32 \vnext, %[so], v1
  .endif
  .if \first && (!\add) && (ag_b == 7)
  v_add_u32 \vnext, 55296, \va
  .endif
  AG_MF ag_b, 1, \bc
  .if ag_b < 4
  buffer_load_dwordx4 v[\bn+4*ag_b:\bn+4*ag_b+3], v0, %[rsw], %[bc] offen offset:ag_b*1024 sc0 sc1
  .endif
  .if ag_b == 4
  s_min_u32 %[bc], %[bo], 127*4096
  .endif
  .if (!\first) && (ag_b == 5)
  v_mov_b32 v6, %[x]
  .endif
  .if (!\first) && (ag_b == 6)
  ds_write_b32 v5, v6
  .endif
  .if \first && \add && (ag_b == 6)
  s_cselect_b32 %[so], 0, %[so]
  .endif
  .if \first && (ag_b == 7)
  s_cmp_le_i32 %[wk], %[x]
  .endif
  .if (!\first) && (ag_b == 7)
  s_cmp_eq_u64 vcc, exec
  .endif
  AG_MF ag_b, 2, \bc
  .if \first && \add && (ag_b == 5)
  s_add_u32 %[so], %[so], 9216
  .endif
  .if \first && (ag_b == \nb-1)
  ds_read_b32 v3, v4
  .endif
  AG_MF ag_b, 3, \bc
  .if ag_b < 8
  ds_read_b128 v[AG_A+4*ag_b:AG_A+4*ag_b+3], \va offset:(ag_b/4)*27648+\kh*4096+(ag_b%%4)*1024
  .else
  ds_read_b128 v[AG_A+32:AG_A+35], \va offset:8192+\kh*27648
  .endif
  .set ag_b, ag_b+1
  .endr
.endm
.macro AG_LOOP nb
  AG_NEEDX 0, v2, \nb
  .if AG_DIAG
  s_mov_b32 %[wx], 0
  s_mov_b32 %[ww], 0
  s_memtime %[c0]
  s_memrealtime %[r0]
  s_waitcnt lgkmcnt(0)
  .endif
.Lag_loop\nb:
  AG_HALF \nb, 44, 92, v2, v7, 1, 1, 0
  s_cbranch_scc1 .Lag_w0\nb
.Lag_w0r\nb:
  AG_HALF \nb, 60, 44, v7, v7, 0, 0, 0
  s_cbranch_scc0 .Lag_x0\nb
.Lag_x0r\nb:
  AG_HALF \nb, 76, 60, v7, v2, 1, 1, 1
  s_cbranch_scc1 .Lag_w1\nb
.Lag_w1r\nb:
  AG_HALF \nb, 92, 76, v2, v2, 0, 0, 1
  s_cbranch_scc0 .Lag_x1\nb
.Lag_x1r\nb:
  s_add_u32 %[t], %[t], 2
  s_cmp_lt_u32 %[t], 64
  s_cbranch_scc1 .Lag_loop\nb
  s_branch .Lag_end
.Lag_w0\nb:
  AG_NEEDW 1
  s_branch .Lag_w0r\nb
.Lag_w1\nb:
  AG_NEEDW 2
  s_branch .Lag_w1r\nb
.Lag_x0\nb:
  AG_NEEDX 1, v7, \nb
  s_branch .Lag_x0r\nb
.Lag_x1\nb:
  AG_NEEDX 2, v2, \nb
  s_branch .Lag_x1r\nb
.endm
  s_nop 4
  s_mov_b32 %[t], 0
  s_mov_b32 %[late], 0
  s_mov_b32 %[so], 0
  s_cmp_gt_i32 %[wk], 0
  s_cbranch_scc1 .Lag_go
  AG_NEEDW 0
.Lag_go:
  .set ag_i, 0
  .rept 12
  s_movk_i32 %[bc], (ag_i/4)*4096
  buffer_load_dwordx4 v[44+4*ag_i:44+4*ag_i+3], v0, %[rsw], %[bc] offen offset:(ag_i%%4)*1024 sc0 sc1
  .set ag_i, ag_i+1
  .endr
  s_movk_i32 %[bo], 3*4096
  s_movk_i32 %[bc], 3*4096
  .set ag_i, 0
  .rept 72
  v_mov_b64 v[AG_C+ag_i:AG_C+ag_i+1], 0
  .set ag_i, ag_i+2
  .endr
  v_mov_b32 v2, v1
  s_cmp_eq_u32 %[host], 0
  s_cbranch_scc1 .Lag_eight
  AG_LOOP 9
.Lag_eight:
  AG_LOOP 8
.Lag_end:
  s_waitcnt vmcnt(0) lgkmcnt(0)
  s_nop 7
)ASM"
            : "={v[112:143]}"(r0), "={v[144:175]}"(r1), "={v[176:207]}"(r2), "={v[208:239]}"(r3), "={v[240:255]}"(r4),
              [t] "=&s"(s_t), [wk] "+s"(s_wk), [bo] "=&s"(s_bo), [bc] "=&s"(s_bc), [x] "=&s"(s_x), [sp] "=&s"(s_sp), [t0] "=&s"(s_t0),
              [late] "=&s"(s_late), [so] "=&s"(s_so), [c] "=&s"(s_c), [c0] "=&s"(s_c0), [wx] "=&s"(s_wx), [ww] "=&s"(s_ww), [r0] "=&s"(s_r0), "+{v0}"(vo)
            : "{v1}"(abase), "{v4}"(polla), "{v5}"(donea), [rsw] "s"(rsw), [rsf] "s"(rsf), [tmo] "s"(tmo.word), [call] "s"(call), [host] "s"(rfl(host ? 1u : 0u))
            : "memory", "vcc", "scc", "v2", "v3", "v6", "v7", "v8", "v9", "v10", "v11", "v12", "v13", "v14", "v15", "v16", "v17", "v18", "v19", "v20", "v21", "v22", "v23", "v24", "v25", "v26", "v27", "v28", "v29", "v30", "v31", "v32", "v33", "v34", "v35", "v36", "v37", "v38", "v39", "v40", "v41", "v42", "v43", "v44", "v45", "v46", "v47", "v48", "v49", "v50", "v51", "v52", "v53", "v54", "v55", "v56", "v57", "v58", "v59", "v60", "v61", "v62", "v63", "v64", "v65", "v66", "v67", "v68", "v69", "v70", "v71", "v72", "v73", "v74", "v75", "v76", "v77", "v78", "v79", "v80", "v81", "v82", "v83", "v84", "v85", "v86", "v87", "v88", "v89", "v90", "v91", "v92", "v93", "v94", "v95", "v96", "v97", "v98", "v99", "v100", "v101", "v102", "v103", "v104", "v105", "v106", "v107");
        f32x4 acc[9][4];
#pragma unroll
        for (int i = 0; i < 144; ++i) acc[i >> 4][(i >> 2) & 3][i & 3] = i < 32 ? r0[i & 31] : i < 64 ? r1[i & 31] : i < 96 ? r2[i & 31] : i < 128 ? r3[i & 31] : r4[i & 15];
        STAMP(3);
        const int ln = (int)(vo >> 4);
        int mx = 0;
#pragma unroll
        for (int n = 0; n < 4; ++n) acc[8][n] *= XSCALE;
        if (host) {
#pragma unroll
            for (int n = 0; n < 4; ++n) { mx = max(max(mx, fbits(acc[8][n][0])), fbits(acc[8][n][1])); mx = max(max(mx, fbits(acc[8][n][2])), fbits(acc[8][n][3])); }
        }
        int m2 = 0;
#pragma unroll
        for (int b = 0; b < 8; ++b)
#pragma unroll
            for (int n = 0; n < 4; ++n) { mx = max(max(mx, fbits(acc[b][n][0])), fbits(acc[b][n][1])); m2 = max(max(m2, fbits(acc[b][n][2])), fbits(acc[b][n][3])); }
        const unsigned wm = wave_max((unsigned)max(mx, m2));
        if (ln == 0) {
            __hip_atomic_fetch_max((LAS unsigned*)(lds + LC_OFF) + C_MAX, wm, __ATOMIC_RELAXED, __HIP_MEMORY_SCOPE_WORKGROUP);
            asm volatile("" ::: "memory");
            const unsigned arrived = __hip_atomic_fetch_add((LAS unsigned*)(lds + LC_OFF) + C_CNT, 1u, __ATOMIC_RELAXED, __HIP_MEMORY_SCOPE_WORKGROUP);
            asm volatile("" ::: "memory");
            const unsigned tm = ctl[C_MAX];
            if (arrived == 3u) __hip_atomic_store((GAS u64*)(me + A1_OFF) + (par * NWG + bx) * 8, ((u64)call << 32) | tm, __ATOMIC_RELAXED, __HIP_MEMORY_SCOPE_SYSTEM);
        }
        STAMP(6);
        const __amdgpu_buffer_rsrc_t rso = mkrsrc(p.out, 8u * 512u * NL);
        const unsigned ovo = (unsigned)(ln & 15) * NL + ct * 256 + wave * 64 + (ln >> 4) * 16;
        unsigned used = 0xffffffffu; Spin sp;
#pragma unroll 1
        for (;;) {
            const u32x2 c = *(volatile LAS u32x2*)(lds + LC_OFF + C_GMAX * 4);
            unsigned st = rfl(c.y);
            const unsigned run = rfl(c.x), g = max(run, wm);
            if (st == 0u && sp.over<4095u>(tmo)) st = 2u;
            if (st == 2u || (g != used && (st != 0u || run != 0u))) {
                if (used != 0xffffffffu) VMWAIT0();
                used = g;
                const float scale = __uint_as_float(g) / 127.0f;
                const float inv = scale > 0.f ? 1.0f / scale : 0.f;
#pragma unroll
                for (int b = 0; b < 9; ++b) {
                    u32x4 o = {0u, 0u, 0u, 0u};
#pragma unroll
                    for (int cc = 0; cc < 4; ++cc) {
                        f32x4 q = acc[b][cc] * inv;
                        asm volatile("" : "+v"(q));
#pragma unroll
                        for (int r = 0; r < 4; ++r) o[r] = __builtin_amdgcn_cvt_pk_u8_f32(q[r], cc, o[r]);
                    }
                    if (st == 2u) o = (u32x4){0x80808080u, 0x80808080u, 0x80808080u, 0x80808080u};
                    if (b < 8 || host) __builtin_amdgcn_raw_buffer_store_b128(o, rso, ovo + (unsigned)(b < 8 ? rtile * 128 + b * 16 : rank * 512 + xrow) * NL, 0, SYS);
                }
            }
            if (st != 0u) break;
            __builtin_amdgcn_s_sleep(1);
        }
        STAMP(5);
    } else {
        const int f = wave & 1, ph = (wave >> 1) & 1, rb = sub * 2 + f;
        STAMP(6);
        const __amdgpu_buffer_rsrc_t rsx = mkrsrc(me + XS_OFF + (size_t)src * XS_SRC + (size_t)rb * KS * UNIT, KS * UNIT);
        const __amdgpu_buffer_rsrc_t rse = mkrsrc(p.x + (size_t)xrow * KDIM + f * 32, host ? 16u * KDIM * 4u - 128u : 0u);
        const __amdgpu_buffer_rsrc_t rsf = mkrsrc(me + XF_OFF + (size_t)(src * 8 + rb) * KS * 16, KS * 16);
        const unsigned ldsb = (unsigned)(size_t)lds;
        const unsigned mbase = ldsb + f * LA_HALF, vpub = ldsb + LC_OFF + (C_FULL + (wave & 3)) * 4, vdone = ldsb + LC_OFF + (C_DONE + (lane & 3)) * 4;
        const unsigned fcall = (nopush && !local) ? 0u : call, hostf = rfl(host ? 1u : 0u);
        unsigned s_cur = ph, s_lim = NST, s_so = 0, s_h, s_g, s_b, s_x, s_y, s_sp, s_t0, s_m0; u64 s_seen, s_c;
        asm volatile(R"ASM(
.macro FD_EXPIRED lout
  s_add_u32 %[sp], %[sp], 1
  s_and_b32 %[x], %[sp], 0xff
  s_cbranch_scc1 8f
  v_mov_b32 v9, 0
  global_load_dword v12, v9, %[tmo] sc1
  s_memrealtime %[c]
  s_waitcnt vmcnt(0) lgkmcnt(0)
  v_readfirstlane_b32 %[x], v12
  s_cmp_eq_u32 %[x], %[call]
  s_cbranch_scc1 \lout
  s_mov_b64 vcc, %[c]
  s_cmp_eq_u32 %[sp], 0x100
  s_cselect_b32 %[t0], vcc_lo, %[t0]
  s_sub_u32 %[x], vcc_lo, %[t0]
  s_cmp_le_u32 %[x], 40000000
  s_cbranch_scc1 8f
  v_mov_b32 v12, %[call]
  global_store_dword v9, v12, %[tmo] sc1
  s_waitcnt vmcnt(0)
  s_branch \lout
8:
.endm
.macro FD_FLAGS
  buffer_load_dwordx4 v[100:103], v0, %[rsf], 0 offen sc0 sc1
  s_waitcnt vmcnt(0)
  v_min_u32 v9, v100, v102
  v_cmp_le_u32 %[seen], %[fcall], v9
.endm
; loads of unit %[y] into the set at r
.macro FD_ISSUE r
  s_lshl_b32 %[x], %[y], 13
  s_lshl_b32 %[b], %[y], 8
  buffer_load_dwordx4 v[\r:\r+3], v0, %[rsx], %[x] offen sc0 sc1
  buffer_load_dwordx4 v[\r+4:\r+7], v0, %[rsx], %[x] offen offset:1024 sc0 sc1
  buffer_load_dwordx3 v[\r+16:\r+18], v1, %[rsx], %[x] offen offset:2048 sc0 sc1
  buffer_load_ushort v[\r+19], v2, %[rsx], %[x] offen offset:2816 sc0 sc1
  s_bitset1_b32 %[x], 12
  buffer_load_dwordx4 v[\r+24:\r+27], v3, %[rse], %[b] offen
  buffer_load_dwordx4 v[\r+28:\r+31], v3, %[rse], %[b] offen offset:64
  buffer_load_dwordx4 v[\r+8:\r+11], v0, %[rsx], %[x] offen sc0 sc1
  buffer_load_dwordx4 v[\r+12:\r+15], v0, %[rsx], %[x] offen offset:1024 sc0 sc1
  buffer_load_dwordx3 v[\r+20:\r+22], v1, %[rsx], %[x] offen offset:2048 sc0 sc1
  buffer_load_ushort v[\r+23], v2, %[rsx], %[x] offen offset:2816 sc0 sc1
.endm
; wait for the flag of unit %[y]
.macro FD_WAITFLAG
  s_mov_b32 %[sp], 0
1:
  FD_FLAGS
  s_bitcmp1_b64 %[seen], %[y]
  s_cbranch_scc1 2f
  s_sleep 1
  FD_EXPIRED 2f
  s_branch 1b
2:
.endm
.macro FD_DEC h, n0, n1, n2
  v_and_or_b32 v[\h], v[\n0], v7, %[b]
  v_alignbit_b32 v12, v[\n0], v[\n0], 3
  v_and_or_b32 v[\h+2], v[\n1], v7, %[b]
  v_alignbit_b32 v13, v[\n1], v[\n1], 3
  v_and_or_b32 v[\h+4], v[\n2], v7, %[b]
  v_alignbit_b32 v14, v[\n2], v[\n2], 3
  v_and_or_b32 v[\h+1], v12, v7, %[b]
  v_lshrrev_b32 v12, 5, v[\n0]
  v_and_or_b32 v[\h+3], v13, v7, %[b]
  v_lshrrev_b32 v13, 5, v[\n1]
  v_and_or_b32 v[\h+5], v14, v7, %[b]
  v_lshlrev_b32 v14, 1, v[\n2]
  v_bfi_b32 v12, v8, v14, v12
  v_lshlrev_b32 v15, 2, v[\n2]
  v_and_or_b32 v[\h+6], v12, v7, %[b]
  v_bfi_b32 v13, v8, v15, v13
  v_and_or_b32 v[\h+7], v13, v7, %[b]
.endm
; exception entry in register e -> address v12, mask v13
.macro FD_EXC e
  v_and_b32 v12, 0x7fe, v[\e]
  v_and_b32 v14, 1, v[\e]
  v_lshrrev_b32 v13, 11, v[\e]
  v_lshl_or_b32 v14, v14, 4, 10
  v_lshl_add_u32 v12, v12, 1, %[x]
  v_lshlrev_b32 v13, v14, v13
.endm
; a half sent raw, or with a window that does not start at a multiple of four
.macro FD_FIX hd, h
  s_bitcmp1_b32 \hd, 7
  s_cbranch_scc0 3f
  buffer_load_dwordx4 v[\h:\h+3], v0, %[rsx], %[x] offen offset:2048 sc0 sc1
  buffer_load_dwordx4 v[\h+4:\h+7], v0, %[rsx], %[x] offen offset:3072 sc0 sc1
  s_waitcnt vmcnt(0)
3:
  s_and_b32 %[b], \hd, 3
  s_mul_i32 %[b], %[b], 0x01010101
  .set fd_i, 0
  .rept 8
  v_add_u32 v[\h+fd_i], %[b], v[\h+fd_i]
  .set fd_i, fd_i+1
  .endr
.endm
; one unit: set r holds the loads of unit cur, set q takes the next one; ob = offset of this parity's ring
.macro FD_STEP r, q, ob, id
  s_add_u32 %[y], %[cur], 2
  s_cmp_ge_u32 %[y], 64
  s_cbranch_scc1 .Lfd_last\id
  s_bitcmp1_b64 %[seen], %[y]
  s_cbranch_scc0 .Lfd_look\id
  FD_ISSUE \q
  s_waitcnt vmcnt(10)
.Lfd_go\id:
  v_readlane_b32 %[h], v101, %[cur]
  v_readlane_b32 %[g], v103, %[cur]
  s_and_b32 %[b], %[h], 0x7c
  s_mul_i32 %[b], %[b], 0x01010101
  FD_DEC 80, \r+16, \r+17, \r+18
  s_and_b32 %[b], %[g], 0x7c
  s_mul_i32 %[b], %[b], 0x01010101
  s_add_u32 m0, %[mb], %[so]
  FD_DEC 88, \r+20, \r+21, \r+22
  s_or_b32 %[x], %[h], %[g]
  s_and_b32 %[x], %[x], 0x83
  s_cbranch_scc1 .Lfd_fix\id
.Lfd_fixed\id:
  s_cmp_ge_u32 %[cur], %[lim]
  s_cbranch_scc1 .Lfd_free\id
.Lfd_freed\id:
  .set fd_i, 0
  .rept 16
  v_perm_b32 v[96+(fd_i%%2)*2], v[80+fd_i], v[\r+fd_i], %[p0]
  v_perm_b32 v[97+(fd_i%%2)*2], v[80+fd_i], v[\r+fd_i], %[p1]
  ds_write_addtid_b32 v[96+(fd_i%%2)*2] offset:\ob+fd_i*512
  ds_write_addtid_b32 v[97+(fd_i%%2)*2] offset:\ob+fd_i*512+256
  .set fd_i, fd_i+1
  .endr
  s_add_u32 %[x], m0, \ob
  FD_EXC \r+19
  s_lshr_b32 %[b], %[h], 8
  s_bfm_b64 exec, %[b], 0
  s_lshr_b32 %[b], %[g], 8
  ds_xor_b32 v12, v13
  s_bfm_b64 exec, %[b], 0
  s_add_u32 %[so], %[so], 9216
  FD_EXC \r+23
  ds_xor_b32 v12, v13 offset:4096
  s_mov_b64 exec, -1
  s_cmp_eq_u32 %[host], 0
  s_cbranch_scc1 .Lfd_pub\id
  v_cvt_pk_bf16_f32 v96, v[\r+24], v[\r+25]
  v_cvt_pk_bf16_f32 v97, v[\r+26], v[\r+27]
  v_cvt_pk_bf16_f32 v98, v[\r+28], v[\r+29]
  v_cvt_pk_bf16_f32 v99, v[\r+30], v[\r+31]
  ds_write_addtid_b32 v96 offset:\ob+8192
  ds_write_addtid_b32 v97 offset:\ob+8192+256
  ds_write_addtid_b32 v98 offset:\ob+8192+512
  ds_write_addtid_b32 v99 offset:\ob+8192+768
.Lfd_pub\id:
  s_add_u32 %[cur], %[cur], 2
  s_cmp_eq_u32 %[so], 27648
  s_cselect_b32 %[so], 0, %[so]
  s_cmp_ge_u32 %[cur], 64
  s_cselect_b32 %[x], 1000, %[cur]
  v_mov_b32 v9, %[x]
  s_waitcnt lgkmcnt(0)
  ds_write_b32 v4, v9
  s_cbranch_scc1 .Lfd_end
  s_cmp_eq_u32 %[y], %[cur]
  s_cbranch_scc0 .Lfd_late\id
.Lfd_next\id:
.endm
; out-of-line parts of a step
.macro FD_SLOW r, q, ob, id
.Lfd_last\id:
  s_waitcnt vmcnt(0)
  s_branch .Lfd_go\id
.Lfd_look\id:
  FD_FLAGS
  s_bitcmp1_b64 %[seen], %[y]
  s_cbranch_scc0 3f
  FD_ISSUE \q
  s_branch .Lfd_go\id
3:
  s_mov_b32 %[y], -1
  s_branch .Lfd_go\id
.Lfd_late\id:
  s_mov_b32 %[y], %[cur]
  FD_WAITFLAG
  FD_ISSUE \q
  s_branch .Lfd_next\id
.Lfd_fix\id:
  s_lshl_b32 %[x], %[cur], 13
  FD_FIX %[h], 80
  s_bitset1_b32 %[x], 12
  FD_FIX %[g], 88
  s_branch .Lfd_fixed\id
.Lfd_free\id:
  s_mov_b32 %[sp], 0
1:
  ds_read_b32 v9, v5
  s_waitcnt lgkmcnt(0)
  v_readlane_b32 %[lim], v9, 0
  v_readlane_b32 %[x], v9, 1
  s_min_u32 %[lim], %[lim], %[x]
  v_readlane_b32 %[x], v9, 2
  s_min_u32 %[lim], %[lim], %[x]
  v_readlane_b32 %[x], v9, 3
  s_min_u32 %[lim], %[lim], %[x]
  s_add_u32 %[lim], %[lim], 6
  s_cmp_lt_u32 %[cur], %[lim]
  s_cbranch_scc1 .Lfd_freed\id
  s_sleep 4
  FD_EXPIRED .Lfd_freed\id
  s_branch 1b
.endm
  s_nop 4
  s_mov_b32 %[m0s], m0
  v_mov_b32 v7, 0x83838383
  v_mov_b32 v8, 0x80808080
  s_mov_b32 %[y], %[cur]
  FD_WAITFLAG
  FD_ISSUE 16
  s_bitcmp1_b32 %[cur], 0
  s_cbranch_scc1 .Lfd_loop1
.Lfd_loop0:
  FD_STEP 16, 48, 0, 0
  FD_STEP 48, 16, 0, 1
  s_branch .Lfd_loop0
.Lfd_loop1:
  FD_STEP 16, 48, 55296, 2
  FD_STEP 48, 16, 55296, 3
  s_branch .Lfd_loop1
  FD_SLOW 16, 48, 0, 0
  FD_SLOW 48, 16, 0, 1
  FD_SLOW 16, 48, 55296, 2
  FD_SLOW 48, 16, 55296, 3
.Lfd_end:
  s_mov_b32 m0, %[m0s]
  s_waitcnt vmcnt(0) lgkmcnt(0)
)ASM"
            : [cur] "+s"(s_cur), [lim] "+s"(s_lim), [so] "+s"(s_so), [h] "=&s"(s_h), [g] "=&s"(s_g), [b] "=&s"(s_b), [x] "=&s"(s_x), [y] "=&s"(s_y), [sp] "=&s"(s_sp),
              [t0] "=&s"(s_t0), [m0s] "=&s"(s_m0), [seen] "=&s"(s_seen), [c] "=&s"(s_c)
            : "{v0}"(lane * 16), "{v1}"(lane * 12), "{v2}"(lane * 2), "{v3}"((lane >> 2) * KDIM * 4 + (lane & 3) * 16), "{v4}"(vpub), "{v5}"(vdone),
              [rsx] "s"(rsx), [rse] "s"(rse), [rsf] "s"(rsf), [tmo] "s"(tmo.word), [call] "s"(call), [fcall] "s"(fcall), [mb] "s"(mbase), [host] "s"(hostf),
              [p0] "s"(0x05010400u), [p1] "s"(0x07030602u)
            : "memory", "vcc", "scc", "v7", "v8", "v9", "v10", "v11", "v12", "v13", "v14", "v15", "v16", "v17", "v18", "v19", "v20", "v21", "v22", "v23", "v24", "v25", "v26", "v27", "v28", "v29", "v30", "v31", "v32", "v33", "v34", "v35", "v36", "v37", "v38", "v39", "v40", "v41", "v42", "v43", "v44", "v45", "v46", "v47", "v48", "v49", "v50", "v51", "v52", "v53", "v54", "v55", "v56", "v57", "v58", "v59", "v60", "v61", "v62", "v63", "v64", "v65", "v66", "v67", "v68", "v69", "v70", "v71", "v72", "v73", "v74", "v75", "v76", "v77", "v78", "v79", "v80", "v81", "v82", "v83", "v84", "v85", "v86", "v87", "v88", "v89", "v90", "v91", "v92", "v93", "v94", "v95", "v96", "v97", "v98", "v99", "v100", "v101", "v102", "v103");
        static_assert(DEADLINE == 40000000u && NST == 6, "literals in the feeder loop");
        if (wave != 4) return;
        if (lane == 0) __hip_atomic_store(seqw, call, __ATOMIC_RELAXED, __HIP_MEMORY_SCOPE_AGENT);
        {
            const GAS u64* a2 = lane < 16 ? (const GAS u64*)(me + (lane < 8 ? A2_OFF : A3_OFF) + (size_t)(par * 8 + (bx & 7)) * 128) + (lane & 7) : (const GAS u64*)(me + TM_OFF);
            unsigned told = 0; Spin sp;
            for (;;) {
                const u64 v = __hip_atomic_load(a2, __ATOMIC_RELAXED, __HIP_MEMORY_SCOPE_SYSTEM);
                const bool in = lane < 16 && (unsigned)(v >> 32) == call;
                unsigned g = in ? (unsigned)v : 0u;
                g = max(g, (unsigned)__builtin_amdgcn_update_dpp(0u, g, 0x111, 0xf, 0xf, true));
                g = max(g, (unsigned)__builtin_amdgcn_update_dpp(0u, g, 0x112, 0xf, 0xf, true));
                g = max(g, (unsigned)__builtin_amdgcn_update_dpp(0u, g, 0x114, 0xf, 0xf, true));
                g = max(g, (unsigned)__builtin_amdgcn_update_dpp(0u, g, 0x118, 0xf, 0xf, true));
                g = __builtin_amdgcn_readlane(g, 15);
                const unsigned im = (unsigned)__builtin_amdgcn_ballot_w64(in);
                unsigned st = (im & 0xffu) == 0xffu ? 1u : 0u;
                if (((im | (im >> 8)) & 0xffu) != 0xffu) g = 0u;
                if ((unsigned)__builtin_amdgcn_readlane((unsigned)v, 16) == call || (st == 0u && sp.over<255u>(tmo))) st = 2u;
                if (st != 0u || g != told) { told = g; *(volatile LAS u32x2*)(lds + LC_OFF + C_GMAX * 4) = (u32x2){g, st}; }
                if (st != 0u) break;
                __builtin_amdgcn_s_sleep(1);
            }
        }
        STAMP(4);
    }
}

}

extern "C" void kernel_launch(int rank, int nranks, ncclComm_t comm,
                              hipStream_t stream,
                              void* const* d_in, const int* in_sizes, int n_in,
                              void* d_out, int out_size,
                              const int* in_dtypes, int out_dtype,
                              void* const* d_out_all,
                              void* const* d_ws_all, size_t ws_bytes) {
    if (nranks != NR || (unsigned)rank >= (unsigned)NR || ws_bytes < WS_NEED || n_in < 2) {
        fprintf(stderr, "unsupported configuration\n");
        return;
    }
    HIP_CHECK(hipFuncSetAttribute((const void*)ag_gemm, hipFuncAttributeMaxDynamicSharedMemorySize, LDS_BYTES));
    Params p;
    p.x = (const float*)d_in[0]; p.w = (const float*)d_in[1]; p.out = (signed char*)d_out;
    for (int i = 0; i < NR; ++i) p.peer[i] = (char*)d_ws_all[(rank + 1 + i) % NR];
    p.rank = rank; p.pad = 0;
    hipLaunchKernelGGL(ag_gemm, dim3(NWG), dim3(512), LDS_BYTES, stream, p);
    HIP_CHECK(hipGetLastError());
}
```

```cpp
#include <hip/hip_runtime.h>
#include <rccl/rccl.h>
#include <cstdio>

#define HIP_CHECK(expr) do { hipError_t _e = (expr); if (_e != hipSuccess) { \
    fprintf(stderr, "HIP error %s at %s:%d\n", hipGetErrorString(_e), __FILE__, __LINE__); \
    return; } } while (0)

namespace {

#define LAS __attribute__((address_space(3)))
#define GAS __attribute__((address_space(1)))
typedef unsigned u32x4 __attribute__((ext_vector_type(4)));
typedef unsigned u32x2 __attribute__((ext_vector_type(2)));
typedef float f32x4 __attribute__((ext_vector_type(4)));
typedef float f32x16 __attribute__((ext_vector_type(16)));
typedef float f32x32 __attribute__((ext_vector_type(32)));
typedef unsigned long long u64;

constexpr int NR = 8;
constexpr int KDIM = 4096, NL = 2048;
constexpr int KS = 64;
constexpr int NWG = 256;
constexpr int UNIT = 8192;
constexpr int EXCAP = 63;

constexpr size_t XS_OFF = 0;
constexpr size_t XS_SRC = (size_t)8 * KS * UNIT;
constexpr size_t WB_OFF = XS_OFF + NR * XS_SRC;
constexpr size_t XF_OFF = WB_OFF + (size_t)32 * KS * 8192;
constexpr size_t WF_OFF = XF_OFF + (size_t)NR * 8 * KS * 16;
constexpr size_t SQ_OFF = WF_OFF + (size_t)32 * KS * 64;
constexpr size_t A1_OFF = SQ_OFF + (size_t)NWG * 128;
constexpr size_t A2_OFF = A1_OFF + (size_t)2 * NWG * 64;
constexpr size_t A3_OFF = A2_OFF + (size_t)2 * 8 * 128;
constexpr size_t TK_OFF = A3_OFF + (size_t)2 * 8 * 128;
constexpr size_t TM_OFF = TK_OFF + 128;
constexpr size_t WS_NEED = TM_OFF + 128;

constexpr int NST = 6, LA_SLOT = 9216, LA_HALF = 3 * LA_SLOT, LA_PAR = 2 * LA_HALF;
constexpr int LC_OFF = 2 * LA_PAR;
constexpr int C_FULL = 0;
constexpr int C_DONE = 4;
constexpr int C_MAX = 8, C_CNT = 9, C_GMAX = 10  , C_SENT = 12, C_ISS = 13, C_SEQ = 14;
constexpr int LJ_OFF = LC_OFF + 64;
constexpr int LDS_BYTES = LJ_OFF + 256;
static_assert(LDS_BYTES > 81920, "one workgroup per CU");

constexpr int CPX = 3;
constexpr int PW = 6;
constexpr int PWIN = 3;
constexpr int SREL = 16, KREL = 10;
constexpr int PEARLY = 2;
constexpr unsigned WDELAY = 20, WDELAY2 = 340, WSTAG = 12;
constexpr int SPECN = 208;
constexpr int CODE_BYTES = 29440;
constexpr int NCOMM = 8 * CPX;
constexpr float XSCALE = 0.5f;
constexpr unsigned DEADLINE = 40000000u;

struct Params {
    const float* x; const float* w; signed char* out;
    char* peer[NR];
    int rank; int pad;
};

__device__ __forceinline__ unsigned rt32() { return (unsigned)__builtin_amdgcn_s_memrealtime(); }
__device__ __forceinline__ unsigned rfl(unsigned v) { return __builtin_amdgcn_readfirstlane(v); }
__device__ __forceinline__ unsigned cvt_pk(float lo, float hi) {
    typedef float f2 __attribute__((ext_vector_type(2))); typedef __bf16 b2 __attribute__((ext_vector_type(2)));
    f2 v = {lo, hi}; return __builtin_bit_cast(unsigned, __builtin_convertvector(v, b2));
}
__device__ __forceinline__ __amdgpu_buffer_rsrc_t mkrsrc(const void* p, unsigned bytes) {
    return __builtin_amdgcn_make_buffer_rsrc((void*)p, 0, bytes, 0x00020000);
}
__device__ __forceinline__ int fbits(float v) { return __float_as_int(v); }
__device__ __forceinline__ unsigned rotr(unsigned v, int n) { return __builtin_amdgcn_alignbit(v, v, n); }
#define VMWAIT0() asm volatile("s_waitcnt vmcnt(0)" ::: "memory")
#define LGKM0() asm volatile("s_waitcnt lgkmcnt(0)" ::: "memory")
#define SCHED() __builtin_amdgcn_sched_barrier(0)
constexpr int SYS = 17;

struct Tmo {
    GAS unsigned* word; unsigned call;
    __device__ __forceinline__ bool hit() const {
        unsigned v;
        asm volatile("v_mov_b32 %0, 0\n\tglobal_load_dword %0, %0, %1 sc1\n\ts_waitcnt vmcnt(0)" : "=&v"(v) : "s"(word) : "memory");
        return rfl(v) == call;
    }
    __device__ __forceinline__ bool expired(unsigned t0) const {
        if (hit() || rt32() - t0 > DEADLINE) {
            unsigned z, c;
            asm volatile("v_mov_b32 %0, 0\n\tv_mov_b32 %1, %3\n\tglobal_store_dword %0, %1, %2 sc1\n\ts_waitcnt vmcnt(0)" : "=&v"(z), "=&v"(c) : "s"(word), "s"(call) : "memory");
            return true;
        }
        return false;
    }
};

struct Spin {
    unsigned n = 0, t0 = 0;
    template <unsigned MASK> __device__ __forceinline__ bool over(const Tmo& tmo) {
        if ((++n & MASK) != 0u) return false;
        if (n == MASK + 1u) { t0 = rt32(); return false; }
        return tmo.expired(t0);
    }
};


__device__ __forceinline__ void w_load(f32x4 (&v)[16], const float* __restrict__ w, bool on, int ct, int t, int q, int lane) {
    const __amdgpu_buffer_rsrc_t rs = mkrsrc(w + (size_t)t * 64 * NL + ct * 256 + q * 64, on ? 63u * NL * 4u + 256u : 0u);
    const int n = lane & 15, g = lane >> 4;
    const unsigned vo = (unsigned)((g >> 1) * 16 + (g & 1) * 2) * NL * 4 + n * 16;
#pragma unroll
    for (int i = 0; i < 16; ++i) v[i] = __builtin_bit_cast(f32x4, __builtin_amdgcn_raw_buffer_load_b128(rs, vo, ((i >> 3) * 32 + ((i >> 1) & 3) * 4 + (i & 1)) * NL * 4, 2));
}
__device__ __forceinline__ void w_store(const f32x4 (&v)[16], char* me, int ct, int t, int q, int lane) {
    const __amdgpu_buffer_rsrc_t rs = mkrsrc(me + WB_OFF + ((size_t)(ct * 4 + q) * KS + t) * 8192, 8192);
#pragma unroll
    for (int kh = 0; kh < 2; ++kh)
#pragma unroll
        for (int c = 0; c < 4; ++c) {
            u32x4 o;
            o.x = cvt_pk(v[kh * 8 + 0][c], v[kh * 8 + 1][c]); o.y = cvt_pk(v[kh * 8 + 2][c], v[kh * 8 + 3][c]);
            o.z = cvt_pk(v[kh * 8 + 4][c], v[kh * 8 + 5][c]); o.w = cvt_pk(v[kh * 8 + 6][c], v[kh * 8 + 7][c]);
            __builtin_amdgcn_raw_buffer_store_b128(o, rs, lane * 16 + c * 1024 + kh * 4096, 0, SYS);
        }
}

__device__ __forceinline__ char* pick_peer(const Params& p, int q) {
    char* r = p.peer[0];
#pragma unroll
    for (int i = 1; i < NR; ++i) { char* c = p.peer[i]; asm volatile("" : "+s"(c)); r = q == i ? c : r; }
    return r;
}

__device__ __forceinline__ void touch(const void* ptr, unsigned junk) {
    unsigned keep;
    asm volatile("s_mov_b32 %0, m0\n\ts_mov_b32 m0, %2\n\ts_nop 0\n\tglobal_load_lds_dword %1, off\n\ts_mov_b32 m0, %0" : "=&s"(keep) : "v"(ptr), "s"(junk) : "memory");
}

__device__ __forceinline__ unsigned wave_max(unsigned g) {
    g = max(g, (unsigned)__builtin_amdgcn_update_dpp(0u, g, 0x111, 0xf, 0xf, true));
    g = max(g, (unsigned)__builtin_amdgcn_update_dpp(0u, g, 0x112, 0xf, 0xf, true));
    g = max(g, (unsigned)__builtin_amdgcn_update_dpp(0u, g, 0x114, 0xf, 0xf, true));
    g = max(g, (unsigned)__builtin_amdgcn_update_dpp(0u, g, 0x118, 0xf, 0xf, true));
    return max(max((unsigned)__builtin_amdgcn_readlane(g, 15), (unsigned)__builtin_amdgcn_readlane(g, 31)), max((unsigned)__builtin_amdgcn_readlane(g, 47), (unsigned)__builtin_amdgcn_readlane(g, 63)));
}

__global__ void __launch_bounds__(512, 2) ag_gemm(Params p) {
    unsigned ka0, ka1;
    asm volatile("s_load_dword %0, %2, 0x0\n\ts_load_dword %1, %2, 0x40" : "=&s"(ka0), "=&s"(ka1) : "s"(__builtin_amdgcn_kernarg_segment_ptr()));
    extern __shared__ __attribute__((aligned(16))) char lds_raw[];
    LAS char* lds = (LAS char*)lds_raw;
    volatile LAS unsigned* ctl = (volatile LAS unsigned*)(lds + LC_OFF);
    const int lane = threadIdx.x & 63, wave = rfl(threadIdx.x >> 6);
    const int bx = blockIdx.x, xcd = bx & 7, idx = bx >> 3;
    const unsigned junk = (unsigned)(size_t)(lds + LJ_OFF);
    if (wave == 7 && idx < 2) {
        static_assert(CODE_BYTES > 3 * 8192 && CODE_BYTES <= 4 * 8192, "four reads");
        unsigned keep, o;
        asm volatile("s_mov_b32 %0, m0\n\ts_mov_b32 m0, %4\n\tv_add_u32 %1, 8192, %2\n\tglobal_load_lds_dword %2, %3\n\tglobal_load_lds_dword %1, %3\n\t"
                     "v_add_u32 %1, 16384, %2\n\tglobal_load_lds_dword %1, %3\n\tv_add_u32 %1, 24576, %2\n\tv_min_u32 %1, %5, %1\n\tglobal_load_lds_dword %1, %3\n\ts_mov_b32 m0, %0"
                     : "=&s"(keep), "=&v"(o) : "v"(lane * 128), "s"(__builtin_amdgcn_s_getpc()), "s"(junk), "i"(CODE_BYTES) : "memory");
    }
    asm volatile("" :: "s"(p.x), "s"(p.w), "s"(p.out), "s"(p.rank), "s"(p.peer[0]), "s"(p.peer[1]), "s"(p.peer[2]), "s"(p.peer[3]),
                       "s"(p.peer[4]), "s"(p.peer[5]), "s"(p.peer[6]), "s"(p.peer[7]), "s"(ka0), "s"(ka1));
    const int rank = p.rank & 7;
    const int rel = idx & 7, src = (rank + 1 + rel) & 7, sub = xcd >> 1, ct = (xcd & 1) * 4 + (idx >> 3), rtile = src * 4 + sub;
    char* me = p.peer[NR - 1];
    const bool local = rel == NR - 1;
    const int jj = ((idx >> 3) + sub) & 3;
    const bool comm = local && jj < CPX;
    const int h = rel * 4 + sub;
    const bool host = h < 8 * CPX;
    const int xrow = (((h >> 3) - (ct & 3)) & 3) * 128 + (h & 7) * 16;
    constexpr int NCW = 32 - CPX;
    static_assert(PEARLY <= PW && PW <= 7, "roles");
    const int ci = local ? 28 + (jj - CPX) : h;
    const int cid = jj * 8 + xcd;
    GAS unsigned* seqw = (GAS unsigned*)(me + SQ_OFF + (size_t)bx * 128);
#define WG_START() do { if (threadIdx.x < 16) ctl[threadIdx.x] = (threadIdx.x == C_FULL + 2 || threadIdx.x == C_FULL + 3) ? 1u : 0u; \
        LGKM0(); __builtin_amdgcn_s_barrier(); } while (0)
#define GET_CALL() const unsigned call = rfl(__hip_atomic_load(seqw, __ATOMIC_RELAXED, __HIP_MEMORY_SCOPE_AGENT)) + 1u; \
    const Tmo tmo{(GAS unsigned*)(me + TM_OFF), call}; const unsigned par = call & 1u; (void)par

    if (__builtin_expect(comm, 1)) {
        if (__builtin_expect(wave < PW, 1)) {
            constexpr unsigned NHU = 16 * KS;
            f32x4 v[8];
            auto load_x = [&](int u) {
                const float* s0 = p.x + (size_t)((u & 7) * 64 + (lane >> 2)) * KDIM + (u >> 3) * 32 + (lane & 3) * 4;
#pragma unroll
                for (int e = 0; e < 8; ++e) v[e] = *(const GAS f32x4*)(s0 + (size_t)(e >> 1) * 16 * KDIM + (e & 1) * 16);
            };
            if (wave >= PEARLY) __builtin_amdgcn_s_sleep(8);
            load_x(wave * NCOMM + cid);
            SCHED();
            WG_START();
            GET_CALL();
            __amdgpu_buffer_rsrc_t rs[NR];
#pragma unroll
            for (int q = 0; q < NR; ++q) rs[q] = mkrsrc(p.peer[q] + XS_OFF + (size_t)rank * XS_SRC, (unsigned)XS_SRC);
            GAS u64* flag = (GAS u64*)(pick_peer(p, lane & 7) + XF_OFF) + rank * 8 * KS * 2;
            GAS unsigned* tk = (GAS unsigned*)(me + TK_OFF);
            LAS char* tail = lds + wave * 2048;
            const unsigned ptag = (unsigned)lane << 1, vo = lane * 16;
            unsigned u = wave * NCOMM + cid;
            int k = wave;
            while (u < NHU) {
                const unsigned fi = (u & 7u) * KS * 2 + (u >> 3), soff = fi * (UNIT / 2);
                u32x4 L[2], H[2], T;
                unsigned sum = 0;
#pragma unroll
                for (int e = 0; e < 8; ++e) {
                    const f32x4 s = v[e] * XSCALE;
                    const unsigned d0 = cvt_pk(s[0], s[1]), d1 = cvt_pk(s[2], s[3]);
                    L[e >> 2][e & 3] = __builtin_amdgcn_perm(d1, d0, 0x06040200u);
                    H[e >> 2][e & 3] = __builtin_amdgcn_perm(d1, d0, 0x07050301u);
                }
                SCHED();
                if (k < PEARLY) {
#pragma unroll
                    for (int j = 0; j < 2; ++j)
#pragma unroll
                        for (int q = 0; q < NR; ++q) __builtin_amdgcn_raw_buffer_store_b128(L[j], rs[q], vo + j * 1024, soff, SYS);
                    SCHED(); asm volatile("s_nop 1" ::: "memory"); SCHED();
                }
#pragma unroll
                for (int e = 0; e < 8; ++e) sum = __builtin_amdgcn_sad_u8(H[e >> 2][e & 3] & 0x7f7f7f7fu, 0u, sum);
                sum += __builtin_amdgcn_update_dpp(0u, sum, 0x111, 0xf, 0xf, true);
                sum += __builtin_amdgcn_update_dpp(0u, sum, 0x112, 0xf, 0xf, true);
                sum += __builtin_amdgcn_update_dpp(0u, sum, 0x114, 0xf, 0xf, true);
                sum += __builtin_amdgcn_update_dpp(0u, sum, 0x118, 0xf, 0xf, true);
                const int tot = __builtin_amdgcn_readlane(sum, 15) + __builtin_amdgcn_readlane(sum, 31) + __builtin_amdgcn_readlane(sum, 47) + __builtin_amdgcn_readlane(sum, 63);
                int bs = (tot - 2970) >> 11;
                bs = bs < 0 ? 0 : (bs > 124 ? 124 : bs);
                const unsigned b4 = (unsigned)bs * 0x01010101u, nb4 = (unsigned)((-bs) & 3) * 0x01010101u;
                unsigned cnt = 0, c[8], sg[8], x[8], rest = 0;
                auto emit = [&](int e) {
                    const u64 m = __builtin_amdgcn_ballot_w64(x[e] != 0u);
                    const unsigned sh = (unsigned)__builtin_ctz(x[e] | 0x80000000u) & 24u;
                    const unsigned pos = min(cnt, (unsigned)EXCAP + 1u) + __builtin_amdgcn_mbcnt_hi((unsigned)(m >> 32), __builtin_amdgcn_mbcnt_lo((unsigned)m, 0u));
                    const unsigned ent = (((x[e] >> sh) & 0x7cu) << 9) | ((unsigned)e << 8) | ((sh & 16u) << 3) | ptag | ((sh >> 3) & 1u);
                    *(LAS unsigned short*)(tail + (x[e] != 0u ? 768u + pos * 2 : 1024u + lane * 2)) = (unsigned short)ent;
                    x[e] &= ~(0xffu << sh);
                    cnt += (unsigned)__builtin_popcountll(m);
                };
#pragma unroll
                for (int e = 0; e < 8; ++e) {
                    const unsigned hh = H[e >> 2][e & 3], hm = hh & 0x7f7f7f7fu;
                    c[e] = (hm + nb4) & 0x03030303u;
                    sg[e] = hh & 0x80808080u;
                    x[e] = hm ^ (c[e] + b4);
                    emit(e);
                    rest |= x[e];
                }
                while (__builtin_amdgcn_ballot_w64(rest != 0u) != 0ull && cnt <= (unsigned)EXCAP) {
                    rest = 0;
#pragma unroll
                    for (int e = 0; e < 8; ++e) { if (__builtin_amdgcn_ballot_w64(x[e] != 0u) != 0ull) emit(e); rest |= x[e]; }
                }
                ((LAS unsigned*)tail)[lane * 3 + 0] = (c[0] | sg[0]) | rotr(c[1] | sg[1], 29) | (c[6] << 5);
                ((LAS unsigned*)tail)[lane * 3 + 1] = (c[2] | sg[2]) | rotr(c[3] | sg[3], 29) | (c[7] << 5);
                ((LAS unsigned*)tail)[lane * 3 + 2] = (c[4] | sg[4]) | rotr(c[5] | sg[5], 29) | (sg[6] >> 1) | (sg[7] >> 2);
                asm volatile("" ::: "memory");
                T = *(volatile LAS u32x4*)(tail + lane * 16);
                unsigned hdr = (unsigned)bs | (cnt << 8);
                if (cnt > (unsigned)EXCAP) { hdr = 0x80u; T = H[0]; }
                unsigned vt = ((hdr & 0x80u) || (unsigned)lane < 48u + ((cnt + 7u) >> 3)) ? vo + 2048 : 0x40000000u;
                asm volatile("" : "+v"(T), "+v"(vt));
                SCHED();
                if (k >= PEARLY) {
                    Spin sp;
                    for (;;) {
                        const u32x2 cc = *(volatile LAS u32x2*)(lds + LC_OFF + C_SENT * 4);
                        if ((int)rfl(cc.x) + PWIN > k && (int)rfl(cc.y) >= k) break;
                        __builtin_amdgcn_s_sleep(1);
                        if (sp.over<1023u>(tmo)) break;
                    }
                    asm volatile("" ::: "memory");
#pragma unroll
                    for (int j = 0; j < 2; ++j)
#pragma unroll
                        for (int q = 0; q < NR; ++q) __builtin_amdgcn_raw_buffer_store_b128(L[j], rs[q], vo + j * 1024, soff, SYS);
                }
#pragma unroll
                for (int q = 0; q < NR; ++q) __builtin_amdgcn_raw_buffer_store_b128(T, rs[q], vt, soff, SYS);
                if (hdr & 0x80u) {
#pragma unroll
                    for (int q = 0; q < NR; ++q) __builtin_amdgcn_raw_buffer_store_b128(H[1], rs[q], vo + 3072, soff, SYS);
                }
                SCHED(); asm volatile("s_nop 1" ::: "memory"); SCHED();
                if (k >= KREL) asm volatile("s_waitcnt vmcnt(%0)" :: "i"(SREL) : "memory");
                if (lane == 0) __hip_atomic_fetch_add((LAS unsigned*)(lds + LC_OFF) + C_ISS, 1u, __ATOMIC_RELAXED, __HIP_MEMORY_SCOPE_WORKGROUP);
                unsigned tv = 0;
                if (lane == 0) tv = __hip_atomic_fetch_add(tk, 1u, __ATOMIC_RELAXED, __HIP_MEMORY_SCOPE_AGENT);
                VMWAIT0();
                if (lane == 0) __hip_atomic_fetch_add((LAS unsigned*)(lds + LC_OFF) + C_SENT, 1u, __ATOMIC_RELAXED, __HIP_MEMORY_SCOPE_WORKGROUP);
                if (lane < NR) __hip_atomic_store(flag + fi, ((u64)hdr << 32) | call, __ATOMIC_RELAXED, __HIP_MEMORY_SCOPE_SYSTEM);
                u = PW * NCOMM + (rfl(tv) - (call - 1u) * NHU);
                if (u < NHU) {
                    load_x((int)u);
                    unsigned kq = 0;
                    if (lane == 0) kq = __hip_atomic_fetch_add((LAS unsigned*)(lds + LC_OFF) + C_SEQ, 1u, __ATOMIC_RELAXED, __HIP_MEMORY_SCOPE_WORKGROUP);
                    k = PW + (int)rfl(kq);
                }
            }
            if (lane == 0) __hip_atomic_fetch_add((LAS unsigned*)(lds + LC_OFF) + C_CNT, 1u, __ATOMIC_RELAXED, __HIP_MEMORY_SCOPE_WORKGROUP);
        } else {
            WG_START();
            if (wave != 7) return;
            GET_CALL();
            if (lane == 0) __hip_atomic_store((GAS u64*)(me + A1_OFF) + (par * NWG + bx) * 8, (u64)call << 32, __ATOMIC_RELAXED, __HIP_MEMORY_SCOPE_SYSTEM);
            { Spin sp; while (rfl(ctl[C_CNT]) < (unsigned)PW) { __builtin_amdgcn_s_sleep(8); if (sp.over<1023u>(tmo)) break; } }
            if (lane == 0) __hip_atomic_store(seqw, call, __ATOMIC_RELAXED, __HIP_MEMORY_SCOPE_AGENT);
            if (cid < 2) {
                const __amdgpu_buffer_rsrc_t rs = mkrsrc(me + A1_OFF + (size_t)par * NWG * 64, NWG * 64);
                GAS u64* dst = (GAS u64*)(pick_peer(p, lane >> 3) + (cid ? A3_OFF : A2_OFF) + (size_t)(par * 8 + (lane & 7)) * 128) + rank;
                unsigned g = 0, sent = 0, off[4]; Spin sp;
#pragma unroll
                for (int j = 0; j < 4; ++j) off[j] = lane * 64 + j * 4096;
                for (;;) {
                    u32x2 a[4];
#pragma unroll
                    for (int j = 0; j < 4; ++j) a[j] = __builtin_amdgcn_raw_buffer_load_b64(rs, off[j], 0, SYS);
                    int nin = 0;
#pragma unroll
                    for (int j = 0; j < 4; ++j) {
                        if (a[j].y == call) { g = max(g, a[j].x); off[j] = 0x40000000u; }
                        nin += __builtin_popcountll(__builtin_amdgcn_ballot_w64(off[j] == 0x40000000u));
                    }
                    if (nin == NWG) break;
                    if (cid && nin >= SPECN && __any(g > sent)) { sent = wave_max(g); __hip_atomic_store(dst, ((u64)call << 32) | sent, __ATOMIC_RELAXED, __HIP_MEMORY_SCOPE_SYSTEM); }
                    asm volatile("" ::: "memory");
                    if (sp.over<255u>(tmo)) break;
                }
                if (cid == 0) {
                    g = wave_max(g);
                    __hip_atomic_store(dst, ((u64)call << 32) | g, __ATOMIC_RELAXED, __HIP_MEMORY_SCOPE_SYSTEM);
                }
            }
        }
        return;
    }

    if (idx == 2 * CPX - 1 && wave < 4) {
        const char* pa = wave == 0 ? p.peer[0] : wave == 1 ? p.peer[2] : wave == 2 ? p.peer[4] : p.peer[6];
        const char* pb = wave == 0 ? p.peer[1] : wave == 1 ? p.peer[3] : wave == 2 ? p.peer[5] : p.peer[7];
        const int w = (lane >> 1) & 7;
        touch(((lane & 1) ? pb : pa) + (w < 4 ? XS_OFF + (size_t)rank * XS_SRC + (size_t)w * (XS_SRC / 4) : XF_OFF + (size_t)rank * 8 * KS * 16), junk);
    }
    if (wave == 3) {
        const size_t o1 = (size_t)rtile * 128 * NL, o2 = (size_t)(rank * 512 + xrow) * NL;
        touch(lane == 0 ? (const char*)p.out + o1 : lane == 1 ? (const char*)p.out + o2 : lane == 2 ? (const char*)(p.x + (size_t)xrow * KDIM) : me + XS_OFF + (size_t)rtile * (XS_SRC / 4), junk);
    }
    const unsigned tstart = rt32();
    WG_START();
    unsigned call, wfl = 0;
    {
        constexpr int NH = 8 * CPX, NN = NCW - NH, TH = KS - 4 * NN;
        static_assert(TH >= NH && TH <= 2 * NH, "at most one unit per wave of a host");
        const int t1 = host ? (wave < 4 ? NH + ci : ci) : TH + (ci - NH) * 4 + (wave >> 2) * 2, q = wave & 3;
        const bool one = !host || t1 < TH;
        while (rt32() - tstart < (host ? (wave < 4 ? WDELAY2 : WDELAY) + ci * WSTAG : WDELAY)) __builtin_amdgcn_s_sleep(2);
        f32x4 va[16], vb[16];
        w_load(va, p.w, one, ct, t1, q, lane);
        if (!host) w_load(vb, p.w, true, ct, t1 + 1, q, lane);
        const unsigned sv = __hip_atomic_load(seqw, __ATOMIC_RELAXED, __HIP_MEMORY_SCOPE_AGENT);
        SCHED();
        if (one) w_store(va, me, ct, t1, q, lane);
        if (!host) w_store(vb, me, ct, t1 + 1, q, lane);
        if (wave < 4 && lane < NH) wfl = __hip_atomic_load((GAS unsigned*)(me + WF_OFF) + ((ct * 4 + wave) * KS + lane) * 16, __ATOMIC_RELAXED, __HIP_MEMORY_SCOPE_SYSTEM);
        VMWAIT0();
        call = rfl(sv) + 1u;
        if (lane < (host ? (one ? 1 : 0) : 2)) __hip_atomic_store((GAS unsigned*)(me + WF_OFF) + ((ct * 4 + q) * KS + t1 + lane) * 16, call, __ATOMIC_RELAXED, __HIP_MEMORY_SCOPE_SYSTEM);
    }
    const Tmo tmo{(GAS unsigned*)(me + TM_OFF), call};
    const unsigned par = call & 1u;
    if (wave < 4) {
        const __amdgpu_buffer_rsrc_t rsw = mkrsrc(me + WB_OFF + (size_t)(ct * 4 + wave) * KS * 8192, KS * 8192);
        const __amdgpu_buffer_rsrc_t rsf = mkrsrc(me + WF_OFF + (size_t)(ct * 4 + wave) * KS * 64, KS * 64);
        const unsigned ldsb = (unsigned)(size_t)lds;
        unsigned vo = lane * 16;
        const unsigned polla = ldsb + LC_OFF + (C_FULL + (lane & 3)) * 4, donea = ldsb + LC_OFF + (C_DONE + wave) * 4;
        f32x32 r0, r1, r2, r3; f32x16 r4;
        static_assert(LA_SLOT == 9216 && LA_HALF == 27648 && LA_PAR == 55296, "literals in the K loop");
        int s_wk = __builtin_ctzll(__builtin_amdgcn_ballot_w64(wfl < call)) - 1;
        unsigned s_wt, s_ws, s_so, s_t, s_bo, s_bc, s_x, s_sp, s_t0; u64 s_c;
        asm volatile(R"ASM(
.set AG_A, 8
.set AG_C, 112
.macro AG_EXPIRED lout
  s_add_u32 %[sp], %[sp], 1
  s_and_b32 %[x], %[sp], 0x3ff
  s_cbranch_scc1 8f
  v_mov_b32 v6, 0
  global_load_dword v3, v6, %[tmo] sc1
  s_memrealtime %[c]
  s_waitcnt vmcnt(0) lgkmcnt(0)
  v_readfirstlane_b32 %[x], v3
  s_cmp_eq_u32 %[x], %[call]
  s_cbranch_scc1 \lout
  s_mov_b64 vcc, %[c]
  s_cmp_eq_u32 %[sp], 0x400
  s_cselect_b32 %[t0], vcc_lo, %[t0]
  s_sub_u32 %[x], vcc_lo, %[t0]
  s_cmp_le_u32 %[x], 40000000
  s_cbranch_scc1 8f
  v_mov_b32 v3, %[call]
  global_store_dword v6, v3, %[tmo] sc1
  s_waitcnt vmcnt(0)
  s_branch \lout
8:
.endm
; wait until w K-step (t + add + 1) is converted; wk = last K-step known to be converted with all before it
.macro AG_NEEDW add
  s_waitcnt lgkmcnt(0)
  s_mov_b32 %[sp], 0
1:
  v_lshlrev_b32 v6, 2, v0
  buffer_load_dword v3, v6, %[rsf], 0 offen sc0 sc1
  s_waitcnt vmcnt(0)
  v_cmp_gt_u32 vcc, %[call], v3
  s_ff1_i32_b64 %[wk], vcc
  s_min_u32 %[wk], %[wk], 1000
  s_sub_i32 %[wk], %[wk], 1
  s_add_u32 %[x], %[t], \add
  s_cmp_gt_i32 %[wk], %[x]
  s_cbranch_scc1 3f
  s_sleep 2
  AG_EXPIRED 2f
  s_branch 1b
2:
  s_movk_i32 %[wk], 1000
3:
  ds_read_b32 v3, v4
  s_add_u32 %[x], %[t], \add
.endm
; visited when wt <= x: one look at the w flags that costs no wait (requested here, read two steps later), else a real wait
.macro AG_WLOOK add, back
  s_cmp_le_i32 %[wk], %[x]
  s_cbranch_scc1 5f
  s_add_u32 %[ws], %[ws], 1
  s_cmp_eq_u32 %[ws], 1
  s_cbranch_scc0 4f
  v_lshlrev_b32 v1, 2, v0
  buffer_load_dword v1, v1, %[rsf], 0 offen sc0 sc1
  s_add_u32 %[wt], %[x], 2
  s_min_i32 %[wt], %[wt], %[wk]
  s_branch \back
4:
  v_cmp_gt_u32 vcc, %[call], v1
  s_ff1_i32_b64 %[wt], vcc
  s_min_u32 %[wt], %[wt], 1000
  s_sub_i32 %[wt], %[wt], 1
  s_max_i32 %[wk], %[wk], %[wt]
  s_mov_b32 %[wt], %[wk]
  s_branch \back
5:
  AG_NEEDW \add
  s_mov_b32 %[wt], %[wk]
  s_branch \back
.endm
; wait until x K-step (t + add) is in the ring, then read its first half
.macro AG_NEEDX add, va, nb
  s_mov_b32 %[sp], 0
1:
  ds_read_b32 v3, v4
  s_add_u32 %[x], %[t], \add
  s_waitcnt lgkmcnt(0)
  v_cmp_lt_u32 vcc, %[x], v3
  s_cmp_eq_u64 vcc, exec
  s_cbranch_scc1 2f
  s_sleep 1
  AG_EXPIRED 2f
  s_branch 1b
2:
  .set ag_b, 0
  .rept 8
  ds_read_b128 v[AG_A+4*ag_b:AG_A+4*ag_b+3], \va offset:(ag_b/4)*27648+(ag_b%%4)*1024
  .set ag_b, ag_b+1
  .endr
  .if \nb == 9
  ds_read_b128 v[AG_A+32:AG_A+35], \va offset:8192
  .endif
.endm
.macro AG_MF b, m, bc
  .set ag_n, \m ^ ((\b & 1) * 3)
  v_mfma_f32_16x16x32_bf16 v[AG_C+16*\b+4*ag_n:AG_C+16*\b+4*ag_n+3], v[\bc+4*ag_n:\bc+4*ag_n+3], v[AG_A+4*\b:AG_A+4*\b+3], v[AG_C+16*\b+4*ag_n:AG_C+16*\b+4*ag_n+3]
.endm
; half K-step on nb row blocks: first = 1 for the first half of step (t + add)
.macro AG_HALF nb, bc, bn, va, vnext, kh, first, add
  s_waitcnt vmcnt(8) lgkmcnt(\nb-3)
  .set ag_b, 0
  .rept \nb
  .if (ag_b == 3) || (ag_b == 6)
  s_waitcnt lgkmcnt(\nb-3)
  .endif
  AG_MF ag_b, 0, \bc
  .if ag_b == 4
  s_add_u32 %[bo], %[bo], 4096
  .endif
  .if \first && (ag_b == 5)
  s_add_u32 %[x], %[t], \add+1
  .endif
  .if \first && \add && (ag_b == 6)
  s_cmp_eq_u32 %[so], 27648
  .endif
  .if (!\first) && (ag_b == 6)
  v_cmp_lt_u32 vcc, %[x], v3
  .endif
  .if \first && \add && (ag_b == 7)
  v_add_u32 \vnext, %[so], v0
  .endif
  .if \first && (!\add) && (ag_b == 7)
  v_add_u32 \vnext, 55296, \va
  .endif
  AG_MF ag_b, 1, \bc
  .if ag_b < 4
  buffer_load_dwordx4 v[\bn+4*ag_b:\bn+4*ag_b+3], v0, %[rsw], %[bc] offen offset:ag_b*1024 sc0 sc1
  .endif
  .if ag_b == 4
  s_min_u32 %[bc], %[bo], 127*4096
  .endif
  .if (!\first) && (ag_b == 5)
  v_mov_b32 v6, %[x]
  .endif
  .if (!\first) && (ag_b == 6)
  ds_write_b32 v5, v6
  .endif
  .if \first && \add && (ag_b == 6)
  s_cselect_b32 %[so], 0, %[so]
  .endif
  .if \first && (ag_b == 7)
  s_cmp_le_i32 %[wt], %[x]
  .endif
  .if (!\first) && (ag_b == 7)
  s_cmp_eq_u64 vcc, exec
  .endif
  AG_MF ag_b, 2, \bc
  .if \first && \add && (ag_b == 5)
  s_add_u32 %[so], %[so], 9216
  .endif
  .if \first && (ag_b == \nb-1)
  ds_read_b32 v3, v4
  .endif
  AG_MF ag_b, 3, \bc
  .if ag_b < 8
  ds_read_b128 v[AG_A+4*ag_b:AG_A+4*ag_b+3], \va offset:(ag_b/4)*27648+\kh*4096+(ag_b%%4)*1024
  .else
  ds_read_b128 v[AG_A+32:AG_A+35], \va offset:8192+\kh*27648
  .endif
  .set ag_b, ag_b+1
  .endr
.endm
.macro AG_LOOP nb
  AG_NEEDX 0, v2, \nb
.Lag_loop\nb:
  AG_HALF \nb, 44, 92, v2, v7, 1, 1, 0
  s_cbranch_scc1 .Lag_w0\nb
.Lag_w0r\nb:
  AG_HALF \nb, 60, 44, v7, v7, 0, 0, 0
  s_cbranch_scc0 .Lag_x0\nb
.Lag_x0r\nb:
  AG_HALF \nb, 76, 60, v7, v2, 1, 1, 1
  s_cbranch_scc1 .Lag_w1\nb
.Lag_w1r\nb:
  AG_HALF \nb, 92, 76, v2, v2, 0, 0, 1
  s_cbranch_scc0 .Lag_x1\nb
.Lag_x1r\nb:
  s_add_u32 %[t], %[t], 2
  s_cmp_lt_u32 %[t], 64
  s_cbranch_scc1 .Lag_loop\nb
  s_branch .Lag_end
.Lag_w0\nb:
  AG_WLOOK 1, .Lag_w0r\nb
.Lag_w1\nb:
  AG_WLOOK 2, .Lag_w1r\nb
.Lag_x0\nb:
  AG_NEEDX 1, v7, \nb
  s_branch .Lag_x0r\nb
.Lag_x1\nb:
  AG_NEEDX 2, v2, \nb
  s_branch .Lag_x1r\nb
.endm
  s_nop 4
  s_mov_b32 %[t], 0
  s_mov_b32 %[so], 0
  s_cmp_gt_i32 %[wk], 0
  s_cbranch_scc1 .Lag_go
  AG_NEEDW 0
.Lag_go:
  s_mov_b32 %[ws], 0
  s_min_i32 %[wt], %[wk], 12
  .set ag_i, 0
  .rept 12
  s_movk_i32 %[bc], (ag_i/4)*4096
  buffer_load_dwordx4 v[44+4*ag_i:44+4*ag_i+3], v0, %[rsw], %[bc] offen offset:(ag_i%%4)*1024 sc0 sc1
  .set ag_i, ag_i+1
  .endr
  s_movk_i32 %[bo], 3*4096
  s_movk_i32 %[bc], 3*4096
  .set ag_i, 0
  .rept 72
  v_mov_b64 v[AG_C+ag_i:AG_C+ag_i+1], 0
  .set ag_i, ag_i+2
  .endr
  v_mov_b32 v2, v0
  s_cmp_eq_u32 %[host], 0
  s_cbranch_scc1 .Lag_eight
  AG_LOOP 9
.Lag_eight:
  AG_LOOP 8
.Lag_end:
  s_waitcnt vmcnt(0) lgkmcnt(0)
  s_nop 7
)ASM"
            : "={v[112:143]}"(r0), "={v[144:175]}"(r1), "={v[176:207]}"(r2), "={v[208:239]}"(r3), "={v[240:255]}"(r4),
              [t] "=&s"(s_t), [wk] "+s"(s_wk), [bo] "=&s"(s_bo), [bc] "=&s"(s_bc), [x] "=&s"(s_x), [sp] "=&s"(s_sp), [t0] "=&s"(s_t0),
              [wt] "=&s"(s_wt), [ws] "=&s"(s_ws), [so] "=&s"(s_so), [c] "=&s"(s_c), "+{v0}"(vo)
            : "{v4}"(polla), "{v5}"(donea), [rsw] "s"(rsw), [rsf] "s"(rsf), [tmo] "s"(tmo.word), [call] "s"(call), [host] "s"(rfl(host ? 1u : 0u))
            : "memory", "vcc", "scc", "v1", "v2", "v3", "v6", "v7", "v8", "v9", "v10", "v11", "v12", "v13", "v14", "v15", "v16", "v17", "v18", "v19", "v20", "v21", "v22", "v23", "v24", "v25", "v26", "v27", "v28", "v29", "v30", "v31", "v32", "v33", "v34", "v35", "v36", "v37", "v38", "v39", "v40", "v41", "v42", "v43", "v44", "v45", "v46", "v47", "v48", "v49", "v50", "v51", "v52", "v53", "v54", "v55", "v56", "v57", "v58", "v59", "v60", "v61", "v62", "v63", "v64", "v65", "v66", "v67", "v68", "v69", "v70", "v71", "v72", "v73", "v74", "v75", "v76", "v77", "v78", "v79", "v80", "v81", "v82", "v83", "v84", "v85", "v86", "v87", "v88", "v89", "v90", "v91", "v92", "v93", "v94", "v95", "v96", "v97", "v98", "v99", "v100", "v101", "v102", "v103", "v104", "v105", "v106", "v107");
        f32x4 acc[9][4];
#pragma unroll
        for (int i = 0; i < 144; ++i) acc[i >> 4][(i >> 2) & 3][i & 3] = i < 32 ? r0[i & 31] : i < 64 ? r1[i & 31] : i < 96 ? r2[i & 31] : i < 128 ? r3[i & 31] : r4[i & 15];
        const int ln = (int)(vo >> 4);
        int mx = 0;
#pragma unroll
        for (int n = 0; n < 4; ++n) acc[8][n] *= XSCALE;
        if (host) {
#pragma unroll
            for (int n = 0; n < 4; ++n) { mx = max(max(mx, fbits(acc[8][n][0])), fbits(acc[8][n][1])); mx = max(max(mx, fbits(acc[8][n][2])), fbits(acc[8][n][3])); }
        }
        int m2 = 0;
#pragma unroll
        for (int b = 0; b < 8; ++b)
#pragma unroll
            for (int n = 0; n < 4; ++n) { mx = max(max(mx, fbits(acc[b][n][0])), fbits(acc[b][n][1])); m2 = max(max(m2, fbits(acc[b][n][2])), fbits(acc[b][n][3])); }
        const unsigned wm = wave_max((unsigned)max(mx, m2));
        if (ln == 0) {
            __hip_atomic_fetch_max((LAS unsigned*)(lds + LC_OFF) + C_MAX, wm, __ATOMIC_RELAXED, __HIP_MEMORY_SCOPE_WORKGROUP);
            asm volatile("" ::: "memory");
            const unsigned arrived = __hip_atomic_fetch_add((LAS unsigned*)(lds + LC_OFF) + C_CNT, 1u, __ATOMIC_RELAXED, __HIP_MEMORY_SCOPE_WORKGROUP);
            asm volatile("" ::: "memory");
            const unsigned tm = ctl[C_MAX];
            if (arrived == 3u) __hip_atomic_store((GAS u64*)(me + A1_OFF) + (par * NWG + bx) * 8, ((u64)call << 32) | tm, __ATOMIC_RELAXED, __HIP_MEMORY_SCOPE_SYSTEM);
        }
        const __amdgpu_buffer_rsrc_t rso = mkrsrc(p.out, 8u * 512u * NL);
        const unsigned ovo = (unsigned)(ln & 15) * NL + ct * 256 + wave * 64 + (ln >> 4) * 16;
        unsigned used = 0xffffffffu; Spin sp;
#pragma unroll 1
        for (;;) {
            const u32x2 c = *(volatile LAS u32x2*)(lds + LC_OFF + C_GMAX * 4);
            unsigned st = rfl(c.y);
            const unsigned run = rfl(c.x), g = max(run, wm);
            if (st == 0u && sp.over<4095u>(tmo)) st = 2u;
            if (st == 2u || (g != used && (st != 0u || run != 0u))) {
                if (used != 0xffffffffu) VMWAIT0();
                used = g;
                const float scale = __uint_as_float(g) / 127.0f;
                const float inv = scale > 0.f ? 1.0f / scale : 0.f;
#pragma unroll
                for (int b = 0; b < 9; ++b) {
                    u32x4 o = {0u, 0u, 0u, 0u};
#pragma unroll
                    for (int cc = 0; cc < 4; ++cc) {
                        f32x4 q = acc[b][cc] * inv;
                        asm volatile("" : "+v"(q));
#pragma unroll
                        for (int r = 0; r < 4; ++r) o[r] = __builtin_amdgcn_cvt_pk_u8_f32(q[r], cc, o[r]);
                    }
                    if (st == 2u) o = (u32x4){0x80808080u, 0x80808080u, 0x80808080u, 0x80808080u};
                    if (b < 8 || host) __builtin_amdgcn_raw_buffer_store_b128(o, rso, ovo + (unsigned)(b < 8 ? rtile * 128 + b * 16 : rank * 512 + xrow) * NL, 0, SYS);
                }
            }
            if (st != 0u) break;
            __builtin_amdgcn_s_sleep(1);
        }
    } else {
        const int f = wave & 1, ph = (wave >> 1) & 1, rb = sub * 2 + f;
        const __amdgpu_buffer_rsrc_t rsx = mkrsrc(me + XS_OFF + (size_t)src * XS_SRC + (size_t)rb * KS * UNIT, KS * UNIT);
        const __amdgpu_buffer_rsrc_t rse = mkrsrc(p.x + (size_t)xrow * KDIM + f * 32, host ? 16u * KDIM * 4u - 128u : 0u);
        const __amdgpu_buffer_rsrc_t rsf = mkrsrc(me + XF_OFF + (size_t)(src * 8 + rb) * KS * 16, KS * 16);
        const unsigned ldsb = (unsigned)(size_t)lds;
        const unsigned mbase = ldsb + f * LA_HALF, vpub = ldsb + LC_OFF + (C_FULL + (wave & 3)) * 4, vdone = ldsb + LC_OFF + (C_DONE + (lane & 3)) * 4;
        const unsigned hostf = rfl(host ? 1u : 0u);
        unsigned s_cur = ph, s_lim = NST, s_so = 0, s_h, s_g, s_b, s_x, s_y, s_sp, s_t0, s_m0; u64 s_seen, s_c;
        asm volatile(R"ASM(
.macro FD_EXPIRED lout
  s_add_u32 %[sp], %[sp], 1
  s_and_b32 %[x], %[sp], 0xff
  s_cbranch_scc1 8f
  v_mov_b32 v9, 0
  global_load_dword v12, v9, %[tmo] sc1
  s_memrealtime %[c]
  s_waitcnt vmcnt(0) lgkmcnt(0)
  v_readfirstlane_b32 %[x], v12
  s_cmp_eq_u32 %[x], %[call]
  s_cbranch_scc1 \lout
  s_mov_b64 vcc, %[c]
  s_cmp_eq_u32 %[sp], 0x100
  s_cselect_b32 %[t0], vcc_lo, %[t0]
  s_sub_u32 %[x], vcc_lo, %[t0]
  s_cmp_le_u32 %[x], 40000000
  s_cbranch_scc1 8f
  v_mov_b32 v12, %[call]
  global_store_dword v9, v12, %[tmo] sc1
  s_waitcnt vmcnt(0)
  s_branch \lout
8:
.endm
.macro FD_FLAGS
  buffer_load_dwordx4 v[100:103], v0, %[rsf], 0 offen sc0 sc1
  s_waitcnt vmcnt(0)
  v_min_u32 v9, v100, v102
  v_cmp_le_u32 %[seen], %[call], v9
.endm
; loads of unit %[y] into the set at r
.macro FD_ISSUE r
  s_lshl_b32 %[x], %[y], 13
  s_lshl_b32 %[b], %[y], 8
  buffer_load_dwordx4 v[\r:\r+3], v0, %[rsx], %[x] offen sc0 sc1
  buffer_load_dwordx4 v[\r+4:\r+7], v0, %[rsx], %[x] offen offset:1024 sc0 sc1
  buffer_load_dwordx3 v[\r+16:\r+18], v1, %[rsx], %[x] offen offset:2048 sc0 sc1
  buffer_load_ushort v[\r+19], v2, %[rsx], %[x] offen offset:2816 sc0 sc1
  s_bitset1_b32 %[x], 12
  buffer_load_dwordx4 v[\r+24:\r+27], v3, %[rse], %[b] offen
  buffer_load_dwordx4 v[\r+28:\r+31], v3, %[rse], %[b] offen offset:64
  buffer_load_dwordx4 v[\r+8:\r+11], v0, %[rsx], %[x] offen sc0 sc1
  buffer_load_dwordx4 v[\r+12:\r+15], v0, %[rsx], %[x] offen offset:1024 sc0 sc1
  buffer_load_dwordx3 v[\r+20:\r+22], v1, %[rsx], %[x] offen offset:2048 sc0 sc1
  buffer_load_ushort v[\r+23], v2, %[rsx], %[x] offen offset:2816 sc0 sc1
.endm
; wait for the flag of unit %[y]
.macro FD_WAITFLAG
  s_mov_b32 %[sp], 0
1:
  FD_FLAGS
  s_bitcmp1_b64 %[seen], %[y]
  s_cbranch_scc1 2f
  s_sleep 1
  FD_EXPIRED 2f
  s_branch 1b
2:
.endm
.macro FD_DEC h, n0, n1, n2
  v_and_or_b32 v[\h], v[\n0], v7, %[b]
  v_alignbit_b32 v12, v[\n0], v[\n0], 3
  v_and_or_b32 v[\h+2], v[\n1], v7, %[b]
  v_alignbit_b32 v13, v[\n1], v[\n1], 3
  v_and_or_b32 v[\h+4], v[\n2], v7, %[b]
  v_alignbit_b32 v14, v[\n2], v[\n2], 3
  v_and_or_b32 v[\h+1], v12, v7, %[b]
  v_lshrrev_b32 v12, 5, v[\n0]
  v_and_or_b32 v[\h+3], v13, v7, %[b]
  v_lshrrev_b32 v13, 5, v[\n1]
  v_and_or_b32 v[\h+5], v14, v7, %[b]
  v_lshlrev_b32 v14, 1, v[\n2]
  v_bfi_b32 v12, v8, v14, v12
  v_lshlrev_b32 v15, 2, v[\n2]
  v_and_or_b32 v[\h+6], v12, v7, %[b]
  v_bfi_b32 v13, v8, v15, v13
  v_and_or_b32 v[\h+7], v13, v7, %[b]
.endm
; exception entry in register e -> address v12, mask v13
.macro FD_EXC e
  v_and_b32 v12, 0x7fe, v[\e]
  v_and_b32 v14, 1, v[\e]
  v_lshrrev_b32 v13, 11, v[\e]
  v_lshl_or_b32 v14, v14, 4, 10
  v_lshl_add_u32 v12, v12, 1, %[x]
  v_lshlrev_b32 v13, v14, v13
.endm
; a half sent raw, or with a window that does not start at a multiple of four
.macro FD_FIX hd, h
  s_bitcmp1_b32 \hd, 7
  s_cbranch_scc0 3f
  buffer_load_dwordx4 v[\h:\h+3], v0, %[rsx], %[x] offen offset:2048 sc0 sc1
  buffer_load_dwordx4 v[\h+4:\h+7], v0, %[rsx], %[x] offen offset:3072 sc0 sc1
  s_waitcnt vmcnt(0)
3:
  s_and_b32 %[b], \hd, 3
  s_mul_i32 %[b], %[b], 0x01010101
  .set fd_i, 0
  .rept 8
  v_add_u32 v[\h+fd_i], %[b], v[\h+fd_i]
  .set fd_i, fd_i+1
  .endr
.endm
; one unit: set r holds the loads of unit cur, set q takes the next one; ob = offset of this parity's ring
.macro FD_STEP r, q, ob, id
  s_add_u32 %[y], %[cur], 2
  s_cmp_ge_u32 %[y], 64
  s_cbranch_scc1 .Lfd_last\id
  s_bitcmp1_b64 %[seen], %[y]
  s_cbranch_scc0 .Lfd_look\id
  FD_ISSUE \q
  s_waitcnt vmcnt(10)
.Lfd_go\id:
  v_readlane_b32 %[h], v101, %[cur]
  v_readlane_b32 %[g], v103, %[cur]
  s_and_b32 %[b], %[h], 0x7c
  s_mul_i32 %[b], %[b], 0x01010101
  FD_DEC 80, \r+16, \r+17, \r+18
  s_and_b32 %[b], %[g], 0x7c
  s_mul_i32 %[b], %[b], 0x01010101
  s_add_u32 m0, %[mb], %[so]
  FD_DEC 88, \r+20, \r+21, \r+22
  s_or_b32 %[x], %[h], %[g]
  s_and_b32 %[x], %[x], 0x83
  s_cbranch_scc1 .Lfd_fix\id
.Lfd_fixed\id:
  s_cmp_ge_u32 %[cur], %[lim]
  s_cbranch_scc1 .Lfd_free\id
.Lfd_freed\id:
  .set fd_i, 0
  .rept 16
  v_perm_b32 v[96+(fd_i%%2)*2], v[80+fd_i], v[\r+fd_i], %[p0]
  v_perm_b32 v[97+(fd_i%%2)*2], v[80+fd_i], v[\r+fd_i], %[p1]
  ds_write_addtid_b32 v[96+(fd_i%%2)*2] offset:\ob+fd_i*512
  ds_write_addtid_b32 v[97+(fd_i%%2)*2] offset:\ob+fd_i*512+256
  .set fd_i, fd_i+1
  .endr
  s_add_u32 %[x], m0, \ob
  FD_EXC \r+19
  s_lshr_b32 %[b], %[h], 8
  s_bfm_b64 exec, %[b], 0
  s_lshr_b32 %[b], %[g], 8
  ds_xor_b32 v12, v13
  s_bfm_b64 exec, %[b], 0
  s_add_u32 %[so], %[so], 9216
  FD_EXC \r+23
  ds_xor_b32 v12, v13 offset:4096
  s_mov_b64 exec, -1
  s_cmp_eq_u32 %[host], 0
  s_cbranch_scc1 .Lfd_pub\id
  v_cvt_pk_bf16_f32 v96, v[\r+24], v[\r+25]
  v_cvt_pk_bf16_f32 v97, v[\r+26], v[\r+27]
  v_cvt_pk_bf16_f32 v98, v[\r+28], v[\r+29]
  v_cvt_pk_bf16_f32 v99, v[\r+30], v[\r+31]
  ds_write_addtid_b32 v96 offset:\ob+8192
  ds_write_addtid_b32 v97 offset:\ob+8192+256
  ds_write_addtid_b32 v98 offset:\ob+8192+512
  ds_write_addtid_b32 v99 offset:\ob+8192+768
.Lfd_pub\id:
  s_add_u32 %[cur], %[cur], 2
  s_cmp_eq_u32 %[so], 27648
  s_cselect_b32 %[so], 0, %[so]
  s_cmp_ge_u32 %[cur], 64
  s_cselect_b32 %[x], 1000, %[cur]
  v_mov_b32 v9, %[x]
  s_waitcnt lgkmcnt(0)
  ds_write_b32 v4, v9
  s_cbranch_scc1 .Lfd_end
  s_cmp_eq_u32 %[y], %[cur]
  s_cbranch_scc0 .Lfd_late\id
.Lfd_next\id:
.endm
; out-of-line parts of a step
.macro FD_SLOW r, q, ob, id
.Lfd_last\id:
  s_waitcnt vmcnt(0)
  s_branch .Lfd_go\id
.Lfd_look\id:
  FD_FLAGS
  s_bitcmp1_b64 %[seen], %[y]
  s_cbranch_scc0 3f
  FD_ISSUE \q
  s_branch .Lfd_go\id
3:
  s_mov_b32 %[y], -1
  s_branch .Lfd_go\id
.Lfd_late\id:
  s_mov_b32 %[y], %[cur]
  FD_WAITFLAG
  FD_ISSUE \q
  s_branch .Lfd_next\id
.Lfd_fix\id:
  s_lshl_b32 %[x], %[cur], 13
  FD_FIX %[h], 80
  s_bitset1_b32 %[x], 12
  FD_FIX %[g], 88
  s_branch .Lfd_fixed\id
.Lfd_free\id:
  s_mov_b32 %[sp], 0
1:
  ds_read_b32 v9, v5
  s_waitcnt lgkmcnt(0)
  v_readlane_b32 %[lim], v9, 0
  v_readlane_b32 %[x], v9, 1
  s_min_u32 %[lim], %[lim], %[x]
  v_readlane_b32 %[x], v9, 2
  s_min_u32 %[lim], %[lim], %[x]
  v_readlane_b32 %[x], v9, 3
  s_min_u32 %[lim], %[lim], %[x]
  s_add_u32 %[lim], %[lim], 6
  s_cmp_lt_u32 %[cur], %[lim]
  s_cbranch_scc1 .Lfd_freed\id
  s_sleep 4
  FD_EXPIRED .Lfd_freed\id
  s_branch 1b
.endm
  s_nop 4
  s_mov_b32 %[m0s], m0
  v_mov_b32 v7, 0x83838383
  v_mov_b32 v8, 0x80808080
  s_mov_b32 %[y], %[cur]
  FD_WAITFLAG
  FD_ISSUE 16
  s_bitcmp1_b32 %[cur], 0
  s_cbranch_scc1 .Lfd_loop1
.Lfd_loop0:
  FD_STEP 16, 48, 0, 0
  FD_STEP 48, 16, 0, 1
  s_branch .Lfd_loop0
.Lfd_loop1:
  FD_STEP 16, 48, 55296, 2
  FD_STEP 48, 16, 55296, 3
  s_branch .Lfd_loop1
  FD_SLOW 16, 48, 0, 0
  FD_SLOW 48, 16, 0, 1
  FD_SLOW 16, 48, 55296, 2
  FD_SLOW 48, 16, 55296, 3
.Lfd_end:
  s_mov_b32 m0, %[m0s]
  s_waitcnt vmcnt(0) lgkmcnt(0)
)ASM"
            : [cur] "+s"(s_cur), [lim] "+s"(s_lim), [so] "+s"(s_so), [h] "=&s"(s_h), [g] "=&s"(s_g), [b] "=&s"(s_b), [x] "=&s"(s_x), [y] "=&s"(s_y), [sp] "=&s"(s_sp),
              [t0] "=&s"(s_t0), [m0s] "=&s"(s_m0), [seen] "=&s"(s_seen), [c] "=&s"(s_c)
            : "{v0}"(lane * 16), "{v1}"(lane * 12), "{v2}"(lane * 2), "{v3}"((lane >> 2) * KDIM * 4 + (lane & 3) * 16), "{v4}"(vpub), "{v5}"(vdone),
              [rsx] "s"(rsx), [rse] "s"(rse), [rsf] "s"(rsf), [tmo] "s"(tmo.word), [call] "s"(call), [mb] "s"(mbase), [host] "s"(hostf),
              [p0] "s"(0x05010400u), [p1] "s"(0x07030602u)
            : "memory", "vcc", "scc", "v7", "v8", "v9", "v10", "v11", "v12", "v13", "v14", "v15", "v16", "v17", "v18", "v19", "v20", "v21", "v22", "v23", "v24", "v25", "v26", "v27", "v28", "v29", "v30", "v31", "v32", "v33", "v34", "v35", "v36", "v37", "v38", "v39", "v40", "v41", "v42", "v43", "v44", "v45", "v46", "v47", "v48", "v49", "v50", "v51", "v52", "v53", "v54", "v55", "v56", "v57", "v58", "v59", "v60", "v61", "v62", "v63", "v64", "v65", "v66", "v67", "v68", "v69", "v70", "v71", "v72", "v73", "v74", "v75", "v76", "v77", "v78", "v79", "v80", "v81", "v82", "v83", "v84", "v85", "v86", "v87", "v88", "v89", "v90", "v91", "v92", "v93", "v94", "v95", "v96", "v97", "v98", "v99", "v100", "v101", "v102", "v103");
        static_assert(DEADLINE == 40000000u && NST == 6, "literals in the feeder loop");
        if (wave != 4) return;
        if (lane == 0) __hip_atomic_store(seqw, call, __ATOMIC_RELAXED, __HIP_MEMORY_SCOPE_AGENT);
        {
            const GAS u64* a2 = lane < 16 ? (const GAS u64*)(me + (lane < 8 ? A2_OFF : A3_OFF) + (size_t)(par * 8 + (bx & 7)) * 128) + (lane & 7) : (const GAS u64*)(me + TM_OFF);
            unsigned told = 0; Spin sp;
            for (;;) {
                const u64 v = __hip_atomic_load(a2, __ATOMIC_RELAXED, __HIP_MEMORY_SCOPE_SYSTEM);
                const bool in = lane < 16 && (unsigned)(v >> 32) == call;
                unsigned g = in ? (unsigned)v : 0u;
                g = max(g, (unsigned)__builtin_amdgcn_update_dpp(0u, g, 0x111, 0xf, 0xf, true));
                g = max(g, (unsigned)__builtin_amdgcn_update_dpp(0u, g, 0x112, 0xf, 0xf, true));
                g = max(g, (unsigned)__builtin_amdgcn_update_dpp(0u, g, 0x114, 0xf, 0xf, true));
                g = max(g, (unsigned)__builtin_amdgcn_update_dpp(0u, g, 0x118, 0xf, 0xf, true));
                g = __builtin_amdgcn_readlane(g, 15);
                const unsigned im = (unsigned)__builtin_amdgcn_ballot_w64(in);
                unsigned st = (im & 0xffu) == 0xffu ? 1u : 0u;
                if (((im | (im >> 8)) & 0xffu) != 0xffu) g = 0u;
                if ((unsigned)__builtin_amdgcn_readlane((unsigned)v, 16) == call || (st == 0u && sp.over<255u>(tmo))) st = 2u;
                if (st != 0u || g != told) { told = g; *(volatile LAS u32x2*)(lds + LC_OFF + C_GMAX * 4) = (u32x2){g, st}; }
                if (st != 0u) break;
                __builtin_amdgcn_s_sleep(1);
            }
        }
    }
}

}

extern "C" void kernel_launch(int rank, int nranks, ncclComm_t comm,
                              hipStream_t stream,
                              void* const* d_in, const int* in_sizes, int n_in,
                              void* d_out, int out_size,
                              const int* in_dtypes, int out_dtype,
                              void* const* d_out_all,
                              void* const* d_ws_all, size_t ws_bytes) {
    if (nranks != NR || (unsigned)rank >= (unsigned)NR || ws_bytes < WS_NEED || n_in < 2) {
        fprintf(stderr, "unsupported configuration\n");
        return;
    }
    HIP_CHECK(hipFuncSetAttribute((const void*)ag_gemm, hipFuncAttributeMaxDynamicSharedMemorySize, LDS_BYTES));
    Params p;
    p.x = (const float*)d_in[0]; p.w = (const float*)d_in[1]; p.out = (signed char*)d_out;
        const size_t base = (ws_bytes - WS_NEED) & ~(size_t)0x1fffff;
    for (int i = 0; i < NR; ++i) p.peer[i] = (char*)d_ws_all[(rank + 1 + i) % NR] + base;
    p.rank = rank; p.pad = 0;
    hipLaunchKernelGGL(ag_gemm, dim3(NWG), dim3(512), LDS_BYTES, stream, p);
    HIP_CHECK(hipGetLastError());
}
```

```cpp
#include <hip/hip_runtime.h>
#include <rccl/rccl.h>
#include <cstdio>

#define HIP_CHECK(expr) do { hipError_t _e = (expr); if (_e != hipSuccess) { \
    fprintf(stderr, "HIP error %s at %s:%d\n", hipGetErrorString(_e), __FILE__, __LINE__); \
    return; } } while (0)

namespace {

#define LAS __attribute__((address_space(3)))
#define GAS __attribute__((address_space(1)))
typedef unsigned u32x4 __attribute__((ext_vector_type(4)));
typedef unsigned u32x2 __attribute__((ext_vector_type(2)));
typedef float f32x4 __attribute__((ext_vector_type(4)));
typedef float f32x16 __attribute__((ext_vector_type(16)));
typedef float f32x32 __attribute__((ext_vector_type(32)));
typedef unsigned long long u64;

constexpr int NR = 8;
constexpr int KDIM = 4096, NL = 2048;
constexpr int KS = 64;
constexpr int NWG = 256;
constexpr int UNIT = 8192;
constexpr int EXCAP = 63;

constexpr size_t XS_OFF = 0;
constexpr size_t XS_SRC = (size_t)8 * KS * UNIT;
constexpr size_t WB_OFF = XS_OFF + NR * XS_SRC;
constexpr size_t XF_OFF = WB_OFF + (size_t)32 * KS * 8192;
constexpr size_t WF_OFF = XF_OFF + (size_t)NR * 8 * KS * 16;
constexpr size_t SQ_OFF = WF_OFF + (size_t)32 * KS * 64;
constexpr size_t A1_OFF = SQ_OFF + (size_t)NWG * 128;
constexpr size_t A2_OFF = A1_OFF + (size_t)2 * NWG * 64;
constexpr size_t A3_OFF = A2_OFF + (size_t)2 * 8 * 128;
constexpr size_t TK_OFF = A3_OFF + (size_t)2 * 8 * 128;
constexpr size_t TM_OFF = TK_OFF + 128;
constexpr size_t DG_OFF = TM_OFF + 128;
constexpr size_t WS_NEED = DG_OFF + (size_t)NWG * 8 * 128 * 8;

constexpr int NST = 6, LA_SLOT = 9216, LA_HALF = 3 * LA_SLOT, LA_PAR = 2 * LA_HALF;
constexpr int LC_OFF = 2 * LA_PAR;
constexpr int C_FULL = 0;
constexpr int C_DONE = 4;
constexpr int C_MAX = 8, C_CNT = 9, C_GMAX = 10  , C_SENT = 12, C_ISS = 13, C_SEQ = 14;
constexpr int LJ_OFF = LC_OFF + 64;
constexpr int LDS_BYTES = LJ_OFF + 256;
static_assert(LDS_BYTES > 81920, "one workgroup per CU");

#define CPX 3
#define PW 6
#define PWIN 3
#define SREL 40
#define KREL 10
#define BIAS 13
#define NSEC 4
#define MDELAY 480
#define PEARLY 2
#define WDELAY 20
#define WDELAY2 340
#define WSTAG 12
#define SPECN 168
#define NWO 4
#define TWO 8
constexpr int CODE_BYTES = NWO ? 32000 : 30000;
#define XDG "0"
constexpr int NCOMM = 8 * CPX - NWO;
constexpr float XSCALE = 0.5f;
constexpr unsigned DEADLINE = 40000000u;

struct Params {
    const float* x; const float* w; signed char* out;
    char* peer[NR];
    int rank; int pad;
};

__device__ __forceinline__ u64 rt() { return __builtin_amdgcn_s_memrealtime(); }
__device__ __forceinline__ unsigned rt32() { return (unsigned)__builtin_amdgcn_s_memrealtime(); }
__device__ __forceinline__ unsigned rfl(unsigned v) { return __builtin_amdgcn_readfirstlane(v); }
__device__ __forceinline__ unsigned cvt_pk(float lo, float hi) {
    typedef float f2 __attribute__((ext_vector_type(2))); typedef __bf16 b2 __attribute__((ext_vector_type(2)));
    f2 v = {lo, hi}; return __builtin_bit_cast(unsigned, __builtin_convertvector(v, b2));
}
__device__ __forceinline__ __amdgpu_buffer_rsrc_t mkrsrc(const void* p, unsigned bytes) {
    return __builtin_amdgcn_make_buffer_rsrc((void*)p, 0, bytes, 0x00020000);
}
__device__ __forceinline__ int fbits(float v) { return __float_as_int(v); }
__device__ __forceinline__ unsigned rotr(unsigned v, int n) { return __builtin_amdgcn_alignbit(v, v, n); }
#define VMWAIT0() asm volatile("s_waitcnt vmcnt(0)" ::: "memory")
#define LGKM0() asm volatile("s_waitcnt lgkmcnt(0)" ::: "memory")
#define SCHED() __builtin_amdgcn_sched_barrier(0)
constexpr int SYS = 17;

struct Tmo {
    GAS unsigned* word; unsigned call;
    __device__ __forceinline__ bool hit() const {
        unsigned v;
        asm volatile("v_mov_b32 %0, 0\n\tglobal_load_dword %0, %0, %1 sc1\n\ts_waitcnt vmcnt(0)" : "=&v"(v) : "s"(word) : "memory");
        return rfl(v) == call;
    }
    __device__ __forceinline__ bool expired(unsigned t0) const {
        if (hit() || rt32() - t0 > DEADLINE) {
            unsigned z, c;
            asm volatile("v_mov_b32 %0, 0\n\tv_mov_b32 %1, %3\n\tglobal_store_dword %0, %1, %2 sc1\n\ts_waitcnt vmcnt(0)" : "=&v"(z), "=&v"(c) : "s"(word), "s"(call) : "memory");
            return true;
        }
        return false;
    }
};

struct Spin {
    unsigned n = 0, t0 = 0;
    template <unsigned MASK> __device__ __forceinline__ bool over(const Tmo& tmo) {
        if ((++n & MASK) != 0u) return false;
        if (n == MASK + 1u) { t0 = rt32(); return false; }
        return tmo.expired(t0);
    }
};

#define PSTAMP(r) do {} while (0)
#define STAMP(i) do {} while (0)

__device__ __forceinline__ void w_load(f32x4 (&v)[16], const float* __restrict__ w, bool on, int ct, int t, int q, int lane) {
    const __amdgpu_buffer_rsrc_t rs = mkrsrc(w + (size_t)t * 64 * NL + ct * 256 + q * 64, on ? 63u * NL * 4u + 256u : 0u);
    const int n = lane & 15, g = lane >> 4;
    const unsigned vo = (unsigned)((g >> 1) * 16 + (g & 1) * 2) * NL * 4 + n * 16;
#pragma unroll
    for (int i = 0; i < 16; ++i) v[i] = __builtin_bit_cast(f32x4, __builtin_amdgcn_raw_buffer_load_b128(rs, vo, ((i >> 3) * 32 + ((i >> 1) & 3) * 4 + (i & 1)) * NL * 4, 2));
}
__device__ __forceinline__ void w_store(const f32x4 (&v)[16], char* me, int ct, int t, int q, int lane) {
    const __amdgpu_buffer_rsrc_t rs = mkrsrc(me + WB_OFF + ((size_t)(ct * 4 + q) * KS + t) * 8192, 8192);
#pragma unroll
    for (int kh = 0; kh < 2; ++kh)
#pragma unroll
        for (int c = 0; c < 4; ++c) {
            u32x4 o;
            o.x = cvt_pk(v[kh * 8 + 0][c], v[kh * 8 + 1][c]); o.y = cvt_pk(v[kh * 8 + 2][c], v[kh * 8 + 3][c]);
            o.z = cvt_pk(v[kh * 8 + 4][c], v[kh * 8 + 5][c]); o.w = cvt_pk(v[kh * 8 + 6][c], v[kh * 8 + 7][c]);
            __builtin_amdgcn_raw_buffer_store_b128(o, rs, lane * 16 + c * 1024 + kh * 4096, 0, SYS);
        }
}

__device__ __forceinline__ char* pick_peer(const Params& p, int q) {
    char* r = p.peer[0];
#pragma unroll
    for (int i = 1; i < NR; ++i) { char* c = p.peer[i]; asm volatile("" : "+s"(c)); r = q == i ? c : r; }
    return r;
}

__device__ __forceinline__ void touch(const void* ptr, unsigned junk) {
    unsigned keep;
    asm volatile("s_mov_b32 %0, m0\n\ts_mov_b32 m0, %2\n\ts_nop 0\n\tglobal_load_lds_dword %1, off\n\ts_mov_b32 m0, %0" : "=&s"(keep) : "v"(ptr), "s"(junk) : "memory");
}

__device__ __forceinline__ unsigned wave_max(unsigned g) {
    g = max(g, (unsigned)__builtin_amdgcn_update_dpp(0u, g, 0x111, 0xf, 0xf, true));
    g = max(g, (unsigned)__builtin_amdgcn_update_dpp(0u, g, 0x112, 0xf, 0xf, true));
    g = max(g, (unsigned)__builtin_amdgcn_update_dpp(0u, g, 0x114, 0xf, 0xf, true));
    g = max(g, (unsigned)__builtin_amdgcn_update_dpp(0u, g, 0x118, 0xf, 0xf, true));
    return max(max((unsigned)__builtin_amdgcn_readlane(g, 15), (unsigned)__builtin_amdgcn_readlane(g, 31)), max((unsigned)__builtin_amdgcn_readlane(g, 47), (unsigned)__builtin_amdgcn_readlane(g, 63)));
}

__global__ void __launch_bounds__(512, 2) ag_gemm(Params p) {
    unsigned ka0, ka1;
    asm volatile("s_load_dword %0, %2, 0x0\n\ts_load_dword %1, %2, 0x40" : "=&s"(ka0), "=&s"(ka1) : "s"(__builtin_amdgcn_kernarg_segment_ptr()));
    extern __shared__ __attribute__((aligned(16))) char lds_raw[];
    LAS char* lds = (LAS char*)lds_raw;
    volatile LAS unsigned* ctl = (volatile LAS unsigned*)(lds + LC_OFF);
    const int lane = threadIdx.x & 63, wave = rfl(threadIdx.x >> 6);
    const int bx = blockIdx.x, xcd = bx & 7, idx = bx >> 3;
    const unsigned junk = (unsigned)(size_t)(lds + LJ_OFF);
    if (wave == 7 && idx < 2) {
        static_assert(CODE_BYTES > 3 * 8192 && CODE_BYTES <= 4 * 8192, "four reads");
        unsigned keep, o;
        asm volatile("s_mov_b32 %0, m0\n\ts_mov_b32 m0, %4\n\tv_add_u32 %1, 8192, %2\n\tglobal_load_lds_dword %2, %3\n\tglobal_load_lds_dword %1, %3\n\t"
                     "v_add_u32 %1, 16384, %2\n\tglobal_load_lds_dword %1, %3\n\tv_add_u32 %1, 24576, %2\n\tv_min_u32 %1, %5, %1\n\tglobal_load_lds_dword %1, %3\n\ts_mov_b32 m0, %0"
                     : "=&s"(keep), "=&v"(o) : "v"(lane * 128), "s"(__builtin_amdgcn_s_getpc()), "s"(junk), "i"(CODE_BYTES) : "memory");
    }
    asm volatile("" :: "s"(p.x), "s"(p.w), "s"(p.out), "s"(p.rank), "s"(p.peer[0]), "s"(p.peer[1]), "s"(p.peer[2]), "s"(p.peer[3]),
                       "s"(p.peer[4]), "s"(p.peer[5]), "s"(p.peer[6]), "s"(p.peer[7]), "s"(ka0), "s"(ka1));
    const int rank = p.rank & 7;
    const int rel = idx & 7, src = (rank + 1 + rel) & 7, sub = xcd & 3, cti = idx >> 3, ct = (xcd >> 2) * 4 + cti;
    char* me = p.peer[NR - 1];
    const bool local = rel == NR - 1, odd = xcd & 1;
    const int rsub = local ? 3 : sub, rtile = src * 4 + rsub;
    const bool comm = local && sub != (cti & 1) * 2;
    const bool wonly = comm && odd && cti == 3;
    const int cid = odd ? (cti == 3 ? 16 : cti * 4) + (xcd >> 1) : 12 + (cti >> 1) * 4 + (xcd >> 1);
    const int h = rel == 6 ? 24 + sub : (odd ? 0 : 12) + rel * 2 + (sub >> 1);
    const bool host = !local && h < 8 * CPX;
    const int xrow = (h & 31) * 16;
    constexpr int NCW = 32 - CPX;
    static_assert(PEARLY <= PW && PW <= 7 && CPX == 3 && NWO == 4, "roles");
    const int ci = local ? 28 : h;
    GAS unsigned* seqw = (GAS unsigned*)(me + SQ_OFF + (size_t)bx * 128);
#define WG_START() do { if (threadIdx.x < 16) ctl[threadIdx.x] = (threadIdx.x == C_FULL + 2 || threadIdx.x == C_FULL + 3) ? 1u : 0u; \
        LGKM0(); __builtin_amdgcn_s_barrier(); } while (0)
#define NOPUSH false
#define GET_CALL() const unsigned call = rfl(__hip_atomic_load(seqw, __ATOMIC_RELAXED, __HIP_MEMORY_SCOPE_AGENT)) + 1u; \
    const Tmo tmo{(GAS unsigned*)(me + TM_OFF), call}; const unsigned par = call & 1u; const bool nopush = NOPUSH; (void)par; (void)nopush
    STAMP(0);

    if (__builtin_expect(comm, 1)) {
        if (__builtin_expect(wave < PW && !wonly, 1)) {
            constexpr unsigned NHU = 16 * KS;
            f32x4 v[8];
            auto load_x = [&](int u) {
                const float* s0 = p.x + (size_t)((u & 7) * 64 + (lane >> 2)) * KDIM + (u >> 3) * 32 + (lane & 3) * 4;
#pragma unroll
                for (int e = 0; e < 8; ++e) v[e] = *(const GAS f32x4*)(s0 + (size_t)(e >> 1) * 16 * KDIM + (e & 1) * 16);
            };
            if (wave >= PEARLY) __builtin_amdgcn_s_sleep(8);
            load_x(wave * NCOMM + cid);
            SCHED();
            WG_START();
            GET_CALL();
            __amdgpu_buffer_rsrc_t rs[NR];
#pragma unroll
            for (int q = 0; q < NR; ++q) rs[q] = mkrsrc(p.peer[q] + XS_OFF + (size_t)rank * XS_SRC, (q == NR - 1 || !nopush) ? (unsigned)XS_SRC : 0u);
            GAS u64* flag = (GAS u64*)(pick_peer(p, lane & 7) + XF_OFF) + rank * 8 * KS * 2;
            GAS unsigned* tk = (GAS unsigned*)(me + TK_OFF);
            LAS char* tail = lds + wave * 2048;
            const unsigned ptag = (unsigned)lane << 1, vo = lane * 16;
            unsigned u = wave * NCOMM + cid;
            int k = wave;
            while (u < NHU) {
                const unsigned fi = (u & 7u) * KS * 2 + (u >> 3), soff = fi * (UNIT / 2);
                const bool own = (u & 7u) >= 6u;
                u32x4 L[2], H[2], T;
                unsigned sum = 0;
#pragma unroll
                for (int e = 0; e < 8; ++e) {
                    const f32x4 s = v[e] * XSCALE;
                    const unsigned d0 = cvt_pk(s[0], s[1]), d1 = cvt_pk(s[2], s[3]);
                    L[e >> 2][e & 3] = __builtin_amdgcn_perm(d1, d0, 0x06040200u);
                    H[e >> 2][e & 3] = __builtin_amdgcn_perm(d1, d0, 0x07050301u);
                }
                SCHED();
                PSTAMP(2);
                if (k < PEARLY) {
#pragma unroll
                    for (int j = 0; j < 2; ++j)
#pragma unroll
                        for (int q = 0; q < NR; ++q) if (q < NR - 1 || own) __builtin_amdgcn_raw_buffer_store_b128(L[j], rs[q], vo + j * 1024, soff, SYS);
                    SCHED(); asm volatile("s_nop 1" ::: "memory"); SCHED();
                }
#pragma unroll
                for (int e = 0; e < 8; ++e) sum = __builtin_amdgcn_sad_u8(H[e >> 2][e & 3] & 0x7f7f7f7fu, 0u, sum);
                sum += __builtin_amdgcn_update_dpp(0u, sum, 0x111, 0xf, 0xf, true);
                sum += __builtin_amdgcn_update_dpp(0u, sum, 0x112, 0xf, 0xf, true);
                sum += __builtin_amdgcn_update_dpp(0u, sum, 0x114, 0xf, 0xf, true);
                sum += __builtin_amdgcn_update_dpp(0u, sum, 0x118, 0xf, 0xf, true);
                const int tot = __builtin_amdgcn_readlane(sum, 15) + __builtin_amdgcn_readlane(sum, 31) + __builtin_amdgcn_readlane(sum, 47) + __builtin_amdgcn_readlane(sum, 63);
                int bs = (tot - 2970) >> 11;
                bs = bs < 0 ? 0 : (bs > 124 ? 124 : bs);
                const unsigned b4 = (unsigned)bs * 0x01010101u, nb4 = (unsigned)((-bs) & 3) * 0x01010101u;
                unsigned cnt = 0, c[8], sg[8], x[8], rest = 0;
                auto emit = [&](int e) {
                    const u64 m = __builtin_amdgcn_ballot_w64(x[e] != 0u);
                    const unsigned sh = (unsigned)__builtin_ctz(x[e] | 0x80000000u) & 24u;
                    const unsigned pos = min(cnt, (unsigned)EXCAP + 1u) + __builtin_amdgcn_mbcnt_hi((unsigned)(m >> 32), __builtin_amdgcn_mbcnt_lo((unsigned)m, 0u));
                    const unsigned ent = (((x[e] >> sh) & 0x7cu) << 9) | ((unsigned)e << 8) | ((sh & 16u) << 3) | ptag | ((sh >> 3) & 1u);
                    *(LAS unsigned short*)(tail + (x[e] != 0u ? 768u + pos * 2 : 1024u + lane * 2)) = (unsigned short)ent;
                    x[e] &= ~(0xffu << sh);
                    cnt += (unsigned)__builtin_popcountll(m);
                };
#pragma unroll
                for (int e = 0; e < 8; ++e) {
                    const unsigned hh = H[e >> 2][e & 3], hm = hh & 0x7f7f7f7fu;
                    c[e] = (hm + nb4) & 0x03030303u;
                    sg[e] = hh & 0x80808080u;
                    x[e] = hm ^ (c[e] + b4);
                    emit(e);
                    rest |= x[e];
                }
                while (__builtin_amdgcn_ballot_w64(rest != 0u) != 0ull && cnt <= (unsigned)EXCAP) {
                    rest = 0;
#pragma unroll
                    for (int e = 0; e < 8; ++e) { if (__builtin_amdgcn_ballot_w64(x[e] != 0u) != 0ull) emit(e); rest |= x[e]; }
                }
                ((LAS unsigned*)tail)[lane * 3 + 0] = (c[0] | sg[0]) | rotr(c[1] | sg[1], 29) | (c[6] << 5);
                ((LAS unsigned*)tail)[lane * 3 + 1] = (c[2] | sg[2]) | rotr(c[3] | sg[3], 29) | (c[7] << 5);
                ((LAS unsigned*)tail)[lane * 3 + 2] = (c[4] | sg[4]) | rotr(c[5] | sg[5], 29) | (sg[6] >> 1) | (sg[7] >> 2);
                asm volatile("" ::: "memory");
                T = *(volatile LAS u32x4*)(tail + lane * 16);
                unsigned hdr = (unsigned)bs | (cnt << 8);
                if (cnt > (unsigned)EXCAP) { hdr = 0x80u; T = H[0]; }
                unsigned vt = ((hdr & 0x80u) || (unsigned)lane < 48u + ((cnt + 7u) >> 3)) ? vo + 2048 : 0x40000000u;
                asm volatile("" : "+v"(T), "+v"(vt));
                PSTAMP(3);
                SCHED();
                if (k >= PEARLY) {
                    Spin sp;
                    for (;;) {
                        const u32x2 cc = *(volatile LAS u32x2*)(lds + LC_OFF + C_SENT * 4);
                        if ((int)rfl(cc.x) + PWIN > k && (int)rfl(cc.y) >= k) break;
                        __builtin_amdgcn_s_sleep(1);
                        if (sp.over<1023u>(tmo)) break;
                    }
                    asm volatile("" ::: "memory");
                    PSTAMP(4);
#pragma unroll
                    for (int j = 0; j < 2; ++j)
#pragma unroll
                        for (int q = 0; q < NR; ++q) if (q < NR - 1 || own) __builtin_amdgcn_raw_buffer_store_b128(L[j], rs[q], vo + j * 1024, soff, SYS);
                }
#pragma unroll
                for (int q = 0; q < NR; ++q) if (q < NR - 1 || own) __builtin_amdgcn_raw_buffer_store_b128(T, rs[q], vt, soff, SYS);
                if (hdr & 0x80u) {
#pragma unroll
                    for (int q = 0; q < NR; ++q) if (q < NR - 1 || own) __builtin_amdgcn_raw_buffer_store_b128(H[1], rs[q], vo + 3072, soff, SYS);
                }
                SCHED(); asm volatile("s_nop 1" ::: "memory"); SCHED();
                if (k >= KREL) asm volatile("s_waitcnt vmcnt(%0)" :: "i"(SREL) : "memory");
                if (lane == 0) __hip_atomic_fetch_add((LAS unsigned*)(lds + LC_OFF) + C_ISS, 1u, __ATOMIC_RELAXED, __HIP_MEMORY_SCOPE_WORKGROUP);
                PSTAMP(5);
                unsigned tv = 0;
                if (lane == 0) tv = __hip_atomic_fetch_add(tk, 1u, __ATOMIC_RELAXED, __HIP_MEMORY_SCOPE_AGENT);
                VMWAIT0();
                PSTAMP(6);
                if (lane == 0) __hip_atomic_fetch_add((LAS unsigned*)(lds + LC_OFF) + C_SENT, 1u, __ATOMIC_RELAXED, __HIP_MEMORY_SCOPE_WORKGROUP);
                if ((lane == NR - 1 && own) || (lane < NR - 1 && !nopush)) __hip_atomic_store(flag + fi, ((u64)hdr << 32) | call, __ATOMIC_RELAXED, __HIP_MEMORY_SCOPE_SYSTEM);
                PSTAMP(7);
                u = PW * NCOMM + (rfl(tv) - (call - 1u) * NHU);
                if (u < NHU) {
                    load_x((int)u);
                    unsigned kq = 0;
                    if (lane == 0) kq = __hip_atomic_fetch_add((LAS unsigned*)(lds + LC_OFF) + C_SEQ, 1u, __ATOMIC_RELAXED, __HIP_MEMORY_SCOPE_WORKGROUP);
                    k = PW + (int)rfl(kq);
                }
            }
            if (lane == 0) __hip_atomic_fetch_add((LAS unsigned*)(lds + LC_OFF) + C_CNT, 1u, __ATOMIC_RELAXED, __HIP_MEMORY_SCOPE_WORKGROUP);
        } else {
            WG_START();
            if (wave != 7 && !wonly) return;
            GET_CALL();
            if (wonly) {
#pragma unroll 1
                for (int e = ((cid - 16) * 8 + wave) * 2; e < TWO * 32; e += (NWO ? NWO : 1) * 16) {
                    f32x4 va[16], vb[16];
                    const int t = KS - TWO + (e >> 5), c2 = (e >> 2) & 7, q = e & 3;
                    w_load(va, p.w, true, c2, t, q, lane); w_load(vb, p.w, true, c2, t, q + 1, lane);
                    SCHED();
                    w_store(va, me, c2, t, q, lane); w_store(vb, me, c2, t, q + 1, lane);
                    VMWAIT0();
                    if (lane < 2) __hip_atomic_store((GAS unsigned*)(me + WF_OFF) + ((c2 * 4 + q + lane) * KS + t) * 16, call, __ATOMIC_RELAXED, __HIP_MEMORY_SCOPE_SYSTEM);
                }
                STAMP(7);
                if (wave != 7) { if (lane == 0) __hip_atomic_fetch_add((LAS unsigned*)(lds + LC_OFF) + C_CNT, 1u, __ATOMIC_RELAXED, __HIP_MEMORY_SCOPE_WORKGROUP); return; }
            }
            if (lane == 0) __hip_atomic_store((GAS u64*)(me + A1_OFF) + (par * NWG + bx) * 8, (u64)call << 32, __ATOMIC_RELAXED, __HIP_MEMORY_SCOPE_SYSTEM);
            { Spin sp; while (rfl(ctl[C_CNT]) < (wonly ? 7u : (unsigned)PW)) { __builtin_amdgcn_s_sleep(8); if (sp.over<1023u>(tmo)) break; } }
            if (lane == 0) __hip_atomic_store(seqw, call, __ATOMIC_RELAXED, __HIP_MEMORY_SCOPE_AGENT);
            if (cid < 2) {
                const __amdgpu_buffer_rsrc_t rs = mkrsrc(me + A1_OFF + (size_t)par * NWG * 64, NWG * 64);
                GAS u64* dst = (GAS u64*)(pick_peer(p, lane >> 3) + (cid ? A3_OFF : A2_OFF) + (size_t)(par * 8 + (lane & 7)) * 128) + rank;
                unsigned g = 0, sent = 0, off[4]; Spin sp;
#pragma unroll
                for (int j = 0; j < 4; ++j) off[j] = lane * 64 + j * 4096;
                for (;;) {
                    u32x2 a[4];
#pragma unroll
                    for (int j = 0; j < 4; ++j) a[j] = __builtin_amdgcn_raw_buffer_load_b64(rs, off[j], 0, SYS);
                    int nin = 0;
#pragma unroll
                    for (int j = 0; j < 4; ++j) {
                        if (a[j].y == call) { g = max(g, a[j].x); off[j] = 0x40000000u; }
                        nin += __builtin_popcountll(__builtin_amdgcn_ballot_w64(off[j] == 0x40000000u));
                    }
                    if (nin == NWG) break;
                    if (cid && nin >= SPECN && __any(g > sent)) { sent = wave_max(g); __hip_atomic_store(dst, ((u64)call << 32) | sent, __ATOMIC_RELAXED, __HIP_MEMORY_SCOPE_SYSTEM); }
                    asm volatile("" ::: "memory");
                    if (sp.over<255u>(tmo)) break;
                }
                if (cid == 0) {
                    g = wave_max(g);
                    __hip_atomic_store(dst, ((u64)call << 32) | g, __ATOMIC_RELAXED, __HIP_MEMORY_SCOPE_SYSTEM);
                    STAMP(2);
                }
            }
        }
        return;
    }

    if (idx == ((BIAS & 2) ? 4 : 5) && wave < 4) {
        const char* pa = wave == 0 ? p.peer[0] : wave == 1 ? p.peer[2] : wave == 2 ? p.peer[4] : p.peer[6];
        const char* pb = wave == 0 ? p.peer[1] : wave == 1 ? p.peer[3] : wave == 2 ? p.peer[5] : p.peer[7];
        const int w = (lane >> 1) & 7;
        touch(((lane & 1) ? pb : pa) + (w < 4 ? XS_OFF + (size_t)rank * XS_SRC + (size_t)w * (XS_SRC / 4) : XF_OFF + (size_t)rank * 8 * KS * 16), junk);
    }
    if (wave == 3) {
        const size_t o1 = (size_t)rtile * 128 * NL, o2 = (size_t)(rank * 512 + xrow) * NL;
        touch(lane == 0 ? (const char*)p.out + o1 : lane == 1 ? (const char*)p.out + o2 : lane == 2 ? (const char*)(p.x + (size_t)xrow * KDIM) : me + XS_OFF + (size_t)rtile * (XS_SRC / 4), junk);
    }
    const unsigned tstart = rt32();
    WG_START();
    unsigned call, wfl = 0;
    {
        constexpr int NH = 8 * CPX, NN = NCW - NH, TH = KS - NSEC * NN - TWO;
        static_assert(TH >= NH && TH <= 2 * NH, "at most one unit per wave of a host");
        const int cm = (BIAS & 6) ? (ci + TH - NH) % NH : ci;
        const bool two = !host && (NSEC == 4 || wave < 4);
        const int cf = (BIAS & 8) ? rel * 4 + sub : (BIAS & 16) ? (ci + 12) % NH : ci;
        const int t1 = host ? (wave < 4 ? NH + cm : cf) : TH + (ci - NH) * NSEC + (wave >> 2) * 2, q = wave & 3;
        const bool one = !host || t1 < TH;
        while (rt32() - tstart < (host ? (wave < 4 ? (one ? WDELAY2 + cm * WSTAG : MDELAY) : WDELAY + cf * WSTAG) : WDELAY)) __builtin_amdgcn_s_sleep(2);
        f32x4 va[16], vb[16];
        w_load(va, p.w, one, ct, t1, q, lane);
        if (two) w_load(vb, p.w, true, ct, t1 + 1, q, lane);
        const unsigned sv = __hip_atomic_load(seqw, __ATOMIC_RELAXED, __HIP_MEMORY_SCOPE_AGENT);
        SCHED();
        if (one) w_store(va, me, ct, t1, q, lane);
        if (two) w_store(vb, me, ct, t1 + 1, q, lane);
        if (wave < 4 && lane < NH) wfl = __hip_atomic_load((GAS unsigned*)(me + WF_OFF) + ((ct * 4 + wave) * KS + lane) * 16, __ATOMIC_RELAXED, __HIP_MEMORY_SCOPE_SYSTEM);
        VMWAIT0();
        call = rfl(sv) + 1u;
        if (lane < (two ? 2 : one ? 1 : 0)) __hip_atomic_store((GAS unsigned*)(me + WF_OFF) + ((ct * 4 + q) * KS + t1 + lane) * 16, call, __ATOMIC_RELAXED, __HIP_MEMORY_SCOPE_SYSTEM);
    }
    const Tmo tmo{(GAS unsigned*)(me + TM_OFF), call};
    const unsigned par = call & 1u;
    const bool nopush = NOPUSH;
    if (wave < 4) {
        STAMP(1);
        const __amdgpu_buffer_rsrc_t rsw = mkrsrc(me + WB_OFF + (size_t)(ct * 4 + wave) * KS * 8192, KS * 8192);
        const __amdgpu_buffer_rsrc_t rsf = mkrsrc(me + WF_OFF + (size_t)(ct * 4 + wave) * KS * 64, KS * 64);
        const unsigned ldsb = (unsigned)(size_t)lds;
        unsigned vo = lane * 16;
        const unsigned polla = ldsb + LC_OFF + (C_FULL + (lane & 3)) * 4, donea = ldsb + LC_OFF + (C_DONE + wave) * 4;
        f32x32 r0, r1, r2, r3; f32x16 r4;
        static_assert(LA_SLOT == 9216 && LA_HALF == 27648 && LA_PAR == 55296, "literals in the K loop");
        int s_wk = __builtin_ctzll(__builtin_amdgcn_ballot_w64(wfl < call)) - 1;
        unsigned s_wt, s_ws, s_so, s_t, s_bo, s_bc, s_x, s_sp, s_t0, s_late; u64 s_c, s_c0, s_r0; unsigned s_wx, s_ww;
        asm volatile(R"ASM(
.set AG_A, 8
.set AG_C, 112
.set AG_DIAG, )ASM" XDG R"ASM(
.macro AG_EXPIRED lout
  s_add_u32 %[sp], %[sp], 1
  s_and_b32 %[x], %[sp], 0x3ff
  s_cbranch_scc1 8f
  v_mov_b32 v6, 0
  global_load_dword v3, v6, %[tmo] sc1
  s_memrealtime %[c]
  s_waitcnt vmcnt(0) lgkmcnt(0)
  v_readfirstlane_b32 %[x], v3
  s_cmp_eq_u32 %[x], %[call]
  s_cbranch_scc1 \lout
  s_mov_b64 vcc, %[c]
  s_cmp_eq_u32 %[sp], 0x400
  s_cselect_b32 %[t0], vcc_lo, %[t0]
  s_sub_u32 %[x], vcc_lo, %[t0]
  s_cmp_le_u32 %[x], 40000000
  s_cbranch_scc1 8f
  v_mov_b32 v3, %[call]
  global_store_dword v6, v3, %[tmo] sc1
  s_waitcnt vmcnt(0)
  s_branch \lout
8:
.endm
.macro AG_TIC
  s_mov_b32 %[sp], 0
  .if AG_DIAG
  s_memrealtime %[c]
  s_waitcnt lgkmcnt(0)
  s_mov_b64 vcc, %[c]
  s_mov_b32 %[t0], vcc_lo
  .endif
.endm
.macro AG_TOC acc
  .if AG_DIAG
  s_memrealtime %[c]
  s_waitcnt lgkmcnt(0)
  s_mov_b64 vcc, %[c]
  s_sub_u32 %[x], vcc_lo, %[t0]
  s_add_u32 \acc, \acc, %[x]
  .endif
.endm
; wait until w K-step (t + add + 1) is converted; wk = last K-step known to be converted with all before it
.macro AG_NEEDW add
  s_waitcnt lgkmcnt(0)
  AG_TIC
1:
  v_lshlrev_b32 v6, 2, v0
  buffer_load_dword v3, v6, %[rsf], 0 offen sc0 sc1
  s_waitcnt vmcnt(0)
  v_cmp_gt_u32 vcc, %[call], v3
  s_ff1_i32_b64 %[wk], vcc
  s_min_u32 %[wk], %[wk], 1000
  s_sub_i32 %[wk], %[wk], 1
  s_add_u32 %[x], %[t], \add
  s_cmp_gt_i32 %[wk], %[x]
  s_cbranch_scc1 3f
  s_sleep 2
  AG_EXPIRED 2f
  s_branch 1b
2:
  s_movk_i32 %[wk], 1000
3:
  AG_TOC %[ww]
  ds_read_b32 v3, v4
  s_add_u32 %[x], %[t], \add
.endm
; visited when wt <= x: one look at the w flags that costs no wait (requested here, read two steps later), else a real wait
.macro AG_WLOOK add, back
  s_cmp_le_i32 %[wk], %[x]
  s_cbranch_scc1 5f
  s_add_u32 %[ws], %[ws], 1
  s_cmp_eq_u32 %[ws], 1
  s_cbranch_scc0 4f
  v_lshlrev_b32 v1, 2, v0
  buffer_load_dword v1, v1, %[rsf], 0 offen sc0 sc1
  s_add_u32 %[wt], %[x], 2
  s_min_i32 %[wt], %[wt], %[wk]
  s_branch \back
4:
  v_cmp_gt_u32 vcc, %[call], v1
  s_ff1_i32_b64 %[wt], vcc
  s_min_u32 %[wt], %[wt], 1000
  s_sub_i32 %[wt], %[wt], 1
  s_max_i32 %[wk], %[wk], %[wt]
  s_mov_b32 %[ws], 0
  s_sub_i32 %[wt], %[wk], 3
  s_branch \back
5:
  AG_NEEDW \add
  s_mov_b32 %[ws], 0
  s_sub_i32 %[wt], %[wk], 3
  s_branch \back
.endm
; wait until x K-step (t + add) is in the ring, then read its first half
.macro AG_NEEDX add, va, nb
  AG_TIC
  s_add_u32 %[late], %[late], 1
1:
  ds_read_b32 v3, v4
  s_add_u32 %[x], %[t], \add
  s_waitcnt lgkmcnt(0)
  v_cmp_lt_u32 vcc, %[x], v3
  s_cmp_eq_u64 vcc, exec
  s_cbranch_scc1 2f
  s_sleep 1
  AG_EXPIRED 2f
  s_branch 1b
2:
  AG_TOC %[wx]
  .set ag_b, 0
  .rept 8
  ds_read_b128 v[AG_A+4*ag_b:AG_A+4*ag_b+3], \va offset:(ag_b/4)*27648+(ag_b%%4)*1024
  .set ag_b, ag_b+1
  .endr
  .if \nb == 9
  ds_read_b128 v[AG_A+32:AG_A+35], \va offset:8192
  .endif
.endm
.macro AG_MF b, m, bc
  .set ag_n, \m ^ ((\b & 1) * 3)
  v_mfma_f32_16x16x32_bf16 v[AG_C+16*\b+4*ag_n:AG_C+16*\b+4*ag_n+3], v[\bc+4*ag_n:\bc+4*ag_n+3], v[AG_A+4*\b:AG_A+4*\b+3], v[AG_C+16*\b+4*ag_n:AG_C+16*\b+4*ag_n+3]
.endm
; half K-step on nb row blocks: first = 1 for the first half of step (t + add)
.macro AG_HALF nb, bc, bn, va, vnext, kh, first, add
  s_waitcnt vmcnt(8) lgkmcnt(\nb-3)
  .set ag_b, 0
  .rept \nb
  .if (ag_b == 3) || (ag_b == 6)
  s_waitcnt lgkmcnt(\nb-3)
  .endif
  AG_MF ag_b, 0, \bc
  .if ag_b == 4
  s_add_u32 %[bo], %[bo], 4096
  .endif
  .if \first && (ag_b == 5)
  s_add_u32 %[x], %[t], \add+1
  .endif
  .if \first && \add && (ag_b == 6)
  s_cmp_eq_u32 %[so], 27648
  .endif
  .if (!\first) && (ag_b == 6)
  v_cmp_lt_u32 vcc, %[x], v3
  .endif
  .if \first && \add && (ag_b == 7)
  v_add_u32 \vnext, %[so], v0
  .endif
  .if \first && (!\add) && (ag_b == 7)
  v_add_u32 \vnext, 55296, \va
  .endif
  AG_MF ag_b, 1, \bc
  .if ag_b < 4
  buffer_load_dwordx4 v[\bn+4*ag_b:\bn+4*ag_b+3], v0, %[rsw], %[bc] offen offset:ag_b*1024 sc0 sc1
  .endif
  .if ag_b == 4
  s_min_u32 %[bc], %[bo], 127*4096
  .endif
  .if (!\first) && (ag_b == 5)
  v_mov_b32 v6, %[x]
  .endif
  .if (!\first) && (ag_b == 6)
  ds_write_b32 v5, v6
  .endif
  .if \first && \add && (ag_b == 6)
  s_cselect_b32 %[so], 0, %[so]
  .endif
  .if \first && (ag_b == 7)
  s_cmp_le_i32 %[wt], %[x]
  .endif
  .if (!\first) && (ag_b == 7)
  s_cmp_eq_u64 vcc, exec
  .endif
  AG_MF ag_b, 2, \bc
  .if \first && \add && (ag_b == 5)
  s_add_u32 %[so], %[so], 9216
  .endif
  .if \first && (ag_b == \nb-1)
  ds_read_b32 v3, v4
  .endif
  AG_MF ag_b, 3, \bc
  .if ag_b < 8
  ds_read_b128 v[AG_A+4*ag_b:AG_A+4*ag_b+3], \va offset:(ag_b/4)*27648+\kh*4096+(ag_b%%4)*1024
  .else
  ds_read_b128 v[AG_A+32:AG_A+35], \va offset:8192+\kh*27648
  .endif
  .set ag_b, ag_b+1
  .endr
.endm
.macro AG_LOOP nb
  AG_NEEDX 0, v2, \nb
  .if AG_DIAG
  s_mov_b32 %[wx], 0
  s_mov_b32 %[ww], 0
  s_memtime %[c0]
  s_memrealtime %[r0]
  s_waitcnt lgkmcnt(0)
  .endif
.Lag_loop\nb:
  AG_HALF \nb, 44, 92, v2, v7, 1, 1, 0
  s_cbranch_scc1 .Lag_w0\nb
.Lag_w0r\nb:
  AG_HALF \nb, 60, 44, v7, v7, 0, 0, 0
  s_cbranch_scc0 .Lag_x0\nb
.Lag_x0r\nb:
  AG_HALF \nb, 76, 60, v7, v2, 1, 1, 1
  s_cbranch_scc1 .Lag_w1\nb
.Lag_w1r\nb:
  AG_HALF \nb, 92, 76, v2, v2, 0, 0, 1
  s_cbranch_scc0 .Lag_x1\nb
.Lag_x1r\nb:
  s_add_u32 %[t], %[t], 2
  s_cmp_lt_u32 %[t], 64
  s_cbranch_scc1 .Lag_loop\nb
  s_branch .Lag_end
.Lag_w0\nb:
  AG_WLOOK 1, .Lag_w0r\nb
.Lag_w1\nb:
  AG_WLOOK 2, .Lag_w1r\nb
.Lag_x0\nb:
  AG_NEEDX 1, v7, \nb
  s_branch .Lag_x0r\nb
.Lag_x1\nb:
  AG_NEEDX 2, v2, \nb
  s_branch .Lag_x1r\nb
.endm
  s_nop 4
  s_mov_b32 %[t], 0
  s_mov_b32 %[late], 0
  s_mov_b32 %[so], 0
  s_cmp_gt_i32 %[wk], 0
  s_cbranch_scc1 .Lag_go
  AG_NEEDW 0
.Lag_go:
  s_mov_b32 %[ws], 0
  s_sub_i32 %[wt], %[wk], 3
  s_min_i32 %[wt], %[wt], 12
  .set ag_i, 0
  .rept 12
  s_movk_i32 %[bc], (ag_i/4)*4096
  buffer_load_dwordx4 v[44+4*ag_i:44+4*ag_i+3], v0, %[rsw], %[bc] offen offset:(ag_i%%4)*1024 sc0 sc1
  .set ag_i, ag_i+1
  .endr
  s_movk_i32 %[bo], 3*4096
  s_movk_i32 %[bc], 3*4096
  .set ag_i, 0
  .rept 72
  v_mov_b64 v[AG_C+ag_i:AG_C+ag_i+1], 0
  .set ag_i, ag_i+2
  .endr
  v_mov_b32 v2, v0
  s_cmp_eq_u32 %[host], 0
  s_cbranch_scc1 .Lag_eight
  AG_LOOP 9
.Lag_eight:
  AG_LOOP 8
.Lag_end:
  s_waitcnt vmcnt(0) lgkmcnt(0)
  s_nop 7
)ASM"
            : "={v[112:143]}"(r0), "={v[144:175]}"(r1), "={v[176:207]}"(r2), "={v[208:239]}"(r3), "={v[240:255]}"(r4),
              [t] "=&s"(s_t), [wk] "+s"(s_wk), [bo] "=&s"(s_bo), [bc] "=&s"(s_bc), [x] "=&s"(s_x), [sp] "=&s"(s_sp), [t0] "=&s"(s_t0),
              [late] "=&s"(s_late), [wt] "=&s"(s_wt), [ws] "=&s"(s_ws), [so] "=&s"(s_so), [c] "=&s"(s_c), [c0] "=&s"(s_c0), [wx] "=&s"(s_wx), [ww] "=&s"(s_ww), [r0] "=&s"(s_r0), "+{v0}"(vo)
            : "{v4}"(polla), "{v5}"(donea), [rsw] "s"(rsw), [rsf] "s"(rsf), [tmo] "s"(tmo.word), [call] "s"(call), [host] "s"(rfl(host ? 1u : 0u))
            : "memory", "vcc", "scc", "v1", "v2", "v3", "v6", "v7", "v8", "v9", "v10", "v11", "v12", "v13", "v14", "v15", "v16", "v17", "v18", "v19", "v20", "v21", "v22", "v23", "v24", "v25", "v26", "v27", "v28", "v29", "v30", "v31", "v32", "v33", "v34", "v35", "v36", "v37", "v38", "v39", "v40", "v41", "v42", "v43", "v44", "v45", "v46", "v47", "v48", "v49", "v50", "v51", "v52", "v53", "v54", "v55", "v56", "v57", "v58", "v59", "v60", "v61", "v62", "v63", "v64", "v65", "v66", "v67", "v68", "v69", "v70", "v71", "v72", "v73", "v74", "v75", "v76", "v77", "v78", "v79", "v80", "v81", "v82", "v83", "v84", "v85", "v86", "v87", "v88", "v89", "v90", "v91", "v92", "v93", "v94", "v95", "v96", "v97", "v98", "v99", "v100", "v101", "v102", "v103", "v104", "v105", "v106", "v107");
        f32x4 acc[9][4];
#pragma unroll
        for (int i = 0; i < 144; ++i) acc[i >> 4][(i >> 2) & 3][i & 3] = i < 32 ? r0[i & 31] : i < 64 ? r1[i & 31] : i < 96 ? r2[i & 31] : i < 128 ? r3[i & 31] : r4[i & 15];
        STAMP(3);
        const int ln = (int)(vo >> 4);
        int mx = 0;
#pragma unroll
        for (int n = 0; n < 4; ++n) acc[8][n] *= XSCALE;
        if (host) {
#pragma unroll
            for (int n = 0; n < 4; ++n) { mx = max(max(mx, fbits(acc[8][n][0])), fbits(acc[8][n][1])); mx = max(max(mx, fbits(acc[8][n][2])), fbits(acc[8][n][3])); }
        }
        int m2 = 0;
#pragma unroll
        for (int b = 0; b < 8; ++b)
#pragma unroll
            for (int n = 0; n < 4; ++n) { mx = max(max(mx, fbits(acc[b][n][0])), fbits(acc[b][n][1])); m2 = max(max(m2, fbits(acc[b][n][2])), fbits(acc[b][n][3])); }
        const unsigned wm = wave_max((unsigned)max(mx, m2));
        if (ln == 0) {
            __hip_atomic_fetch_max((LAS unsigned*)(lds + LC_OFF) + C_MAX, wm, __ATOMIC_RELAXED, __HIP_MEMORY_SCOPE_WORKGROUP);
            asm volatile("" ::: "memory");
            const unsigned arrived = __hip_atomic_fetch_add((LAS unsigned*)(lds + LC_OFF) + C_CNT, 1u, __ATOMIC_RELAXED, __HIP_MEMORY_SCOPE_WORKGROUP);
            asm volatile("" ::: "memory");
            const unsigned tm = ctl[C_MAX];
            if (arrived == 3u) __hip_atomic_store((GAS u64*)(me + A1_OFF) + (par * NWG + bx) * 8, ((u64)call << 32) | tm, __ATOMIC_RELAXED, __HIP_MEMORY_SCOPE_SYSTEM);
        }
        STAMP(6);
        const __amdgpu_buffer_rsrc_t rso = mkrsrc(p.out, 8u * 512u * NL);
        const unsigned ovo = (unsigned)(ln & 15) * NL + ct * 256 + wave * 64 + (ln >> 4) * 16;
        unsigned used = 0xffffffffu; Spin sp;
#pragma unroll 1
        for (;;) {
            const u32x2 c = *(volatile LAS u32x2*)(lds + LC_OFF + C_GMAX * 4);
            unsigned st = rfl(c.y);
            const unsigned run = rfl(c.x), g = max(run, wm);
            if (st == 0u && sp.over<4095u>(tmo)) st = 2u;
            if (st == 2u || (g != used && (st != 0u || run != 0u))) {
                if (used != 0xffffffffu) VMWAIT0();
                used = g;
                const float scale = __uint_as_float(g) / 127.0f;
                const float inv = scale > 0.f ? 1.0f / scale : 0.f;
#pragma unroll
                for (int b = 0; b < 9; ++b) {
                    u32x4 o = {0u, 0u, 0u, 0u};
#pragma unroll
                    for (int cc = 0; cc < 4; ++cc) {
                        f32x4 q = acc[b][cc] * inv;
                        asm volatile("" : "+v"(q));
#pragma unroll
                        for (int r = 0; r < 4; ++r) o[r] = __builtin_amdgcn_cvt_pk_u8_f32(q[r], cc, o[r]);
                    }
                    if (st == 2u) o = (u32x4){0x80808080u, 0x80808080u, 0x80808080u, 0x80808080u};
                    if (b < 8 || host) __builtin_amdgcn_raw_buffer_store_b128(o, rso, ovo + (unsigned)(b < 8 ? rtile * 128 + b * 16 : rank * 512 + xrow) * NL, 0, SYS);
                }
            }
            if (st != 0u) break;
            __builtin_amdgcn_s_sleep(1);
        }
        STAMP(5);
    } else {
        const int f = wave & 1, ph = (wave >> 1) & 1, rb = rsub * 2 + f;
        STAMP(6);
        const __amdgpu_buffer_rsrc_t rsx = mkrsrc(me + XS_OFF + (size_t)src * XS_SRC + (size_t)rb * KS * UNIT, KS * UNIT);
        const __amdgpu_buffer_rsrc_t rse = mkrsrc(p.x + (size_t)xrow * KDIM + f * 32, host ? 16u * KDIM * 4u - 128u : 0u);
        const __amdgpu_buffer_rsrc_t rsf = mkrsrc(me + XF_OFF + (size_t)(src * 8 + rb) * KS * 16, KS * 16);
        const unsigned ldsb = (unsigned)(size_t)lds;
        const unsigned mbase = ldsb + f * LA_HALF, vpub = ldsb + LC_OFF + (C_FULL + (wave & 3)) * 4, vdone = ldsb + LC_OFF + (C_DONE + (lane & 3)) * 4;
        const unsigned fcall = (nopush && !local) ? 0u : call, hostf = rfl(host ? 1u : 0u);
        unsigned s_cur = ph, s_lim = NST, s_so = 0, s_h, s_g, s_b, s_x, s_y, s_sp, s_t0, s_m0; u64 s_seen, s_c;
        asm volatile(R"ASM(
.macro FD_EXPIRED lout
  s_add_u32 %[sp], %[sp], 1
  s_and_b32 %[x], %[sp], 0xff
  s_cbranch_scc1 8f
  v_mov_b32 v9, 0
  global_load_dword v12, v9, %[tmo] sc1
  s_memrealtime %[c]
  s_waitcnt vmcnt(0) lgkmcnt(0)
  v_readfirstlane_b32 %[x], v12
  s_cmp_eq_u32 %[x], %[call]
  s_cbranch_scc1 \lout
  s_mov_b64 vcc, %[c]
  s_cmp_eq_u32 %[sp], 0x100
  s_cselect_b32 %[t0], vcc_lo, %[t0]
  s_sub_u32 %[x], vcc_lo, %[t0]
  s_cmp_le_u32 %[x], 40000000
  s_cbranch_scc1 8f
  v_mov_b32 v12, %[call]
  global_store_dword v9, v12, %[tmo] sc1
  s_waitcnt vmcnt(0)
  s_branch \lout
8:
.endm
.macro FD_FLAGS
  buffer_load_dwordx4 v[100:103], v0, %[rsf], 0 offen sc0 sc1
  s_waitcnt vmcnt(0)
  v_min_u32 v9, v100, v102
  v_cmp_le_u32 %[seen], %[fcall], v9
.endm
; loads of unit %[y] into the set at r
.macro FD_ISSUE r
  s_lshl_b32 %[x], %[y], 13
  s_lshl_b32 %[b], %[y], 8
  buffer_load_dwordx4 v[\r:\r+3], v0, %[rsx], %[x] offen sc0 sc1
  buffer_load_dwordx4 v[\r+4:\r+7], v0, %[rsx], %[x] offen offset:1024 sc0 sc1
  buffer_load_dwordx3 v[\r+16:\r+18], v1, %[rsx], %[x] offen offset:2048 sc0 sc1
  buffer_load_ushort v[\r+19], v2, %[rsx], %[x] offen offset:2816 sc0 sc1
  s_bitset1_b32 %[x], 12
  buffer_load_dwordx4 v[\r+24:\r+27], v3, %[rse], %[b] offen
  buffer_load_dwordx4 v[\r+28:\r+31], v3, %[rse], %[b] offen offset:64
  buffer_load_dwordx4 v[\r+8:\r+11], v0, %[rsx], %[x] offen sc0 sc1
  buffer_load_dwordx4 v[\r+12:\r+15], v0, %[rsx], %[x] offen offset:1024 sc0 sc1
  buffer_load_dwordx3 v[\r+20:\r+22], v1, %[rsx], %[x] offen offset:2048 sc0 sc1
  buffer_load_ushort v[\r+23], v2, %[rsx], %[x] offen offset:2816 sc0 sc1
.endm
; wait for the flag of unit %[y]
.macro FD_WAITFLAG
  s_mov_b32 %[sp], 0
1:
  FD_FLAGS
  s_bitcmp1_b64 %[seen], %[y]
  s_cbranch_scc1 2f
  s_sleep 1
  FD_EXPIRED 2f
  s_branch 1b
2:
.endm
.macro FD_DEC h, n0, n1, n2
  v_and_or_b32 v[\h], v[\n0], v7, %[b]
  v_alignbit_b32 v12, v[\n0], v[\n0], 3
  v_and_or_b32 v[\h+2], v[\n1], v7, %[b]
  v_alignbit_b32 v13, v[\n1], v[\n1], 3
  v_and_or_b32 v[\h+4], v[\n2], v7, %[b]
  v_alignbit_b32 v14, v[\n2], v[\n2], 3
  v_and_or_b32 v[\h+1], v12, v7, %[b]
  v_lshrrev_b32 v12, 5, v[\n0]
  v_and_or_b32 v[\h+3], v13, v7, %[b]
  v_lshrrev_b32 v13, 5, v[\n1]
  v_and_or_b32 v[\h+5], v14, v7, %[b]
  v_lshlrev_b32 v14, 1, v[\n2]
  v_bfi_b32 v12, v8, v14, v12
  v_lshlrev_b32 v15, 2, v[\n2]
  v_and_or_b32 v[\h+6], v12, v7, %[b]
  v_bfi_b32 v13, v8, v15, v13
  v_and_or_b32 v[\h+7], v13, v7, %[b]
.endm
; exception entry in register e -> address v12, mask v13
.macro FD_EXC e
  v_and_b32 v12, 0x7fe, v[\e]
  v_and_b32 v14, 1, v[\e]
  v_lshrrev_b32 v13, 11, v[\e]
  v_lshl_or_b32 v14, v14, 4, 10
  v_lshl_add_u32 v12, v12, 1, %[x]
  v_lshlrev_b32 v13, v14, v13
.endm
; a half sent raw, or with a window that does not start at a multiple of four
.macro FD_FIX hd, h
  s_bitcmp1_b32 \hd, 7
  s_cbranch_scc0 3f
  buffer_load_dwordx4 v[\h:\h+3], v0, %[rsx], %[x] offen offset:2048 sc0 sc1
  buffer_load_dwordx4 v[\h+4:\h+7], v0, %[rsx], %[x] offen offset:3072 sc0 sc1
  s_waitcnt vmcnt(0)
3:
  s_and_b32 %[b], \hd, 3
  s_mul_i32 %[b], %[b], 0x01010101
  .set fd_i, 0
  .rept 8
  v_add_u32 v[\h+fd_i], %[b], v[\h+fd_i]
  .set fd_i, fd_i+1
  .endr
.endm
; one unit: set r holds the loads of unit cur, set q takes the next one; ob = offset of this parity's ring
.macro FD_STEP r, q, ob, id
  s_add_u32 %[y], %[cur], 2
  s_cmp_ge_u32 %[y], 64
  s_cbranch_scc1 .Lfd_last\id
  s_bitcmp1_b64 %[seen], %[y]
  s_cbranch_scc0 .Lfd_look\id
  FD_ISSUE \q
  s_waitcnt vmcnt(10)
.Lfd_go\id:
  v_readlane_b32 %[h], v101, %[cur]
  v_readlane_b32 %[g], v103, %[cur]
  s_and_b32 %[b], %[h], 0x7c
  s_mul_i32 %[b], %[b], 0x01010101
  FD_DEC 80, \r+16, \r+17, \r+18
  s_and_b32 %[b], %[g], 0x7c
  s_mul_i32 %[b], %[b], 0x01010101
  s_add_u32 m0, %[mb], %[so]
  FD_DEC 88, \r+20, \r+21, \r+22
  s_or_b32 %[x], %[h], %[g]
  s_and_b32 %[x], %[x], 0x83
  s_cbranch_scc1 .Lfd_fix\id
.Lfd_fixed\id:
  s_cmp_ge_u32 %[cur], %[lim]
  s_cbranch_scc1 .Lfd_free\id
.Lfd_freed\id:
  .set fd_i, 0
  .rept 16
  v_perm_b32 v[96+(fd_i%%2)*2], v[80+fd_i], v[\r+fd_i], %[p0]
  v_perm_b32 v[97+(fd_i%%2)*2], v[80+fd_i], v[\r+fd_i], %[p1]
  ds_write_addtid_b32 v[96+(fd_i%%2)*2] offset:\ob+fd_i*512
  ds_write_addtid_b32 v[97+(fd_i%%2)*2] offset:\ob+fd_i*512+256
  .set fd_i, fd_i+1
  .endr
  s_add_u32 %[x], m0, \ob
  FD_EXC \r+19
  s_lshr_b32 %[b], %[h], 8
  s_bfm_b64 exec, %[b], 0
  s_lshr_b32 %[b], %[g], 8
  ds_xor_b32 v12, v13
  s_bfm_b64 exec, %[b], 0
  s_add_u32 %[so], %[so], 9216
  FD_EXC \r+23
  ds_xor_b32 v12, v13 offset:4096
  s_mov_b64 exec, -1
  s_cmp_eq_u32 %[host], 0
  s_cbranch_scc1 .Lfd_pub\id
  v_cvt_pk_bf16_f32 v96, v[\r+24], v[\r+25]
  v_cvt_pk_bf16_f32 v97, v[\r+26], v[\r+27]
  v_cvt_pk_bf16_f32 v98, v[\r+28], v[\r+29]
  v_cvt_pk_bf16_f32 v99, v[\r+30], v[\r+31]
  ds_write_addtid_b32 v96 offset:\ob+8192
  ds_write_addtid_b32 v97 offset:\ob+8192+256
  ds_write_addtid_b32 v98 offset:\ob+8192+512
  ds_write_addtid_b32 v99 offset:\ob+8192+768
.Lfd_pub\id:
  s_add_u32 %[cur], %[cur], 2
  s_cmp_eq_u32 %[so], 27648
  s_cselect_b32 %[so], 0, %[so]
  s_cmp_ge_u32 %[cur], 64
  s_cselect_b32 %[x], 1000, %[cur]
  v_mov_b32 v9, %[x]
  s_waitcnt lgkmcnt(0)
  ds_write_b32 v4, v9
  s_cbranch_scc1 .Lfd_end
  s_cmp_eq_u32 %[y], %[cur]
  s_cbranch_scc0 .Lfd_late\id
.Lfd_next\id:
.endm
; out-of-line parts of a step
.macro FD_SLOW r, q, ob, id
.Lfd_last\id:
  s_waitcnt vmcnt(0)
  s_branch .Lfd_go\id
.Lfd_look\id:
  FD_FLAGS
  s_bitcmp1_b64 %[seen], %[y]
  s_cbranch_scc0 3f
  FD_ISSUE \q
  s_branch .Lfd_go\id
3:
  s_mov_b32 %[y], -1
  s_branch .Lfd_go\id
.Lfd_late\id:
  s_mov_b32 %[y], %[cur]
  FD_WAITFLAG
  FD_ISSUE \q
  s_branch .Lfd_next\id
.Lfd_fix\id:
  s_lshl_b32 %[x], %[cur], 13
  FD_FIX %[h], 80
  s_bitset1_b32 %[x], 12
  FD_FIX %[g], 88
  s_branch .Lfd_fixed\id
.Lfd_free\id:
  s_mov_b32 %[sp], 0
1:
  ds_read_b32 v9, v5
  s_waitcnt lgkmcnt(0)
  v_readlane_b32 %[lim], v9, 0
  v_readlane_b32 %[x], v9, 1
  s_min_u32 %[lim], %[lim], %[x]
  v_readlane_b32 %[x], v9, 2
  s_min_u32 %[lim], %[lim], %[x]
  v_readlane_b32 %[x], v9, 3
  s_min_u32 %[lim], %[lim], %[x]
  s_add_u32 %[lim], %[lim], 6
  s_cmp_lt_u32 %[cur], %[lim]
  s_cbranch_scc1 .Lfd_freed\id
  s_sleep 4
  FD_EXPIRED .Lfd_freed\id
  s_branch 1b
.endm
  s_nop 4
  s_mov_b32 %[m0s], m0
  v_mov_b32 v7, 0x83838383
  v_mov_b32 v8, 0x80808080
  s_mov_b32 %[y], %[cur]
  FD_WAITFLAG
  FD_ISSUE 16
  s_bitcmp1_b32 %[cur], 0
  s_cbranch_scc1 .Lfd_loop1
.Lfd_loop0:
  FD_STEP 16, 48, 0, 0
  FD_STEP 48, 16, 0, 1
  s_branch .Lfd_loop0
.Lfd_loop1:
  FD_STEP 16, 48, 55296, 2
  FD_STEP 48, 16, 55296, 3
  s_branch .Lfd_loop1
  FD_SLOW 16, 48, 0, 0
  FD_SLOW 48, 16, 0, 1
  FD_SLOW 16, 48, 55296, 2
  FD_SLOW 48, 16, 55296, 3
.Lfd_end:
  s_mov_b32 m0, %[m0s]
  s_waitcnt vmcnt(0) lgkmcnt(0)
)ASM"
            : [cur] "+s"(s_cur), [lim] "+s"(s_lim), [so] "+s"(s_so), [h] "=&s"(s_h), [g] "=&s"(s_g), [b] "=&s"(s_b), [x] "=&s"(s_x), [y] "=&s"(s_y), [sp] "=&s"(s_sp),
              [t0] "=&s"(s_t0), [m0s] "=&s"(s_m0), [seen] "=&s"(s_seen), [c] "=&s"(s_c)
            : "{v0}"(lane * 16), "{v1}"(lane * 12), "{v2}"(lane * 2), "{v3}"((lane >> 2) * KDIM * 4 + (lane & 3) * 16), "{v4}"(vpub), "{v5}"(vdone),
              [rsx] "s"(rsx), [rse] "s"(rse), [rsf] "s"(rsf), [tmo] "s"(tmo.word), [call] "s"(call), [fcall] "s"(fcall), [mb] "s"(mbase), [host] "s"(hostf),
              [p0] "s"(0x05010400u), [p1] "s"(0x07030602u)
            : "memory", "vcc", "scc", "v7", "v8", "v9", "v10", "v11", "v12", "v13", "v14", "v15", "v16", "v17", "v18", "v19", "v20", "v21", "v22", "v23", "v24", "v25", "v26", "v27", "v28", "v29", "v30", "v31", "v32", "v33", "v34", "v35", "v36", "v37", "v38", "v39", "v40", "v41", "v42", "v43", "v44", "v45", "v46", "v47", "v48", "v49", "v50", "v51", "v52", "v53", "v54", "v55", "v56", "v57", "v58", "v59", "v60", "v61", "v62", "v63", "v64", "v65", "v66", "v67", "v68", "v69", "v70", "v71", "v72", "v73", "v74", "v75", "v76", "v77", "v78", "v79", "v80", "v81", "v82", "v83", "v84", "v85", "v86", "v87", "v88", "v89", "v90", "v91", "v92", "v93", "v94", "v95", "v96", "v97", "v98", "v99", "v100", "v101", "v102", "v103");
        static_assert(DEADLINE == 40000000u && NST == 6, "literals in the feeder loop");
        if (wave != 4) return;
        if (lane == 0) __hip_atomic_store(seqw, call, __ATOMIC_RELAXED, __HIP_MEMORY_SCOPE_AGENT);
        {
            const GAS u64* a2 = lane < 16 ? (const GAS u64*)(me + (lane < 8 ? A2_OFF : A3_OFF) + (size_t)(par * 8 + (bx & 7)) * 128) + (lane & 7) : (const GAS u64*)(me + TM_OFF);
            unsigned told = 0; Spin sp;
            for (;;) {
                const u64 v = __hip_atomic_load(a2, __ATOMIC_RELAXED, __HIP_MEMORY_SCOPE_SYSTEM);
                const bool in = lane < 16 && (unsigned)(v >> 32) == call;
                unsigned g = in ? (unsigned)v : 0u;
                g = max(g, (unsigned)__builtin_amdgcn_update_dpp(0u, g, 0x111, 0xf, 0xf, true));
                g = max(g, (unsigned)__builtin_amdgcn_update_dpp(0u, g, 0x112, 0xf, 0xf, true));
                g = max(g, (unsigned)__builtin_amdgcn_update_dpp(0u, g, 0x114, 0xf, 0xf, true));
                g = max(g, (unsigned)__builtin_amdgcn_update_dpp(0u, g, 0x118, 0xf, 0xf, true));
                g = __builtin_amdgcn_readlane(g, 15);
                const unsigned im = (unsigned)__builtin_amdgcn_ballot_w64(in);
                unsigned st = (im & 0xffu) == 0xffu ? 1u : 0u;
                if (((im | (im >> 8)) & 0xffu) != 0xffu) g = 0u;
                if ((unsigned)__builtin_amdgcn_readlane((unsigned)v, 16) == call || (st == 0u && sp.over<255u>(tmo))) st = 2u;
                if (st != 0u || g != told) { told = g; *(volatile LAS u32x2*)(lds + LC_OFF + C_GMAX * 4) = (u32x2){g, st}; }
                if (st != 0u) break;
                __builtin_amdgcn_s_sleep(1);
            }
        }
        STAMP(4);
    }
}

}

extern "C" void kernel_launch(int rank, int nranks, ncclComm_t comm,
                              hipStream_t stream,
                              void* const* d_in, const int* in_sizes, int n_in,
                              void* d_out, int out_size,
                              const int* in_dtypes, int out_dtype,
                              void* const* d_out_all,
                              void* const* d_ws_all, size_t ws_bytes) {
    if (nranks != NR || (unsigned)rank >= (unsigned)NR || ws_bytes < WS_NEED || n_in < 2) {
        fprintf(stderr, "unsupported configuration\n");
        return;
    }
    HIP_CHECK(hipFuncSetAttribute((const void*)ag_gemm, hipFuncAttributeMaxDynamicSharedMemorySize, LDS_BYTES));
    Params p;
    p.x = (const float*)d_in[0]; p.w = (const float*)d_in[1]; p.out = (signed char*)d_out;
#define WSEND 1
    const size_t base = WSEND ? (ws_bytes - WS_NEED) & ~(size_t)0x1fffff : 0;
    for (int i = 0; i < NR; ++i) p.peer[i] = (char*)d_ws_all[(rank + 1 + i) % NR] + base;
    p.rank = rank; p.pad = 0;
    hipLaunchKernelGGL(ag_gemm, dim3(NWG), dim3(512), LDS_BYTES, stream, p);
    HIP_CHECK(hipGetLastError());
}
```

```cpp
#include <hip/hip_runtime.h>
#include <rccl/rccl.h>
#include <cstdio>

#define HIP_CHECK(expr) do { hipError_t _e = (expr); if (_e != hipSuccess) { \
    fprintf(stderr, "HIP error %s at %s:%d\n", hipGetErrorString(_e), __FILE__, __LINE__); \
    return; } } while (0)

namespace {

#define LAS __attribute__((address_space(3)))
#define GAS __attribute__((address_space(1)))
typedef unsigned u32x4 __attribute__((ext_vector_type(4)));
typedef unsigned u32x2 __attribute__((ext_vector_type(2)));
typedef float f32x4 __attribute__((ext_vector_type(4)));
typedef float f32x16 __attribute__((ext_vector_type(16)));
typedef float f32x32 __attribute__((ext_vector_type(32)));
typedef unsigned long long u64;

constexpr int NR = 8;
constexpr int KDIM = 4096, NL = 2048;
constexpr int KS = 64;
constexpr int NWG = 256;
constexpr int UNIT = 8192;
constexpr int EXCAP = 63;

constexpr size_t XS_OFF = 0;
constexpr size_t XS_SRC = (size_t)8 * KS * UNIT;
constexpr size_t WB_OFF = XS_OFF + NR * XS_SRC;
constexpr size_t XF_OFF = WB_OFF + (size_t)32 * KS * 8192;
constexpr size_t WF_OFF = XF_OFF + (size_t)NR * 8 * KS * 16;
constexpr size_t SQ_OFF = WF_OFF + (size_t)32 * KS * 64;
constexpr size_t A1_OFF = SQ_OFF + (size_t)NWG * 128;
constexpr size_t A2_OFF = A1_OFF + (size_t)2 * NWG * 64;
constexpr size_t A3_OFF = A2_OFF + (size_t)2 * 8 * 128;
constexpr size_t TK_OFF = A3_OFF + (size_t)2 * 8 * 128;
constexpr size_t TM_OFF = TK_OFF + 128;
constexpr size_t WS_NEED = TM_OFF + 128;

constexpr int NST = 6, LA_SLOT = 9216, LA_HALF = 3 * LA_SLOT, LA_PAR = 2 * LA_HALF;
constexpr int LC_OFF = 2 * LA_PAR;
constexpr int C_FULL = 0;
constexpr int C_DONE = 4;
constexpr int C_MAX = 8, C_CNT = 9, C_GMAX = 10  , C_SENT = 12, C_ISS = 13, C_SEQ = 14;
constexpr int LJ_OFF = LC_OFF + 64;
constexpr int LDS_BYTES = LJ_OFF + 256;
static_assert(LDS_BYTES > 81920, "one workgroup per CU");

constexpr int CPX = 3;
constexpr int PW = 6;
constexpr int PWIN = 3;
constexpr int SREL = 14, KREL = 10;
constexpr int PEARLY = 2;
constexpr unsigned WDELAY = 20, WDELAY2 = 340, WSTAG = 12;
constexpr int SPECN = 168;
constexpr int CODE_BYTES = 30208;
constexpr int NCOMM = 8 * CPX;
constexpr float XSCALE = 0.5f;
constexpr unsigned DEADLINE = 40000000u;

struct Params {
    const float* x; const float* w; signed char* out;
    char* peer[NR];
    int rank; int pad;
};

__device__ __forceinline__ unsigned rt32() { return (unsigned)__builtin_amdgcn_s_memrealtime(); }
__device__ __forceinline__ unsigned rfl(unsigned v) { return __builtin_amdgcn_readfirstlane(v); }
__device__ __forceinline__ unsigned cvt_pk(float lo, float hi) {
    typedef float f2 __attribute__((ext_vector_type(2))); typedef __bf16 b2 __attribute__((ext_vector_type(2)));
    f2 v = {lo, hi}; return __builtin_bit_cast(unsigned, __builtin_convertvector(v, b2));
}
__device__ __forceinline__ __amdgpu_buffer_rsrc_t mkrsrc(const void* p, unsigned bytes) {
    return __builtin_amdgcn_make_buffer_rsrc((void*)p, 0, bytes, 0x00020000);
}
__device__ __forceinline__ int fbits(float v) { return __float_as_int(v); }
__device__ __forceinline__ unsigned rotr(unsigned v, int n) { return __builtin_amdgcn_alignbit(v, v, n); }
#define VMWAIT0() asm volatile("s_waitcnt vmcnt(0)" ::: "memory")
#define LGKM0() asm volatile("s_waitcnt lgkmcnt(0)" ::: "memory")
#define SCHED() __builtin_amdgcn_sched_barrier(0)
constexpr int SYS = 17;

struct Tmo {
    GAS unsigned* word; unsigned call;
    __device__ __forceinline__ bool hit() const {
        unsigned v;
        asm volatile("v_mov_b32 %0, 0\n\tglobal_load_dword %0, %0, %1 sc1\n\ts_waitcnt vmcnt(0)" : "=&v"(v) : "s"(word) : "memory");
        return rfl(v) == call;
    }
    __device__ __forceinline__ bool expired(unsigned t0) const {
        if (hit() || rt32() - t0 > DEADLINE) {
            unsigned z, c;
            asm volatile("v_mov_b32 %0, 0\n\tv_mov_b32 %1, %3\n\tglobal_store_dword %0, %1, %2 sc1\n\ts_waitcnt vmcnt(0)" : "=&v"(z), "=&v"(c) : "s"(word), "s"(call) : "memory");
            return true;
        }
        return false;
    }
};

struct Spin {
    unsigned n = 0, t0 = 0;
    template <unsigned MASK> __device__ __forceinline__ bool over(const Tmo& tmo) {
        if ((++n & MASK) != 0u) return false;
        if (n == MASK + 1u) { t0 = rt32(); return false; }
        return tmo.expired(t0);
    }
};


__device__ __forceinline__ void w_load(f32x4 (&v)[16], const float* __restrict__ w, bool on, int ct, int t, int q, int lane) {
    const __amdgpu_buffer_rsrc_t rs = mkrsrc(w + (size_t)t * 64 * NL + ct * 256 + q * 64, on ? 63u * NL * 4u + 256u : 0u);
    const int n = lane & 15, g = lane >> 4;
    const unsigned vo = (unsigned)((g >> 1) * 16 + (g & 1) * 2) * NL * 4 + n * 16;
#pragma unroll
    for (int i = 0; i < 16; ++i) v[i] = __builtin_bit_cast(f32x4, __builtin_amdgcn_raw_buffer_load_b128(rs, vo, ((i >> 3) * 32 + ((i >> 1) & 3) * 4 + (i & 1)) * NL * 4, 2));
}
__device__ __forceinline__ void w_store(const f32x4 (&v)[16], char* me, int ct, int t, int q, int lane) {
    const __amdgpu_buffer_rsrc_t rs = mkrsrc(me + WB_OFF + ((size_t)(ct * 4 + q) * KS + t) * 8192, 8192);
#pragma unroll
    for (int kh = 0; kh < 2; ++kh)
#pragma unroll
        for (int c = 0; c < 4; ++c) {
            u32x4 o;
            o.x = cvt_pk(v[kh * 8 + 0][c], v[kh * 8 + 1][c]); o.y = cvt_pk(v[kh * 8 + 2][c], v[kh * 8 + 3][c]);
            o.z = cvt_pk(v[kh * 8 + 4][c], v[kh * 8 + 5][c]); o.w = cvt_pk(v[kh * 8 + 6][c], v[kh * 8 + 7][c]);
            __builtin_amdgcn_raw_buffer_store_b128(o, rs, lane * 16 + c * 1024 + kh * 4096, 0, SYS);
        }
}

__device__ __forceinline__ char* pick_peer(const Params& p, int q) {
    char* r = p.peer[0];
#pragma unroll
    for (int i = 1; i < NR; ++i) { char* c = p.peer[i]; asm volatile("" : "+s"(c)); r = q == i ? c : r; }
    return r;
}

__device__ __forceinline__ void touch(const void* ptr, unsigned junk) {
    unsigned keep;
    asm volatile("s_mov_b32 %0, m0\n\ts_mov_b32 m0, %2\n\ts_nop 0\n\tglobal_load_lds_dword %1, off\n\ts_mov_b32 m0, %0" : "=&s"(keep) : "v"(ptr), "s"(junk) : "memory");
}

__device__ __forceinline__ unsigned wave_max(unsigned g) {
    g = max(g, (unsigned)__builtin_amdgcn_update_dpp(0u, g, 0x111, 0xf, 0xf, true));
    g = max(g, (unsigned)__builtin_amdgcn_update_dpp(0u, g, 0x112, 0xf, 0xf, true));
    g = max(g, (unsigned)__builtin_amdgcn_update_dpp(0u, g, 0x114, 0xf, 0xf, true));
    g = max(g, (unsigned)__builtin_amdgcn_update_dpp(0u, g, 0x118, 0xf, 0xf, true));
    return max(max((unsigned)__builtin_amdgcn_readlane(g, 15), (unsigned)__builtin_amdgcn_readlane(g, 31)), max((unsigned)__builtin_amdgcn_readlane(g, 47), (unsigned)__builtin_amdgcn_readlane(g, 63)));
}

__global__ void __launch_bounds__(512, 2) ag_gemm(Params p) {
    unsigned ka0, ka1;
    asm volatile("s_load_dword %0, %2, 0x0\n\ts_load_dword %1, %2, 0x40" : "=&s"(ka0), "=&s"(ka1) : "s"(__builtin_amdgcn_kernarg_segment_ptr()));
    extern __shared__ __attribute__((aligned(16))) char lds_raw[];
    LAS char* lds = (LAS char*)lds_raw;
    volatile LAS unsigned* ctl = (volatile LAS unsigned*)(lds + LC_OFF);
    const int lane = threadIdx.x & 63, wave = rfl(threadIdx.x >> 6);
    const int bx = blockIdx.x, xcd = bx & 7, idx = bx >> 3;
    const unsigned junk = (unsigned)(size_t)(lds + LJ_OFF);
    if (wave == 7 && idx < 2) {
        static_assert(CODE_BYTES > 3 * 8192 && CODE_BYTES <= 4 * 8192, "four reads");
        unsigned keep, o;
        asm volatile("s_mov_b32 %0, m0\n\ts_mov_b32 m0, %4\n\tv_add_u32 %1, 8192, %2\n\tglobal_load_lds_dword %2, %3\n\tglobal_load_lds_dword %1, %3\n\t"
                     "v_add_u32 %1, 16384, %2\n\tglobal_load_lds_dword %1, %3\n\tv_add_u32 %1, 24576, %2\n\tv_min_u32 %1, %5, %1\n\tglobal_load_lds_dword %1, %3\n\ts_mov_b32 m0, %0"
                     : "=&s"(keep), "=&v"(o) : "v"(lane * 128), "s"(__builtin_amdgcn_s_getpc()), "s"(junk), "i"(CODE_BYTES) : "memory");
    }
    asm volatile("" :: "s"(p.x), "s"(p.w), "s"(p.out), "s"(p.rank), "s"(p.peer[0]), "s"(p.peer[1]), "s"(p.peer[2]), "s"(p.peer[3]),
                       "s"(p.peer[4]), "s"(p.peer[5]), "s"(p.peer[6]), "s"(p.peer[7]), "s"(ka0), "s"(ka1));
    const int rank = p.rank & 7;
    const int rel = idx & 7, src = (rank + 1 + rel) & 7, sub = xcd & 3, ct = (xcd >> 2) * 4 + (idx >> 3);
    char* me = p.peer[NR - 1];
    const bool local = rel == NR - 1;
    const int rsub = local ? 3 : sub, rtile = src * 4 + rsub;
    const int jj = ((idx >> 3) + sub) & 3;
    const bool comm = local && jj < CPX;
    const int h = rel * 4 + sub;
    const bool host = h < 8 * CPX;
    const int xrow = h * 16;
    constexpr int NCW = 32 - CPX;
    static_assert(PEARLY <= PW && PW <= 7 && CPX == 3, "roles");
    const int ci = local ? 28 : h;
    const int cid = jj * 8 + xcd;
    GAS unsigned* seqw = (GAS unsigned*)(me + SQ_OFF + (size_t)bx * 128);
#define WG_START() do { if (threadIdx.x < 16) ctl[threadIdx.x] = (threadIdx.x == C_FULL + 2 || threadIdx.x == C_FULL + 3) ? 1u : 0u; \
        LGKM0(); __builtin_amdgcn_s_barrier(); } while (0)
#define GET_CALL() const unsigned call = rfl(__hip_atomic_load(seqw, __ATOMIC_RELAXED, __HIP_MEMORY_SCOPE_AGENT)) + 1u; \
    const Tmo tmo{(GAS unsigned*)(me + TM_OFF), call}; const unsigned par = call & 1u; (void)par

    if (__builtin_expect(comm, 1)) {
        if (__builtin_expect(wave < PW, 1)) {
            constexpr unsigned NHU = 16 * KS;
            f32x4 v[8];
            auto load_x = [&](int u) {
                const float* s0 = p.x + (size_t)((u & 7) * 64 + (lane >> 2)) * KDIM + (u >> 3) * 32 + (lane & 3) * 4;
#pragma unroll
                for (int e = 0; e < 8; ++e) v[e] = *(const GAS f32x4*)(s0 + (size_t)(e >> 1) * 16 * KDIM + (e & 1) * 16);
            };
            if (wave >= PEARLY) __builtin_amdgcn_s_sleep(8);
            load_x(wave * NCOMM + cid);
            SCHED();
            WG_START();
            GET_CALL();
            __amdgpu_buffer_rsrc_t rs[NR];
#pragma unroll
            for (int q = 0; q < NR; ++q) rs[q] = mkrsrc(p.peer[q] + XS_OFF + (size_t)rank * XS_SRC, (unsigned)XS_SRC);
            GAS u64* flag = (GAS u64*)(pick_peer(p, lane & 7) + XF_OFF) + rank * 8 * KS * 2;
            GAS unsigned* tk = (GAS unsigned*)(me + TK_OFF);
            LAS char* tail = lds + wave * 2048;
            const unsigned ptag = (unsigned)lane << 1, vo = lane * 16;
            unsigned u = wave * NCOMM + cid;
            int k = wave;
            while (u < NHU) {
                const unsigned fi = (u & 7u) * KS * 2 + (u >> 3), soff = fi * (UNIT / 2);
                const bool own = (u & 7u) >= 6u;
                u32x4 L[2], H[2], T;
                unsigned sum = 0;
#pragma unroll
                for (int e = 0; e < 8; ++e) {
                    const f32x4 s = v[e] * XSCALE;
                    const unsigned d0 = cvt_pk(s[0], s[1]), d1 = cvt_pk(s[2], s[3]);
                    L[e >> 2][e & 3] = __builtin_amdgcn_perm(d1, d0, 0x06040200u);
                    H[e >> 2][e & 3] = __builtin_amdgcn_perm(d1, d0, 0x07050301u);
                }
                SCHED();
                if (k < PEARLY) {
#pragma unroll
                    for (int j = 0; j < 2; ++j)
#pragma unroll
                        for (int q = 0; q < NR; ++q) if (q < NR - 1 || own) __builtin_amdgcn_raw_buffer_store_b128(L[j], rs[q], vo + j * 1024, soff, SYS);
                    SCHED(); asm volatile("s_nop 1" ::: "memory"); SCHED();
                }
#pragma unroll
                for (int e = 0; e < 8; ++e) sum = __builtin_amdgcn_sad_u8(H[e >> 2][e & 3] & 0x7f7f7f7fu, 0u, sum);
                sum += __builtin_amdgcn_update_dpp(0u, sum, 0x111, 0xf, 0xf, true);
                sum += __builtin_amdgcn_update_dpp(0u, sum, 0x112, 0xf, 0xf, true);
                sum += __builtin_amdgcn_update_dpp(0u, sum, 0x114, 0xf, 0xf, true);
                sum += __builtin_amdgcn_update_dpp(0u, sum, 0x118, 0xf, 0xf, true);
                const int tot = __builtin_amdgcn_readlane(sum, 15) + __builtin_amdgcn_readlane(sum, 31) + __builtin_amdgcn_readlane(sum, 47) + __builtin_amdgcn_readlane(sum, 63);
                int bs = (tot - 2970) >> 11;
                bs = bs < 0 ? 0 : (bs > 124 ? 124 : bs);
                const unsigned b4 = (unsigned)bs * 0x01010101u, nb4 = (unsigned)((-bs) & 3) * 0x01010101u;
                unsigned cnt = 0, c[8], sg[8], x[8], rest = 0;
                auto emit = [&](int e) {
                    const u64 m = __builtin_amdgcn_ballot_w64(x[e] != 0u);
                    const unsigned sh = (unsigned)__builtin_ctz(x[e] | 0x80000000u) & 24u;
                    const unsigned pos = min(cnt, (unsigned)EXCAP + 1u) + __builtin_amdgcn_mbcnt_hi((unsigned)(m >> 32), __builtin_amdgcn_mbcnt_lo((unsigned)m, 0u));
                    const unsigned ent = (((x[e] >> sh) & 0x7cu) << 9) | ((unsigned)e << 8) | ((sh & 16u) << 3) | ptag | ((sh >> 3) & 1u);
                    *(LAS unsigned short*)(tail + (x[e] != 0u ? 768u + pos * 2 : 1024u + lane * 2)) = (unsigned short)ent;
                    x[e] &= ~(0xffu << sh);
                    cnt += (unsigned)__builtin_popcountll(m);
                };
#pragma unroll
                for (int e = 0; e < 8; ++e) {
                    const unsigned hh = H[e >> 2][e & 3], hm = hh & 0x7f7f7f7fu;
                    c[e] = (hm + nb4) & 0x03030303u;
                    sg[e] = hh & 0x80808080u;
                    x[e] = hm ^ (c[e] + b4);
                    emit(e);
                    rest |= x[e];
                }
                while (__builtin_amdgcn_ballot_w64(rest != 0u) != 0ull && cnt <= (unsigned)EXCAP) {
                    rest = 0;
#pragma unroll
                    for (int e = 0; e < 8; ++e) { if (__builtin_amdgcn_ballot_w64(x[e] != 0u) != 0ull) emit(e); rest |= x[e]; }
                }
                ((LAS unsigned*)tail)[lane * 3 + 0] = (c[0] | sg[0]) | rotr(c[1] | sg[1], 29) | (c[6] << 5);
                ((LAS unsigned*)tail)[lane * 3 + 1] = (c[2] | sg[2]) | rotr(c[3] | sg[3], 29) | (c[7] << 5);
                ((LAS unsigned*)tail)[lane * 3 + 2] = (c[4] | sg[4]) | rotr(c[5] | sg[5], 29) | (sg[6] >> 1) | (sg[7] >> 2);
                asm volatile("" ::: "memory");
                T = *(volatile LAS u32x4*)(tail + lane * 16);
                unsigned hdr = (unsigned)bs | (cnt << 8);
                if (cnt > (unsigned)EXCAP) { hdr = 0x80u; T = H[0]; }
                unsigned vt = ((hdr & 0x80u) || (unsigned)lane < 48u + ((cnt + 7u) >> 3)) ? vo + 2048 : 0x40000000u;
                asm volatile("" : "+v"(T), "+v"(vt));
                SCHED();
                if (k >= PEARLY) {
                    Spin sp;
                    for (;;) {
                        const u32x2 cc = *(volatile LAS u32x2*)(lds + LC_OFF + C_SENT * 4);
                        if ((int)rfl(cc.x) + PWIN > k && (int)rfl(cc.y) >= k) break;
                        __builtin_amdgcn_s_sleep(1);
                        if (sp.over<1023u>(tmo)) break;
                    }
                    asm volatile("" ::: "memory");
#pragma unroll
                    for (int j = 0; j < 2; ++j)
#pragma unroll
                        for (int q = 0; q < NR; ++q) if (q < NR - 1 || own) __builtin_amdgcn_raw_buffer_store_b128(L[j], rs[q], vo + j * 1024, soff, SYS);
                }
#pragma unroll
                for (int q = 0; q < NR; ++q) if (q < NR - 1 || own) __builtin_amdgcn_raw_buffer_store_b128(T, rs[q], vt, soff, SYS);
                if (hdr & 0x80u) {
#pragma unroll
                    for (int q = 0; q < NR; ++q) if (q < NR - 1 || own) __builtin_amdgcn_raw_buffer_store_b128(H[1], rs[q], vo + 3072, soff, SYS);
                }
                SCHED(); asm volatile("s_nop 1" ::: "memory"); SCHED();
                if (k >= KREL) asm volatile("s_waitcnt vmcnt(%0)" :: "i"(SREL) : "memory");
                if (lane == 0) __hip_atomic_fetch_add((LAS unsigned*)(lds + LC_OFF) + C_ISS, 1u, __ATOMIC_RELAXED, __HIP_MEMORY_SCOPE_WORKGROUP);
                unsigned tv = 0;
                if (lane == 0) tv = __hip_atomic_fetch_add(tk, 1u, __ATOMIC_RELAXED, __HIP_MEMORY_SCOPE_AGENT);
                VMWAIT0();
                if (lane == 0) __hip_atomic_fetch_add((LAS unsigned*)(lds + LC_OFF) + C_SENT, 1u, __ATOMIC_RELAXED, __HIP_MEMORY_SCOPE_WORKGROUP);
                if (lane < (own ? NR : NR - 1)) __hip_atomic_store(flag + fi, ((u64)hdr << 32) | call, __ATOMIC_RELAXED, __HIP_MEMORY_SCOPE_SYSTEM);
                u = PW * NCOMM + (rfl(tv) - (call - 1u) * NHU);
                if (u < NHU) {
                    load_x((int)u);
                    unsigned kq = 0;
                    if (lane == 0) kq = __hip_atomic_fetch_add((LAS unsigned*)(lds + LC_OFF) + C_SEQ, 1u, __ATOMIC_RELAXED, __HIP_MEMORY_SCOPE_WORKGROUP);
                    k = PW + (int)rfl(kq);
                }
            }
            if (lane == 0) __hip_atomic_fetch_add((LAS unsigned*)(lds + LC_OFF) + C_CNT, 1u, __ATOMIC_RELAXED, __HIP_MEMORY_SCOPE_WORKGROUP);
        } else {
            WG_START();
            if (wave != 7) return;
            GET_CALL();
            if (lane == 0) __hip_atomic_store((GAS u64*)(me + A1_OFF) + (par * NWG + bx) * 8, (u64)call << 32, __ATOMIC_RELAXED, __HIP_MEMORY_SCOPE_SYSTEM);
            { Spin sp; while (rfl(ctl[C_CNT]) < (unsigned)PW) { __builtin_amdgcn_s_sleep(8); if (sp.over<1023u>(tmo)) break; } }
            if (lane == 0) __hip_atomic_store(seqw, call, __ATOMIC_RELAXED, __HIP_MEMORY_SCOPE_AGENT);
            if (cid < 2) {
                const __amdgpu_buffer_rsrc_t rs = mkrsrc(me + A1_OFF + (size_t)par * NWG * 64, NWG * 64);
                GAS u64* dst = (GAS u64*)(pick_peer(p, lane >> 3) + (cid ? A3_OFF : A2_OFF) + (size_t)(par * 8 + (lane & 7)) * 128) + rank;
                unsigned g = 0, sent = 0, off[4]; Spin sp;
#pragma unroll
                for (int j = 0; j < 4; ++j) off[j] = lane * 64 + j * 4096;
                for (;;) {
                    u32x2 a[4];
#pragma unroll
                    for (int j = 0; j < 4; ++j) a[j] = __builtin_amdgcn_raw_buffer_load_b64(rs, off[j], 0, SYS);
                    int nin = 0;
#pragma unroll
                    for (int j = 0; j < 4; ++j) {
                        if (a[j].y == call) { g = max(g, a[j].x); off[j] = 0x40000000u; }
                        nin += __builtin_popcountll(__builtin_amdgcn_ballot_w64(off[j] == 0x40000000u));
                    }
                    if (nin == NWG) break;
                    if (cid && nin >= SPECN && __any(g > sent)) { sent = wave_max(g); __hip_atomic_store(dst, ((u64)call << 32) | sent, __ATOMIC_RELAXED, __HIP_MEMORY_SCOPE_SYSTEM); }
                    asm volatile("" ::: "memory");
                    if (sp.over<255u>(tmo)) break;
                }
                if (cid == 0) {
                    g = wave_max(g);
                    __hip_atomic_store(dst, ((u64)call << 32) | g, __ATOMIC_RELAXED, __HIP_MEMORY_SCOPE_SYSTEM);
                }
            }
        }
        return;
    }

    if (idx == 2 * CPX - 1 && wave < 4) {
        const char* pa = wave == 0 ? p.peer[0] : wave == 1 ? p.peer[2] : wave == 2 ? p.peer[4] : p.peer[6];
        const char* pb = wave == 0 ? p.peer[1] : wave == 1 ? p.peer[3] : wave == 2 ? p.peer[5] : p.peer[7];
        const int w = (lane >> 1) & 7;
        touch(((lane & 1) ? pb : pa) + (w < 4 ? XS_OFF + (size_t)rank * XS_SRC + (size_t)w * (XS_SRC / 4) : XF_OFF + (size_t)rank * 8 * KS * 16), junk);
    }
    if (wave == 3) {
        const size_t o1 = (size_t)rtile * 128 * NL, o2 = (size_t)(rank * 512 + xrow) * NL;
        touch(lane == 0 ? (const char*)p.out + o1 : lane == 1 ? (const char*)p.out + o2 : lane == 2 ? (const char*)(p.x + (size_t)xrow * KDIM) : me + XS_OFF + (size_t)rtile * (XS_SRC / 4), junk);
    }
    const unsigned tstart = rt32();
    WG_START();
    unsigned call, wfl = 0;
    {
        constexpr int NH = 8 * CPX, NN = NCW - NH, TH = KS - 4 * NN;
        static_assert(TH >= NH && TH <= 2 * NH, "at most one unit per wave of a host");
        const int cm = (((xcd & 1) ? 0 : NH / 2) + (5 - rel) * 2 + (sub >> 1) + TH - NH) % NH;
        const int t1 = host ? (wave < 4 ? NH + cm : ci) : TH + (ci - NH) * 4 + (wave >> 2) * 2, q = wave & 3;
        const bool one = !host || t1 < TH;
        while (rt32() - tstart < (host ? (wave < 4 ? WDELAY2 + cm * WSTAG : WDELAY + ci * WSTAG) : WDELAY)) __builtin_amdgcn_s_sleep(2);
        f32x4 va[16], vb[16];
        w_load(va, p.w, one, ct, t1, q, lane);
        if (!host) w_load(vb, p.w, true, ct, t1 + 1, q, lane);
        const unsigned sv = __hip_atomic_load(seqw, __ATOMIC_RELAXED, __HIP_MEMORY_SCOPE_AGENT);
        SCHED();
        if (one) w_store(va, me, ct, t1, q, lane);
        if (!host) w_store(vb, me, ct, t1 + 1, q, lane);
        if (wave < 4 && lane < NH) wfl = __hip_atomic_load((GAS unsigned*)(me + WF_OFF) + ((ct * 4 + wave) * KS + lane) * 16, __ATOMIC_RELAXED, __HIP_MEMORY_SCOPE_SYSTEM);
        VMWAIT0();
        call = rfl(sv) + 1u;
        if (lane < (host ? (one ? 1 : 0) : 2)) __hip_atomic_store((GAS unsigned*)(me + WF_OFF) + ((ct * 4 + q) * KS + t1 + lane) * 16, call, __ATOMIC_RELAXED, __HIP_MEMORY_SCOPE_SYSTEM);
    }
    const Tmo tmo{(GAS unsigned*)(me + TM_OFF), call};
    const unsigned par = call & 1u;
    if (wave < 4) {
        const __amdgpu_buffer_rsrc_t rsw = mkrsrc(me + WB_OFF + (size_t)(ct * 4 + wave) * KS * 8192, KS * 8192);
        const __amdgpu_buffer_rsrc_t rsf = mkrsrc(me + WF_OFF + (size_t)(ct * 4 + wave) * KS * 64, KS * 64);
        const unsigned ldsb = (unsigned)(size_t)lds;
        unsigned vo = lane * 16;
        const unsigned polla = ldsb + LC_OFF + (C_FULL + (lane & 3)) * 4, donea = ldsb + LC_OFF + (C_DONE + wave) * 4;
        f32x32 r0, r1, r2, r3; f32x16 r4;
        static_assert(LA_SLOT == 9216 && LA_HALF == 27648 && LA_PAR == 55296, "literals in the K loop");
        int s_wk = __builtin_ctzll(__builtin_amdgcn_ballot_w64(wfl < call)) - 1;
        unsigned s_wt, s_ws, s_so, s_t, s_bo, s_bc, s_x, s_sp, s_t0; u64 s_c;
        asm volatile(R"ASM(
.set AG_A, 8
.set AG_C, 112
.macro AG_EXPIRED lout
  s_add_u32 %[sp], %[sp], 1
  s_and_b32 %[x], %[sp], 0x3ff
  s_cbranch_scc1 8f
  v_mov_b32 v6, 0
  global_load_dword v3, v6, %[tmo] sc1
  s_memrealtime %[c]
  s_waitcnt vmcnt(0) lgkmcnt(0)
  v_readfirstlane_b32 %[x], v3
  s_cmp_eq_u32 %[x], %[call]
  s_cbranch_scc1 \lout
  s_mov_b64 vcc, %[c]
  s_cmp_eq_u32 %[sp], 0x400
  s_cselect_b32 %[t0], vcc_lo, %[t0]
  s_sub_u32 %[x], vcc_lo, %[t0]
  s_cmp_le_u32 %[x], 40000000
  s_cbranch_scc1 8f
  v_mov_b32 v3, %[call]
  global_store_dword v6, v3, %[tmo] sc1
  s_waitcnt vmcnt(0)
  s_branch \lout
8:
.endm
; wait until w K-step (t + add + 1) is converted; wk = last K-step known to be converted with all before it
.macro AG_NEEDW add
  s_waitcnt lgkmcnt(0)
  s_mov_b32 %[sp], 0
1:
  v_lshlrev_b32 v6, 2, v0
  buffer_load_dword v3, v6, %[rsf], 0 offen sc0 sc1
  s_waitcnt vmcnt(0)
  v_cmp_gt_u32 vcc, %[call], v3
  s_ff1_i32_b64 %[wk], vcc
  s_min_u32 %[wk], %[wk], 1000
  s_sub_i32 %[wk], %[wk], 1
  s_add_u32 %[x], %[t], \add
  s_cmp_gt_i32 %[wk], %[x]
  s_cbranch_scc1 3f
  s_sleep 2
  AG_EXPIRED 2f
  s_branch 1b
2:
  s_movk_i32 %[wk], 1000
3:
  ds_read_b32 v3, v4
  s_add_u32 %[x], %[t], \add
.endm
; visited when wt <= x: one look at the w flags that costs no wait (requested here, read two steps later), else a real wait
.macro AG_WLOOK add, back
  s_cmp_le_i32 %[wk], %[x]
  s_cbranch_scc1 5f
  s_add_u32 %[ws], %[ws], 1
  s_cmp_eq_u32 %[ws], 1
  s_cbranch_scc0 4f
  v_lshlrev_b32 v1, 2, v0
  buffer_load_dword v1, v1, %[rsf], 0 offen sc0 sc1
  s_add_u32 %[wt], %[x], 2
  s_min_i32 %[wt], %[wt], %[wk]
  s_branch \back
4:
  v_cmp_gt_u32 vcc, %[call], v1
  s_ff1_i32_b64 %[wt], vcc
  s_min_u32 %[wt], %[wt], 1000
  s_sub_i32 %[wt], %[wt], 1
  s_max_i32 %[wk], %[wk], %[wt]
  s_mov_b32 %[ws], 0
  s_sub_i32 %[wt], %[wk], 3
  s_branch \back
5:
  AG_NEEDW \add
  s_mov_b32 %[ws], 0
  s_sub_i32 %[wt], %[wk], 3
  s_branch \back
.endm
; wait until x K-step (t + add) is in the ring, then read its first half
.macro AG_NEEDX add, va, nb
  s_mov_b32 %[sp], 0
1:
  ds_read_b32 v3, v4
  s_add_u32 %[x], %[t], \add
  s_waitcnt lgkmcnt(0)
  v_cmp_lt_u32 vcc, %[x], v3
  s_cmp_eq_u64 vcc, exec
  s_cbranch_scc1 2f
  s_sleep 1
  AG_EXPIRED 2f
  s_branch 1b
2:
  .set ag_b, 0
  .rept 8
  ds_read_b128 v[AG_A+4*ag_b:AG_A+4*ag_b+3], \va offset:(ag_b/4)*27648+(ag_b%%4)*1024
  .set ag_b, ag_b+1
  .endr
  .if \nb == 9
  ds_read_b128 v[AG_A+32:AG_A+35], \va offset:8192
  .endif
.endm
.macro AG_MF b, m, bc
  .set ag_n, \m ^ ((\b & 1) * 3)
  v_mfma_f32_16x16x32_bf16 v[AG_C+16*\b+4*ag_n:AG_C+16*\b+4*ag_n+3], v[\bc+4*ag_n:\bc+4*ag_n+3], v[AG_A+4*\b:AG_A+4*\b+3], v[AG_C+16*\b+4*ag_n:AG_C+16*\b+4*ag_n+3]
.endm
; half K-step on nb row blocks: first = 1 for the first half of step (t + add)
.macro AG_HALF nb, bc, bn, va, vnext, kh, first, add
  s_waitcnt vmcnt(8) lgkmcnt(\nb-3)
  .set ag_b, 0
  .rept \nb
  .if (ag_b == 3) || (ag_b == 6)
  s_waitcnt lgkmcnt(\nb-3)
  .endif
  AG_MF ag_b, 0, \bc
  .if ag_b == 4
  s_add_u32 %[bo], %[bo], 4096
  .endif
  .if \first && (ag_b == 5)
  s_add_u32 %[x], %[t], \add+1
  .endif
  .if \first && \add && (ag_b == 6)
  s_cmp_eq_u32 %[so], 27648
  .endif
  .if (!\first) && (ag_b == 6)
  v_cmp_lt_u32 vcc, %[x], v3
  .endif
  .if \first && \add && (ag_b == 7)
  v_add_u32 \vnext, %[so], v0
  .endif
  .if \first && (!\add) && (ag_b == 7)
  v_add_u32 \vnext, 55296, \va
  .endif
  AG_MF ag_b, 1, \bc
  .if ag_b < 4
  buffer_load_dwordx4 v[\bn+4*ag_b:\bn+4*ag_b+3], v0, %[rsw], %[bc] offen offset:ag_b*1024 sc0 sc1
  .endif
  .if ag_b == 4
  s_min_u32 %[bc], %[bo], 127*4096
  .endif
  .if (!\first) && (ag_b == 5)
  v_mov_b32 v6, %[x]
  .endif
  .if (!\first) && (ag_b == 6)
  ds_write_b32 v5, v6
  .endif
  .if \first && \add && (ag_b == 6)
  s_cselect_b32 %[so], 0, %[so]
  .endif
  .if \first && (ag_b == 7)
  s_cmp_le_i32 %[wt], %[x]
  .endif
  .if (!\first) && (ag_b == 7)
  s_cmp_eq_u64 vcc, exec
  .endif
  AG_MF ag_b, 2, \bc
  .if \first && \add && (ag_b == 5)
  s_add_u32 %[so], %[so], 9216
  .endif
  .if \first && (ag_b == \nb-1)
  ds_read_b32 v3, v4
  .endif
  AG_MF ag_b, 3, \bc
  .if ag_b < 8
  ds_read_b128 v[AG_A+4*ag_b:AG_A+4*ag_b+3], \va offset:(ag_b/4)*27648+\kh*4096+(ag_b%%4)*1024
  .else
  ds_read_b128 v[AG_A+32:AG_A+35], \va offset:8192+\kh*27648
  .endif
  .set ag_b, ag_b+1
  .endr
.endm
.macro AG_LOOP nb
  AG_NEEDX 0, v2, \nb
.Lag_loop\nb:
  AG_HALF \nb, 44, 92, v2, v7, 1, 1, 0
  s_cbranch_scc1 .Lag_w0\nb
.Lag_w0r\nb:
  AG_HALF \nb, 60, 44, v7, v7, 0, 0, 0
  s_cbranch_scc0 .Lag_x0\nb
.Lag_x0r\nb:
  AG_HALF \nb, 76, 60, v7, v2, 1, 1, 1
  s_cbranch_scc1 .Lag_w1\nb
.Lag_w1r\nb:
  AG_HALF \nb, 92, 76, v2, v2, 0, 0, 1
  s_cbranch_scc0 .Lag_x1\nb
.Lag_x1r\nb:
  s_add_u32 %[t], %[t], 2
  s_cmp_lt_u32 %[t], 64
  s_cbranch_scc1 .Lag_loop\nb
  s_branch .Lag_end
.Lag_w0\nb:
  AG_WLOOK 1, .Lag_w0r\nb
.Lag_w1\nb:
  AG_WLOOK 2, .Lag_w1r\nb
.Lag_x0\nb:
  AG_NEEDX 1, v7, \nb
  s_branch .Lag_x0r\nb
.Lag_x1\nb:
  AG_NEEDX 2, v2, \nb
  s_branch .Lag_x1r\nb
.endm
  s_nop 4
  s_mov_b32 %[t], 0
  s_mov_b32 %[so], 0
  s_cmp_gt_i32 %[wk], 0
  s_cbranch_scc1 .Lag_go
  AG_NEEDW 0
.Lag_go:
  s_mov_b32 %[ws], 0
  s_sub_i32 %[wt], %[wk], 3
  s_min_i32 %[wt], %[wt], 12
  .set ag_i, 0
  .rept 12
  s_movk_i32 %[bc], (ag_i/4)*4096
  buffer_load_dwordx4 v[44+4*ag_i:44+4*ag_i+3], v0, %[rsw], %[bc] offen offset:(ag_i%%4)*1024 sc0 sc1
  .set ag_i, ag_i+1
  .endr
  s_movk_i32 %[bo], 3*4096
  s_movk_i32 %[bc], 3*4096
  .set ag_i, 0
  .rept 72
  v_mov_b64 v[AG_C+ag_i:AG_C+ag_i+1], 0
  .set ag_i, ag_i+2
  .endr
  v_mov_b32 v2, v0
  s_cmp_eq_u32 %[host], 0
  s_cbranch_scc1 .Lag_eight
  AG_LOOP 9
.Lag_eight:
  AG_LOOP 8
.Lag_end:
  s_waitcnt vmcnt(0) lgkmcnt(0)
  s_nop 7
)ASM"
            : "={v[112:143]}"(r0), "={v[144:175]}"(r1), "={v[176:207]}"(r2), "={v[208:239]}"(r3), "={v[240:255]}"(r4),
              [t] "=&s"(s_t), [wk] "+s"(s_wk), [bo] "=&s"(s_bo), [bc] "=&s"(s_bc), [x] "=&s"(s_x), [sp] "=&s"(s_sp), [t0] "=&s"(s_t0),
              [wt] "=&s"(s_wt), [ws] "=&s"(s_ws), [so] "=&s"(s_so), [c] "=&s"(s_c), "+{v0}"(vo)
            : "{v4}"(polla), "{v5}"(donea), [rsw] "s"(rsw), [rsf] "s"(rsf), [tmo] "s"(tmo.word), [call] "s"(call), [host] "s"(rfl(host ? 1u : 0u))
            : "memory", "vcc", "scc", "v1", "v2", "v3", "v6", "v7", "v8", "v9", "v10", "v11", "v12", "v13", "v14", "v15", "v16", "v17", "v18", "v19", "v20", "v21", "v22", "v23", "v24", "v25", "v26", "v27", "v28", "v29", "v30", "v31", "v32", "v33", "v34", "v35", "v36", "v37", "v38", "v39", "v40", "v41", "v42", "v43", "v44", "v45", "v46", "v47", "v48", "v49", "v50", "v51", "v52", "v53", "v54", "v55", "v56", "v57", "v58", "v59", "v60", "v61", "v62", "v63", "v64", "v65", "v66", "v67", "v68", "v69", "v70", "v71", "v72", "v73", "v74", "v75", "v76", "v77", "v78", "v79", "v80", "v81", "v82", "v83", "v84", "v85", "v86", "v87", "v88", "v89", "v90", "v91", "v92", "v93", "v94", "v95", "v96", "v97", "v98", "v99", "v100", "v101", "v102", "v103", "v104", "v105", "v106", "v107");
        f32x4 acc[9][4];
#pragma unroll
        for (int i = 0; i < 144; ++i) acc[i >> 4][(i >> 2) & 3][i & 3] = i < 32 ? r0[i & 31] : i < 64 ? r1[i & 31] : i < 96 ? r2[i & 31] : i < 128 ? r3[i & 31] : r4[i & 15];
        const int ln = (int)(vo >> 4);
        int mx = 0;
#pragma unroll
        for (int n = 0; n < 4; ++n) acc[8][n] *= XSCALE;
        if (host) {
#pragma unroll
            for (int n = 0; n < 4; ++n) { mx = max(max(mx, fbits(acc[8][n][0])), fbits(acc[8][n][1])); mx = max(max(mx, fbits(acc[8][n][2])), fbits(acc[8][n][3])); }
        }
        int m2 = 0;
#pragma unroll
        for (int b = 0; b < 8; ++b)
#pragma unroll
            for (int n = 0; n < 4; ++n) { mx = max(max(mx, fbits(acc[b][n][0])), fbits(acc[b][n][1])); m2 = max(max(m2, fbits(acc[b][n][2])), fbits(acc[b][n][3])); }
        const unsigned wm = wave_max((unsigned)max(mx, m2));
        if (ln == 0) {
            __hip_atomic_fetch_max((LAS unsigned*)(lds + LC_OFF) + C_MAX, wm, __ATOMIC_RELAXED, __HIP_MEMORY_SCOPE_WORKGROUP);
            asm volatile("" ::: "memory");
            const unsigned arrived = __hip_atomic_fetch_add((LAS unsigned*)(lds + LC_OFF) + C_CNT, 1u, __ATOMIC_RELAXED, __HIP_MEMORY_SCOPE_WORKGROUP);
            asm volatile("" ::: "memory");
            const unsigned tm = ctl[C_MAX];
            if (arrived == 3u) __hip_atomic_store((GAS u64*)(me + A1_OFF) + (par * NWG + bx) * 8, ((u64)call << 32) | tm, __ATOMIC_RELAXED, __HIP_MEMORY_SCOPE_SYSTEM);
        }
        const __amdgpu_buffer_rsrc_t rso = mkrsrc(p.out, 8u * 512u * NL);
        const unsigned ovo = (unsigned)(ln & 15) * NL + ct * 256 + wave * 64 + (ln >> 4) * 16;
        unsigned used = 0xffffffffu; Spin sp;
#pragma unroll 1
        for (;;) {
            const u32x2 c = *(volatile LAS u32x2*)(lds + LC_OFF + C_GMAX * 4);
            unsigned st = rfl(c.y);
            const unsigned run = rfl(c.x), g = max(run, wm);
            if (st == 0u && sp.over<4095u>(tmo)) st = 2u;
            if (st == 2u || (g != used && (st != 0u || run != 0u))) {
                if (used != 0xffffffffu) VMWAIT0();
                used = g;
                const float scale = __uint_as_float(g) / 127.0f;
                const float inv = scale > 0.f ? 1.0f / scale : 0.f;
#pragma unroll
                for (int b = 0; b < 9; ++b) {
                    u32x4 o = {0u, 0u, 0u, 0u};
#pragma unroll
                    for (int cc = 0; cc < 4; ++cc) {
                        f32x4 q = acc[b][cc] * inv;
                        asm volatile("" : "+v"(q));
#pragma unroll
                        for (int r = 0; r < 4; ++r) o[r] = __builtin_amdgcn_cvt_pk_u8_f32(q[r], cc, o[r]);
                    }
                    if (st == 2u) o = (u32x4){0x80808080u, 0x80808080u, 0x80808080u, 0x80808080u};
                    if (b < 8 || host) __builtin_amdgcn_raw_buffer_store_b128(o, rso, ovo + (unsigned)(b < 8 ? rtile * 128 + b * 16 : rank * 512 + xrow) * NL, 0, SYS);
                }
            }
            if (st != 0u) break;
            __builtin_amdgcn_s_sleep(1);
        }
    } else {
        const int f = wave & 1, ph = (wave >> 1) & 1, rb = rsub * 2 + f;
        const __amdgpu_buffer_rsrc_t rsx = mkrsrc(me + XS_OFF + (size_t)src * XS_SRC + (size_t)rb * KS * UNIT, KS * UNIT);
        const __amdgpu_buffer_rsrc_t rse = mkrsrc(p.x + (size_t)xrow * KDIM + f * 32, host ? 16u * KDIM * 4u - 128u : 0u);
        const __amdgpu_buffer_rsrc_t rsf = mkrsrc(me + XF_OFF + (size_t)(src * 8 + rb) * KS * 16, KS * 16);
        const unsigned ldsb = (unsigned)(size_t)lds;
        const unsigned mbase = ldsb + f * LA_HALF, vpub = ldsb + LC_OFF + (C_FULL + (wave & 3)) * 4, vdone = ldsb + LC_OFF + (C_DONE + (lane & 3)) * 4;
        const unsigned hostf = rfl(host ? 1u : 0u);
        unsigned s_cur = ph, s_lim = NST, s_so = 0, s_h, s_g, s_b, s_x, s_y, s_sp, s_t0, s_m0; u64 s_seen, s_c;
        asm volatile(R"ASM(
.macro FD_EXPIRED lout
  s_add_u32 %[sp], %[sp], 1
  s_and_b32 %[x], %[sp], 0xff
  s_cbranch_scc1 8f
  v_mov_b32 v9, 0
  global_load_dword v12, v9, %[tmo] sc1
  s_memrealtime %[c]
  s_waitcnt vmcnt(0) lgkmcnt(0)
  v_readfirstlane_b32 %[x], v12
  s_cmp_eq_u32 %[x], %[call]
  s_cbranch_scc1 \lout
  s_mov_b64 vcc, %[c]
  s_cmp_eq_u32 %[sp], 0x100
  s_cselect_b32 %[t0], vcc_lo, %[t0]
  s_sub_u32 %[x], vcc_lo, %[t0]
  s_cmp_le_u32 %[x], 40000000
  s_cbranch_scc1 8f
  v_mov_b32 v12, %[call]
  global_store_dword v9, v12, %[tmo] sc1
  s_waitcnt vmcnt(0)
  s_branch \lout
8:
.endm
.macro FD_FLAGS
  buffer_load_dwordx4 v[100:103], v0, %[rsf], 0 offen sc0 sc1
  s_waitcnt vmcnt(0)
  v_min_u32 v9, v100, v102
  v_cmp_le_u32 %[seen], %[call], v9
.endm
; loads of unit %[y] into the set at r
.macro FD_ISSUE r
  s_lshl_b32 %[x], %[y], 13
  s_lshl_b32 %[b], %[y], 8
  buffer_load_dwordx4 v[\r:\r+3], v0, %[rsx], %[x] offen sc0 sc1
  buffer_load_dwordx4 v[\r+4:\r+7], v0, %[rsx], %[x] offen offset:1024 sc0 sc1
  buffer_load_dwordx3 v[\r+16:\r+18], v1, %[rsx], %[x] offen offset:2048 sc0 sc1
  buffer_load_ushort v[\r+19], v2, %[rsx], %[x] offen offset:2816 sc0 sc1
  s_bitset1_b32 %[x], 12
  buffer_load_dwordx4 v[\r+24:\r+27], v3, %[rse], %[b] offen
  buffer_load_dwordx4 v[\r+28:\r+31], v3, %[rse], %[b] offen offset:64
  buffer_load_dwordx4 v[\r+8:\r+11], v0, %[rsx], %[x] offen sc0 sc1
  buffer_load_dwordx4 v[\r+12:\r+15], v0, %[rsx], %[x] offen offset:1024 sc0 sc1
  buffer_load_dwordx3 v[\r+20:\r+22], v1, %[rsx], %[x] offen offset:2048 sc0 sc1
  buffer_load_ushort v[\r+23], v2, %[rsx], %[x] offen offset:2816 sc0 sc1
.endm
; wait for the flag of unit %[y]
.macro FD_WAITFLAG
  s_mov_b32 %[sp], 0
1:
  FD_FLAGS
  s_bitcmp1_b64 %[seen], %[y]
  s_cbranch_scc1 2f
  s_sleep 1
  FD_EXPIRED 2f
  s_branch 1b
2:
.endm
.macro FD_DEC h, n0, n1, n2
  v_and_or_b32 v[\h], v[\n0], v7, %[b]
  v_alignbit_b32 v12, v[\n0], v[\n0], 3
  v_and_or_b32 v[\h+2], v[\n1], v7, %[b]
  v_alignbit_b32 v13, v[\n1], v[\n1], 3
  v_and_or_b32 v[\h+4], v[\n2], v7, %[b]
  v_alignbit_b32 v14, v[\n2], v[\n2], 3
  v_and_or_b32 v[\h+1], v12, v7, %[b]
  v_lshrrev_b32 v12, 5, v[\n0]
  v_and_or_b32 v[\h+3], v13, v7, %[b]
  v_lshrrev_b32 v13, 5, v[\n1]
  v_and_or_b32 v[\h+5], v14, v7, %[b]
  v_lshlrev_b32 v14, 1, v[\n2]
  v_bfi_b32 v12, v8, v14, v12
  v_lshlrev_b32 v15, 2, v[\n2]
  v_and_or_b32 v[\h+6], v12, v7, %[b]
  v_bfi_b32 v13, v8, v15, v13
  v_and_or_b32 v[\h+7], v13, v7, %[b]
.endm
; exception entry in register e -> address v12, mask v13
.macro FD_EXC e
  v_and_b32 v12, 0x7fe, v[\e]
  v_and_b32 v14, 1, v[\e]
  v_lshrrev_b32 v13, 11, v[\e]
  v_lshl_or_b32 v14, v14, 4, 10
  v_lshl_add_u32 v12, v12, 1, %[x]
  v_lshlrev_b32 v13, v14, v13
.endm
; a half sent raw, or with a window that does not start at a multiple of four
.macro FD_FIX hd, h
  s_bitcmp1_b32 \hd, 7
  s_cbranch_scc0 3f
  buffer_load_dwordx4 v[\h:\h+3], v0, %[rsx], %[x] offen offset:2048 sc0 sc1
  buffer_load_dwordx4 v[\h+4:\h+7], v0, %[rsx], %[x] offen offset:3072 sc0 sc1
  s_waitcnt vmcnt(0)
3:
  s_and_b32 %[b], \hd, 3
  s_mul_i32 %[b], %[b], 0x01010101
  .set fd_i, 0
  .rept 8
  v_add_u32 v[\h+fd_i], %[b], v[\h+fd_i]
  .set fd_i, fd_i+1
  .endr
.endm
; one unit: set r holds the loads of unit cur, set q takes the next one; ob = offset of this parity's ring
.macro FD_STEP r, q, ob, id
  s_add_u32 %[y], %[cur], 2
  s_cmp_ge_u32 %[y], 64
  s_cbranch_scc1 .Lfd_last\id
  s_bitcmp1_b64 %[seen], %[y]
  s_cbranch_scc0 .Lfd_look\id
  FD_ISSUE \q
  s_waitcnt vmcnt(10)
.Lfd_go\id:
  v_readlane_b32 %[h], v101, %[cur]
  v_readlane_b32 %[g], v103, %[cur]
  s_and_b32 %[b], %[h], 0x7c
  s_mul_i32 %[b], %[b], 0x01010101
  FD_DEC 80, \r+16, \r+17, \r+18
  s_and_b32 %[b], %[g], 0x7c
  s_mul_i32 %[b], %[b], 0x01010101
  s_add_u32 m0, %[mb], %[so]
  FD_DEC 88, \r+20, \r+21, \r+22
  s_or_b32 %[x], %[h], %[g]
  s_and_b32 %[x], %[x], 0x83
  s_cbranch_scc1 .Lfd_fix\id
.Lfd_fixed\id:
  s_cmp_ge_u32 %[cur], %[lim]
  s_cbranch_scc1 .Lfd_free\id
.Lfd_freed\id:
  .set fd_i, 0
  .rept 16
  v_perm_b32 v[96+(fd_i%%2)*2], v[80+fd_i], v[\r+fd_i], %[p0]
  v_perm_b32 v[97+(fd_i%%2)*2], v[80+fd_i], v[\r+fd_i], %[p1]
  ds_write_addtid_b32 v[96+(fd_i%%2)*2] offset:\ob+fd_i*512
  ds_write_addtid_b32 v[97+(fd_i%%2)*2] offset:\ob+fd_i*512+256
  .set fd_i, fd_i+1
  .endr
  s_add_u32 %[x], m0, \ob
  FD_EXC \r+19
  s_lshr_b32 %[b], %[h], 8
  s_bfm_b64 exec, %[b], 0
  s_lshr_b32 %[b], %[g], 8
  ds_xor_b32 v12, v13
  s_bfm_b64 exec, %[b], 0
  s_add_u32 %[so], %[so], 9216
  FD_EXC \r+23
  ds_xor_b32 v12, v13 offset:4096
  s_mov_b64 exec, -1
  s_cmp_eq_u32 %[host], 0
  s_cbranch_scc1 .Lfd_pub\id
  v_cvt_pk_bf16_f32 v96, v[\r+24], v[\r+25]
  v_cvt_pk_bf16_f32 v97, v[\r+26], v[\r+27]
  v_cvt_pk_bf16_f32 v98, v[\r+28], v[\r+29]
  v_cvt_pk_bf16_f32 v99, v[\r+30], v[\r+31]
  ds_write_addtid_b32 v96 offset:\ob+8192
  ds_write_addtid_b32 v97 offset:\ob+8192+256
  ds_write_addtid_b32 v98 offset:\ob+8192+512
  ds_write_addtid_b32 v99 offset:\ob+8192+768
.Lfd_pub\id:
  s_add_u32 %[cur], %[cur], 2
  s_cmp_eq_u32 %[so], 27648
  s_cselect_b32 %[so], 0, %[so]
  s_cmp_ge_u32 %[cur], 64
  s_cselect_b32 %[x], 1000, %[cur]
  v_mov_b32 v9, %[x]
  s_waitcnt lgkmcnt(0)
  ds_write_b32 v4, v9
  s_cbranch_scc1 .Lfd_end
  s_cmp_eq_u32 %[y], %[cur]
  s_cbranch_scc0 .Lfd_late\id
.Lfd_next\id:
.endm
; out-of-line parts of a step
.macro FD_SLOW r, q, ob, id
.Lfd_last\id:
  s_waitcnt vmcnt(0)
  s_branch .Lfd_go\id
.Lfd_look\id:
  FD_FLAGS
  s_bitcmp1_b64 %[seen], %[y]
  s_cbranch_scc0 3f
  FD_ISSUE \q
  s_branch .Lfd_go\id
3:
  s_mov_b32 %[y], -1
  s_branch .Lfd_go\id
.Lfd_late\id:
  s_mov_b32 %[y], %[cur]
  FD_WAITFLAG
  FD_ISSUE \q
  s_branch .Lfd_next\id
.Lfd_fix\id:
  s_lshl_b32 %[x], %[cur], 13
  FD_FIX %[h], 80
  s_bitset1_b32 %[x], 12
  FD_FIX %[g], 88
  s_branch .Lfd_fixed\id
.Lfd_free\id:
  s_mov_b32 %[sp], 0
1:
  ds_read_b32 v9, v5
  s_waitcnt lgkmcnt(0)
  v_readlane_b32 %[lim], v9, 0
  v_readlane_b32 %[x], v9, 1
  s_min_u32 %[lim], %[lim], %[x]
  v_readlane_b32 %[x], v9, 2
  s_min_u32 %[lim], %[lim], %[x]
  v_readlane_b32 %[x], v9, 3
  s_min_u32 %[lim], %[lim], %[x]
  s_add_u32 %[lim], %[lim], 6
  s_cmp_lt_u32 %[cur], %[lim]
  s_cbranch_scc1 .Lfd_freed\id
  s_sleep 4
  FD_EXPIRED .Lfd_freed\id
  s_branch 1b
.endm
  s_nop 4
  s_mov_b32 %[m0s], m0
  v_mov_b32 v7, 0x83838383
  v_mov_b32 v8, 0x80808080
  s_mov_b32 %[y], %[cur]
  FD_WAITFLAG
  FD_ISSUE 16
  s_bitcmp1_b32 %[cur], 0
  s_cbranch_scc1 .Lfd_loop1
.Lfd_loop0:
  FD_STEP 16, 48, 0, 0
  FD_STEP 48, 16, 0, 1
  s_branch .Lfd_loop0
.Lfd_loop1:
  FD_STEP 16, 48, 55296, 2
  FD_STEP 48, 16, 55296, 3
  s_branch .Lfd_loop1
  FD_SLOW 16, 48, 0, 0
  FD_SLOW 48, 16, 0, 1
  FD_SLOW 16, 48, 55296, 2
  FD_SLOW 48, 16, 55296, 3
.Lfd_end:
  s_mov_b32 m0, %[m0s]
  s_waitcnt vmcnt(0) lgkmcnt(0)
)ASM"
            : [cur] "+s"(s_cur), [lim] "+s"(s_lim), [so] "+s"(s_so), [h] "=&s"(s_h), [g] "=&s"(s_g), [b] "=&s"(s_b), [x] "=&s"(s_x), [y] "=&s"(s_y), [sp] "=&s"(s_sp),
              [t0] "=&s"(s_t0), [m0s] "=&s"(s_m0), [seen] "=&s"(s_seen), [c] "=&s"(s_c)
            : "{v0}"(lane * 16), "{v1}"(lane * 12), "{v2}"(lane * 2), "{v3}"((lane >> 2) * KDIM * 4 + (lane & 3) * 16), "{v4}"(vpub), "{v5}"(vdone),
              [rsx] "s"(rsx), [rse] "s"(rse), [rsf] "s"(rsf), [tmo] "s"(tmo.word), [call] "s"(call), [mb] "s"(mbase), [host] "s"(hostf),
              [p0] "s"(0x05010400u), [p1] "s"(0x07030602u)
            : "memory", "vcc", "scc", "v7", "v8", "v9", "v10", "v11", "v12", "v13", "v14", "v15", "v16", "v17", "v18", "v19", "v20", "v21", "v22", "v23", "v24", "v25", "v26", "v27", "v28", "v29", "v30", "v31", "v32", "v33", "v34", "v35", "v36", "v37", "v38", "v39", "v40", "v41", "v42", "v43", "v44", "v45", "v46", "v47", "v48", "v49", "v50", "v51", "v52", "v53", "v54", "v55", "v56", "v57", "v58", "v59", "v60", "v61", "v62", "v63", "v64", "v65", "v66", "v67", "v68", "v69", "v70", "v71", "v72", "v73", "v74", "v75", "v76", "v77", "v78", "v79", "v80", "v81", "v82", "v83", "v84", "v85", "v86", "v87", "v88", "v89", "v90", "v91", "v92", "v93", "v94", "v95", "v96", "v97", "v98", "v99", "v100", "v101", "v102", "v103");
        static_assert(DEADLINE == 40000000u && NST == 6, "literals in the feeder loop");
        if (wave != 4) return;
        if (lane == 0) __hip_atomic_store(seqw, call, __ATOMIC_RELAXED, __HIP_MEMORY_SCOPE_AGENT);
        {
            const GAS u64* a2 = lane < 16 ? (const GAS u64*)(me + (lane < 8 ? A2_OFF : A3_OFF) + (size_t)(par * 8 + (bx & 7)) * 128) + (lane & 7) : (const GAS u64*)(me + TM_OFF);
            unsigned told = 0; Spin sp;
            for (;;) {
                const u64 v = __hip_atomic_load(a2, __ATOMIC_RELAXED, __HIP_MEMORY_SCOPE_SYSTEM);
                const bool in = lane < 16 && (unsigned)(v >> 32) == call;
                unsigned g = in ? (unsigned)v : 0u;
                g = max(g, (unsigned)__builtin_amdgcn_update_dpp(0u, g, 0x111, 0xf, 0xf, true));
                g = max(g, (unsigned)__builtin_amdgcn_update_dpp(0u, g, 0x112, 0xf, 0xf, true));
                g = max(g, (unsigned)__builtin_amdgcn_update_dpp(0u, g, 0x114, 0xf, 0xf, true));
                g = max(g, (unsigned)__builtin_amdgcn_update_dpp(0u, g, 0x118, 0xf, 0xf, true));
                g = __builtin_amdgcn_readlane(g, 15);
                const unsigned im = (unsigned)__builtin_amdgcn_ballot_w64(in);
                unsigned st = (im & 0xffu) == 0xffu ? 1u : 0u;
                if (((im | (im >> 8)) & 0xffu) != 0xffu) g = 0u;
                if ((unsigned)__builtin_amdgcn_readlane((unsigned)v, 16) == call || (st == 0u && sp.over<255u>(tmo))) st = 2u;
                if (st != 0u || g != told) { told = g; *(volatile LAS u32x2*)(lds + LC_OFF + C_GMAX * 4) = (u32x2){g, st}; }
                if (st != 0u) break;
                __builtin_amdgcn_s_sleep(1);
            }
        }
    }
}

}

extern "C" void kernel_launch(int rank, int nranks, ncclComm_t comm,
                              hipStream_t stream,
                              void* const* d_in, const int* in_sizes, int n_in,
                              void* d_out, int out_size,
                              const int* in_dtypes, int out_dtype,
                              void* const* d_out_all,
                              void* const* d_ws_all, size_t ws_bytes) {
    if (nranks != NR || (unsigned)rank >= (unsigned)NR || ws_bytes < WS_NEED || n_in < 2) {
        fprintf(stderr, "unsupported configuration\n");
        return;
    }
    HIP_CHECK(hipFuncSetAttribute((const void*)ag_gemm, hipFuncAttributeMaxDynamicSharedMemorySize, LDS_BYTES));
    Params p;
    p.x = (const float*)d_in[0]; p.w = (const float*)d_in[1]; p.out = (signed char*)d_out;
    const size_t base = (ws_bytes - WS_NEED) & ~(size_t)0x1fffff;
    for (int i = 0; i < NR; ++i) p.peer[i] = (char*)d_ws_all[(rank + 1 + i) % NR] + base;
    p.rank = rank; p.pad = 0;
    hipLaunchKernelGGL(ag_gemm, dim3(NWG), dim3(512), LDS_BYTES, stream, p);
    HIP_CHECK(hipGetLastError());
}
```

```cpp
#include <hip/hip_runtime.h>
#include <rccl/rccl.h>
#include <cstdio>

#define HIP_CHECK(expr) do { hipError_t _e = (expr); if (_e != hipSuccess) { \
    fprintf(stderr, "HIP error %s at %s:%d\n", hipGetErrorString(_e), __FILE__, __LINE__); \
    return; } } while (0)

namespace {

#define LAS __attribute__((address_space(3)))
#define GAS __attribute__((address_space(1)))
typedef unsigned u32x4 __attribute__((ext_vector_type(4)));
typedef unsigned u32x2 __attribute__((ext_vector_type(2)));
typedef float f32x4 __attribute__((ext_vector_type(4)));
typedef float f32x16 __attribute__((ext_vector_type(16)));
typedef float f32x32 __attribute__((ext_vector_type(32)));
typedef unsigned long long u64;

constexpr int NR = 8;
constexpr int KDIM = 4096, NL = 2048;
constexpr int KS = 64;
constexpr int NWG = 256;
constexpr int UNIT = 8192;
constexpr int EXCAP = 63;

constexpr size_t XS_OFF = 0;
constexpr size_t XS_SRC = (size_t)8 * KS * UNIT;
constexpr size_t WB_OFF = XS_OFF + NR * XS_SRC;
constexpr size_t XF_OFF = WB_OFF + (size_t)32 * KS * 8192;
constexpr size_t WF_OFF = XF_OFF + (size_t)NR * 8 * KS * 16;
constexpr size_t SQ_OFF = WF_OFF + (size_t)32 * KS * 64;
constexpr size_t A1_OFF = SQ_OFF + (size_t)NWG * 128;
constexpr size_t A2_OFF = A1_OFF + (size_t)2 * NWG * 64;
constexpr size_t A3_OFF = A2_OFF + (size_t)2 * 8 * 128;
constexpr size_t TK_OFF = A3_OFF + (size_t)2 * 8 * 128;
constexpr size_t TM_OFF = TK_OFF + 128;
constexpr size_t WS_NEED = TM_OFF + 128;

constexpr int NST = 6, LA_SLOT = 9216, LA_HALF = 3 * LA_SLOT, LA_PAR = 2 * LA_HALF;
constexpr int LC_OFF = 2 * LA_PAR;
constexpr int C_FULL = 0;
constexpr int C_DONE = 4;
constexpr int C_MAX = 8, C_CNT = 9, C_GMAX = 10  , C_SENT = 12, C_ISS = 13, C_SEQ = 14;
constexpr int LJ_OFF = LC_OFF + 64;
constexpr int LDS_BYTES = LJ_OFF + 256;
static_assert(LDS_BYTES > 81920, "one workgroup per CU");

constexpr int CPX = 3;
constexpr int NWO = 4, TWO = 8;
constexpr int PW = 6;
constexpr int PWIN = 3;
constexpr int PEARLY = 2;
constexpr unsigned WDELAY = 20, WDELAY2 = 340, WSTAG = 12, MDELAY = 560;
constexpr int SPECN = 168;
constexpr int CODE_BYTES = 32000;
constexpr int NCOMM = 8 * CPX - NWO;
constexpr float XSCALE = 0.5f;
constexpr unsigned DEADLINE = 40000000u;

struct Params {
    const float* x; const float* w; signed char* out;
    char* peer[NR];
    int rank; int pad;
};

__device__ __forceinline__ unsigned rt32() { return (unsigned)__builtin_amdgcn_s_memrealtime(); }
__device__ __forceinline__ unsigned rfl(unsigned v) { return __builtin_amdgcn_readfirstlane(v); }
__device__ __forceinline__ unsigned cvt_pk(float lo, float hi) {
    typedef float f2 __attribute__((ext_vector_type(2))); typedef __bf16 b2 __attribute__((ext_vector_type(2)));
    f2 v = {lo, hi}; return __builtin_bit_cast(unsigned, __builtin_convertvector(v, b2));
}
__device__ __forceinline__ __amdgpu_buffer_rsrc_t mkrsrc(const void* p, unsigned bytes) {
    return __builtin_amdgcn_make_buffer_rsrc((void*)p, 0, bytes, 0x00020000);
}
__device__ __forceinline__ int fbits(float v) { return __float_as_int(v); }
__device__ __forceinline__ unsigned rotr(unsigned v, int n) { return __builtin_amdgcn_alignbit(v, v, n); }
#define VMWAIT0() asm volatile("s_waitcnt vmcnt(0)" ::: "memory")
#define LGKM0() asm volatile("s_waitcnt lgkmcnt(0)" ::: "memory")
#define SCHED() __builtin_amdgcn_sched_barrier(0)
constexpr int SYS = 17;

struct Tmo {
    GAS unsigned* word; unsigned call;
    __device__ __forceinline__ bool hit() const {
        unsigned v;
        asm volatile("v_mov_b32 %0, 0\n\tglobal_load_dword %0, %0, %1 sc1\n\ts_waitcnt vmcnt(0)" : "=&v"(v) : "s"(word) : "memory");
        return rfl(v) == call;
    }
    __device__ __forceinline__ bool expired(unsigned t0) const {
        if (hit() || rt32() - t0 > DEADLINE) {
            unsigned z, c;
            asm volatile("v_mov_b32 %0, 0\n\tv_mov_b32 %1, %3\n\tglobal_store_dword %0, %1, %2 sc1\n\ts_waitcnt vmcnt(0)" : "=&v"(z), "=&v"(c) : "s"(word), "s"(call) : "memory");
            return true;
        }
        return false;
    }
};

struct Spin {
    unsigned n = 0, t0 = 0;
    template <unsigned MASK> __device__ __forceinline__ bool over(const Tmo& tmo) {
        if ((++n & MASK) != 0u) return false;
        if (n == MASK + 1u) { t0 = rt32(); return false; }
        return tmo.expired(t0);
    }
};


__device__ __forceinline__ void w_load(f32x4 (&v)[16], const float* __restrict__ w, bool on, int ct, int t, int q, int lane) {
    const __amdgpu_buffer_rsrc_t rs = mkrsrc(w + (size_t)t * 64 * NL + ct * 256 + q * 64, on ? 63u * NL * 4u + 256u : 0u);
    const int n = lane & 15, g = lane >> 4;
    const unsigned vo = (unsigned)((g >> 1) * 16 + (g & 1) * 2) * NL * 4 + n * 16;
#pragma unroll
    for (int i = 0; i < 16; ++i) v[i] = __builtin_bit_cast(f32x4, __builtin_amdgcn_raw_buffer_load_b128(rs, vo, ((i >> 3) * 32 + ((i >> 1) & 3) * 4 + (i & 1)) * NL * 4, 2));
}
__device__ __forceinline__ void w_store(const f32x4 (&v)[16], char* me, int ct, int t, int q, int lane) {
    const __amdgpu_buffer_rsrc_t rs = mkrsrc(me + WB_OFF + ((size_t)(ct * 4 + q) * KS + t) * 8192, 8192);
#pragma unroll
    for (int kh = 0; kh < 2; ++kh)
#pragma unroll
        for (int c = 0; c < 4; ++c) {
            u32x4 o;
            o.x = cvt_pk(v[kh * 8 + 0][c], v[kh * 8 + 1][c]); o.y = cvt_pk(v[kh * 8 + 2][c], v[kh * 8 + 3][c]);
            o.z = cvt_pk(v[kh * 8 + 4][c], v[kh * 8 + 5][c]); o.w = cvt_pk(v[kh * 8 + 6][c], v[kh * 8 + 7][c]);
            __builtin_amdgcn_raw_buffer_store_b128(o, rs, lane * 16 + c * 1024 + kh * 4096, 0, SYS);
        }
}

__device__ __forceinline__ char* pick_peer(const Params& p, int q) {
    char* r = p.peer[0];
#pragma unroll
    for (int i = 1; i < NR; ++i) { char* c = p.peer[i]; asm volatile("" : "+s"(c)); r = q == i ? c : r; }
    return r;
}

__device__ __forceinline__ void touch(const void* ptr, unsigned junk) {
    unsigned keep;
    asm volatile("s_mov_b32 %0, m0\n\ts_mov_b32 m0, %2\n\ts_nop 0\n\tglobal_load_lds_dword %1, off\n\ts_mov_b32 m0, %0" : "=&s"(keep) : "v"(ptr), "s"(junk) : "memory");
}

__device__ __forceinline__ unsigned wave_max(unsigned g) {
    g = max(g, (unsigned)__builtin_amdgcn_update_dpp(0u, g, 0x111, 0xf, 0xf, true));
    g = max(g, (unsigned)__builtin_amdgcn_update_dpp(0u, g, 0x112, 0xf, 0xf, true));
    g = max(g, (unsigned)__builtin_amdgcn_update_dpp(0u, g, 0x114, 0xf, 0xf, true));
    g = max(g, (unsigned)__builtin_amdgcn_update_dpp(0u, g, 0x118, 0xf, 0xf, true));
    return max(max((unsigned)__builtin_amdgcn_readlane(g, 15), (unsigned)__builtin_amdgcn_readlane(g, 31)), max((unsigned)__builtin_amdgcn_readlane(g, 47), (unsigned)__builtin_amdgcn_readlane(g, 63)));
}

__global__ void __launch_bounds__(512, 2) ag_gemm(Params p) {
    unsigned ka0, ka1;
    asm volatile("s_load_dword %0, %2, 0x0\n\ts_load_dword %1, %2, 0x40" : "=&s"(ka0), "=&s"(ka1) : "s"(__builtin_amdgcn_kernarg_segment_ptr()));
    extern __shared__ __attribute__((aligned(16))) char lds_raw[];
    LAS char* lds = (LAS char*)lds_raw;
    volatile LAS unsigned* ctl = (volatile LAS unsigned*)(lds + LC_OFF);
    const int lane = threadIdx.x & 63, wave = rfl(threadIdx.x >> 6);
    const int bx = blockIdx.x, xcd = bx & 7, idx = bx >> 3;
    const unsigned junk = (unsigned)(size_t)(lds + LJ_OFF);
    if (wave == 7 && idx < 2) {
        static_assert(CODE_BYTES > 3 * 8192 && CODE_BYTES <= 4 * 8192, "four reads");
        unsigned keep, o;
        asm volatile("s_mov_b32 %0, m0\n\ts_mov_b32 m0, %4\n\tv_add_u32 %1, 8192, %2\n\tglobal_load_lds_dword %2, %3\n\tglobal_load_lds_dword %1, %3\n\t"
                     "v_add_u32 %1, 16384, %2\n\tglobal_load_lds_dword %1, %3\n\tv_add_u32 %1, 24576, %2\n\tv_min_u32 %1, %5, %1\n\tglobal_load_lds_dword %1, %3\n\ts_mov_b32 m0, %0"
                     : "=&s"(keep), "=&v"(o) : "v"(lane * 128), "s"(__builtin_amdgcn_s_getpc()), "s"(junk), "i"(CODE_BYTES) : "memory");
    }
    asm volatile("" :: "s"(p.x), "s"(p.w), "s"(p.out), "s"(p.rank), "s"(p.peer[0]), "s"(p.peer[1]), "s"(p.peer[2]), "s"(p.peer[3]),
                       "s"(p.peer[4]), "s"(p.peer[5]), "s"(p.peer[6]), "s"(p.peer[7]), "s"(ka0), "s"(ka1));
    const int rank = p.rank & 7;
    const int rel = idx & 7, src = (rank + 1 + rel) & 7, sub = xcd & 3, cti = idx >> 3, ct = (xcd >> 2) * 4 + cti;
    char* me = p.peer[NR - 1];
    const bool local = rel == NR - 1;
    const int rsub = local ? 3 : sub, rtile = src * 4 + rsub;
    const int jj = (sub - cti - 1) & 3;
    const bool comm = local && jj < CPX;
    const bool wonly = comm && jj == 2 && (xcd & 1);
    const int cid = jj == 2 ? 16 + (xcd >> 1) : jj * 8 + xcd;
    const int h = rel * 4 + sub;
    const bool host = h < 8 * CPX;
    const int xrow = h * 16;
    constexpr int NCW = 32 - CPX;
    static_assert(PEARLY <= PW && PW <= 7 && CPX == 3 && NWO == 4, "roles");
    const int ci = local ? 28 : h;
    GAS unsigned* seqw = (GAS unsigned*)(me + SQ_OFF + (size_t)bx * 128);
#define WG_START() do { if (threadIdx.x < 16) ctl[threadIdx.x] = (threadIdx.x == C_FULL + 2 || threadIdx.x == C_FULL + 3) ? 1u : 0u; \
        LGKM0(); __builtin_amdgcn_s_barrier(); } while (0)
#define GET_CALL() const unsigned call = rfl(__hip_atomic_load(seqw, __ATOMIC_RELAXED, __HIP_MEMORY_SCOPE_AGENT)) + 1u; \
    const Tmo tmo{(GAS unsigned*)(me + TM_OFF), call}; const unsigned par = call & 1u; (void)par

    if (__builtin_expect(comm, 1)) {
        if (__builtin_expect(wave < PW && !wonly, 1)) {
            constexpr unsigned NHU = 16 * KS;
            f32x4 v[8];
            auto load_x = [&](int u) {
                const float* s0 = p.x + (size_t)((u & 7) * 64 + (lane >> 2)) * KDIM + (u >> 3) * 32 + (lane & 3) * 4;
#pragma unroll
                for (int e = 0; e < 8; ++e) v[e] = *(const GAS f32x4*)(s0 + (size_t)(e >> 1) * 16 * KDIM + (e & 1) * 16);
            };
            if (wave >= PEARLY) __builtin_amdgcn_s_sleep(8);
            load_x(wave * NCOMM + cid);
            SCHED();
            WG_START();
            GET_CALL();
            __amdgpu_buffer_rsrc_t rs[NR];
#pragma unroll
            for (int q = 0; q < NR; ++q) rs[q] = mkrsrc(p.peer[q] + XS_OFF + (size_t)rank * XS_SRC, (unsigned)XS_SRC);
            GAS u64* flag = (GAS u64*)(pick_peer(p, lane & 7) + XF_OFF) + rank * 8 * KS * 2;
            GAS unsigned* tk = (GAS unsigned*)(me + TK_OFF);
            LAS char* tail = lds + wave * 2048;
            const unsigned ptag = (unsigned)lane << 1, vo = lane * 16;
            unsigned u = wave * NCOMM + cid;
            int k = wave;
            while (u < NHU) {
                const unsigned fi = (u & 7u) * KS * 2 + (u >> 3), soff = fi * (UNIT / 2);
                const bool own = (u & 7u) >= 6u;
                u32x4 L[2], H[2], T;
                unsigned sum = 0;
#pragma unroll
                for (int e = 0; e < 8; ++e) {
                    const f32x4 s = v[e] * XSCALE;
                    const unsigned d0 = cvt_pk(s[0], s[1]), d1 = cvt_pk(s[2], s[3]);
                    L[e >> 2][e & 3] = __builtin_amdgcn_perm(d1, d0, 0x06040200u);
                    H[e >> 2][e & 3] = __builtin_amdgcn_perm(d1, d0, 0x07050301u);
                }
                SCHED();
                if (k < PEARLY) {
#pragma unroll
                    for (int j = 0; j < 2; ++j)
#pragma unroll
                        for (int q = 0; q < NR; ++q) if (q < NR - 1 || own) __builtin_amdgcn_raw_buffer_store_b128(L[j], rs[q], vo + j * 1024, soff, SYS);
                    SCHED(); asm volatile("s_nop 1" ::: "memory"); SCHED();
                }
#pragma unroll
                for (int e = 0; e < 8; ++e) sum = __builtin_amdgcn_sad_u8(H[e >> 2][e & 3] & 0x7f7f7f7fu, 0u, sum);
                sum += __builtin_amdgcn_update_dpp(0u, sum, 0x111, 0xf, 0xf, true);
                sum += __builtin_amdgcn_update_dpp(0u, sum, 0x112, 0xf, 0xf, true);
                sum += __builtin_amdgcn_update_dpp(0u, sum, 0x114, 0xf, 0xf, true);
                sum += __builtin_amdgcn_update_dpp(0u, sum, 0x118, 0xf, 0xf, true);
                const int tot = __builtin_amdgcn_readlane(sum, 15) + __builtin_amdgcn_readlane(sum, 31) + __builtin_amdgcn_readlane(sum, 47) + __builtin_amdgcn_readlane(sum, 63);
                int bs = (tot - 2970) >> 11;
                bs = bs < 0 ? 0 : (bs > 124 ? 124 : bs);
                const unsigned b4 = (unsigned)bs * 0x01010101u, nb4 = (unsigned)((-bs) & 3) * 0x01010101u;
                unsigned cnt = 0, c[8], sg[8], x[8], rest = 0;
                auto emit = [&](int e) {
                    const u64 m = __builtin_amdgcn_ballot_w64(x[e] != 0u);
                    const unsigned sh = (unsigned)__builtin_ctz(x[e] | 0x80000000u) & 24u;
                    const unsigned pos = min(cnt, (unsigned)EXCAP + 1u) + __builtin_amdgcn_mbcnt_hi((unsigned)(m >> 32), __builtin_amdgcn_mbcnt_lo((unsigned)m, 0u));
                    const unsigned ent = (((x[e] >> sh) & 0x7cu) << 9) | ((unsigned)e << 8) | ((sh & 16u) << 3) | ptag | ((sh >> 3) & 1u);
                    *(LAS unsigned short*)(tail + (x[e] != 0u ? 768u + pos * 2 : 1024u + lane * 2)) = (unsigned short)ent;
                    x[e] &= ~(0xffu << sh);
                    cnt += (unsigned)__builtin_popcountll(m);
                };
#pragma unroll
                for (int e = 0; e < 8; ++e) {
                    const unsigned hh = H[e >> 2][e & 3], hm = hh & 0x7f7f7f7fu;
                    c[e] = (hm + nb4) & 0x03030303u;
                    sg[e] = hh & 0x80808080u;
                    x[e] = hm ^ (c[e] + b4);
                    emit(e);
                    rest |= x[e];
                }
                while (__builtin_amdgcn_ballot_w64(rest != 0u) != 0ull && cnt <= (unsigned)EXCAP) {
                    rest = 0;
#pragma unroll
                    for (int e = 0; e < 8; ++e) { if (__builtin_amdgcn_ballot_w64(x[e] != 0u) != 0ull) emit(e); rest |= x[e]; }
                }
                ((LAS unsigned*)tail)[lane * 3 + 0] = (c[0] | sg[0]) | rotr(c[1] | sg[1], 29) | (c[6] << 5);
                ((LAS unsigned*)tail)[lane * 3 + 1] = (c[2] | sg[2]) | rotr(c[3] | sg[3], 29) | (c[7] << 5);
                ((LAS unsigned*)tail)[lane * 3 + 2] = (c[4] | sg[4]) | rotr(c[5] | sg[5], 29) | (sg[6] >> 1) | (sg[7] >> 2);
                asm volatile("" ::: "memory");
                T = *(volatile LAS u32x4*)(tail + lane * 16);
                unsigned hdr = (unsigned)bs | (cnt << 8);
                if (cnt > (unsigned)EXCAP) { hdr = 0x80u; T = H[0]; }
                unsigned vt = ((hdr & 0x80u) || (unsigned)lane < 48u + ((cnt + 7u) >> 3)) ? vo + 2048 : 0x40000000u;
                asm volatile("" : "+v"(T), "+v"(vt));
                SCHED();
                if (k >= PEARLY) {
                    Spin sp;
                    for (;;) {
                        const u32x2 cc = *(volatile LAS u32x2*)(lds + LC_OFF + C_SENT * 4);
                        if ((int)rfl(cc.x) + PWIN > k && (int)rfl(cc.y) >= k) break;
                        __builtin_amdgcn_s_sleep(1);
                        if (sp.over<1023u>(tmo)) break;
                    }
                    asm volatile("" ::: "memory");
#pragma unroll
                    for (int j = 0; j < 2; ++j)
#pragma unroll
                        for (int q = 0; q < NR; ++q) if (q < NR - 1 || own) __builtin_amdgcn_raw_buffer_store_b128(L[j], rs[q], vo + j * 1024, soff, SYS);
                }
#pragma unroll
                for (int q = 0; q < NR; ++q) if (q < NR - 1 || own) __builtin_amdgcn_raw_buffer_store_b128(T, rs[q], vt, soff, SYS);
                if (hdr & 0x80u) {
#pragma unroll
                    for (int q = 0; q < NR; ++q) if (q < NR - 1 || own) __builtin_amdgcn_raw_buffer_store_b128(H[1], rs[q], vo + 3072, soff, SYS);
                }
                SCHED(); asm volatile("s_nop 1" ::: "memory"); SCHED();
                if (lane == 0) __hip_atomic_fetch_add((LAS unsigned*)(lds + LC_OFF) + C_ISS, 1u, __ATOMIC_RELAXED, __HIP_MEMORY_SCOPE_WORKGROUP);
                unsigned tv = 0;
                if (lane == 0) tv = __hip_atomic_fetch_add(tk, 1u, __ATOMIC_RELAXED, __HIP_MEMORY_SCOPE_AGENT);
                VMWAIT0();
                if (lane == 0) __hip_atomic_fetch_add((LAS unsigned*)(lds + LC_OFF) + C_SENT, 1u, __ATOMIC_RELAXED, __HIP_MEMORY_SCOPE_WORKGROUP);
                if (lane < (own ? NR : NR - 1)) __hip_atomic_store(flag + fi, ((u64)hdr << 32) | call, __ATOMIC_RELAXED, __HIP_MEMORY_SCOPE_SYSTEM);
                u = PW * NCOMM + (rfl(tv) - (call - 1u) * NHU);
                if (u < NHU) {
                    load_x((int)u);
                    unsigned kq = 0;
                    if (lane == 0) kq = __hip_atomic_fetch_add((LAS unsigned*)(lds + LC_OFF) + C_SEQ, 1u, __ATOMIC_RELAXED, __HIP_MEMORY_SCOPE_WORKGROUP);
                    k = PW + (int)rfl(kq);
                }
            }
            if (lane == 0) __hip_atomic_fetch_add((LAS unsigned*)(lds + LC_OFF) + C_CNT, 1u, __ATOMIC_RELAXED, __HIP_MEMORY_SCOPE_WORKGROUP);
        } else {
            WG_START();
            if (wave != 7 && !wonly) return;
            GET_CALL();
            if (wonly) {
#pragma unroll 1
                for (int e = ((cid - 16) * 8 + wave) * 2; e < TWO * 32; e += NWO * 16) {
                    f32x4 va[16], vb[16];
                    const int t = KS - TWO + (e >> 5), c2 = (e >> 2) & 7, q = e & 3;
                    w_load(va, p.w, true, c2, t, q, lane); w_load(vb, p.w, true, c2, t, q + 1, lane);
                    SCHED();
                    w_store(va, me, c2, t, q, lane); w_store(vb, me, c2, t, q + 1, lane);
                    VMWAIT0();
                    if (lane < 2) __hip_atomic_store((GAS unsigned*)(me + WF_OFF) + ((c2 * 4 + q + lane) * KS + t) * 16, call, __ATOMIC_RELAXED, __HIP_MEMORY_SCOPE_SYSTEM);
                }
                if (wave != 7) { if (lane == 0) __hip_atomic_fetch_add((LAS unsigned*)(lds + LC_OFF) + C_CNT, 1u, __ATOMIC_RELAXED, __HIP_MEMORY_SCOPE_WORKGROUP); return; }
            }
            if (lane == 0) __hip_atomic_store((GAS u64*)(me + A1_OFF) + (par * NWG + bx) * 8, (u64)call << 32, __ATOMIC_RELAXED, __HIP_MEMORY_SCOPE_SYSTEM);
            { Spin sp; while (rfl(ctl[C_CNT]) < (wonly ? 7u : (unsigned)PW)) { __builtin_amdgcn_s_sleep(8); if (sp.over<1023u>(tmo)) break; } }
            if (lane == 0) __hip_atomic_store(seqw, call, __ATOMIC_RELAXED, __HIP_MEMORY_SCOPE_AGENT);
            if (cid < 2) {
                const __amdgpu_buffer_rsrc_t rs = mkrsrc(me + A1_OFF + (size_t)par * NWG * 64, NWG * 64);
                GAS u64* dst = (GAS u64*)(pick_peer(p, lane >> 3) + (cid ? A3_OFF : A2_OFF) + (size_t)(par * 8 + (lane & 7)) * 128) + rank;
                unsigned g = 0, sent = 0, off[4]; Spin sp;
#pragma unroll
                for (int j = 0; j < 4; ++j) off[j] = lane * 64 + j * 4096;
                for (;;) {
                    u32x2 a[4];
#pragma unroll
                    for (int j = 0; j < 4; ++j) a[j] = __builtin_amdgcn_raw_buffer_load_b64(rs, off[j], 0, SYS);
                    int nin = 0;
#pragma unroll
                    for (int j = 0; j < 4; ++j) {
                        if (a[j].y == call) { g = max(g, a[j].x); off[j] = 0x40000000u; }
                        nin += __builtin_popcountll(__builtin_amdgcn_ballot_w64(off[j] == 0x40000000u));
                    }
                    if (nin == NWG) break;
                    if (cid && nin >= SPECN && __any(g > sent)) { sent = wave_max(g); __hip_atomic_store(dst, ((u64)call << 32) | sent, __ATOMIC_RELAXED, __HIP_MEMORY_SCOPE_SYSTEM); }
                    asm volatile("" ::: "memory");
                    if (sp.over<255u>(tmo)) break;
                }
                if (cid == 0) {
                    g = wave_max(g);
                    __hip_atomic_store(dst, ((u64)call << 32) | g, __ATOMIC_RELAXED, __HIP_MEMORY_SCOPE_SYSTEM);
                }
            }
        }
        return;
    }

    if (idx == 2 * CPX - 1 && wave < 4) {
        const char* pa = wave == 0 ? p.peer[0] : wave == 1 ? p.peer[2] : wave == 2 ? p.peer[4] : p.peer[6];
        const char* pb = wave == 0 ? p.peer[1] : wave == 1 ? p.peer[3] : wave == 2 ? p.peer[5] : p.peer[7];
        const int w = (lane >> 1) & 7;
        touch(((lane & 1) ? pb : pa) + (w < 4 ? XS_OFF + (size_t)rank * XS_SRC + (size_t)w * (XS_SRC / 4) : XF_OFF + (size_t)rank * 8 * KS * 16), junk);
    }
    if (wave == 3) {
        const size_t o1 = (size_t)rtile * 128 * NL, o2 = (size_t)(rank * 512 + xrow) * NL;
        touch(lane == 0 ? (const char*)p.out + o1 : lane == 1 ? (const char*)p.out + o2 : lane == 2 ? (const char*)(p.x + (size_t)xrow * KDIM) : me + XS_OFF + (size_t)rtile * (XS_SRC / 4), junk);
    }
    const unsigned tstart = rt32();
    WG_START();
    unsigned call, wfl = 0;
    {
        constexpr int NH = 8 * CPX, NN = NCW - NH, TH = KS - 4 * NN - TWO;
        static_assert(TH >= NH && TH <= 2 * NH, "at most one unit per wave of a host");
        const bool two = !host;
        const int t1 = host ? (wave < 4 ? NH + ci : ci) : TH + (ci - NH) * 4 + (wave >> 2) * 2, q = wave & 3;
        const bool one = !host || t1 < TH;
        while (rt32() - tstart < (host ? (wave < 4 ? (one ? WDELAY2 + ci * WSTAG : MDELAY) : WDELAY + ci * WSTAG) : WDELAY)) __builtin_amdgcn_s_sleep(2);
        f32x4 va[16], vb[16];
        w_load(va, p.w, one, ct, t1, q, lane);
        if (two) w_load(vb, p.w, true, ct, t1 + 1, q, lane);
        const unsigned sv = __hip_atomic_load(seqw, __ATOMIC_RELAXED, __HIP_MEMORY_SCOPE_AGENT);
        SCHED();
        if (one) w_store(va, me, ct, t1, q, lane);
        if (two) w_store(vb, me, ct, t1 + 1, q, lane);
        if (wave < 4 && lane < NH) wfl = __hip_atomic_load((GAS unsigned*)(me + WF_OFF) + ((ct * 4 + wave) * KS + lane) * 16, __ATOMIC_RELAXED, __HIP_MEMORY_SCOPE_SYSTEM);
        VMWAIT0();
        call = rfl(sv) + 1u;
        if (lane < (two ? 2 : one ? 1 : 0)) __hip_atomic_store((GAS unsigned*)(me + WF_OFF) + ((ct * 4 + q) * KS + t1 + lane) * 16, call, __ATOMIC_RELAXED, __HIP_MEMORY_SCOPE_SYSTEM);
    }
    const Tmo tmo{(GAS unsigned*)(me + TM_OFF), call};
    const unsigned par = call & 1u;
    if (wave < 4) {
        const __amdgpu_buffer_rsrc_t rsw = mkrsrc(me + WB_OFF + (size_t)(ct * 4 + wave) * KS * 8192, KS * 8192);
        const __amdgpu_buffer_rsrc_t rsf = mkrsrc(me + WF_OFF + (size_t)(ct * 4 + wave) * KS * 64, KS * 64);
        const unsigned ldsb = (unsigned)(size_t)lds;
        unsigned vo = lane * 16;
        const unsigned polla = ldsb + LC_OFF + (C_FULL + (lane & 3)) * 4, donea = ldsb + LC_OFF + (C_DONE + wave) * 4;
        f32x32 r0, r1, r2, r3; f32x16 r4;
        static_assert(LA_SLOT == 9216 && LA_HALF == 27648 && LA_PAR == 55296, "literals in the K loop");
        int s_wk = __builtin_ctzll(__builtin_amdgcn_ballot_w64(wfl < call)) - 1;
        unsigned s_wt, s_ws, s_so, s_t, s_bo, s_bc, s_x, s_sp, s_t0; u64 s_c;
        asm volatile(R"ASM(
.set AG_A, 8
.set AG_C, 112
.macro AG_EXPIRED lout
  s_add_u32 %[sp], %[sp], 1
  s_and_b32 %[x], %[sp], 0x3ff
  s_cbranch_scc1 8f
  v_mov_b32 v6, 0
  global_load_dword v3, v6, %[tmo] sc1
  s_memrealtime %[c]
  s_waitcnt vmcnt(0) lgkmcnt(0)
  v_readfirstlane_b32 %[x], v3
  s_cmp_eq_u32 %[x], %[call]
  s_cbranch_scc1 \lout
  s_mov_b64 vcc, %[c]
  s_cmp_eq_u32 %[sp], 0x400
  s_cselect_b32 %[t0], vcc_lo, %[t0]
  s_sub_u32 %[x], vcc_lo, %[t0]
  s_cmp_le_u32 %[x], 40000000
  s_cbranch_scc1 8f
  v_mov_b32 v3, %[call]
  global_store_dword v6, v3, %[tmo] sc1
  s_waitcnt vmcnt(0)
  s_branch \lout
8:
.endm
; wait until w K-step (t + add + 1) is converted; wk = last K-step known to be converted with all before it
.macro AG_NEEDW add
  s_waitcnt lgkmcnt(0)
  s_mov_b32 %[sp], 0
1:
  v_lshlrev_b32 v6, 2, v0
  buffer_load_dword v3, v6, %[rsf], 0 offen sc0 sc1
  s_waitcnt vmcnt(0)
  v_cmp_gt_u32 vcc, %[call], v3
  s_ff1_i32_b64 %[wk], vcc
  s_min_u32 %[wk], %[wk], 1000
  s_sub_i32 %[wk], %[wk], 1
  s_add_u32 %[x], %[t], \add
  s_cmp_gt_i32 %[wk], %[x]
  s_cbranch_scc1 3f
  s_sleep 2
  AG_EXPIRED 2f
  s_branch 1b
2:
  s_movk_i32 %[wk], 1000
3:
  ds_read_b32 v3, v4
  s_add_u32 %[x], %[t], \add
.endm
; visited when wt <= x: one look at the w flags that costs no wait (requested here, read two steps later), else a real wait
.macro AG_WLOOK add, back
  s_cmp_le_i32 %[wk], %[x]
  s_cbranch_scc1 5f
  s_add_u32 %[ws], %[ws], 1
  s_cmp_eq_u32 %[ws], 1
  s_cbranch_scc0 4f
  v_lshlrev_b32 v1, 2, v0
  buffer_load_dword v1, v1, %[rsf], 0 offen sc0 sc1
  s_add_u32 %[wt], %[x], 2
  s_min_i32 %[wt], %[wt], %[wk]
  s_branch \back
4:
  v_cmp_gt_u32 vcc, %[call], v1
  s_ff1_i32_b64 %[wt], vcc
  s_min_u32 %[wt], %[wt], 1000
  s_sub_i32 %[wt], %[wt], 1
  s_max_i32 %[wk], %[wk], %[wt]
  s_mov_b32 %[ws], 0
  s_sub_i32 %[wt], %[wk], 3
  s_branch \back
5:
  AG_NEEDW \add
  s_mov_b32 %[ws], 0
  s_sub_i32 %[wt], %[wk], 3
  s_branch \back
.endm
; wait until x K-step (t + add) is in the ring, then read its first half
.macro AG_NEEDX add, va, nb
  s_mov_b32 %[sp], 0
1:
  ds_read_b32 v3, v4
  s_add_u32 %[x], %[t], \add
  s_waitcnt lgkmcnt(0)
  v_cmp_lt_u32 vcc, %[x], v3
  s_cmp_eq_u64 vcc, exec
  s_cbranch_scc1 2f
  s_sleep 1
  AG_EXPIRED 2f
  s_branch 1b
2:
  .set ag_b, 0
  .rept 8
  ds_read_b128 v[AG_A+4*ag_b:AG_A+4*ag_b+3], \va offset:(ag_b/4)*27648+(ag_b%%4)*1024
  .set ag_b, ag_b+1
  .endr
  .if \nb == 9
  ds_read_b128 v[AG_A+32:AG_A+35], \va offset:8192
  .endif
.endm
.macro AG_MF b, m, bc
  .set ag_n, \m ^ ((\b & 1) * 3)
  v_mfma_f32_16x16x32_bf16 v[AG_C+16*\b+4*ag_n:AG_C+16*\b+4*ag_n+3], v[\bc+4*ag_n:\bc+4*ag_n+3], v[AG_A+4*\b:AG_A+4*\b+3], v[AG_C+16*\b+4*ag_n:AG_C+16*\b+4*ag_n+3]
.endm
; half K-step on nb row blocks: first = 1 for the first half of step (t + add)
.macro AG_HALF nb, bc, bn, va, vnext, kh, first, add
  s_waitcnt vmcnt(8) lgkmcnt(\nb-3)
  .set ag_b, 0
  .rept \nb
  .if (ag_b == 3) || (ag_b == 6)
  s_waitcnt lgkmcnt(\nb-3)
  .endif
  AG_MF ag_b, 0, \bc
  .if ag_b == 4
  s_add_u32 %[bo], %[bo], 4096
  .endif
  .if \first && (ag_b == 5)
  s_add_u32 %[x], %[t], \add+1
  .endif
  .if \first && \add && (ag_b == 6)
  s_cmp_eq_u32 %[so], 27648
  .endif
  .if (!\first) && (ag_b == 6)
  v_cmp_lt_u32 vcc, %[x], v3
  .endif
  .if \first && \add && (ag_b == 7)
  v_add_u32 \vnext, %[so], v0
  .endif
  .if \first && (!\add) && (ag_b == 7)
  v_add_u32 \vnext, 55296, \va
  .endif
  AG_MF ag_b, 1, \bc
  .if ag_b < 4
  buffer_load_dwordx4 v[\bn+4*ag_b:\bn+4*ag_b+3], v0, %[rsw], %[bc] offen offset:ag_b*1024 sc0 sc1
  .endif
  .if ag_b == 4
  s_min_u32 %[bc], %[bo], 127*4096
  .endif
  .if (!\first) && (ag_b == 5)
  v_mov_b32 v6, %[x]
  .endif
  .if (!\first) && (ag_b == 6)
  ds_write_b32 v5, v6
  .endif
  .if \first && \add && (ag_b == 6)
  s_cselect_b32 %[so], 0, %[so]
  .endif
  .if \first && (ag_b == 7)
  s_cmp_le_i32 %[wt], %[x]
  .endif
  .if (!\first) && (ag_b == 7)
  s_cmp_eq_u64 vcc, exec
  .endif
  AG_MF ag_b, 2, \bc
  .if \first && \add && (ag_b == 5)
  s_add_u32 %[so], %[so], 9216
  .endif
  .if \first && (ag_b == \nb-1)
  ds_read_b32 v3, v4
  .endif
  AG_MF ag_b, 3, \bc
  .if ag_b < 8
  ds_read_b128 v[AG_A+4*ag_b:AG_A+4*ag_b+3], \va offset:(ag_b/4)*27648+\kh*4096+(ag_b%%4)*1024
  .else
  ds_read_b128 v[AG_A+32:AG_A+35], \va offset:8192+\kh*27648
  .endif
  .set ag_b, ag_b+1
  .endr
.endm
.macro AG_LOOP nb
  AG_NEEDX 0, v2, \nb
.Lag_loop\nb:
  AG_HALF \nb, 44, 92, v2, v7, 1, 1, 0
  s_cbranch_scc1 .Lag_w0\nb
.Lag_w0r\nb:
  AG_HALF \nb, 60, 44, v7, v7, 0, 0, 0
  s_cbranch_scc0 .Lag_x0\nb
.Lag_x0r\nb:
  AG_HALF \nb, 76, 60, v7, v2, 1, 1, 1
  s_cbranch_scc1 .Lag_w1\nb
.Lag_w1r\nb:
  AG_HALF \nb, 92, 76, v2, v2, 0, 0, 1
  s_cbranch_scc0 .Lag_x1\nb
.Lag_x1r\nb:
  s_add_u32 %[t], %[t], 2
  s_cmp_lt_u32 %[t], 64
  s_cbranch_scc1 .Lag_loop\nb
  s_branch .Lag_end
.Lag_w0\nb:
  AG_WLOOK 1, .Lag_w0r\nb
.Lag_w1\nb:
  AG_WLOOK 2, .Lag_w1r\nb
.Lag_x0\nb:
  AG_NEEDX 1, v7, \nb
  s_branch .Lag_x0r\nb
.Lag_x1\nb:
  AG_NEEDX 2, v2, \nb
  s_branch .Lag_x1r\nb
.endm
  s_nop 4
  s_mov_b32 %[t], 0
  s_mov_b32 %[so], 0
  s_cmp_gt_i32 %[wk], 0
  s_cbranch_scc1 .Lag_go
  AG_NEEDW 0
.Lag_go:
  s_mov_b32 %[ws], 0
  s_sub_i32 %[wt], %[wk], 3
  s_min_i32 %[wt], %[wt], 12
  .set ag_i, 0
  .rept 12
  s_movk_i32 %[bc], (ag_i/4)*4096
  buffer_load_dwordx4 v[44+4*ag_i:44+4*ag_i+3], v0, %[rsw], %[bc] offen offset:(ag_i%%4)*1024 sc0 sc1
  .set ag_i, ag_i+1
  .endr
  s_movk_i32 %[bo], 3*4096
  s_movk_i32 %[bc], 3*4096
  .set ag_i, 0
  .rept 72
  v_mov_b64 v[AG_C+ag_i:AG_C+ag_i+1], 0
  .set ag_i, ag_i+2
  .endr
  v_mov_b32 v2, v0
  s_cmp_eq_u32 %[host], 0
  s_cbranch_scc1 .Lag_eight
  AG_LOOP 9
.Lag_eight:
  AG_LOOP 8
.Lag_end:
  s_waitcnt vmcnt(0) lgkmcnt(0)
  s_nop 7
)ASM"
            : "={v[112:143]}"(r0), "={v[144:175]}"(r1), "={v[176:207]}"(r2), "={v[208:239]}"(r3), "={v[240:255]}"(r4),
              [t] "=&s"(s_t), [wk] "+s"(s_wk), [bo] "=&s"(s_bo), [bc] "=&s"(s_bc), [x] "=&s"(s_x), [sp] "=&s"(s_sp), [t0] "=&s"(s_t0),
              [wt] "=&s"(s_wt), [ws] "=&s"(s_ws), [so] "=&s"(s_so), [c] "=&s"(s_c), "+{v0}"(vo)
            : "{v4}"(polla), "{v5}"(donea), [rsw] "s"(rsw), [rsf] "s"(rsf), [tmo] "s"(tmo.word), [call] "s"(call), [host] "s"(rfl(host ? 1u : 0u))
            : "memory", "vcc", "scc", "v1", "v2", "v3", "v6", "v7", "v8", "v9", "v10", "v11", "v12", "v13", "v14", "v15", "v16", "v17", "v18", "v19", "v20", "v21", "v22", "v23", "v24", "v25", "v26", "v27", "v28", "v29", "v30", "v31", "v32", "v33", "v34", "v35", "v36", "v37", "v38", "v39", "v40", "v41", "v42", "v43", "v44", "v45", "v46", "v47", "v48", "v49", "v50", "v51", "v52", "v53", "v54", "v55", "v56", "v57", "v58", "v59", "v60", "v61", "v62", "v63", "v64", "v65", "v66", "v67", "v68", "v69", "v70", "v71", "v72", "v73", "v74", "v75", "v76", "v77", "v78", "v79", "v80", "v81", "v82", "v83", "v84", "v85", "v86", "v87", "v88", "v89", "v90", "v91", "v92", "v93", "v94", "v95", "v96", "v97", "v98", "v99", "v100", "v101", "v102", "v103", "v104", "v105", "v106", "v107");
        f32x4 acc[9][4];
#pragma unroll
        for (int i = 0; i < 144; ++i) acc[i >> 4][(i >> 2) & 3][i & 3] = i < 32 ? r0[i & 31] : i < 64 ? r1[i & 31] : i < 96 ? r2[i & 31] : i < 128 ? r3[i & 31] : r4[i & 15];
        const int ln = (int)(vo >> 4);
        int mx = 0;
#pragma unroll
        for (int n = 0; n < 4; ++n) acc[8][n] *= XSCALE;
        if (host) {
#pragma unroll
            for (int n = 0; n < 4; ++n) { mx = max(max(mx, fbits(acc[8][n][0])), fbits(acc[8][n][1])); mx = max(max(mx, fbits(acc[8][n][2])), fbits(acc[8][n][3])); }
        }
        int m2 = 0;
#pragma unroll
        for (int b = 0; b < 8; ++b)
#pragma unroll
            for (int n = 0; n < 4; ++n) { mx = max(max(mx, fbits(acc[b][n][0])), fbits(acc[b][n][1])); m2 = max(max(m2, fbits(acc[b][n][2])), fbits(acc[b][n][3])); }
        const unsigned wm = wave_max((unsigned)max(mx, m2));
        if (ln == 0) {
            __hip_atomic_fetch_max((LAS unsigned*)(lds + LC_OFF) + C_MAX, wm, __ATOMIC_RELAXED, __HIP_MEMORY_SCOPE_WORKGROUP);
            asm volatile("" ::: "memory");
            const unsigned arrived = __hip_atomic_fetch_add((LAS unsigned*)(lds + LC_OFF) + C_CNT, 1u, __ATOMIC_RELAXED, __HIP_MEMORY_SCOPE_WORKGROUP);
            asm volatile("" ::: "memory");
            const unsigned tm = ctl[C_MAX];
            if (arrived == 3u) __hip_atomic_store((GAS u64*)(me + A1_OFF) + (par * NWG + bx) * 8, ((u64)call << 32) | tm, __ATOMIC_RELAXED, __HIP_MEMORY_SCOPE_SYSTEM);
        }
        const __amdgpu_buffer_rsrc_t rso = mkrsrc(p.out, 8u * 512u * NL);
        const unsigned ovo = (unsigned)(ln & 15) * NL + ct * 256 + wave * 64 + (ln >> 4) * 16;
        unsigned used = 0xffffffffu; Spin sp;
#pragma unroll 1
        for (;;) {
            const u32x2 c = *(volatile LAS u32x2*)(lds + LC_OFF + C_GMAX * 4);
            unsigned st = rfl(c.y);
            const unsigned run = rfl(c.x), g = max(run, wm);
            if (st == 0u && sp.over<4095u>(tmo)) st = 2u;
            if (st == 2u || (g != used && (st != 0u || run != 0u))) {
                if (used != 0xffffffffu) VMWAIT0();
                used = g;
                const float scale = __uint_as_float(g) / 127.0f;
                const float inv = scale > 0.f ? 1.0f / scale : 0.f;
#pragma unroll
                for (int b = 0; b < 9; ++b) {
                    u32x4 o = {0u, 0u, 0u, 0u};
#pragma unroll
                    for (int cc = 0; cc < 4; ++cc) {
                        f32x4 q = acc[b][cc] * inv;
                        asm volatile("" : "+v"(q));
#pragma unroll
                        for (int r = 0; r < 4; ++r) o[r] = __builtin_amdgcn_cvt_pk_u8_f32(q[r], cc, o[r]);
                    }
                    if (st == 2u) o = (u32x4){0x80808080u, 0x80808080u, 0x80808080u, 0x80808080u};
                    if (b < 8 || host) __builtin_amdgcn_raw_buffer_store_b128(o, rso, ovo + (unsigned)(b < 8 ? rtile * 128 + b * 16 : rank * 512 + xrow) * NL, 0, SYS);
                }
            }
            if (st != 0u) break;
            __builtin_amdgcn_s_sleep(1);
        }
    } else {
        const int f = wave & 1, ph = (wave >> 1) & 1, rb = rsub * 2 + f;
        const __amdgpu_buffer_rsrc_t rsx = mkrsrc(me + XS_OFF + (size_t)src * XS_SRC + (size_t)rb * KS * UNIT, KS * UNIT);
        const __amdgpu_buffer_rsrc_t rse = mkrsrc(p.x + (size_t)xrow * KDIM + f * 32, host ? 16u * KDIM * 4u - 128u : 0u);
        const __amdgpu_buffer_rsrc_t rsf = mkrsrc(me + XF_OFF + (size_t)(src * 8 + rb) * KS * 16, KS * 16);
        const unsigned ldsb = (unsigned)(size_t)lds;
        const unsigned mbase = ldsb + f * LA_HALF, vpub = ldsb + LC_OFF + (C_FULL + (wave & 3)) * 4, vdone = ldsb + LC_OFF + (C_DONE + (lane & 3)) * 4;
        const unsigned hostf = rfl(host ? 1u : 0u);
        unsigned s_cur = ph, s_lim = NST, s_so = 0, s_h, s_g, s_b, s_x, s_y, s_sp, s_t0, s_m0; u64 s_seen, s_c;
        asm volatile(R"ASM(
.macro FD_EXPIRED lout
  s_add_u32 %[sp], %[sp], 1
  s_and_b32 %[x], %[sp], 0xff
  s_cbranch_scc1 8f
  v_mov_b32 v9, 0
  global_load_dword v12, v9, %[tmo] sc1
  s_memrealtime %[c]
  s_waitcnt vmcnt(0) lgkmcnt(0)
  v_readfirstlane_b32 %[x], v12
  s_cmp_eq_u32 %[x], %[call]
  s_cbranch_scc1 \lout
  s_mov_b64 vcc, %[c]
  s_cmp_eq_u32 %[sp], 0x100
  s_cselect_b32 %[t0], vcc_lo, %[t0]
  s_sub_u32 %[x], vcc_lo, %[t0]
  s_cmp_le_u32 %[x], 40000000
  s_cbranch_scc1 8f
  v_mov_b32 v12, %[call]
  global_store_dword v9, v12, %[tmo] sc1
  s_waitcnt vmcnt(0)
  s_branch \lout
8:
.endm
.macro FD_FLAGS
  buffer_load_dwordx4 v[100:103], v0, %[rsf], 0 offen sc0 sc1
  s_waitcnt vmcnt(0)
  v_min_u32 v9, v100, v102
  v_cmp_le_u32 %[seen], %[call], v9
.endm
; loads of unit %[y] into the set at r
.macro FD_ISSUE r
  s_lshl_b32 %[x], %[y], 13
  s_lshl_b32 %[b], %[y], 8
  buffer_load_dwordx4 v[\r:\r+3], v0, %[rsx], %[x] offen sc0 sc1
  buffer_load_dwordx4 v[\r+4:\r+7], v0, %[rsx], %[x] offen offset:1024 sc0 sc1
  buffer_load_dwordx3 v[\r+16:\r+18], v1, %[rsx], %[x] offen offset:2048 sc0 sc1
  buffer_load_ushort v[\r+19], v2, %[rsx], %[x] offen offset:2816 sc0 sc1
  s_bitset1_b32 %[x], 12
  buffer_load_dwordx4 v[\r+24:\r+27], v3, %[rse], %[b] offen
  buffer_load_dwordx4 v[\r+28:\r+31], v3, %[rse], %[b] offen offset:64
  buffer_load_dwordx4 v[\r+8:\r+11], v0, %[rsx], %[x] offen sc0 sc1
  buffer_load_dwordx4 v[\r+12:\r+15], v0, %[rsx], %[x] offen offset:1024 sc0 sc1
  buffer_load_dwordx3 v[\r+20:\r+22], v1, %[rsx], %[x] offen offset:2048 sc0 sc1
  buffer_load_ushort v[\r+23], v2, %[rsx], %[x] offen offset:2816 sc0 sc1
.endm
; wait for the flag of unit %[y]
.macro FD_WAITFLAG
  s_mov_b32 %[sp], 0
1:
  FD_FLAGS
  s_bitcmp1_b64 %[seen], %[y]
  s_cbranch_scc1 2f
  s_sleep 1
  FD_EXPIRED 2f
  s_branch 1b
2:
.endm
.macro FD_DEC h, n0, n1, n2
  v_and_or_b32 v[\h], v[\n0], v7, %[b]
  v_alignbit_b32 v12, v[\n0], v[\n0], 3
  v_and_or_b32 v[\h+2], v[\n1], v7, %[b]
  v_alignbit_b32 v13, v[\n1], v[\n1], 3
  v_and_or_b32 v[\h+4], v[\n2], v7, %[b]
  v_alignbit_b32 v14, v[\n2], v[\n2], 3
  v_and_or_b32 v[\h+1], v12, v7, %[b]
  v_lshrrev_b32 v12, 5, v[\n0]
  v_and_or_b32 v[\h+3], v13, v7, %[b]
  v_lshrrev_b32 v13, 5, v[\n1]
  v_and_or_b32 v[\h+5], v14, v7, %[b]
  v_lshlrev_b32 v14, 1, v[\n2]
  v_bfi_b32 v12, v8, v14, v12
  v_lshlrev_b32 v15, 2, v[\n2]
  v_and_or_b32 v[\h+6], v12, v7, %[b]
  v_bfi_b32 v13, v8, v15, v13
  v_and_or_b32 v[\h+7], v13, v7, %[b]
.endm
; exception entry in register e -> address v12, mask v13
.macro FD_EXC e
  v_and_b32 v12, 0x7fe, v[\e]
  v_and_b32 v14, 1, v[\e]
  v_lshrrev_b32 v13, 11, v[\e]
  v_lshl_or_b32 v14, v14, 4, 10
  v_lshl_add_u32 v12, v12, 1, %[x]
  v_lshlrev_b32 v13, v14, v13
.endm
; a half sent raw, or with a window that does not start at a multiple of four
.macro FD_FIX hd, h
  s_bitcmp1_b32 \hd, 7
  s_cbranch_scc0 3f
  buffer_load_dwordx4 v[\h:\h+3], v0, %[rsx], %[x] offen offset:2048 sc0 sc1
  buffer_load_dwordx4 v[\h+4:\h+7], v0, %[rsx], %[x] offen offset:3072 sc0 sc1
  s_waitcnt vmcnt(0)
3:
  s_and_b32 %[b], \hd, 3
  s_mul_i32 %[b], %[b], 0x01010101
  .set fd_i, 0
  .rept 8
  v_add_u32 v[\h+fd_i], %[b], v[\h+fd_i]
  .set fd_i, fd_i+1
  .endr
.endm
; one unit: set r holds the loads of unit cur, set q takes the next one; ob = offset of this parity's ring
.macro FD_STEP r, q, ob, id
  s_add_u32 %[y], %[cur], 2
  s_cmp_ge_u32 %[y], 64
  s_cbranch_scc1 .Lfd_last\id
  s_bitcmp1_b64 %[seen], %[y]
  s_cbranch_scc0 .Lfd_look\id
  FD_ISSUE \q
  s_waitcnt vmcnt(10)
.Lfd_go\id:
  v_readlane_b32 %[h], v101, %[cur]
  v_readlane_b32 %[g], v103, %[cur]
  s_and_b32 %[b], %[h], 0x7c
  s_mul_i32 %[b], %[b], 0x01010101
  FD_DEC 80, \r+16, \r+17, \r+18
  s_and_b32 %[b], %[g], 0x7c
  s_mul_i32 %[b], %[b], 0x01010101
  s_add_u32 m0, %[mb], %[so]
  FD_DEC 88, \r+20, \r+21, \r+22
  s_or_b32 %[x], %[h], %[g]
  s_and_b32 %[x], %[x], 0x83
  s_cbranch_scc1 .Lfd_fix\id
.Lfd_fixed\id:
  s_cmp_ge_u32 %[cur], %[lim]
  s_cbranch_scc1 .Lfd_free\id
.Lfd_freed\id:
  .set fd_i, 0
  .rept 16
  v_perm_b32 v[96+(fd_i%%2)*2], v[80+fd_i], v[\r+fd_i], %[p0]
  v_perm_b32 v[97+(fd_i%%2)*2], v[80+fd_i], v[\r+fd_i], %[p1]
  ds_write_addtid_b32 v[96+(fd_i%%2)*2] offset:\ob+fd_i*512
  ds_write_addtid_b32 v[97+(fd_i%%2)*2] offset:\ob+fd_i*512+256
  .set fd_i, fd_i+1
  .endr
  s_add_u32 %[x], m0, \ob
  FD_EXC \r+19
  s_lshr_b32 %[b], %[h], 8
  s_bfm_b64 exec, %[b], 0
  s_lshr_b32 %[b], %[g], 8
  ds_xor_b32 v12, v13
  s_bfm_b64 exec, %[b], 0
  s_add_u32 %[so], %[so], 9216
  FD_EXC \r+23
  ds_xor_b32 v12, v13 offset:4096
  s_mov_b64 exec, -1
  s_cmp_eq_u32 %[host], 0
  s_cbranch_scc1 .Lfd_pub\id
  v_cvt_pk_bf16_f32 v96, v[\r+24], v[\r+25]
  v_cvt_pk_bf16_f32 v97, v[\r+26], v[\r+27]
  v_cvt_pk_bf16_f32 v98, v[\r+28], v[\r+29]
  v_cvt_pk_bf16_f32 v99, v[\r+30], v[\r+31]
  ds_write_addtid_b32 v96 offset:\ob+8192
  ds_write_addtid_b32 v97 offset:\ob+8192+256
  ds_write_addtid_b32 v98 offset:\ob+8192+512
  ds_write_addtid_b32 v99 offset:\ob+8192+768
.Lfd_pub\id:
  s_add_u32 %[cur], %[cur], 2
  s_cmp_eq_u32 %[so], 27648
  s_cselect_b32 %[so], 0, %[so]
  s_cmp_ge_u32 %[cur], 64
  s_cselect_b32 %[x], 1000, %[cur]
  v_mov_b32 v9, %[x]
  s_waitcnt lgkmcnt(0)
  ds_write_b32 v4, v9
  s_cbranch_scc1 .Lfd_end
  s_cmp_eq_u32 %[y], %[cur]
  s_cbranch_scc0 .Lfd_late\id
.Lfd_next\id:
.endm
; out-of-line parts of a step
.macro FD_SLOW r, q, ob, id
.Lfd_last\id:
  s_waitcnt vmcnt(0)
  s_branch .Lfd_go\id
.Lfd_look\id:
  FD_FLAGS
  s_bitcmp1_b64 %[seen], %[y]
  s_cbranch_scc0 3f
  FD_ISSUE \q
  s_branch .Lfd_go\id
3:
  s_mov_b32 %[y], -1
  s_branch .Lfd_go\id
.Lfd_late\id:
  s_mov_b32 %[y], %[cur]
  FD_WAITFLAG
  FD_ISSUE \q
  s_branch .Lfd_next\id
.Lfd_fix\id:
  s_lshl_b32 %[x], %[cur], 13
  FD_FIX %[h], 80
  s_bitset1_b32 %[x], 12
  FD_FIX %[g], 88
  s_branch .Lfd_fixed\id
.Lfd_free\id:
  s_mov_b32 %[sp], 0
1:
  ds_read_b32 v9, v5
  s_waitcnt lgkmcnt(0)
  v_readlane_b32 %[lim], v9, 0
  v_readlane_b32 %[x], v9, 1
  s_min_u32 %[lim], %[lim], %[x]
  v_readlane_b32 %[x], v9, 2
  s_min_u32 %[lim], %[lim], %[x]
  v_readlane_b32 %[x], v9, 3
  s_min_u32 %[lim], %[lim], %[x]
  s_add_u32 %[lim], %[lim], 6
  s_cmp_lt_u32 %[cur], %[lim]
  s_cbranch_scc1 .Lfd_freed\id
  s_sleep 4
  FD_EXPIRED .Lfd_freed\id
  s_branch 1b
.endm
  s_nop 4
  s_mov_b32 %[m0s], m0
  v_mov_b32 v7, 0x83838383
  v_mov_b32 v8, 0x80808080
  s_mov_b32 %[y], %[cur]
  FD_WAITFLAG
  FD_ISSUE 16
  s_bitcmp1_b32 %[cur], 0
  s_cbranch_scc1 .Lfd_loop1
.Lfd_loop0:
  FD_STEP 16, 48, 0, 0
  FD_STEP 48, 16, 0, 1
  s_branch .Lfd_loop0
.Lfd_loop1:
  FD_STEP 16, 48, 55296, 2
  FD_STEP 48, 16, 55296, 3
  s_branch .Lfd_loop1
  FD_SLOW 16, 48, 0, 0
  FD_SLOW 48, 16, 0, 1
  FD_SLOW 16, 48, 55296, 2
  FD_SLOW 48, 16, 55296, 3
.Lfd_end:
  s_mov_b32 m0, %[m0s]
  s_waitcnt vmcnt(0) lgkmcnt(0)
)ASM"
            : [cur] "+s"(s_cur), [lim] "+s"(s_lim), [so] "+s"(s_so), [h] "=&s"(s_h), [g] "=&s"(s_g), [b] "=&s"(s_b), [x] "=&s"(s_x), [y] "=&s"(s_y), [sp] "=&s"(s_sp),
              [t0] "=&s"(s_t0), [m0s] "=&s"(s_m0), [seen] "=&s"(s_seen), [c] "=&s"(s_c)
            : "{v0}"(lane * 16), "{v1}"(lane * 12), "{v2}"(lane * 2), "{v3}"((lane >> 2) * KDIM * 4 + (lane & 3) * 16), "{v4}"(vpub), "{v5}"(vdone),
              [rsx] "s"(rsx), [rse] "s"(rse), [rsf] "s"(rsf), [tmo] "s"(tmo.word), [call] "s"(call), [mb] "s"(mbase), [host] "s"(hostf),
              [p0] "s"(0x05010400u), [p1] "s"(0x07030602u)
            : "memory", "vcc", "scc", "v7", "v8", "v9", "v10", "v11", "v12", "v13", "v14", "v15", "v16", "v17", "v18", "v19", "v20", "v21", "v22", "v23", "v24", "v25", "v26", "v27", "v28", "v29", "v30", "v31", "v32", "v33", "v34", "v35", "v36", "v37", "v38", "v39", "v40", "v41", "v42", "v43", "v44", "v45", "v46", "v47", "v48", "v49", "v50", "v51", "v52", "v53", "v54", "v55", "v56", "v57", "v58", "v59", "v60", "v61", "v62", "v63", "v64", "v65", "v66", "v67", "v68", "v69", "v70", "v71", "v72", "v73", "v74", "v75", "v76", "v77", "v78", "v79", "v80", "v81", "v82", "v83", "v84", "v85", "v86", "v87", "v88", "v89", "v90", "v91", "v92", "v93", "v94", "v95", "v96", "v97", "v98", "v99", "v100", "v101", "v102", "v103");
        static_assert(DEADLINE == 40000000u && NST == 6, "literals in the feeder loop");
        if (wave != 4) return;
        if (lane == 0) __hip_atomic_store(seqw, call, __ATOMIC_RELAXED, __HIP_MEMORY_SCOPE_AGENT);
        {
            const GAS u64* a2 = lane < 16 ? (const GAS u64*)(me + (lane < 8 ? A2_OFF : A3_OFF) + (size_t)(par * 8 + (bx & 7)) * 128) + (lane & 7) : (const GAS u64*)(me + TM_OFF);
            unsigned told = 0; Spin sp;
            for (;;) {
                const u64 v = __hip_atomic_load(a2, __ATOMIC_RELAXED, __HIP_MEMORY_SCOPE_SYSTEM);
                const bool in = lane < 16 && (unsigned)(v >> 32) == call;
                unsigned g = in ? (unsigned)v : 0u;
                g = max(g, (unsigned)__builtin_amdgcn_update_dpp(0u, g, 0x111, 0xf, 0xf, true));
                g = max(g, (unsigned)__builtin_amdgcn_update_dpp(0u, g, 0x112, 0xf, 0xf, true));
                g = max(g, (unsigned)__builtin_amdgcn_update_dpp(0u, g, 0x114, 0xf, 0xf, true));
                g = max(g, (unsigned)__builtin_amdgcn_update_dpp(0u, g, 0x118, 0xf, 0xf, true));
                g = __builtin_amdgcn_readlane(g, 15);
                const unsigned im = (unsigned)__builtin_amdgcn_ballot_w64(in);
                unsigned st = (im & 0xffu) == 0xffu ? 1u : 0u;
                if (((im | (im >> 8)) & 0xffu) != 0xffu) g = 0u;
                if ((unsigned)__builtin_amdgcn_readlane((unsigned)v, 16) == call || (st == 0u && sp.over<255u>(tmo))) st = 2u;
                if (st != 0u || g != told) { told = g; *(volatile LAS u32x2*)(lds + LC_OFF + C_GMAX * 4) = (u32x2){g, st}; }
                if (st != 0u) break;
                __builtin_amdgcn_s_sleep(1);
            }
        }
    }
}

}

extern "C" void kernel_launch(int rank, int nranks, ncclComm_t comm,
                              hipStream_t stream,
                              void* const* d_in, const int* in_sizes, int n_in,
                              void* d_out, int out_size,
                              const int* in_dtypes, int out_dtype,
                              void* const* d_out_all,
                              void* const* d_ws_all, size_t ws_bytes) {
    if (nranks != NR || (unsigned)rank >= (unsigned)NR || ws_bytes < WS_NEED || n_in < 2) {
        fprintf(stderr, "unsupported configuration\n");
        return;
    }
    HIP_CHECK(hipFuncSetAttribute((const void*)ag_gemm, hipFuncAttributeMaxDynamicSharedMemorySize, LDS_BYTES));
    Params p;
    p.x = (const float*)d_in[0]; p.w = (const float*)d_in[1]; p.out = (signed char*)d_out;
    const size_t base = (ws_bytes - WS_NEED) & ~(size_t)0x1fffff;
    for (int i = 0; i < NR; ++i) p.peer[i] = (char*)d_ws_all[(rank + 1 + i) % NR] + base;
    p.rank = rank; p.pad = 0;
    hipLaunchKernelGGL(ag_gemm, dim3(NWG), dim3(512), LDS_BYTES, stream, p);
    HIP_CHECK(hipGetLastError());
}
```
